```python
import numpy as np

kernel_name = "gpu_ag_gemm_m4096_k4096_n4096_bf16_none"


ABSMAX_RTOL = 0.008

COMPUTE_DTYPE = "bfloat16"
EPILOGUE = "none"
IN_QUANT = None
OUT_QUANT = None
IN_DTYPES = {"x": "bfloat16", "w_mat": "bfloat16"}
WORLD_SIZE = 8

def _to_bf16(x):
    u = np.ascontiguousarray(x, dtype=np.float32).view(np.uint32)
    return ((u + 0x7FFF + ((u >> 16) & 1)) & 0xFFFF0000).view(np.float32)

def setup_inputs():

    x = _to_bf16(np.random.randn(4096, 4096).astype(np.float32))
    w_mat = _to_bf16((np.random.randn(4096, 4096) / np.sqrt(4096)).astype(np.float32))
    return {"x": x, "w_mat": w_mat}

def _epilogue(y):
    return y

def reference(x, w_mat):
    y = (x @ w_mat).astype(np.float32)
    return _epilogue(y)
```

```cpp
#include <hip/hip_runtime.h>
#include <rccl/rccl.h>
#include <cstdio>
#include <cstdint>

#define HIP_CHECK(expr) do { hipError_t _e = (expr); if (_e != hipSuccess) { \
    fprintf(stderr, "HIP error %s at %s:%d\n", hipGetErrorString(_e), __FILE__, __LINE__); \
    return; } } while (0)

namespace agk {

typedef unsigned int u32x4 __attribute__((ext_vector_type(4)));
typedef unsigned int u32x2 __attribute__((ext_vector_type(2)));
typedef short s16x4 __attribute__((ext_vector_type(4)));
typedef short s16x8 __attribute__((ext_vector_type(8)));
typedef __bf16 bf16x8 __attribute__((ext_vector_type(8)));
typedef float f32x4 __attribute__((ext_vector_type(4)));
#define LAS __attribute__((address_space(3)))
#define GAS __attribute__((address_space(1)))
typedef GAS unsigned int gu32;

constexpr int NR = 8;
constexpr int KD = 4096;
constexpr int MSH = 512;
constexpr int NSH = 512;
constexpr int NSTRIP = 32;
#define AGK_HS 21
constexpr int HS = AGK_HS;
constexpr int AS = NSTRIP - HS;
constexpr int SEGB = 8;
constexpr int SEGA = HS - SEGB;
constexpr int NKC = 16;
constexpr int NSTEP = 64;

#define AGK_CPL 16
constexpr int CPL = AGK_CPL;
constexpr int NCOMM = 7 * CPL;

constexpr int NA_T = (3 * SEGA + 7) / 8;
constexpr int NC_T = (4 * AS + 7) / 8;
constexpr int NB_T = 3 * SEGB / 4;
constexpr int OWN_FILL = (NA_T * 8 - 3 * SEGA) + (NC_T * 8 - 4 * AS);
constexpr int NO_T = (HS - OWN_FILL) / 8;
static_assert(NO_T * 8 + OWN_FILL == HS, "own strips must pack exactly");
static_assert((NO_T + NA_T + NC_T) * 8 + NB_T * 4 == 128, "strip count");
constexpr int NBIG_T = NO_T + NA_T + NC_T;
constexpr int NTILE_WG = NBIG_T * 16 + NB_T * 16;
constexpr int NWG = NCOMM + NTILE_WG;
static_assert(NWG <= 512, "all workgroups must be co-resident (2 per CU)");
static_assert(NCOMM % 8 == 0, "keep blockIdx % 8 meaningful");

constexpr size_t STRIP_BYTES = 16ull * KD * 2;
constexpr int NXSLOT = 3 * HS + 4 * AS;
constexpr size_t XS_BYTES = (size_t)NXSLOT * STRIP_BYTES;
constexpr size_t WSL_BYTES = (size_t)KD * 128 * 2;
constexpr size_t BUF_BYTES = XS_BYTES + 4 * WSL_BYTES;
constexpr size_t CTL_OFF = 2 * BUF_BYTES;
constexpr size_t XFLAG_OFF = CTL_OFF;
constexpr size_t WFLAG_OFF = XFLAG_OFF + 8 * 32 * 16 * 4;
constexpr size_t OCNT_OFF = WFLAG_OFF + 4 * 128 * 4;
constexpr size_t LDONE_OFF = OCNT_OFF + 4 * 128;
constexpr size_t TMO_OFF = LDONE_OFF + 4 * 128;
constexpr size_t CALL_OFF = TMO_OFF + 128;
constexpr size_t DBG_OFF = CALL_OFF + 512 * 4;
constexpr size_t WS_NEED = DBG_OFF + 512 * 64;

constexpr unsigned long long DEADLINE_TICKS = 30000000ull;
constexpr int LDS_BYTES = 81920;
constexpr unsigned OUT_ADDS = (NBIG_T + NB_T) * 2 * 4;

struct Params {
    const char* x;
    const char* w;
    char* ws[NR];
    char* out[NR];
    int rank;
    unsigned char strip[128];
};

__device__ __forceinline__ unsigned ld_sys(const gu32* p) { return __hip_atomic_load(p, __ATOMIC_RELAXED, __HIP_MEMORY_SCOPE_SYSTEM); }
__device__ __forceinline__ unsigned ld_agent(const gu32* p) { return __hip_atomic_load(p, __ATOMIC_RELAXED, __HIP_MEMORY_SCOPE_AGENT); }
__device__ __forceinline__ void st_sys(gu32* p, unsigned v) { __hip_atomic_store(p, v, __ATOMIC_RELAXED, __HIP_MEMORY_SCOPE_SYSTEM); }
__device__ __forceinline__ void st_agent(gu32* p, unsigned v) { __hip_atomic_store(p, v, __ATOMIC_RELAXED, __HIP_MEMORY_SCOPE_AGENT); }
__device__ __forceinline__ unsigned long long now() { return __builtin_amdgcn_s_memrealtime(); }

__device__ __forceinline__ void st16_wt(GAS char* p, u32x4 v) {
    asm volatile("global_store_dwordx4 %0, %1, off sc0 sc1\n\ts_nop 1" :: "v"(p), "v"(v) : "memory");
}
template <int N> __device__ __forceinline__ void wait_vm() { asm volatile("s_waitcnt vmcnt(%0)" :: "i"(N) : "memory"); }
template <int N> __device__ __forceinline__ void wait_vm_lgkm() { asm volatile("s_waitcnt vmcnt(%0) lgkmcnt(0)" :: "i"(N) : "memory"); }
__device__ __forceinline__ void bar() { asm volatile("s_barrier" ::: "memory"); }

__device__ __forceinline__ void glds(const GAS char* src, unsigned lds_dst) {
    unsigned keep;
#define AGK_LDPOL "sc0 sc1"
    asm volatile("s_mov_b32 %0, m0\n\ts_mov_b32 m0, %2\n\ts_nop 0\n\tglobal_load_lds_dwordx4 %1, off " AGK_LDPOL "\n\ts_mov_b32 m0, %0"
                 : "=&s"(keep) : "v"(src), "s"(lds_dst) : "memory");
}

__device__ __forceinline__ unsigned pack_bf16(float lo, float hi) {
    typedef float f32x2 __attribute__((ext_vector_type(2)));
    typedef __bf16 bf16x2 __attribute__((ext_vector_type(2)));
    f32x2 v = {lo, hi};
    bf16x2 r = __builtin_convertvector(v, bf16x2);
    return __builtin_bit_cast(unsigned, r);
}

__device__ __forceinline__ int xslot(int j, int s) { return j < 4 ? (j - 1) * HS + s : 3 * HS + (j - 4) * AS + (s - HS); }

#define STAMP(i) do { } while (0)

__device__ __forceinline__ void comm_role(const Params& P, LAS char* smem, unsigned call, int b) {
    GAS char* myws = (GAS char*)P.ws[0];
    const int j = 1 + b % 7, u = b / 7;
    const int lane = threadIdx.x & 63;
    const int wv = __builtin_amdgcn_readfirstlane(threadIdx.x >> 6);
    GAS char* dws = (GAS char*)P.ws[j];
    GAS char* dbuf = dws + (size_t)(call & 1) * BUF_BYTES;
    const GAS char* xin = (const GAS char*)P.x;
    const GAS char* win = (const GAS char*)P.w;
    LAS unsigned* cnt = (LAS unsigned*)smem;
    if (threadIdx.x < 64) cnt[threadIdx.x] = 0;
    __syncthreads();
    const int npiece = j < 4 ? HS * NKC : 128 + AS * NKC;

    auto locate = [&](int p, const GAS char*& src, GAS char*& dst, gu32*& flag, int& sstep, int& dstep) {
        if (j >= 4 && p < 128) {
            const int k = p * 32 + wv * 8 + (lane >> 4), c16 = lane & 15;
            src = win + (size_t)k * (NSH * 2) + (j & 3) * 256 + c16 * 16;
            dst = dbuf + XS_BYTES + (size_t)(j - 4) * WSL_BYTES + (size_t)(c16 >> 3) * (WSL_BYTES / 2) + (size_t)k * 128 + (c16 & 7) * 16;
            sstep = 4 * NSH * 2; dstep = 4 * 128;
            flag = (gu32*)(dws + WFLAG_OFF) + (j - 4) * 128 + p;
        } else {
            int s, kc;
            if (j < 4) {
                if (p < SEGA * NKC) { kc = p / SEGA; s = p - kc * SEGA; }
                else { const int q = p - SEGA * NKC; kc = q / SEGB; s = SEGA + q - kc * SEGB; }
            } else { const int q = p - 128; kc = q / AS; s = HS + q - kc * AS; }
            const int row = wv * 4 + (lane >> 5), c16 = lane & 31;
            src = xin + (size_t)(s * 16 + row) * (KD * 2) + kc * 512 + c16 * 16;
            dst = dbuf + (size_t)xslot(j, s) * STRIP_BYTES + kc * 8192 + row * 512 + c16 * 16;
            sstep = 2 * KD * 2; dstep = 2 * 512;
            flag = (gu32*)(dws + XFLAG_OFF) + (j * 32 + s) * NKC + kc;
        }
    };

    const GAS char* src; GAS char* dst; gu32* flag; int sstep, dstep;
    u32x4 v0, v1;
    int p = u;
    locate(p, src, dst, flag, sstep, dstep);
    v0 = *(const GAS u32x4*)src; v1 = *(const GAS u32x4*)(src + sstep);
    for (int it = 0; p < npiece; ++it) {
        st16_wt(dst, v0); st16_wt(dst + dstep, v1);
        gu32* f = flag;
        p += CPL;
        if (p < npiece) {
            locate(p, src, dst, flag, sstep, dstep);
            v0 = *(const GAS u32x4*)src; v1 = *(const GAS u32x4*)(src + sstep);
        }
        wait_vm<0>();
        if (lane == 0) {
            const unsigned old = __hip_atomic_fetch_add(cnt + it, 1u, __ATOMIC_RELAXED, __HIP_MEMORY_SCOPE_WORKGROUP);
            if (old == 3) st_sys(f, call);
        }
    }
    STAMP(1);
    if (b == 0) {
        if (threadIdx.x < 4) {
            const gu32* c = (const gu32*)(myws + OCNT_OFF + threadIdx.x * 128);
            gu32* tmo = (gu32*)(myws + TMO_OFF);
            const unsigned need = call * OUT_ADDS;
            const unsigned long long t0 = now();
            bool bad = false;
            while (ld_sys(c) < need) {
                if (ld_agent(tmo) == call || now() - t0 > DEADLINE_TICKS) { st_agent(tmo, call); bad = true; break; }
                __builtin_amdgcn_s_sleep(1);
            }
            if (bad) {
                __builtin_amdgcn_s_sleep(127);
                __hip_atomic_store((GAS unsigned short*)(P.out[0] + (size_t)(P.rank * MSH) * (NSH * 2)), (unsigned short)0x7FC0, __ATOMIC_RELAXED, __HIP_MEMORY_SCOPE_AGENT);
            }
        }
        STAMP(2);
    }
}

template <int SPW, int NSLOT>
__device__ __forceinline__ void tile_role(const Params& P, LAS char* smem, unsigned call, int first_strip, int cg, int ch) {
    constexpr int D = NSLOT - 1;
    constexpr int NI = 2 * SPW + 2;
    constexpr int A_BYTES = SPW * 4 * 2048;
    constexpr int SLOT_BYTES = A_BYTES + 8192;
    static_assert(SLOT_BYTES * NSLOT <= LDS_BYTES, "LDS ring");
    GAS char* myws = (GAS char*)P.ws[0];
    const GAS char* mybuf = myws + (size_t)(call & 1) * BUF_BYTES;
    gu32* tmo = (gu32*)(myws + TMO_OFF);
    const int lane = threadIdx.x & 63;
    const int wv = __builtin_amdgcn_readfirstlane(threadIdx.x >> 6);
    const unsigned lds0 = (unsigned)(size_t)smem;

    const GAS char* asrc[SPW][2];
    int ajump[SPW];
    int sj[SPW], ss[SPW];
#pragma unroll
    for (int i = 0; i < SPW; ++i) {
        const int code = P.strip[first_strip + wv * SPW + i];
        sj[i] = code >> 5; ss[i] = code & 31;
#pragma unroll
        for (int h = 0; h < 2; ++h) {
            const int row = h * 8 + (lane >> 3);
            const int c16 = (lane & 7) ^ ((row >> 1) & 7);
            asrc[i][h] = sj[i] == 0 ? (const GAS char*)P.x + (size_t)(ss[i] * 16 + row) * (KD * 2) + c16 * 16
                                    : mybuf + (size_t)xslot(sj[i], ss[i]) * STRIP_BYTES + row * 512 + c16 * 16;
        }
        ajump[i] = sj[i] == 0 ? 128 : 8192 - 384;
    }
    const bool wremote = cg >= 4;
    const GAS char* bsrc[2];
#pragma unroll
    for (int h = 0; h < 2; ++h) {
        const int k = wv * 16 + h * 8 + (lane >> 3);
        const int f = ((k >> 1) & 1) | (((k >> 3) & 1) << 1);
        const int c16 = (lane & 7) ^ (f << 1);
        bsrc[h] = wremote ? mybuf + XS_BYTES + (size_t)(cg - 4) * WSL_BYTES + (size_t)ch * (WSL_BYTES / 2) + k * 128 + c16 * 16
                          : (const GAS char*)P.w + (size_t)k * (NSH * 2) + cg * 256 + ch * 128 + c16 * 16;
    }
    const int bstep = wremote ? 64 * 128 : 64 * NSH * 2;

    const gu32* fptr; bool fown;
    {
        const int i = (SPW == 2 && (lane & 16)) ? SPW - 1 : 0;
        const int fj = (SPW == 2 && (lane & 16)) ? sj[SPW - 1] : sj[0];
        const int fs = (SPW == 2 && (lane & 16)) ? ss[SPW - 1] : ss[0];
        (void)i;
        if (lane < 32) { fptr = (const gu32*)(myws + XFLAG_OFF) + (fj * 32 + fs) * NKC + (lane & 15); fown = fj == 0; }
        else { fptr = (const gu32*)(myws + WFLAG_OFF) + (wremote ? cg - 4 : 0) * 128 + (lane - 32) * 2 + (wv >> 1); fown = !wremote; }
    }
    int mark = 0;
    bool dead = false;
    auto ensure = [&](int t) {
        if (t < mark) return;
        const unsigned long long t0 = now();
        for (;;) {
            const int wbase = t & 32;
            const gu32* fp = lane < 32 ? fptr : fptr + wbase * 2;
            const unsigned v = ld_sys(fp);
            const unsigned long long m = __ballot(fown || v >= call);
            const unsigned mx = (unsigned)m & ((unsigned)m >> 16) & 0xFFFFu;
            const unsigned mw = (unsigned)(m >> 32);
            const int c0 = t >> 2;
            const int xr = (c0 + __builtin_ctz(~(mx >> c0))) * 4;
            const int w0 = t & 31;
            const unsigned long long mw64 = (unsigned long long)mw >> w0;
            const int wr = t + __builtin_ctzll(~mw64);
            mark = xr < wr ? xr : wr;
            if (t < mark) break;
            if (__builtin_amdgcn_readfirstlane(ld_agent(tmo)) == call || now() - t0 > DEADLINE_TICKS) {
                if (lane == 0) st_agent(tmo, call);
                dead = true; mark = NSTEP; break;
            }
            __builtin_amdgcn_s_sleep(4);
        }
    };
    auto issue = [&](int t, int slot) {
        const unsigned base = lds0 + slot * SLOT_BYTES;
#pragma unroll
        for (int i = 0; i < SPW; ++i) {
#pragma unroll
            for (int h = 0; h < 2; ++h) glds(asrc[i][h], base + (wv * SPW + i) * 2048 + h * 1024);
            const int inc = (t & 3) == 3 ? ajump[i] : 128;
            asrc[i][0] += inc; asrc[i][1] += inc;
        }
#pragma unroll
        for (int h = 0; h < 2; ++h) { glds(bsrc[h], base + A_BYTES + wv * 2048 + h * 1024); bsrc[h] += bstep; }
    };

    const int fr = lane & 15, g = lane >> 4;
    unsigned aoff[2];
#pragma unroll
    for (int kk = 0; kk < 2; ++kk) aoff[kk] = wv * SPW * 2048 + fr * 128 + (((kk * 4 + g) ^ ((fr >> 1) & 7)) << 4);
    unsigned boff[4];
    {
        const int q = fr >> 2, pp = lane & 3, f = (q >> 1) | ((g & 1) << 1);
#pragma unroll
        for (int nb = 0; nb < 4; ++nb) boff[nb] = A_BYTES + (8 * g + q) * 128 + ((nb ^ f) << 5) + pp * 8;
    }
    f32x4 acc[SPW][4];
#pragma unroll
    for (int i = 0; i < SPW; ++i)
#pragma unroll
        for (int nb = 0; nb < 4; ++nb) acc[i][nb] = (f32x4){0.f, 0.f, 0.f, 0.f};

    auto compute = [&](int slot) {
        LAS char* sb = smem + slot * SLOT_BYTES;
#pragma unroll
        for (int kk = 0; kk < 2; ++kk) {
            bf16x8 af[SPW];
#pragma unroll
            for (int i = 0; i < SPW; ++i) af[i] = *(LAS bf16x8*)(sb + aoff[kk] + i * 2048);
#pragma unroll
            for (int nb = 0; nb < 4; ++nb) {
                const s16x4 lo = __builtin_amdgcn_ds_read_tr16_b64_v4i16((LAS s16x4*)(sb + boff[nb] + kk * 4096));
                const s16x4 hi = __builtin_amdgcn_ds_read_tr16_b64_v4i16((LAS s16x4*)(sb + boff[nb] + kk * 4096 + 512));
                const s16x8 bs = __builtin_shufflevector(lo, hi, 0, 1, 2, 3, 4, 5, 6, 7);
                const bf16x8 bfr = __builtin_bit_cast(bf16x8, bs);
#pragma unroll
                for (int i = 0; i < SPW; ++i) acc[i][nb] = __builtin_amdgcn_mfma_f32_16x16x32_bf16(bfr, af[i], acc[i][nb], 0, 0, 0);
            }
        }
    };

    ensure(0);
    STAMP(1);
#pragma unroll
    for (int t = 0; t < D; ++t) { ensure(t); issue(t, t); }
    int cs = 0, is = D;
    for (int t = 0; t < NSTEP - D; ++t) {
        wait_vm_lgkm<(D - 1) * NI>();
        bar();
        ensure(t + D);
        issue(t + D, is);
        compute(cs);
        cs = cs + 1 == NSLOT ? 0 : cs + 1;
        is = is + 1 == NSLOT ? 0 : is + 1;
    }
#pragma unroll
    for (int i = 0; i < D; ++i) {
        if (i == 0) wait_vm<(D - 1) * NI>();
        else if (i == 1) wait_vm<(D > 1 ? D - 2 : 0) * NI>();
        else if (i == 2) wait_vm<(D > 2 ? D - 3 : 0) * NI>();
        else if (i == 3) wait_vm<(D > 3 ? D - 4 : 0) * NI>();
        else wait_vm<0>();
        bar();
        compute(cs);
        cs = cs + 1 == NSLOT ? 0 : cs + 1;
    }
    STAMP(2);
    asm volatile("s_waitcnt lgkmcnt(0)" ::: "memory");
    bar();
    dead = dead || __builtin_amdgcn_readfirstlane(ld_agent(tmo)) == call;
    LAS char* eb = smem + wv * (SPW * 2048);
#pragma unroll
    for (int i = 0; i < SPW; ++i)
#pragma unroll
        for (int nb = 0; nb < 4; ++nb) {
            u32x2 o;
            o.x = pack_bf16(acc[i][nb][0], acc[i][nb][1]);
            o.y = pack_bf16(acc[i][nb][2], acc[i][nb][3]);
            if (dead) { o.x = 0x7FC07FC0u; o.y = 0x7FC07FC0u; }
            *(LAS u32x2*)(eb + i * 2048 + fr * 128 + ((((nb * 2 + (g >> 1)) ^ (fr & 7))) << 4) + (g & 1) * 8) = o;
        }
    GAS char* obase = (GAS char*)P.out[wremote ? cg : 0] + (cg & 3) * 256 + ch * 128 + (lane & 7) * 16;
#pragma unroll
    for (int i = 0; i < SPW; ++i)
#pragma unroll
        for (int h = 0; h < 2; ++h) {
            const int row = h * 8 + (lane >> 3);
            const u32x4 v = *(LAS u32x4*)(eb + i * 2048 + row * 128 + (((lane & 7) ^ (row & 7)) << 4));
            const int grow = (P.rank ^ sj[i]) * MSH + ss[i] * 16 + row;
            GAS char* dst = obase + (size_t)grow * (NSH * 2);
            if (wremote) st16_wt(dst, v); else *(GAS u32x4*)dst = v;
        }
    if (wremote) {
        wait_vm<0>();
        if (lane == 0) __hip_atomic_fetch_add((gu32*)((GAS char*)P.ws[cg] + OCNT_OFF + (cg - 4) * 128), 1u, __ATOMIC_RELAXED, __HIP_MEMORY_SCOPE_SYSTEM);
    }
    STAMP(3);
}

__global__ void __launch_bounds__(256, 2) ag_gemm_kernel(const Params P) {
    __shared__ __attribute__((aligned(1024))) char smem_raw[LDS_BYTES];
    LAS char* smem = (LAS char*)smem_raw;
    GAS char* myws = (GAS char*)P.ws[0];
    const int b = blockIdx.x;
    gu32* cc = (gu32*)(myws + CALL_OFF) + b;
    const unsigned call = __builtin_amdgcn_readfirstlane(ld_agent(cc)) + 1u;
    STAMP(0);
    if (b < NCOMM) {
        comm_role(P, smem, call, b);
    } else {
        const int i = b - NCOMM;
        const int cg = i & 7, r = i >> 3, ch = r & 1, tr = r >> 1;
        if (tr < NBIG_T) tile_role<2, 3>(P, smem, call, tr * 8, cg, ch);
        else tile_role<1, 5>(P, smem, call, NBIG_T * 8 + (tr - NBIG_T) * 4, cg, ch);
    }
    __syncthreads();
    if (threadIdx.x == 0) st_agent(cc, call);
}

}

extern "C" void kernel_launch(int rank, int nranks, ncclComm_t comm,
                              hipStream_t stream,
                              void* const* d_in, const int* in_sizes, int n_in,
                              void* d_out, int out_size,
                              const int* in_dtypes, int out_dtype,
                              void* const* d_out_all,
                              void* const* d_ws_all, size_t ws_bytes) {
    using namespace agk;
    (void)comm; (void)in_dtypes; (void)out_dtype; (void)d_out;
    if (nranks != NR || (unsigned)rank >= (unsigned)NR || n_in < 2 || ws_bytes < WS_NEED ||
        in_sizes[0] != MSH * KD || in_sizes[1] != KD * NSH || out_size != NR * MSH * NSH) {
        fprintf(stderr, "ag_gemm: unexpected problem shape\n");
        return;
    }
    Params P;
    P.x = (const char*)d_in[0];
    P.w = (const char*)d_in[1];
    P.rank = rank;
    for (int j = 0; j < NR; ++j) { P.ws[j] = (char*)d_ws_all[rank ^ j]; P.out[j] = (char*)d_out_all[rank ^ j]; }
    int n = 0, own = 0;
    for (int i = 0; i < NO_T * 8; ++i) P.strip[n++] = (unsigned char)(own++);
    for (int j = 1; j < 4; ++j) for (int s = 0; s < SEGA; ++s) P.strip[n++] = (unsigned char)((j << 5) | s);
    while (n % 8) P.strip[n++] = (unsigned char)(own++);
    for (int j = 4; j < 8; ++j) for (int s = HS; s < NSTRIP; ++s) P.strip[n++] = (unsigned char)((j << 5) | s);
    while (n % 8) P.strip[n++] = (unsigned char)(own++);
    for (int j = 1; j < 4; ++j) for (int s = SEGA; s < HS; ++s) P.strip[n++] = (unsigned char)((j << 5) | s);
    hipLaunchKernelGGL(ag_gemm_kernel, dim3(NWG), dim3(256), 0, stream, P);
    HIP_CHECK(hipGetLastError());
}
```

```cpp
#include <hip/hip_runtime.h>
#include <rccl/rccl.h>
#include <cstdio>
#include <cstdint>

#define HIP_CHECK(expr) do { hipError_t _e = (expr); if (_e != hipSuccess) { \
    fprintf(stderr, "HIP error %s at %s:%d\n", hipGetErrorString(_e), __FILE__, __LINE__); \
    return; } } while (0)

namespace agk {

typedef unsigned int u32x4 __attribute__((ext_vector_type(4)));
typedef unsigned int u32x2 __attribute__((ext_vector_type(2)));
typedef short s16x4 __attribute__((ext_vector_type(4)));
typedef short s16x8 __attribute__((ext_vector_type(8)));
typedef __bf16 bf16x8 __attribute__((ext_vector_type(8)));
typedef float f32x4 __attribute__((ext_vector_type(4)));
#define LAS __attribute__((address_space(3)))
#define GAS __attribute__((address_space(1)))
typedef GAS unsigned int gu32;

constexpr int NR = 8;
constexpr int KD = 4096;
constexpr int MSH = 512;
constexpr int NSH = 512;
constexpr int NSTRIP = 32;
constexpr int HS = 21;
constexpr int AS = NSTRIP - HS;
constexpr int SEGB = 8;
constexpr int SEGA = HS - SEGB;
constexpr int NKC = 16;
constexpr int NSTEP = 64;
constexpr int NWP = 128;

#define AGK_CPL 8
constexpr int CPL = AGK_CPL;
constexpr int NCOMM = 7 * CPL;
constexpr int NP_IN = HS * NKC;
constexpr int NP_X = NWP + AS * NKC;

constexpr int NA_T = (3 * SEGA + 7) / 8;
constexpr int NC_T = (4 * AS + 7) / 8;
constexpr int NB_T = 3 * SEGB / 4;
constexpr int FILL_A = NA_T * 8 - 3 * SEGA, FILL_C = NC_T * 8 - 4 * AS;
constexpr int NO_T = (HS - FILL_A - FILL_C) / 8;
static_assert(NO_T * 8 + FILL_A + FILL_C == HS, "own strips must pack exactly");
static_assert((NO_T + NA_T + NC_T) * 8 + NB_T * 4 == 128, "strip count");
constexpr int NBIG_T = NO_T + NA_T + NC_T;
constexpr int NWG = 256;
enum { R_IDLE = 0, R_COMM = 1, R_T128 = 2, R_T128H = 3, R_T64H = 4 };

constexpr size_t STRIP_BYTES = 16ull * KD * 2;
constexpr int NXSLOT = 3 * HS + 4 * AS;
constexpr size_t XS_BYTES = (size_t)NXSLOT * STRIP_BYTES;
constexpr size_t WSL_BYTES = (size_t)KD * 128 * 2;
constexpr size_t BUF_BYTES = XS_BYTES + 4 * WSL_BYTES;
constexpr size_t CTL_OFF = 2 * BUF_BYTES;
constexpr int XF_STRIDE = 64;
constexpr size_t XFLAG_OFF = CTL_OFF;
constexpr size_t WFLAG_OFF = XFLAG_OFF + 8 * 32 * XF_STRIDE * 4;
constexpr size_t OCNT_OFF = WFLAG_OFF + 4 * NWP * 4;
constexpr size_t LDONE_OFF = OCNT_OFF + 4 * 256;
constexpr size_t TICK_OFF = LDONE_OFF + 4 * 256;
constexpr size_t TMO_OFF = TICK_OFF + 8 * 256;
constexpr size_t DUMMY_OFF = TMO_OFF + 256;
constexpr size_t CALL_OFF = DUMMY_OFF + 256 * 64;
constexpr size_t DBG_OFF = CALL_OFF + 256 * 64;
constexpr size_t WS_NEED = DBG_OFF + 256 * 128;

constexpr unsigned long long DEADLINE_TICKS = 30000000ull;
constexpr int LDS_BYTES = 163840;

struct Params {
    const char* x;
    const char* w;
    char* ws[NR];
    char* out[NR];
    int rank;
    unsigned out_adds;
    unsigned role[NWG];
    unsigned role2[NWG];
    unsigned char strip[128];
};

__device__ __forceinline__ unsigned ld_sys(const gu32* p) { return __hip_atomic_load(p, __ATOMIC_RELAXED, __HIP_MEMORY_SCOPE_SYSTEM); }
__device__ __forceinline__ unsigned ld_agent(const gu32* p) { return __hip_atomic_load(p, __ATOMIC_RELAXED, __HIP_MEMORY_SCOPE_AGENT); }
__device__ __forceinline__ void st_agent(gu32* p, unsigned v) { __hip_atomic_store(p, v, __ATOMIC_RELAXED, __HIP_MEMORY_SCOPE_AGENT); }
__device__ __forceinline__ unsigned long long now() { return __builtin_amdgcn_s_memrealtime(); }
__device__ __forceinline__ unsigned rfl(unsigned v) { return __builtin_amdgcn_readfirstlane(v); }

__device__ __forceinline__ void st16_wt(GAS char* p, u32x4 v) {
    asm volatile("global_store_dwordx4 %0, %1, off sc0 sc1\n\ts_nop 1" :: "v"(p), "v"(v) : "memory");
}
template <int N> __device__ __forceinline__ void wait_vm() { asm volatile("s_waitcnt vmcnt(%0)" :: "i"(N) : "memory"); }
__device__ __forceinline__ void wait_lgkm() { asm volatile("" ::: "memory"); __builtin_amdgcn_s_waitcnt(0xC07F); asm volatile("" ::: "memory"); }
#define AGK_SL1 3
#define AGK_SL2 2
#define AGK_SL3 24
#define AGK_FAST_SPINS 40
#define PIN() __builtin_amdgcn_sched_barrier(0)
__device__ __forceinline__ void bar() { asm volatile("s_barrier" ::: "memory"); }

__device__ __forceinline__ void glds(const GAS char* src, unsigned lds_dst) {
    unsigned keep;
#define AGK_LDPOL "sc0 sc1"
    asm volatile("s_mov_b32 %0, m0\n\ts_mov_b32 m0, %2\n\ts_nop 0\n\tglobal_load_lds_dwordx4 %1, off " AGK_LDPOL "\n\ts_mov_b32 m0, %0"
                 : "=&s"(keep) : "v"(src), "s"(lds_dst) : "memory");
}

__device__ __forceinline__ unsigned pack_bf16(float lo, float hi) {
    typedef float f32x2 __attribute__((ext_vector_type(2)));
    typedef __bf16 bf16x2 __attribute__((ext_vector_type(2)));
    f32x2 v = {lo, hi};
    bf16x2 r = __builtin_convertvector(v, bf16x2);
    return __builtin_bit_cast(unsigned, r);
}

__device__ __forceinline__ int xslot(int j, int s) { return j < 4 ? (j - 1) * HS + s : 3 * HS + (j - 4) * AS + (s - HS); }

#define STAMP(i) do { } while (0)

template <int N, int OFF> __device__ __forceinline__ void st_ld(u32x4& v, unsigned vdst, const GAS char* dst, unsigned vsrc, const GAS char* src) {
    asm volatile("s_waitcnt vmcnt(%5)\n\tglobal_store_dwordx4 %1, %0, %2 offset:%6 sc0 sc1\n\ts_nop 1\n\tglobal_load_dwordx4 %0, %3, %4"
                 : "+v"(v) : "v"(vdst), "s"(dst), "v"(vsrc), "s"(src), "i"(N), "i"(OFF) : "memory");
}
template <int N, int OFF> __device__ __forceinline__ void st_only(u32x4& v, unsigned vdst, const GAS char* dst) {
    asm volatile("s_waitcnt vmcnt(%3)\n\tglobal_store_dwordx4 %1, %0, %2 offset:%4 sc0 sc1\n\ts_nop 1"
                 : "+v"(v) : "v"(vdst), "s"(dst), "i"(N), "i"(OFF) : "memory");
}
__device__ __forceinline__ void flag_st(const gu32* f, unsigned val, unsigned zero) {
    asm volatile("s_mov_b64 exec, 1\n\tglobal_store_dword %0, %1, %2 sc0 sc1\n\ts_mov_b64 exec, -1" :: "v"(zero), "v"(val), "s"(f) : "memory");
}
__device__ __forceinline__ void draw(unsigned& t, const gu32* tk, unsigned one, unsigned zero) {
    asm volatile("s_mov_b64 exec, 1\n\tglobal_atomic_add %0, %1, %2, %3 sc0\n\ts_mov_b64 exec, -1" : "+v"(t) : "v"(zero), "v"(one), "s"(tk) : "memory");
}
__device__ __forceinline__ void add_local(const gu32* c, unsigned one, unsigned zero) {
    asm volatile("s_mov_b64 exec, 1\n\tglobal_atomic_add %0, %1, %2\n\ts_mov_b64 exec, -1" :: "v"(zero), "v"(one), "s"(c) : "memory");
}

struct Piece { unsigned soff, doff, foff; int sstep; };

__device__ __forceinline__ Piece locate(int j, int p) {
    Piece r;
    if (j >= 4 && p < NWP) {
        r.soff = 0x80000000u | (unsigned)(p * 32 * (NSH * 2) + (j & 3) * 256);
        r.doff = (unsigned)XS_BYTES + (unsigned)(j - 4) * (unsigned)WSL_BYTES + p * 8192;
        r.foff = (unsigned)(WFLAG_OFF - CTL_OFF) + ((j - 4) * NWP + (p & 1) * 64 + (p >> 1)) * 4;
        r.sstep = 4 * NSH * 2;
    } else {
        int s, kc;
        if (j < 4) {
            if (p < SEGA * NKC) { kc = p / SEGA; s = p - kc * SEGA; }
            else { const int q = p - SEGA * NKC; kc = q / SEGB; s = SEGA + q - kc * SEGB; }
        } else { const int q = p - NWP; kc = q / AS; s = HS + q - kc * AS; }
        r.soff = (unsigned)(s * 16 * (KD * 2) + kc * 512);
        r.doff = (unsigned)xslot(j, s) * (unsigned)STRIP_BYTES + kc * 8192;
        r.foff = ((j * 32 + s) * XF_STRIDE + kc) * 4;
        r.sstep = 2 * KD * 2;
    }
    return r;
}

__device__ __forceinline__ unsigned comm_role(const Params& P, const gu32* cc, int b, int j, int u) {
    GAS char* myws = (GAS char*)P.ws[0];
    const int lane = threadIdx.x & 63;
    const int npiece = j < 4 ? NP_IN : NP_X;
    const unsigned vdst = lane * 16;
    const unsigned vx = (lane >> 5) * (KD * 2) + (lane & 31) * 16, vw = (lane >> 4) * (NSH * 2) + (lane & 15) * 16;
    auto srcp = [&](const Piece& q) { return (q.soff >> 31) ? (const GAS char*)P.w + (q.soff & 0x7FFFFFFFu) : (const GAS char*)P.x + q.soff; };
    int pB = u + CPL;
    Piece A = locate(j, u);
    u32x4 v[8];
    {
        const GAS char* s = srcp(A);
        const unsigned vs = (A.soff >> 31) ? vw : vx;
#pragma unroll
        for (int k = 0; k < 8; ++k) asm volatile("global_load_dwordx4 %0, %1, %2" : "=v"(v[k]) : "v"(vs), "s"(s + k * A.sstep) : "memory");
    }
    const unsigned call = rfl(ld_agent(cc)) + 1u;
    wait_vm<0>();
    STAMP(0);
    const GAS char* dbuf = (const GAS char*)P.ws[j] + (size_t)(call & 1) * BUF_BYTES + 4096;
    const GAS char* dctl = (const GAS char*)P.ws[j] + CTL_OFF;
    const gu32* tk = (const gu32*)(myws + TICK_OFF + j * 256);
    const unsigned base = (call - 1u) * (unsigned)npiece - 2u * CPL;
    unsigned one = 1, zero = 0, callv = call, t = 0;
    asm volatile("" : "+v"(one), "+v"(zero), "+v"(callv), "+v"(t));
    const gu32* fprev = (const gu32*)(myws + DUMMY_OFF + b * 64);
    while (pB < npiece) {
        const Piece B = locate(j, pB);
        const GAS char* d = dbuf + A.doff;
        const GAS char* s = srcp(B);
        const unsigned vs = (B.soff >> 31) ? vw : vx;
        draw(t, tk, one, zero);
        st_ld<16, -4096>(v[0], vdst, d, vs, s);
        st_ld<16, -3072>(v[1], vdst, d, vs, s + 1 * B.sstep);
        st_ld<16, -2048>(v[2], vdst, d, vs, s + 2 * B.sstep);
        st_ld<16, -1024>(v[3], vdst, d, vs, s + 3 * B.sstep);
        st_ld<16, 0>(v[4], vdst, d, vs, s + 4 * B.sstep);
        st_ld<16, 1024>(v[5], vdst, d, vs, s + 5 * B.sstep);
        st_ld<16, 2048>(v[6], vdst, d, vs, s + 6 * B.sstep);
        wait_vm<15>();
        flag_st(fprev, callv, zero);
        st_ld<63, 3072>(v[7], vdst, d, vs, s + 7 * B.sstep);
        fprev = (const gu32*)(dctl + A.foff);
        asm volatile("s_waitcnt vmcnt(17)" : "+v"(t) :: "memory");
        pB = (int)(rfl(t) - base);
        A = B;
    }
    {
        const GAS char* d = dbuf + A.doff;
        draw(t, tk, one, zero);
        st_only<16, -4096>(v[0], vdst, d);
        st_only<15, -3072>(v[1], vdst, d);
        st_only<14, -2048>(v[2], vdst, d);
        st_only<13, -1024>(v[3], vdst, d);
        st_only<12, 0>(v[4], vdst, d);
        st_only<11, 1024>(v[5], vdst, d);
        st_only<10, 2048>(v[6], vdst, d);
        wait_vm<8>();
        flag_st(fprev, callv, zero);
        st_only<63, 3072>(v[7], vdst, d);
        if (j >= 4) add_local((const gu32*)(myws + LDONE_OFF + (j - 4) * 256), one, zero);
        wait_vm<0>();
        flag_st((const gu32*)(dctl + A.foff), callv, zero);
    }
    STAMP(1);
    if (j == 4 && u == 0) {
        if (lane < 4) {
            const gu32* c = (const gu32*)(myws + OCNT_OFF + lane * 256);
            gu32* tmo = (gu32*)(myws + TMO_OFF);
            const unsigned need = call * P.out_adds;
            const unsigned long long t0 = now();
            bool bad = false;
            unsigned spin = 0;
            while (ld_sys(c) < need) {
                if ((++spin & 63) == 0 && (ld_agent(tmo) == call || now() - t0 > DEADLINE_TICKS)) { st_agent(tmo, call); bad = true; break; }
                __builtin_amdgcn_s_sleep(1);
            }
            if (bad) {
                __builtin_amdgcn_s_sleep(127);
                __hip_atomic_store((GAS unsigned short*)(P.out[0] + (size_t)(P.rank * MSH) * (NSH * 2)), (unsigned short)0x7FC0, __ATOMIC_RELAXED, __HIP_MEMORY_SCOPE_AGENT);
            }
        }
        STAMP(2);
    }
    return call;
}

template <int S, int NH, int NSLOT>
__device__ __forceinline__ void tile_role(const Params& P, LAS char* smem, unsigned call, int first_strip, int cg, int ch, bool defer, bool more) {
    constexpr int D = NSLOT - 1;
    constexpr int SPL = S / 4;
    constexpr int NA = 2 * SPL, NB = 2 * NH, NI = NA + NB;
    constexpr int A_BYTES = S * 2048, SLOT_BYTES = A_BYTES + NH * 8192;
    constexpr int SPW = S / 2, NFR = NH * 2;
    static_assert(SLOT_BYTES * NSLOT <= LDS_BYTES, "LDS ring");
    static_assert(D * (NI + 2) <= 48, "vmcnt range");
    GAS char* myws = (GAS char*)P.ws[0];
    gu32* tmo = (gu32*)(myws + TMO_OFF);
    const int lane = threadIdx.x & 63;
    const int wv = rfl(threadIdx.x >> 6);
    const bool wremote = cg >= 4;

    if (wv >= 4) {
        const int lw = wv - 4;
        const GAS char* mybuf = myws + (size_t)(call & 1) * BUF_BYTES;
        const unsigned lds0 = (unsigned)(size_t)smem;
        const GAS char* asrc[NA];
        int ajump[SPL], sj[SPL], ss[SPL];
#pragma unroll
        for (int i = 0; i < SPL; ++i) {
            const int code = P.strip[first_strip + lw * SPL + i];
            sj[i] = code >> 5; ss[i] = code & 31;
#pragma unroll
            for (int h = 0; h < 2; ++h) {
                const int row = h * 8 + (lane >> 3);
                const int c16 = (lane & 7) ^ ((row >> 1) & 7);
                asrc[i * 2 + h] = sj[i] == 0 ? (const GAS char*)P.x + (size_t)(ss[i] * 16 + row) * (KD * 2) + c16 * 16
                                             : mybuf + (size_t)xslot(sj[i], ss[i]) * STRIP_BYTES + row * 512 + c16 * 16;
            }
            ajump[i] = sj[i] == 0 ? 128 : 8192 - 384;
        }
        const GAS char* bsrc[NB];
        const int bhalf = NH == 2 ? (lw >> 1) : ch;
        const int kb0 = NH == 2 ? (lw & 1) * 4 : lw * 2;
#pragma unroll
        for (int i = 0; i < NB; ++i) {
            const int k = (kb0 + i) * 8 + (lane >> 3);
            const int f = ((k >> 1) & 1) | (((k >> 3) & 1) << 1);
            const int c16 = (lane & 7) ^ (f << 1);
            bsrc[i] = wremote ? mybuf + XS_BYTES + (size_t)(cg - 4) * WSL_BYTES + k * 256 + bhalf * 128 + c16 * 16
                              : (const GAS char*)P.w + (size_t)k * (NSH * 2) + cg * 256 + bhalf * 128 + c16 * 16;
        }
        const int bstep = wremote ? 64 * 256 : 64 * NSH * 2;
        const unsigned ldsA = lds0 + lw * SPL * 2048;
        const unsigned ldsB = lds0 + A_BYTES + (NH == 2 ? (lw >> 1) * 8192 : 0) + kb0 * 1024;

        const gu32* fx; unsigned long long xown;
        {
            const int fi = (SPL == 2 && (lane & 16)) ? SPL - 1 : 0;
            const int fj = fi ? sj[SPL - 1] : sj[0], fs = fi ? ss[SPL - 1] : ss[0];
            fx = (const gu32*)(myws + XFLAG_OFF) + (fj * 32 + fs) * XF_STRIDE + (lane & 15);
            xown = __ballot(fj == 0 || lane >= 16 * SPL);
        }
        const gu32* fw = (const gu32*)(myws + WFLAG_OFF) + (wremote ? cg - 4 : 0) * NWP + (kb0 >> 2) * 64 + lane;
        const unsigned long long xact = ~xown;
        int xmark = xact == 0 ? NSTEP : 0, wmark = wremote ? 0 : NSTEP;
        int mark = xmark < wmark ? xmark : wmark;
        bool dead = false;
        asm volatile("v_mov_b32 v250, 0\n\tv_mov_b32 v251, 0" ::: "v250", "v251", "memory");
        auto poll = [&]() {
            asm volatile("s_cmp_lt_i32 %3, 64\n\ts_cselect_b64 exec, %2, 0\n\tglobal_load_dword v250, %0, off sc0 sc1\n\t"
                         "s_cmp_lt_i32 %4, 64\n\ts_cselect_b64 exec, -1, 0\n\tglobal_load_dword v251, %1, off sc0 sc1\n\ts_mov_b64 exec, -1"
                         :: "v"(fx), "v"(fw), "s"(xact), "s"(xmark), "s"(wmark) : "v250", "v251", "scc", "memory");
        };
        auto eval = [&]() {
            unsigned long long bx, bw;
            asm volatile("v_cmp_le_u32 %0, %2, v250\n\tv_cmp_gt_u32 %1, %2, v251" : "=&s"(bx), "=&s"(bw) : "s"(call) : "memory");
            const unsigned m = (unsigned)(bx | xown);
            xmark = rfl(__builtin_ctz(~(m & (m >> 16) & 0xFFFFu)) * 4);
            if (wmark < NSTEP) wmark = rfl(bw ? __builtin_ctzll(bw) : NSTEP);
            mark = xmark < wmark ? xmark : wmark;
        };
        auto ensure = [&](int t) {
            if (t < mark) return;
            unsigned long long t0 = 0;
            unsigned spin = 0;
            for (;;) {
                eval();
                if (t < mark) break;
                wait_vm<48>();
                poll();
                if (++spin < AGK_FAST_SPINS) __builtin_amdgcn_s_sleep(AGK_SL1); else __builtin_amdgcn_s_sleep(AGK_SL3);
                if ((spin & 63) == 0) {
                    if (t0 == 0) t0 = now();
                    if (rfl(ld_agent(tmo)) == call || now() - t0 > DEADLINE_TICKS) {
                        if (lane == 0) st_agent(tmo, call);
                        dead = true; mark = xmark = wmark = NSTEP; break;
                    }
                }
            }
        };
        auto issue = [&](int t, int slot) {
            const unsigned so = slot * SLOT_BYTES;
            if (!dead) {
#pragma unroll
                for (int i = 0; i < NA; ++i) glds(asrc[i], ldsA + so + i * 1024);
#pragma unroll
                for (int i = 0; i < NB; ++i) glds(bsrc[i], ldsB + so + i * 1024);
            } else {
                const u32x4 nan = {0x7FC07FC0u, 0x7FC07FC0u, 0x7FC07FC0u, 0x7FC07FC0u};
#pragma unroll
                for (int i = 0; i < NA; ++i) *(LAS u32x4*)(size_t)(ldsA + so + i * 1024 + lane * 16) = nan;
#pragma unroll
                for (int i = 0; i < NB; ++i) *(LAS u32x4*)(size_t)(ldsB + so + i * 1024 + lane * 16) = nan;
                wait_lgkm();
            }
#pragma unroll
            for (int i = 0; i < NA; ++i) asrc[i] += (t & 3) == 3 ? ajump[i >> 1] : 128;
#pragma unroll
            for (int i = 0; i < NB; ++i) bsrc[i] += bstep;
        };
        int is = 0;
        bool polled = mark < NSTEP;
#pragma nounroll
        for (int t = 0; t < NSTEP; ++t) {
            if (t >= D) {
                if (t == D) { wait_vm<(D - 1) * NI>(); bar(); }
                if (polled) wait_vm<(D - 2) * (NI + 1)>(); else wait_vm<(D - 2) * NI>();
                bar();
            }
            ensure(t);
            if (t == 0 && lw == 0) STAMP(1);
            issue(t, is);
            is = is + 1 == NSLOT ? 0 : is + 1;
            polled = mark < NSTEP;
            poll();
        }
        if (lw == 0) STAMP(4);
#pragma unroll
        for (int i = 0; i < D; ++i) {
            if (i == 0) wait_vm<(D - 2) * NI>();
            else if (i == 1) wait_vm<(D > 3 ? D - 3 : 0) * NI>();
            else if (i == 2) wait_vm<(D > 4 ? D - 4 : 0) * NI>();
            else if (i == 3) wait_vm<(D > 5 ? D - 5 : 0) * NI>();
            else if (i == 4) wait_vm<(D > 6 ? D - 6 : 0) * NI>();
            else if (i == 5) wait_vm<(D > 7 ? D - 7 : 0) * NI>();
            else wait_vm<0>();
            bar();
        }
        if (more) bar();
    } else {
        const int fr = lane & 15, g = lane >> 4;
        const int wm = wv >> 1, wn = wv & 1;
        unsigned aoff[2];
#pragma unroll
        for (int kk = 0; kk < 2; ++kk) aoff[kk] = wm * SPW * 2048 + fr * 128 + (((kk * 4 + g) ^ ((fr >> 1) & 7)) << 4);
        unsigned boff[NFR];
        {
            const int q = fr >> 2, pp = lane & 3, f = (q >> 1) | ((g & 1) << 1);
#pragma unroll
            for (int nb = 0; nb < NFR; ++nb) {
                const int gn = wn * NFR + nb;
                boff[nb] = A_BYTES + (gn >> 2) * 8192 + (8 * g + q) * 128 + (((gn & 3) ^ f) << 5) + pp * 8;
            }
        }
        int rowbase[SPW];
#pragma unroll
        for (int i = 0; i < SPW; ++i) {
            const int code = P.strip[first_strip + wm * SPW + i];
            rowbase[i] = rfl((P.rank ^ (code >> 5)) * MSH + (code & 31) * 16);
        }
        GAS char* outp = (GAS char*)P.out[wremote ? cg : 0];
        GAS char* cntp = (GAS char*)P.ws[cg] + OCNT_OFF + (cg & 3) * 256;
        asm volatile("" : "+s"(outp), "+s"(cntp));
        f32x4 acc[SPW][NFR];
#pragma unroll
        for (int i = 0; i < SPW; ++i)
#pragma unroll
            for (int nb = 0; nb < NFR; ++nb) acc[i][nb] = (f32x4){0.f, 0.f, 0.f, 0.f};
        struct Frag { bf16x8 a[SPW]; bf16x8 b[NFR]; };
        auto rd = [&](Frag& F, int slot, int kk) {
            LAS char* sb = smem + slot * SLOT_BYTES;
#pragma unroll
            for (int i = 0; i < SPW; ++i) F.a[i] = *(LAS bf16x8*)(sb + aoff[kk] + i * 2048);
#pragma unroll
            for (int nb = 0; nb < NFR; ++nb) {
                const s16x4 lo = __builtin_amdgcn_ds_read_tr16_b64_v4i16((LAS s16x4*)(sb + boff[nb] + kk * 4096));
                const s16x4 hi = __builtin_amdgcn_ds_read_tr16_b64_v4i16((LAS s16x4*)(sb + boff[nb] + kk * 4096 + 512));
                F.b[nb] = __builtin_bit_cast(bf16x8, __builtin_shufflevector(lo, hi, 0, 1, 2, 3, 4, 5, 6, 7));
            }
        };
        auto mm = [&](const Frag& F) {
#pragma unroll
            for (int nb = 0; nb < NFR; ++nb)
#pragma unroll
                for (int i = 0; i < SPW; ++i) acc[i][nb] = __builtin_amdgcn_mfma_f32_16x16x32_bf16(F.b[nb], F.a[i], acc[i][nb], 0, 0, 0);
        };
        Frag F0, F1;
        int cs = 0;
        bar();
        rd(F0, 0, 0);
        wait_lgkm();
#pragma nounroll
        for (int t = 0; t < NSTEP - 1; ++t) {
            bar();
            const int ns = cs + 1 == NSLOT ? 0 : cs + 1;
            rd(F1, cs, 1);
            PIN();
            mm(F0);
            PIN();
            rd(F0, ns, 0);
            PIN();
            mm(F1);
            PIN();
            wait_lgkm();
            cs = ns;
        }
        bar();
        rd(F1, cs, 1);
        PIN();
        mm(F0);
        PIN();
        mm(F1);
        if (wv == 0) STAMP(2);
        constexpr int ROWB = NFR * 32;
        constexpr int CPR = ROWB / 16;
        constexpr int ESLOT = (NSTEP - 1) % NSLOT == 0 ? 1 : 0;
        LAS char* eb = smem + ESLOT * SLOT_BYTES + wv * (SPW * 16 * ROWB);
        static_assert(4 * SPW * 16 * ROWB <= SLOT_BYTES, "epilogue space");
#pragma unroll
        for (int i = 0; i < SPW; ++i)
#pragma unroll
            for (int nb = 0; nb < NFR; ++nb) {
                u32x2 o;
                o.x = pack_bf16(acc[i][nb][0], acc[i][nb][1]);
                o.y = pack_bf16(acc[i][nb][2], acc[i][nb][3]);
                *(LAS u32x2*)(eb + (i * 16 + fr) * ROWB + (((nb * 2 + (g >> 1)) ^ (fr & (CPR - 1))) << 4) + (g & 1) * 8) = o;
            }
        constexpr int RPI = 64 / CPR;
        constexpr int NST = SPW * 16 / RPI;
        u32x4 ov[NST];
        GAS char* od[NST];
        GAS char* obase = outp + (cg & 3) * 256 + (NH == 2 ? wn * 128 : ch * 128 + wn * 64) + (lane & (CPR - 1)) * 16;
#pragma unroll
        for (int n = 0; n < NST; ++n) {
            const int row = n * RPI + lane / CPR;
            ov[n] = *(LAS u32x4*)(eb + row * ROWB + (((lane & (CPR - 1)) ^ (row & (CPR - 1))) << 4));
            od[n] = obase + (size_t)(rowbase[(n * RPI) / 16] + (row & 15)) * (NSH * 2);
        }
        if (more) { wait_lgkm(); bar(); }
        if (defer) {
            const gu32* c = (const gu32*)(myws + LDONE_OFF + (cg - 4) * 256);
            const unsigned long long t0 = now();
            unsigned spin = 0;
            while (rfl(ld_agent(c)) < call * CPL) {
                if ((++spin & 15) == 0 && (rfl(ld_agent(tmo)) == call || now() - t0 > DEADLINE_TICKS)) {
                    if (lane == 0) st_agent(tmo, call);
#pragma unroll
                    for (int n = 0; n < NST; ++n) ov[n] = (u32x4){0x7FC07FC0u, 0x7FC07FC0u, 0x7FC07FC0u, 0x7FC07FC0u};
                    break;
                }
                __builtin_amdgcn_s_sleep(8);
            }
        }
        if (wv == 0) STAMP(5);
#define AGK_WIN_A 2
#define AGK_WIN_C 1
        constexpr int WIN = NH == 2 ? AGK_WIN_A : (S == 8 ? AGK_WIN_C : 8);
        if (wremote) {
#pragma unroll
            for (int n = 0; n < NST; ++n) { if (n >= WIN) wait_vm<WIN - 1>(); st16_wt(od[n], ov[n]); }
        } else {
#pragma unroll
            for (int n = 0; n < NST; ++n) st16_wt(od[n], ov[n]);
        }
        if (wremote) {
            wait_vm<0>();
            if (lane == 0) __hip_atomic_fetch_add((gu32*)cntp, 1u, __ATOMIC_RELAXED, __HIP_MEMORY_SCOPE_SYSTEM);
        }
        if (wv == 0) STAMP(3);
    }
}

__global__ void __launch_bounds__(512, 1) __attribute__((amdgpu_num_vgpr(240))) ag_gemm_kernel(const Params P) {
    __shared__ __attribute__((aligned(1024))) char smem_raw[LDS_BYTES];
    LAS char* smem = (LAS char*)smem_raw;
    GAS char* myws = (GAS char*)P.ws[0];
    const int b = blockIdx.x;
    gu32* cc = (gu32*)(myws + CALL_OFF + b * 64);
    const unsigned role = rfl(P.role[b]);
    const int type = role & 15;
    if (type == R_COMM) {
        if (threadIdx.x >= 64) return;
        const unsigned call = comm_role(P, cc, b, (role >> 4) & 7, role >> 12);
        if (threadIdx.x == 0) st_agent(cc, call);
    } else if (type != R_IDLE) {
        const unsigned call = rfl(ld_agent(cc)) + 1u;
        if (threadIdx.x == 0) STAMP(0);
        unsigned cur = role, nxt = rfl(P.role2[b]);
        for (;;) {
            const int ty = cur & 15, fs = (cur >> 4) & 255, cg = (cur >> 12) & 7, ch = (cur >> 15) & 1;
            const bool defer = (cur >> 16) & 1, more = nxt != R_IDLE;
            if (ty == R_T128) tile_role<8, 2, 5>(P, smem, call, fs, cg, 0, defer, more);
            else if (ty == R_T128H) tile_role<8, 1, 6>(P, smem, call, fs, cg, ch, defer, more);
            else tile_role<4, 1, 8>(P, smem, call, fs, cg, ch, defer, more);
            if (!more) break;
            cur = nxt; nxt = R_IDLE;
        }
        if (threadIdx.x == 256) st_agent(cc, call);
    }
}

}

extern "C" void kernel_launch(int rank, int nranks, ncclComm_t comm,
                              hipStream_t stream,
                              void* const* d_in, const int* in_sizes, int n_in,
                              void* d_out, int out_size,
                              const int* in_dtypes, int out_dtype,
                              void* const* d_out_all,
                              void* const* d_ws_all, size_t ws_bytes) {
    using namespace agk;
    (void)comm; (void)in_dtypes; (void)out_dtype; (void)d_out;
    if (nranks != NR || (unsigned)rank >= (unsigned)NR || n_in < 2 || ws_bytes < WS_NEED ||
        in_sizes[0] != MSH * KD || in_sizes[1] != KD * NSH || out_size != NR * MSH * NSH) {
        fprintf(stderr, "ag_gemm: unexpected problem shape\n");
        return;
    }
    Params P;
    P.x = (const char*)d_in[0];
    P.w = (const char*)d_in[1];
    P.rank = rank;
    for (int j = 0; j < NR; ++j) { P.ws[j] = (char*)d_ws_all[rank ^ j]; P.out[j] = (char*)d_out_all[rank ^ j]; }
    int n = 0, own = 0;
    for (int i = 0; i < NO_T * 8; ++i) P.strip[n++] = (unsigned char)(own++);
    for (int s = 0; s < SEGA; ++s) for (int j = 1; j < 4; ++j) P.strip[n++] = (unsigned char)((j << 5) | s);
    while (n % 8) P.strip[n++] = (unsigned char)(own++);
    for (int s = HS; s < NSTRIP; ++s) for (int j = 4; j < 8; ++j) P.strip[n++] = (unsigned char)((j << 5) | s);
    while (n % 8) P.strip[n++] = (unsigned char)(own++);
    for (int s = SEGA; s < HS; ++s) for (int j = 1; j < 4; ++j) P.strip[n++] = (unsigned char)((j << 5) | s);
    int fill[8] = {0, 0, 0, 0, 0, 0, 0, 0};
    for (int i = 0; i < NWG; ++i) { P.role[i] = R_IDLE; P.role2[i] = R_IDLE; }
    bool ok = true;
    auto put = [&](int k, unsigned role) {
        if (fill[k] < NWG / 8) { P.role[fill[k]++ * 8 + k] = role; return; }
        for (int i = k; i < NWG; i += 8) {
            const unsigned r = P.role[i];
            if ((r & 15) == R_T128 && ((r >> 4) & 255) < (unsigned)((NO_T + NA_T) * 8) && ((r >> 12) & 7) < 4 && P.role2[i] == R_IDLE) { P.role2[i] = role; return; }
        }
        ok = false;
    };
    auto tile = [&](int half, int type, int fs, int cg, int ch, bool defer) {
        const int k = (cg & 3) + 4 * half;
        put(k, (unsigned)type | (fs << 4) | (cg << 12) | (ch << 15) | ((defer ? 1u : 0u) << 16));
    };
    for (int i = 0; i < NCOMM; ++i) put(i & 7, R_COMM | ((1 + i % 7) << 4) | ((i / 7) << 12));
    unsigned adds = 0;
    const int fsC = (NO_T + NA_T) * 8, fsB = NBIG_T * 8;
    for (int cg = 0; cg < 8; ++cg) {
        const bool rem = cg >= 4;
        for (int r = 0; r < NO_T + NA_T; ++r) {
            const int half = r == NO_T + NA_T - 1 ? (rem ? 1 : 0) : (r & 1);
            tile(half, R_T128, r * 8, cg, 0, rem);
            if (cg == 4) adds += 4;
        }
        for (int r = 0; r < NC_T; ++r) {
            if (!rem) tile(r & 1, R_T128, fsC + r * 8, cg, 0, false);
            else for (int ch = 0; ch < 2; ++ch) { tile(r & 1, R_T128H, fsC + r * 8, cg, ch, false); if (cg == 4) adds += 4; }
        }
    }
    for (int cg = 7; cg >= 0; --cg) {
        for (int r = NB_T / 2 - 1; r >= 0; --r) for (int ch = 0; ch < 2; ++ch) {
            const int half = r < 2 ? r : ch;
            for (int q = 1; q >= 0; --q) { tile(half, R_T64H, fsB + r * 8 + q * 4, cg, ch, false); if (cg == 4) adds += 4; }
        }
    }
    P.out_adds = adds;
    if (!ok) { fprintf(stderr, "ag_gemm: role table overflow\n"); return; }
    hipLaunchKernelGGL(ag_gemm_kernel, dim3(NWG), dim3(512), 0, stream, P);
    HIP_CHECK(hipGetLastError());
}
```

```cpp
#include <hip/hip_runtime.h>
#include <rccl/rccl.h>
#include <cstdio>
#include <cstdint>

#define HIP_CHECK(expr) do { hipError_t _e = (expr); if (_e != hipSuccess) { \
    fprintf(stderr, "HIP error %s at %s:%d\n", hipGetErrorString(_e), __FILE__, __LINE__); \
    return; } } while (0)

namespace agk {

typedef unsigned int u32x4 __attribute__((ext_vector_type(4)));
typedef unsigned int u32x2 __attribute__((ext_vector_type(2)));
typedef short s16x4 __attribute__((ext_vector_type(4)));
typedef short s16x8 __attribute__((ext_vector_type(8)));
typedef __bf16 bf16x8 __attribute__((ext_vector_type(8)));
typedef float f32x4 __attribute__((ext_vector_type(4)));
#define LAS __attribute__((address_space(3)))
#define GAS __attribute__((address_space(1)))
typedef GAS unsigned int gu32;

constexpr int NR = 8;
constexpr int KD = 4096;
constexpr int MSH = 512;
constexpr int NSH = 512;
constexpr int NSTRIP = 32;
#define AGK_HS 21
constexpr int HS = AGK_HS;
constexpr int AS = NSTRIP - HS;
constexpr int NKC = 16;
constexpr int NSTEP = 64;
constexpr int NWP = 128;

#define AGK_NCOMM 56
#define AGK_NCW 3
constexpr int NCOMM = AGK_NCOMM;
constexpr int NCW = AGK_NCW;
constexpr int WPL = NCOMM * NCW / 7;

constexpr int NEARLY_T = 8;
constexpr int NP_IN = HS * NKC;
constexpr int NP_XI = NWP + AS * NKC;
constexpr int NPMAX = NP_IN > NP_XI ? NP_IN : NP_XI;
constexpr int NWG = 256;
enum { R_IDLE = 0, R_COMM = 1, R_T128 = 2, R_T128H = 3, R_T64H = 4, R_UNPACK = 5 };

constexpr size_t STRIP_BYTES = 16ull * KD * 2;
constexpr int NXSLOT = 3 * HS + 4 * AS;
constexpr size_t XS_BYTES = (size_t)NXSLOT * STRIP_BYTES;
constexpr size_t WSL_BYTES = (size_t)KD * 128 * 2;
constexpr size_t PK_STRIDE = 6144 + 128;
constexpr size_t PK_OFF = XS_BYTES + 4 * WSL_BYTES;
constexpr int XF_STRIDE = 64;
constexpr size_t XFLAG_OFF = PK_OFF + (size_t)(3 * NP_IN + 4 * NP_XI) * PK_STRIDE;
constexpr size_t WFLAG_OFF = XFLAG_OFF + 8 * 32 * XF_STRIDE * 4;
constexpr size_t PFLAG_OFF = WFLAG_OFF + 4 * NWP * 4;
constexpr size_t BUF_BYTES = (PFLAG_OFF + 8 * NPMAX * 8 + 4095) / 4096 * 4096;
constexpr size_t STASH_OFF = 2 * BUF_BYTES;
constexpr size_t STASH_BYTES = (size_t)NEARLY_T * 8 * 4096;
constexpr size_t OCNT_OFF = STASH_OFF + 4 * STASH_BYTES;
constexpr size_t SCNT_OFF = OCNT_OFF + 4 * 256;
constexpr size_t TICK_OFF = SCNT_OFF + 4 * 256;
constexpr size_t TMO_OFF = TICK_OFF + 8 * 256;
constexpr size_t DUMMY_OFF = TMO_OFF + 256;
constexpr size_t CALL_OFF = DUMMY_OFF + 256 * 64;
constexpr size_t DBG_OFF = CALL_OFF + 256 * 64;
constexpr size_t DBG2_OFF = DBG_OFF + 256 * 128;
constexpr size_t WS_NEED = DBG2_OFF + 8 * 512 * 32;
static_assert(WS_NEED <= 65ull << 20, "workspace");

constexpr unsigned long long DEADLINE_TICKS = 30000000ull;
constexpr int LDS_BYTES = 163840;

struct Params {
    const char* x;
    const char* w;
    char* ws[NR];
    char* out[NR];
    int rank;
    unsigned out_adds;
    unsigned npx;
    unsigned nunpack;
    unsigned role[NWG];
    unsigned orow[NEARLY_T * 4];
    unsigned ord[NP_IN + NP_XI + 1];
    unsigned char strip[128];
};

__device__ __forceinline__ unsigned ld_sys(const gu32* p) { return __hip_atomic_load(p, __ATOMIC_RELAXED, __HIP_MEMORY_SCOPE_SYSTEM); }
__device__ __forceinline__ unsigned ld_agent(const gu32* p) { return __hip_atomic_load(p, __ATOMIC_RELAXED, __HIP_MEMORY_SCOPE_AGENT); }
__device__ __forceinline__ void st_agent(gu32* p, unsigned v) { __hip_atomic_store(p, v, __ATOMIC_RELAXED, __HIP_MEMORY_SCOPE_AGENT); }
__device__ __forceinline__ unsigned long long now() { return __builtin_amdgcn_s_memrealtime(); }
__device__ __forceinline__ unsigned rfl(unsigned v) { return __builtin_amdgcn_readfirstlane(v); }

__device__ __forceinline__ void st16_wt(GAS char* p, u32x4 v) {
    asm volatile("global_store_dwordx4 %0, %1, off sc0 sc1\n\ts_nop 1" :: "v"(p), "v"(v) : "memory");
}
template <int N> __device__ __forceinline__ void wait_vm() { asm volatile("s_waitcnt vmcnt(%0)" :: "i"(N) : "memory"); }
__device__ __forceinline__ void wait_lgkm() { asm volatile("" ::: "memory"); __builtin_amdgcn_s_waitcnt(0xC07F); asm volatile("" ::: "memory"); }
#define AGK_W3 28
#define AGK_SL1 6
#define AGK_SL2 2
#define AGK_SL3 6
#define AGK_FAST_SPINS 40
#define PIN() __builtin_amdgcn_sched_barrier(0)
__device__ __forceinline__ void bar() { asm volatile("s_barrier" ::: "memory"); }

#define AGK_LDPOL "sc0 sc1"
__device__ __forceinline__ void glds2(const GAS char* base, unsigned v0, unsigned v1, unsigned lds_dst) {
    asm volatile("s_mov_b32 m0, %3\n\ts_nop 3\n\tglobal_load_lds_dwordx4 %1, %0 " AGK_LDPOL "\n\tglobal_load_lds_dwordx4 %2, %0 offset:1024 " AGK_LDPOL
                 :: "s"(base), "v"(v0), "v"(v1), "s"(lds_dst) : "m0", "memory");
}

__device__ __forceinline__ unsigned pack_bf16(float lo, float hi) {
    typedef float f32x2 __attribute__((ext_vector_type(2)));
    typedef __bf16 bf16x2 __attribute__((ext_vector_type(2)));
    f32x2 v = {lo, hi};
    bf16x2 r = __builtin_convertvector(v, bf16x2);
    return __builtin_bit_cast(unsigned, r);
}

__device__ __forceinline__ int xslot(int j, int s) { return j < 4 ? (j - 1) * HS + s : 3 * HS + (j - 4) * AS + (s - HS); }

#define STAMP(i) do { } while (0)
#define PSTAMP(j, p, i, t) do { } while (0)

enum { K_X = 0, K_W = 1, K_OUT = 2 };
struct Piece { int kind, raw; unsigned soff, doff, foff, when; };

__device__ __forceinline__ Piece locate(const Params& P, int j, int p) {
    Piece r;
    if (j >= 4 && p >= NP_XI) {
        r.kind = K_OUT; r.raw = 1;
        r.soff = (unsigned)((j - 4) * STASH_BYTES) + (p - NP_XI) * 8192;
        r.doff = p - NP_XI; r.foff = 0;
        const unsigned e = P.ord[NP_IN + NP_XI];
        r.when = (e >> 16) + (((p - NP_XI) * (e & 0xFFFFu)) >> 4);
        return r;
    }
    const unsigned e = P.ord[j < 4 ? p : NP_IN + p];
    r.when = e >> 16;
    const int s = (e >> 4) & 31, kc = e & 15, wp = (e >> 10) & 1 ? (int)(e & 127) : -1;
    r.raw = (e >> 9) & 1;
    if (wp >= 0) {
        r.kind = K_W;
        r.soff = (unsigned)(wp * 32 * (NSH * 2) + (j & 3) * 256);
        r.doff = (unsigned)XS_BYTES + (unsigned)(j - 4) * (unsigned)WSL_BYTES + wp * 8192;
        r.foff = (unsigned)WFLAG_OFF + ((j - 4) * NWP + (wp & 1) * 64 + (wp >> 1)) * 4;
    } else {
        r.kind = K_X;
        r.soff = (unsigned)(s * 16 * (KD * 2) + kc * 512);
        r.doff = (unsigned)xslot(j, s) * (unsigned)STRIP_BYTES + kc * 8192;
        r.foff = (unsigned)XFLAG_OFF + ((j * 32 + s) * XF_STRIDE + kc) * 4;
    }
    return r;
}
__device__ __forceinline__ unsigned pk_off(int j, int p) { return (unsigned)PK_OFF + (unsigned)((j < 4 ? (j - 1) * NP_IN : 3 * NP_IN + (j - 4) * NP_XI) + p) * (unsigned)PK_STRIDE; }
__device__ __forceinline__ unsigned pf_off(int j, int p) { return (unsigned)PFLAG_OFF + (unsigned)(j * NPMAX + p) * 8u; }

template <int N> __device__ __forceinline__ void st_w(const u32x4& v, unsigned voff, const GAS char* dst) {
    asm volatile("s_waitcnt vmcnt(%3)\n\ts_nop 3\n\tglobal_store_dwordx4 %1, %0, %2 sc0 sc1\n\ts_nop 1" :: "v"(v), "v"(voff), "s"(dst), "i"(N) : "memory");
}
template <int N, int OFF> __device__ __forceinline__ void st_wo(const u32x4& v, unsigned voff, const GAS char* dst) {
    asm volatile("s_waitcnt vmcnt(%3)\n\ts_nop 3\n\tglobal_store_dwordx4 %1, %0, %2 offset:%4 sc0 sc1\n\ts_nop 1" :: "v"(v), "v"(voff), "s"(dst), "i"(N), "i"(OFF) : "memory");
}
template <int OFF> __device__ __forceinline__ void ld_c(u32x4& v, unsigned voff, const GAS char* src) {
    asm volatile("s_nop 4\n\tglobal_load_dwordx4 %0, %1, %2 offset:%3 sc0 sc1" : "=v"(v) : "v"(voff), "s"(src), "i"(OFF) : "memory");
}
__device__ __forceinline__ void glds_s(unsigned voff, const GAS char* src, unsigned lds_dst) {
    unsigned keep;
    asm volatile("s_mov_b32 %0, m0\n\ts_mov_b32 m0, %3\n\ts_nop 2\n\tglobal_load_lds_dwordx4 %1, %2 sc0 sc1\n\ts_mov_b32 m0, %0"
                 : "=&s"(keep) : "v"(voff), "s"(src), "s"(lds_dst) : "memory");
}
__device__ __forceinline__ void flag_st(const GAS char* f, unsigned val, unsigned zero) {
    asm volatile("s_mov_b64 exec, 1\n\ts_nop 3\n\tglobal_store_dword %0, %1, %2 sc0 sc1\n\ts_mov_b64 exec, -1" :: "v"(zero), "v"(val), "s"(f) : "memory");
}
__device__ __forceinline__ void flag_st64(const GAS char* f, u32x2 val, unsigned zero) {
    asm volatile("s_mov_b64 exec, 1\n\ts_nop 3\n\tglobal_store_dwordx2 %0, %1, %2 sc0 sc1\n\ts_mov_b64 exec, -1" :: "v"(zero), "v"(val), "s"(f) : "memory");
}
__device__ __forceinline__ void add_sys(const GAS char* c, unsigned one, unsigned zero) {
    asm volatile("s_mov_b64 exec, 1\n\ts_nop 3\n\tglobal_atomic_add %0, %1, %2 sc1\n\ts_mov_b64 exec, -1" :: "v"(zero), "v"(one), "s"(c) : "memory");
}
__device__ __forceinline__ unsigned perm(unsigned hi, unsigned lo, unsigned sel) { return __builtin_amdgcn_perm(hi, lo, sel); }
__device__ __forceinline__ unsigned bfi(unsigned m, unsigned a, unsigned b) { return (a & m) | (b & ~m); }
__device__ __forceinline__ unsigned pkmax(unsigned a, unsigned b) {
    typedef unsigned short u16x2 __attribute__((ext_vector_type(2)));
    return __builtin_bit_cast(unsigned, __builtin_elementwise_max(__builtin_bit_cast(u16x2, a), __builtin_bit_cast(u16x2, b)));
}
template <int CTRL> __device__ __forceinline__ unsigned dppmax(unsigned v) { return max(v, (unsigned)__builtin_amdgcn_update_dpp(0, (int)v, CTRL, 0xF, 0xF, true)); }
constexpr int MAXEXC = 16;

__device__ __forceinline__ void fetch_piece(int kind, unsigned soff, const GAS char* sx, const GAS char* sw, const GAS char* ss,
                                            unsigned vx, unsigned vw, unsigned vl, unsigned lds) {
    const bool kx = kind == K_X, kw = kind == K_W;
    const GAS char* s = (kx ? sx : kw ? sw : ss) + soff;
    const unsigned vs = kx ? vx : kw ? vw : vl;
    const int step = kx ? 2 * KD * 2 : kw ? 4 * NSH * 2 : 1024;
#pragma unroll
    for (int k = 0; k < 8; ++k) glds_s(vs, s + k * step, lds + k * 1024);
}

__device__ __forceinline__ unsigned comm_role(const Params& P, LAS char* smem, const gu32* cc, int wslot, int j, int u) {
    const unsigned tstart = (unsigned)now();
    GAS char* myws = (GAS char*)P.ws[0];
    const int lane = threadIdx.x & 63;
    const int npiece = j < 4 ? NP_IN : (int)P.npx;
    const unsigned vlin = lane * 16;
    const unsigned vx = (lane >> 5) * (KD * 2) + (lane & 31) * 16, vw = (lane >> 4) * (NSH * 2) + (lane & 15) * 16;
    const unsigned lds0 = (unsigned)(size_t)smem;
    const GAS char* const srcx = (const GAS char*)P.x;
    const GAS char* const srcw = (const GAS char*)P.w;
    const GAS char* const srcs = myws + STASH_OFF;
#define AGK_FETCH(q, slot) fetch_piece((q).kind, (q).soff, srcx, srcw, srcs, vx, vw, vlin, lds0 + (slot) * 8192)
    int pA = u;
    unsigned ccv;
    asm volatile("global_load_dword %0, %1, off sc1" : "=v"(ccv) : "v"(cc) : "memory");
    Piece A = locate(P, j, pA), B = locate(P, j, pA + WPL);
    AGK_FETCH(A, 0); AGK_FETCH(B, 1);
    asm volatile("v_mov_b32 v251, 0" ::: "v251", "memory");
    const GAS char* pws = (const GAS char*)P.ws[j];
    const GAS char* outp = (const GAS char*)P.out[j] + (j & 3) * 256;
    asm volatile("" : "+s"(pws), "+s"(outp));
    asm volatile("s_waitcnt vmcnt(8)" : "+v"(ccv) :: "memory");
    const unsigned call = rfl(ccv) + 1u;
    if (wslot == 0) STAMP(0);
    const GAS char* dbuf = pws + (size_t)(call & 1) * BUF_BYTES;
    const GAS char* ocnt = pws + OCNT_OFF + (j & 3) * 256;
    const GAS char* scnt = myws + SCNT_OFF + (j & 3) * 256;
    gu32* tmo = (gu32*)(myws + TMO_OFF);
    const unsigned sneed = call * 8u;
    unsigned one = 1, zero = 0, callv = call, v4 = lane * 4;
    asm volatile("" : "+v"(one), "+v"(zero), "+v"(callv), "+v"(v4));
    int slot = 0;
    bool poison = false, defA = false, defB = false;
    auto stashed = [&](unsigned piece) {
        unsigned sv;
        asm volatile("s_nop 3\n\tv_readlane_b32 %0, v251, %1" : "=s"(sv) : "s"(piece >> 2) : "memory");
        return sv >= sneed;
    };
    for (;;) {
        if (defA) {
            unsigned long long t0 = 0;
            for (unsigned spin = 1; !stashed(A.doff); ++spin) {
                if ((spin & 63) == 0) {
                    if (t0 == 0) t0 = now();
                    if (rfl(ld_agent(tmo)) == call || now() - t0 > DEADLINE_TICKS) { st_agent(tmo, call); poison = true; break; }
                }
                asm volatile("s_sleep 2\n\ts_mov_b64 exec, 0xff\n\ts_nop 3\n\tglobal_load_dword v251, %1, %0 sc1\n\ts_mov_b64 exec, -1\n\ts_waitcnt vmcnt(0)"
                             :: "s"(scnt), "v"(v4) : "v251", "memory");
            }
            AGK_FETCH(A, slot);
            wait_vm<0>();
        }
        if (defB && stashed(B.doff)) { AGK_FETCH(B, slot ^ 1); defB = false; }
        u32x4 v[8];
#pragma unroll
        for (int k = 0; k < 8; ++k) v[k] = *(LAS u32x4*)(smem + slot * 8192 + k * 1024 + lane * 16);
        bool raw = A.raw;
        u32x2 ev = {0, 0};
        unsigned top = 0, cnt = 0; unsigned long long bm = 0;
        if (!raw) {
            unsigned mx = pkmax(pkmax(pkmax(v[0].x & 0x7FFF7FFFu, v[0].y & 0x7FFF7FFFu), pkmax(v[0].z & 0x7FFF7FFFu, v[0].w & 0x7FFF7FFFu)),
                                pkmax(pkmax(v[4].x & 0x7FFF7FFFu, v[4].y & 0x7FFF7FFFu), pkmax(v[4].z & 0x7FFF7FFFu, v[4].w & 0x7FFF7FFFu)));
            mx = max(mx >> 16, mx & 0xFFFFu) >> 8;
            mx = dppmax<0x143>(dppmax<0x142>(dppmax<0x118>(dppmax<0x114>(dppmax<0x112>(dppmax<0x111>(mx))))));
            top = max((unsigned)__builtin_amdgcn_readlane((int)mx, 63), 7u);
            const unsigned T = (top | 0x80u) * 0x01010101u;
            unsigned accO = 0, accA = ~0u;
            u32x4 L[4], N[2];
#pragma unroll
            for (int k = 0; k < 8; ++k) {
                const unsigned Ha = perm(v[k].y, v[k].x, 0x07050301u), Hb = perm(v[k].w, v[k].z, 0x07050301u);
                const unsigned da = T - (Ha & 0x7F7F7F7Fu), db = T - (Hb & 0x7F7F7F7Fu);
                accO |= da | db; accA &= da & db;
                L[k >> 1][(k & 1) * 2] = perm(v[k].y, v[k].x, 0x06040200u);
                L[k >> 1][(k & 1) * 2 + 1] = perm(v[k].w, v[k].z, 0x06040200u);
                N[k >> 2][k & 3] = bfi(0x0F0F0F0Fu, bfi(0x07070707u, db, Hb >> 4), bfi(0x80808080u, Ha, da << 4));
            }
            bm = __ballot(((accO & 0x78787878u) | (~accA & 0x80808080u)) != 0);
            if (bm) {
                unsigned nb = 0, ib = 0;
#pragma unroll
                for (int i = 0; i < 16; ++i) {
                    const unsigned H = (i & 1) ? perm(v[i >> 1].w, v[i >> 1].z, 0x07050301u) : perm(v[i >> 1].y, v[i >> 1].x, 0x07050301u);
                    const bool bad = ((T - (H & 0x7F7F7F7Fu)) & 0xF8F8F8F8u) != 0x80808080u;
                    ev.y = bad ? H : ev.y; ib = bad ? (unsigned)i : ib; nb += bad;
                }
                ev.x = lane | (ib << 8);
                cnt = __builtin_popcountll(bm);
                raw = cnt > MAXEXC || __ballot(nb > 1) != 0;
            }
            if (!raw) { v[0] = L[0]; v[1] = L[1]; v[2] = L[2]; v[3] = L[3]; v[4] = N[0]; v[5] = N[1]; }
        } else if (poison && A.kind == K_OUT) {
#pragma unroll
            for (int k = 0; k < 8; ++k) v[k] = (u32x4){0x7FC07FC0u, 0x7FC07FC0u, 0x7FC07FC0u, 0x7FC07FC0u};
        }
        const GAS char *d0, *d1; unsigned vd = vlin; int dstep = 1024;
        if (!raw) { d0 = dbuf + pk_off(j, pA); d1 = d0 + 4096; }
        else if (A.kind == K_OUT) {
            const unsigned rows = P.orow[A.doff];
            d0 = outp + (rows & 0xFFFFu) * (NSH * 2);
            d1 = outp + (rows >> 16) * (NSH * 2);
            vd = vw; dstep = 4 * NSH * 2;
        } else { d0 = dbuf + A.doff; d1 = d0 + 4096; }
        const int pC = pA + 2 * WPL;
        const Piece C = locate(P, j, pC < npiece ? pC : u);
        asm volatile("" :: "v"(v[0]), "v"(v[1]), "v"(v[2]), "v"(v[3]), "v"(v[4]), "v"(v[5]), "v"(v[6]), "v"(v[7]), "v"(ev), "s"(d0), "s"(d1), "s"(C.soff) : "memory");
        for (int rest = (int)(A.when - ((unsigned)now() - tstart)); rest > 2; rest -= 3) __builtin_amdgcn_s_sleep(1);
        st_w<63>(v[0], vd, d0);
        st_w<63>(v[1], vd, d0 + dstep);
        st_w<63>(v[2], vd, d0 + 2 * dstep);
        st_w<63>(v[3], vd, d0 + 3 * dstep);
        st_w<63>(v[4], vd, d1);
        st_w<63>(v[5], vd, d1 + dstep);
        if (raw) {
            st_w<63>(v[6], vd, d1 + 2 * dstep);
            st_w<63>(v[7], vd, d1 + 3 * dstep);
        } else if (cnt) {
            const unsigned idx = __builtin_amdgcn_mbcnt_hi((unsigned)(bm >> 32), __builtin_amdgcn_mbcnt_lo((unsigned)bm, 0)) * 8;
            asm volatile("s_mov_b64 exec, %3\n\ts_nop 3\n\tglobal_store_dwordx2 %0, %1, %2 offset:2048 sc0 sc1\n\ts_mov_b64 exec, -1" :: "v"(idx), "v"(ev), "s"(d1), "s"(bm) : "memory");
        }
        const bool defC = C.kind == K_OUT && !stashed(C.doff);
        asm volatile("s_mov_b64 exec, 0xff\n\ts_nop 3\n\tglobal_load_dword v251, %1, %0 sc1\n\ts_mov_b64 exec, -1" :: "s"(scnt), "v"(v4) : "v251", "memory");
        AGK_FETCH(C, defC ? 2 * (NCW - wslot) : slot);
        wait_vm<9>();
        if (!A.raw) { u32x2 f = {callv, top | (cnt << 8) | (raw ? 1u << 16 : 0u)}; flag_st64(dbuf + pf_off(j, pA), f, zero); }
        if (A.kind == K_OUT) add_sys(ocnt, one, zero);
        else if (raw) flag_st(dbuf + A.foff, callv, zero);
        PSTAMP(j, pA, 0, tstart + A.when); PSTAMP(j, pA, 1, tb); PSTAMP(j, pA, 2, now());
        pA += WPL; A = B; B = C; defA = defB; defB = defC; slot ^= 1;
        if (pA >= npiece) break;
    }
    STAMP(j < 4 ? 1 : 3);
    if (j == 1 && u == 0) {
        if (lane < 4) {
            const gu32* c = (const gu32*)(myws + OCNT_OFF + lane * 256);
            const unsigned need = call * P.out_adds;
            const unsigned long long t0 = now();
            bool bad = false;
            unsigned spin = 0;
            while (ld_sys(c) < need) {
                if ((++spin & 63) == 0 && (ld_agent(tmo) == call || now() - t0 > DEADLINE_TICKS)) { st_agent(tmo, call); bad = true; break; }
                __builtin_amdgcn_s_sleep(1);
            }
            if (bad) {
                __builtin_amdgcn_s_sleep(127);
                __hip_atomic_store((GAS unsigned short*)(P.out[0] + (size_t)(P.rank * MSH) * (NSH * 2)), (unsigned short)0x7FC0, __ATOMIC_RELAXED, __HIP_MEMORY_SCOPE_AGENT);
            }
        }
        STAMP(2);
    }
    return call;
}

__device__ __forceinline__ void unpack_role(const Params& P, GAS char* myws, unsigned call, int w, int nw) {
    const GAS char* buf = myws + (size_t)(call & 1) * BUF_BYTES;
    gu32* tmo = (gu32*)(myws + TMO_OFF);
    const int lane = threadIdx.x & 63;
    const unsigned vlin = lane * 16;
    unsigned zero = 0, callv = call;
    asm volatile("" : "+v"(zero), "+v"(callv));
    unsigned pend = 0;
    int pj = 0, pp = 0;
    for (int q = w; q < NPMAX * 7; q += nw) {
        const int p = q / 7, j = 1 + q - p * 7;
        if (p >= (j < 4 ? NP_IN : NP_XI)) continue;
        const Piece A = locate(P, j, p);
        if (A.raw) continue;
        const GAS unsigned long long* pf = (const GAS unsigned long long*)(buf + pf_off(j, p));
        unsigned long long fv, t0 = 0;
        unsigned spin = 0;
        bool dead = false;
        for (;;) {
            fv = __hip_atomic_load(pf, __ATOMIC_RELAXED, __HIP_MEMORY_SCOPE_SYSTEM);
            const unsigned seen = rfl((unsigned)fv);
            if (pend) { wait_vm<0>(); flag_st(buf + pend, callv, zero); pend = 0; PSTAMP(pj, pp, 6, now()); }
            if (seen == call) break;
            if ((++spin & 63) == 0) {
                if (t0 == 0) t0 = now();
                if (rfl(ld_agent(tmo)) == call || now() - t0 > DEADLINE_TICKS) { st_agent(tmo, call); dead = true; break; }
            }
            __builtin_amdgcn_s_sleep(2);
        }
        if (dead) return;
        PSTAMP(j, p, 3, now()); PSTAMP(j, p, 7, spin);
        const unsigned meta = rfl((unsigned)(fv >> 32));
        if (meta >> 16) continue;
        const GAS char* s = buf + pk_off(j, p) + 4096;
        const GAS char* d = buf + A.doff + 4096;
        const int cnt = (meta >> 8) & 255;
        u32x4 L[4], N[2]; u32x2 ex = {0, 0};
        ld_c<0>(N[0], vlin, s); ld_c<1024>(N[1], vlin, s);
        ld_c<-4096>(L[0], vlin, s); ld_c<-3072>(L[1], vlin, s); ld_c<-2048>(L[2], vlin, s); ld_c<-1024>(L[3], vlin, s);
        if (cnt) asm volatile("s_nop 4\n\tglobal_load_dwordx2 %0, %1, %2 offset:2048 sc0 sc1" : "=v"(ex) : "v"((lane & (MAXEXC - 1)) * 8), "s"(s) : "memory");
        asm volatile("s_waitcnt vmcnt(0)" : "+v"(L[0]), "+v"(L[1]), "+v"(L[2]), "+v"(L[3]), "+v"(N[0]), "+v"(N[1]), "+v"(ex) :: "memory");
        PSTAMP(j, p, 4, now());
        const unsigned T7 = (meta & 127) * 0x01010101u;
        unsigned H[16];
#pragma unroll
        for (int k = 0; k < 8; ++k) {
            const unsigned n = N[k >> 2][k & 3];
            H[2 * k] = bfi(0x80808080u, n, T7 - ((n >> 4) & 0x07070707u));
            H[2 * k + 1] = bfi(0x80808080u, n << 4, T7 - (n & 0x07070707u));
        }
        if (cnt) {
            unsigned tag = 255, val = 0;
            for (int e = 0; e < cnt; ++e) {
                const unsigned id = __builtin_amdgcn_readlane(ex.x, e), hv = __builtin_amdgcn_readlane(ex.y, e);
                const bool mine = (id & 63) == (unsigned)lane;
                tag = mine ? id >> 8 : tag; val = mine ? hv : val;
            }
#pragma unroll
            for (int i = 0; i < 16; ++i) H[i] = tag == (unsigned)i ? val : H[i];
        }
#define AGK_DST(k, off) { const unsigned l0 = L[k >> 1][(k & 1) * 2], l1 = L[k >> 1][(k & 1) * 2 + 1]; u32x4 o; \
            o.x = perm(H[2 * k], l0, 0x05010400u); o.y = perm(H[2 * k], l0, 0x07030602u); \
            o.z = perm(H[2 * k + 1], l1, 0x05010400u); o.w = perm(H[2 * k + 1], l1, 0x07030602u); st_wo<63, off>(o, vlin, d); }
        AGK_DST(0, -4096) AGK_DST(1, -3072) AGK_DST(2, -2048) AGK_DST(3, -1024) AGK_DST(4, 0) AGK_DST(5, 1024) AGK_DST(6, 2048) AGK_DST(7, 3072)
        PSTAMP(j, p, 5, now());
        wait_vm<0>(); flag_st(buf + A.foff, callv, zero);
        PSTAMP(j, p, 6, now());
    }
    if (pend) { wait_vm<0>(); flag_st(buf + pend, callv, zero); PSTAMP(pj, pp, 6, now()); }
}

template <int S, int NH, int NSLOT>
__device__ __forceinline__ void tile_role(const Params& P, LAS char* smem, unsigned call, int first_strip, int cg, int ch, bool stash, int win) {
    constexpr int D = NSLOT - 1;
    constexpr int SPL = S / 4;
    constexpr int NA = 2 * SPL, NB = 2 * NH, NI = NA + NB;
    constexpr int A_BYTES = S * 2048, SLOT_BYTES = A_BYTES + NH * 8192;
    constexpr int SPW = S / 2, NFR = NH * 2;
    static_assert(SLOT_BYTES * NSLOT <= LDS_BYTES, "LDS ring");
    static_assert(D * (NI + 2) <= 48, "vmcnt range");
    GAS char* myws = (GAS char*)P.ws[0];
    gu32* tmo = (gu32*)(myws + TMO_OFF);
    const int lane = threadIdx.x & 63;
    const int wv = rfl(threadIdx.x >> 6);
    const bool wremote = cg >= 4;

    if (wv >= 4) {
        const int lw = wv - 4;
        const GAS char* mybuf = myws + (size_t)(call & 1) * BUF_BYTES;
        const unsigned lds0 = (unsigned)(size_t)smem;
        const GAS char* asrc[SPL];
        unsigned av0[SPL], av1[SPL];
        int ajump[SPL], sj[SPL], ss[SPL];
        const unsigned r0 = lane >> 3, r1 = 8 + r0;
#pragma unroll
        for (int i = 0; i < SPL; ++i) {
            const int code = rfl(P.strip[first_strip + lw * SPL + i]);
            sj[i] = code >> 5; ss[i] = code & 31;
            const unsigned pitch = sj[i] == 0 ? KD * 2 : 512;
            av0[i] = r0 * pitch + ((lane & 7) ^ (r0 >> 1)) * 16;
            av1[i] = r1 * pitch + ((lane & 7) ^ (r1 >> 1)) * 16 - 1024;
            asrc[i] = sj[i] == 0 ? (const GAS char*)P.x + (unsigned)ss[i] * (16 * KD * 2) : mybuf + (unsigned)xslot(sj[i], ss[i]) * (unsigned)STRIP_BYTES;
            ajump[i] = sj[i] == 0 ? 128 : 8192 - 384;
        }
        const int bhalf = NH == 2 ? (lw >> 1) : ch;
        const int kb0 = NH == 2 ? (lw & 1) * 4 : lw * 2;
        const unsigned bpitch = wremote ? 256 : NSH * 2;
        const unsigned bv0 = r0 * bpitch + ((lane & 7) ^ (r0 & 2)) * 16, bv1 = r1 * bpitch + ((lane & 7) ^ ((r0 & 2) | 4)) * 16 - 1024;
        const GAS char* bsrc = (wremote ? mybuf + XS_BYTES + (unsigned)(cg - 4) * (unsigned)WSL_BYTES : (const GAS char*)P.w + cg * 256) + bhalf * 128 + kb0 * 8 * bpitch;
        const unsigned ldsA = lds0 + lw * SPL * 2048;
        const unsigned ldsB = lds0 + A_BYTES + (NH == 2 ? (lw >> 1) * 8192 : 0) + kb0 * 1024;

        const gu32* fx; unsigned long long xown;
        {
            const int fi = (SPL == 2 && (lane & 16)) ? SPL - 1 : 0;
            const int fj = fi ? sj[SPL - 1] : sj[0], fs = fi ? ss[SPL - 1] : ss[0];
            fx = (const gu32*)(mybuf + XFLAG_OFF) + (fj * 32 + fs) * XF_STRIDE + (lane & 15);
            xown = __ballot(fj == 0 || lane >= 16 * SPL);
        }
        const gu32* fw = (const gu32*)(mybuf + WFLAG_OFF) + (wremote ? cg - 4 : 0) * NWP + (kb0 >> 2) * 64 + lane;
        const unsigned long long xact = ~xown;
        int xmark = xact == 0 ? NSTEP : 0, wmark = wremote ? 0 : NSTEP;
        int mark = xmark < wmark ? xmark : wmark;
        bool dead = false;
        asm volatile("v_mov_b32 v250, 0\n\tv_mov_b32 v251, 0" ::: "v250", "v251", "memory");
        auto poll = [&]() {
            if (xmark < NSTEP) asm volatile("s_mov_b64 exec, %1\n\tglobal_load_dword v250, %0, off sc0 sc1\n\ts_mov_b64 exec, -1" :: "v"(fx), "s"(xact) : "v250", "memory");
            if (wmark < NSTEP) asm volatile("global_load_dword v251, %0, off sc0 sc1" :: "v"(fw) : "v251", "memory");
        };
        auto eval = [&]() {
            unsigned long long bx, bw;
            asm volatile("v_cmp_le_u32 %0, %2, v250\n\tv_cmp_gt_u32 %1, %2, v251" : "=&s"(bx), "=&s"(bw) : "s"(call) : "memory");
            const unsigned m = (unsigned)(bx | xown);
            xmark = rfl(__builtin_ctz(~(m & (m >> 16) & 0xFFFFu)) * 4);
            if (wmark < NSTEP) wmark = rfl(bw ? __builtin_ctzll(bw) : NSTEP);
            mark = xmark < wmark ? xmark : wmark;
        };
        auto ensure = [&](int t) {
            if (t < mark) return;
            unsigned long long t0 = 0;
            unsigned spin = 0;
            for (;;) {
                eval();
                if (t < mark) break;
                wait_vm<48>();
                poll();
                if (++spin < AGK_FAST_SPINS) __builtin_amdgcn_s_sleep(AGK_SL1); else __builtin_amdgcn_s_sleep(AGK_SL3);
                if ((spin & 63) == 0) {
                    if (t0 == 0) t0 = now();
                    if (rfl(ld_agent(tmo)) == call || now() - t0 > DEADLINE_TICKS) {
                        if (lane == 0) st_agent(tmo, call);
                        dead = true; mark = xmark = wmark = NSTEP; break;
                    }
                }
            }
        };
        auto issue = [&](int t, int slot) {
            const unsigned so = slot * SLOT_BYTES;
            if (!dead) {
#pragma unroll
                for (int i = 0; i < SPL; ++i) glds2(asrc[i], av0[i], av1[i], ldsA + so + i * 2048);
#pragma unroll
                for (int i = 0; i < NH; ++i) glds2(bsrc + i * 16 * bpitch, bv0, bv1, ldsB + so + i * 2048);
            } else {
                const u32x4 nan = {0x7FC07FC0u, 0x7FC07FC0u, 0x7FC07FC0u, 0x7FC07FC0u};
#pragma unroll
                for (int i = 0; i < NA; ++i) *(LAS u32x4*)(size_t)(ldsA + so + i * 1024 + lane * 16) = nan;
#pragma unroll
                for (int i = 0; i < NB; ++i) *(LAS u32x4*)(size_t)(ldsB + so + i * 1024 + lane * 16) = nan;
                wait_lgkm();
            }
#pragma unroll
            for (int i = 0; i < SPL; ++i) asrc[i] += (t & 3) == 3 ? ajump[i] : 128;
            bsrc += 64 * bpitch;
        };
        int is = 0;
#define AGK_LOOK 3
        unsigned h1 = 0, h2 = 0;
        constexpr unsigned HM = (1u << (D - 1)) - 1;
#pragma nounroll
        for (int t = 0; t < NSTEP; ++t) {
            if (t >= D) {
                if (t == D) { wait_vm<(D - 1) * NI>(); bar(); }
                if ((h2 & HM) == HM) wait_vm<(D - 2) * (NI + 2) + 2>(); else if ((h1 & HM) == HM) wait_vm<(D - 2) * (NI + 1) + 1>(); else wait_vm<(D - 2) * NI>();
                bar();
            }
            ensure(t);
            if (t == 0 && lw == 0) STAMP(1);
            issue(t, is);
            is = is + 1 == NSLOT ? 0 : is + 1;
            h1 <<= 1; h2 <<= 1;
            if (mark < NSTEP && mark - t <= AGK_LOOK) {
                eval();
                if (mark < NSTEP && mark - t <= AGK_LOOK) { h1 |= 1; h2 |= xmark < NSTEP && wmark < NSTEP; poll(); }
            }
        }
        if (lw == 0) STAMP(4);
#pragma unroll
        for (int i = 0; i < D; ++i) {
            if (i == 0) wait_vm<(D - 2) * NI>();
            else if (i == 1) wait_vm<(D > 3 ? D - 3 : 0) * NI>();
            else if (i == 2) wait_vm<(D > 4 ? D - 4 : 0) * NI>();
            else if (i == 3) wait_vm<(D > 5 ? D - 5 : 0) * NI>();
            else if (i == 4) wait_vm<(D > 6 ? D - 6 : 0) * NI>();
            else if (i == 5) wait_vm<(D > 7 ? D - 7 : 0) * NI>();
            else wait_vm<0>();
            bar();
        }
    } else {
        const int fr = lane & 15, g = lane >> 4;
        const int wm = wv >> 1, wn = wv & 1;
        unsigned aoff[2];
#pragma unroll
        for (int kk = 0; kk < 2; ++kk) aoff[kk] = wm * SPW * 2048 + fr * 128 + (((kk * 4 + g) ^ ((fr >> 1) & 7)) << 4);
        unsigned boff[NFR];
        {
            const int q = fr >> 2, pp = lane & 3, f = (q >> 1) | ((g & 1) << 1);
#pragma unroll
            for (int nb = 0; nb < NFR; ++nb) {
                const int gn = wn * NFR + nb;
                boff[nb] = A_BYTES + (gn >> 2) * 8192 + (8 * g + q) * 128 + (((gn & 3) ^ f) << 5) + pp * 8;
            }
        }
        int rowbase[SPW];
#pragma unroll
        for (int i = 0; i < SPW; ++i) {
            const int code = P.strip[first_strip + wm * SPW + i];
            rowbase[i] = stash ? (first_strip + wm * SPW + i) * 16 : rfl((P.rank ^ (code >> 5)) * MSH + (code & 31) * 16);
        }
        GAS char* outp = stash ? myws + STASH_OFF + (cg & 3) * STASH_BYTES : (GAS char*)P.out[wremote ? cg : 0] + (cg & 3) * 256;
        GAS char* cntp = stash ? myws + SCNT_OFF + (cg & 3) * 256 + (first_strip >> 3) * 4 : (GAS char*)P.ws[cg] + OCNT_OFF + (cg & 3) * 256;
        const int orow = stash ? 256 : NSH * 2;
        asm volatile("" : "+s"(outp), "+s"(cntp));
        f32x4 acc[SPW][NFR];
#pragma unroll
        for (int i = 0; i < SPW; ++i)
#pragma unroll
            for (int nb = 0; nb < NFR; ++nb) acc[i][nb] = (f32x4){0.f, 0.f, 0.f, 0.f};
        struct Frag { bf16x8 a[SPW]; bf16x8 b[NFR]; };
        auto rd = [&](Frag& F, int slot, int kk) {
            LAS char* sb = smem + slot * SLOT_BYTES;
#pragma unroll
            for (int i = 0; i < SPW; ++i) F.a[i] = *(LAS bf16x8*)(sb + aoff[kk] + i * 2048);
#pragma unroll
            for (int nb = 0; nb < NFR; ++nb) {
                const s16x4 lo = __builtin_amdgcn_ds_read_tr16_b64_v4i16((LAS s16x4*)(sb + boff[nb] + kk * 4096));
                const s16x4 hi = __builtin_amdgcn_ds_read_tr16_b64_v4i16((LAS s16x4*)(sb + boff[nb] + kk * 4096 + 512));
                F.b[nb] = __builtin_bit_cast(bf16x8, __builtin_shufflevector(lo, hi, 0, 1, 2, 3, 4, 5, 6, 7));
            }
        };
        auto mm = [&](const Frag& F) {
#pragma unroll
            for (int nb = 0; nb < NFR; ++nb)
#pragma unroll
                for (int i = 0; i < SPW; ++i) acc[i][nb] = __builtin_amdgcn_mfma_f32_16x16x32_bf16(F.b[nb], F.a[i], acc[i][nb], 0, 0, 0);
        };
        Frag F0, F1;
        int cs = 0;
        bar();
        rd(F0, 0, 0);
        wait_lgkm();
#pragma nounroll
        for (int t = 0; t < NSTEP - 1; ++t) {
            bar();
            const int ns = cs + 1 == NSLOT ? 0 : cs + 1;
            constexpr int NRD = SPW + 2 * NFR, NMM = SPW * NFR, RPG = (NRD + NMM - 1) / NMM;
            rd(F1, cs, 1);
            mm(F0);
#pragma unroll
            for (int i = 0; i < NMM; ++i) { __builtin_amdgcn_sched_group_barrier(0x008, 1, 0); if (i * RPG < NRD) __builtin_amdgcn_sched_group_barrier(0x100, RPG, 0); }
            PIN();
            rd(F0, ns, 0);
            mm(F1);
#pragma unroll
            for (int i = 0; i < NMM; ++i) { __builtin_amdgcn_sched_group_barrier(0x008, 1, 1); if (i * RPG < NRD) __builtin_amdgcn_sched_group_barrier(0x100, RPG, 1); }
            PIN();
            wait_lgkm();
            cs = ns;
        }
        bar();
        rd(F1, cs, 1);
        PIN();
        mm(F0);
        PIN();
        mm(F1);
        if (wv == 0) STAMP(2);
        constexpr int ROWB = NFR * 32;
        constexpr int CPR = ROWB / 16;
        constexpr int ESLOT = (NSTEP - 1) % NSLOT == 0 ? 1 : 0;
        LAS char* eb = smem + ESLOT * SLOT_BYTES + wv * (SPW * 16 * ROWB);
        static_assert(4 * SPW * 16 * ROWB <= SLOT_BYTES, "epilogue space");
#pragma unroll
        for (int i = 0; i < SPW; ++i)
#pragma unroll
            for (int nb = 0; nb < NFR; ++nb) {
                u32x2 o;
                o.x = pack_bf16(acc[i][nb][0], acc[i][nb][1]);
                o.y = pack_bf16(acc[i][nb][2], acc[i][nb][3]);
                *(LAS u32x2*)(eb + (i * 16 + fr) * ROWB + (((nb * 2 + (g >> 1)) ^ (fr & (CPR - 1))) << 4) + (g & 1) * 8) = o;
            }
        constexpr int RPI = 64 / CPR;
        constexpr int NST = SPW * 16 / RPI;
        u32x4 ov[NST];
        GAS char* od[NST];
        GAS char* obase = outp + (NH == 2 ? wn * 128 : ch * 128 + wn * 64) + (lane & (CPR - 1)) * 16;
#pragma unroll
        for (int n = 0; n < NST; ++n) {
            const int row = n * RPI + lane / CPR;
            ov[n] = *(LAS u32x4*)(eb + row * ROWB + (((lane & (CPR - 1)) ^ (row & (CPR - 1))) << 4));
            od[n] = obase + (size_t)((rowbase[(n * RPI) / 16] + (row & 15)) * orow);
        }
        if (wv == 0) STAMP(5);
        if (win == 1) {
#pragma unroll
            for (int n = 0; n < NST; ++n) { if (n >= 1) wait_vm<0>(); st16_wt(od[n], ov[n]); }
        } else if (win == 2) {
#pragma unroll
            for (int n = 0; n < NST; ++n) { if (n >= 2) wait_vm<1>(); st16_wt(od[n], ov[n]); }
        } else {
#pragma unroll
            for (int n = 0; n < NST; ++n) st16_wt(od[n], ov[n]);
        }
        if (wremote) {
            wait_vm<0>();
            if (lane == 0) __hip_atomic_fetch_add((gu32*)cntp, stash ? (unsigned)NH : 1u, __ATOMIC_RELAXED, __HIP_MEMORY_SCOPE_SYSTEM);
        }
        if (wv == 0) STAMP(3);
    }
}

__global__ void __launch_bounds__(512, 1) __attribute__((amdgpu_num_vgpr(240))) ag_gemm_kernel(const Params P) {
    __shared__ __attribute__((aligned(1024))) char smem_raw[LDS_BYTES];
    LAS char* smem = (LAS char*)smem_raw;
    GAS char* myws = (GAS char*)P.ws[0];
    const int b = blockIdx.x;
    gu32* cc = (gu32*)(myws + CALL_OFF + b * 64);
    const unsigned role = rfl(P.role[b]);
    const int type = role & 15;
    if (type == R_COMM || type == R_UNPACK) {
        const int wv = rfl(threadIdx.x >> 6);
        const int nun = P.nunpack & 255, cun = P.nunpack >> 8;
        unsigned call;
        if (type == R_COMM && wv < NCW) {
            const int g = wv * NCOMM + (int)(role >> 4);
            if (g >= 7 * WPL) return;
            call = comm_role(P, smem + wv * 16384, cc + wv, wv, 1 + g % 7, g / 7);
        } else {
            if (type == R_COMM && wv >= NCW + cun) return;
            call = rfl(ld_agent(cc + wv)) + 1u;
            const int w = type == R_COMM ? nun * 8 + (wv - NCW) * NCOMM + (int)(role >> 4) : (int)(role >> 4) + nun * wv;
            unpack_role(P, myws, call, w, nun * 8 + cun * NCOMM);
            if (wv == 7) STAMP(4);
        }
        if ((threadIdx.x & 63) == 0) st_agent(cc + wv, call);
    } else if (type != R_IDLE) {
        const unsigned call = rfl(ld_agent(cc)) + 1u;
        if (threadIdx.x == 0) STAMP(0);
        const int fs = (role >> 4) & 255, cg = (role >> 12) & 7, ch = (role >> 15) & 1, win = (role >> 18) & 3;
        const bool stash = (role >> 16) & 1;
        if (type == R_T128) tile_role<8, 2, 5>(P, smem, call, fs, cg, 0, stash, win);
        else if (type == R_T128H) tile_role<8, 1, 6>(P, smem, call, fs, cg, ch, stash, win);
        else tile_role<4, 1, 8>(P, smem, call, fs, cg, ch, stash, win);
        if (threadIdx.x == 256) st_agent(cc, call);
    }
}

}

#define TUNE(i, dflt) (dflt)

extern "C" void kernel_launch(int rank, int nranks, ncclComm_t comm,
                              hipStream_t stream,
                              void* const* d_in, const int* in_sizes, int n_in,
                              void* d_out, int out_size,
                              const int* in_dtypes, int out_dtype,
                              void* const* d_out_all,
                              void* const* d_ws_all, size_t ws_bytes) {
    using namespace agk;
    (void)comm; (void)in_dtypes; (void)out_dtype; (void)d_out;
    if (nranks != NR || (unsigned)rank >= (unsigned)NR || n_in < 2 || ws_bytes < WS_NEED ||
        in_sizes[0] != MSH * KD || in_sizes[1] != KD * NSH || out_size != NR * MSH * NSH) {
        fprintf(stderr, "ag_gemm: unexpected problem shape\n");
        return;
    }
    Params P;
    P.x = (const char*)d_in[0];
    P.w = (const char*)d_in[1];
    P.rank = rank;
    for (int j = 0; j < NR; ++j) { P.ws[j] = (char*)d_ws_all[rank ^ j]; P.out[j] = (char*)d_out_all[rank ^ j]; }
    int sega = TUNE(0, 11);
    int blate = TUNE(1, 9);
    int wlead = TUNE(2, 4);
    int rt_b = TUNE(3, 2), rt_c = TUNE(4, 2), rt_a = TUNE(5, 0), rt_w = TUNE(6, 0);
    int split_c = TUNE(7, 0);
    int split_a = TUNE(8, 0);
    int small_b = TUNE(9, 0);
    const int segb = HS - sega;
    const int na_t = (3 * sega + 7) / 8, nc_t = (4 * AS + 7) / 8, nb_t = (3 * segb + 7) / 8, no_t = 16 - na_t - nc_t - nb_t;
    const int ne_t = no_t + na_t;
    bool ok = no_t >= 0 && ne_t <= NEARLY_T;
    int n = 0, own = 0;
    for (int i = 0; i < no_t * 8; ++i) P.strip[n++] = (unsigned char)(own++);
    for (int s = 0; s < sega; ++s) for (int j = 1; j < 4; ++j) P.strip[n++] = (unsigned char)((j << 5) | s);
    while (n % 8) P.strip[n++] = (unsigned char)(own++);
    for (int s = HS; s < NSTRIP; ++s) for (int j = 4; j < 8; ++j) P.strip[n++] = (unsigned char)((j << 5) | s);
    while (n % 8) P.strip[n++] = (unsigned char)(own++);
    for (int s = sega; s < HS; ++s) for (int j = 1; j < 4; ++j) P.strip[n++] = (unsigned char)((j << 5) | s);
    while (n % 8) P.strip[n++] = (unsigned char)(own++);
    ok = ok && n == 128 && own == HS;
    for (int i = 0; i < NEARLY_T * 8; ++i) {
        const unsigned row = (unsigned)((rank ^ (P.strip[i] >> 5)) * MSH + (P.strip[i] & 31) * 16);
        P.orow[i / 2] = (i & 1) ? P.orow[i / 2] | (row << 16) : row;
    }
    int q = 0;
    const int t_first = TUNE(16, 100) * 16, t_pk = TUNE(17, 180), t_raw = TUNE(18, 236);
    const int rh_a = TUNE(19, 0), rh_w = TUNE(20, 0), afirst = TUNE(21, 0);
    int tq = t_first;
    auto emit = [&](unsigned e) { if (q < NP_IN + NP_XI) P.ord[q] = e | (unsigned)(tq >> 4) << 16; ++q; tq += (e & 512u) ? t_raw : t_pk; };
    auto xchunk = [&](int s0, int s1, int kc, int rt, int rh) { for (int s = s0; s < s1; ++s) emit((unsigned)kc | (s << 4) | (kc >= NKC - rt || kc < rh ? 512u : 0u)); };
    auto wchunk = [&](int kc) { for (int i = 0; i < 8; ++i) emit((unsigned)(kc * 8 + i) | 1024u | (kc >= NKC - rt_w || kc < rh_w ? 512u : 0u)); };
    for (int ka = 0, kb = 0; kb < NKC; ) {
        if (ka < NKC && kb * (NKC - afirst) >= (ka - afirst) * (NKC - blate)) xchunk(0, sega, ka++, rt_a, rh_a);
        else xchunk(sega, HS, kb++, rt_b, 0);
    }
    ok = ok && q == NP_IN;
    tq = t_first;
    for (int k = -wlead; k < NKC; ++k) {
        if (k + wlead < NKC) wchunk(k + wlead);
        if (k >= 0) xchunk(HS, NSTRIP, k, rt_c, 0);
    }
    ok = ok && q == NP_IN + NP_XI;
    P.ord[NP_IN + NP_XI] = (unsigned)t_raw | (unsigned)(tq >> 4) << 16;
    int fill[8] = {0, 0, 0, 0, 0, 0, 0, 0};
    for (int i = 0; i < NWG; ++i) P.role[i] = R_IDLE;
    auto put = [&](int k, unsigned role) { if (fill[k] < NWG / 8) P.role[fill[k]++ * 8 + k] = role; else ok = false; };
    auto code = [&](int type, int fs, int cg, int ch, bool stash) { return (unsigned)type | (fs << 4) | (cg << 12) | (ch << 15) | ((stash ? 1u : 0u) << 16); };
    for (int i = 0; i < NCOMM; ++i) put(i & 7, R_COMM | (i << 4));
    unsigned adds = ne_t * 4;
    for (int cg = 0; cg < 8; ++cg) {
        const bool rem = cg >= 4;
        for (int r = 0; r < 16; ++r) {
            const int k = (cg & 3) + 4 * (r & 1);
            const bool early = r < ne_t, last = r >= ne_t + nc_t;
            const bool split = last || (rem ? (early ? r >= ne_t - split_a : r - ne_t < split_c) : (!early && r - ne_t < TUNE(22, 0)));
            const bool small = last && r - ne_t - nc_t < (rem ? small_b >> 4 : small_b & 15);
            const unsigned win = (unsigned)(!rem || early ? 0 : last ? TUNE(12, 0) : split ? TUNE(13, 1) : TUNE(11, 2)) << 18;
            int nw = 0;
            if (small) for (int h = 0; h < 4; ++h) { put(k, win | code(R_T64H, r * 8 + (h >> 1) * 4, cg, h & 1, false)); nw += 4; }
            else if (split) for (int ch = 0; ch < 2; ++ch) { put(k, win | code(R_T128H, r * 8, cg, ch, early && rem)); nw += 4; }
            else { put(k, win | code(R_T128, r * 8, cg, 0, early && rem)); nw += 4; }
            if (cg == 4 && !early) adds += nw;
        }
    }
    unsigned nun = 0;
    for (int i = 0; i < NWG; ++i) if (P.role[i] == R_IDLE && nun < (unsigned)TUNE(10, 64)) P.role[i] = R_UNPACK | (nun++ << 4);
    P.out_adds = adds; P.npx = NP_XI + ne_t * 4; P.nunpack = nun | (unsigned)TUNE(14, 0) << 8;
    ok = ok && P.nunpack > 0 && (int)(P.nunpack >> 8) <= 8 - NCW;
    if (!ok) { fprintf(stderr, "ag_gemm: role table overflow\n"); return; }
    hipLaunchKernelGGL(ag_gemm_kernel, dim3(NWG), dim3(512), 0, stream, P);
    HIP_CHECK(hipGetLastError());
}
```

```cpp
#include <hip/hip_runtime.h>
#include <rccl/rccl.h>
#include <cstdio>
#include <cstdint>

#define HIP_CHECK(expr) do { hipError_t _e = (expr); if (_e != hipSuccess) { \
    fprintf(stderr, "HIP error %s at %s:%d\n", hipGetErrorString(_e), __FILE__, __LINE__); \
    return; } } while (0)

namespace agk {

typedef unsigned int u32x4 __attribute__((ext_vector_type(4)));
typedef unsigned int u32x2 __attribute__((ext_vector_type(2)));
typedef short s16x4 __attribute__((ext_vector_type(4)));
typedef short s16x8 __attribute__((ext_vector_type(8)));
typedef __bf16 bf16x8 __attribute__((ext_vector_type(8)));
typedef float f32x4 __attribute__((ext_vector_type(4)));
#define LAS __attribute__((address_space(3)))
#define GAS __attribute__((address_space(1)))
typedef GAS unsigned int gu32;

constexpr int NR = 8;
constexpr int KD = 4096;
constexpr int MSH = 512;
constexpr int NSH = 512;
constexpr int NSTRIP = 32;
#define AGK_HS 21
constexpr int HS = AGK_HS;
constexpr int AS = NSTRIP - HS;
constexpr int NKC = 16;
constexpr int NSTEP = 64;
constexpr int NWP = 128;

#define AGK_NCOMM 56
#define AGK_NCW 4
constexpr int NCOMM = AGK_NCOMM;
constexpr int NCW = AGK_NCW;
constexpr int WPL = NCOMM * NCW / 7;

constexpr int NEARLY_T = 8;
constexpr int NP_IN = HS * NKC;
constexpr int NP_XI = NWP + AS * NKC;
constexpr int NPMAX = NP_IN > NP_XI ? NP_IN : NP_XI;
constexpr int NWG = 256;
enum { R_IDLE = 0, R_COMM = 1, R_T128 = 2, R_T128H = 3, R_T64H = 4, R_UNPACK = 5 };

constexpr size_t STRIP_BYTES = 16ull * KD * 2;
constexpr int NXSLOT = 3 * HS + 4 * AS;
constexpr size_t XS_BYTES = (size_t)NXSLOT * STRIP_BYTES;
constexpr size_t WSL_BYTES = 8 * STRIP_BYTES;
constexpr int XF_STRIDE = 256;
constexpr size_t XFLAG_OFF = XS_BYTES + 4 * WSL_BYTES;
constexpr size_t WFLAG_OFF = XFLAG_OFF + 8 * 32 * XF_STRIDE;
constexpr size_t PFLAG_OFF = WFLAG_OFF + 4 * NWP * 4;
constexpr size_t BUF_BYTES = (PFLAG_OFF + 8 * NPMAX * 8 + 4095) / 4096 * 4096;
constexpr size_t STASH_OFF = 2 * BUF_BYTES;
constexpr size_t STASH_BYTES = (size_t)NEARLY_T * 8 * 4096;
constexpr size_t OCNT_OFF = STASH_OFF + 4 * STASH_BYTES;
constexpr size_t SCNT_OFF = OCNT_OFF + 4 * 256;
constexpr size_t TMO_OFF = SCNT_OFF + 4 * 256;
constexpr size_t CALL_OFF = TMO_OFF + 256;
constexpr size_t DBG_OFF = CALL_OFF + 256 * 64;
constexpr size_t DBG2_OFF = DBG_OFF + 256 * 128;
constexpr size_t WS_NEED = DBG2_OFF + 8 * 512 * 32;
static_assert(BUF_BYTES % 4096 == 0, "alignment");
static_assert(WS_NEED <= 65ull << 20, "workspace");

constexpr unsigned long long DEADLINE_TICKS = 30000000ull;
constexpr int LDS_BYTES = 163840;

struct Params {
    const char* x;
    const char* w;
    char* ws[NR];
    char* out[NR];
    int rank;
    unsigned out_adds;
    unsigned npx;
    unsigned nunpack;
    unsigned role[NWG];
    unsigned orow[NEARLY_T * 4];
    unsigned ord[NP_IN + NP_XI + 1];
    unsigned char strip[128];
};

__device__ __forceinline__ unsigned ld_sys(const gu32* p) { return __hip_atomic_load(p, __ATOMIC_RELAXED, __HIP_MEMORY_SCOPE_SYSTEM); }
__device__ __forceinline__ unsigned ld_agent(const gu32* p) { return __hip_atomic_load(p, __ATOMIC_RELAXED, __HIP_MEMORY_SCOPE_AGENT); }
__device__ __forceinline__ void st_agent(gu32* p, unsigned v) { __hip_atomic_store(p, v, __ATOMIC_RELAXED, __HIP_MEMORY_SCOPE_AGENT); }
__device__ __forceinline__ unsigned long long now() { return __builtin_amdgcn_s_memrealtime(); }
__device__ __forceinline__ unsigned rfl(unsigned v) { return __builtin_amdgcn_readfirstlane(v); }

__device__ __forceinline__ void st16_wt(GAS char* p, u32x4 v) {
    asm volatile("global_store_dwordx4 %0, %1, off sc0 sc1\n\ts_nop 1" :: "v"(p), "v"(v) : "memory");
}
template <int N> __device__ __forceinline__ void wait_vm() { asm volatile("s_waitcnt vmcnt(%0)" :: "i"(N) : "memory"); }
__device__ __forceinline__ void wait_lgkm() { asm volatile("" ::: "memory"); __builtin_amdgcn_s_waitcnt(0xC07F); asm volatile("" ::: "memory"); }
#define AGK_SL1 6
#define PIN() __builtin_amdgcn_sched_barrier(0)
__device__ __forceinline__ void bar() { asm volatile("s_barrier" ::: "memory"); }

#define AGK_LDPOL "sc0 sc1"
__device__ __forceinline__ void glds2(const GAS char* base, unsigned v0, unsigned v1, unsigned lds_dst) {
    asm volatile("s_mov_b32 m0, %3\n\ts_nop 3\n\tglobal_load_lds_dwordx4 %1, %0 " AGK_LDPOL "\n\tglobal_load_lds_dwordx4 %2, %0 offset:1024 " AGK_LDPOL
                 :: "s"(base), "v"(v0), "v"(v1), "s"(lds_dst) : "m0", "memory");
}
template <int OFF> __device__ __forceinline__ void glds2o(const GAS char* base, unsigned v0, unsigned v1) {
    asm volatile("global_load_lds_dwordx4 %1, %0 offset:%3 " AGK_LDPOL "\n\tglobal_load_lds_dwordx4 %2, %0 offset:%4 " AGK_LDPOL
                 :: "s"(base), "v"(v0), "v"(v1), "i"(OFF), "i"(OFF + 1024) : "memory");
}
__device__ __forceinline__ void wait_vm_var(unsigned n) {
    asm volatile("s_getpc_b64 vcc\n\ts_add_u32 vcc_lo, vcc_lo, %0\n\ts_addc_u32 vcc_hi, vcc_hi, 0\n\ts_setpc_b64 vcc\n\t"
                 ".set agk_n, 0\n\t.rept 48\n\ts_waitcnt vmcnt(agk_n)\n\ts_branch 1f\n\t.set agk_n, agk_n + 1\n\t.endr\n1:"
                 :: "s"(n * 8 + 12) : "vcc", "scc", "memory");
}

__device__ __forceinline__ unsigned pack_bf16(float lo, float hi) {
    typedef float f32x2 __attribute__((ext_vector_type(2)));
    typedef __bf16 bf16x2 __attribute__((ext_vector_type(2)));
    f32x2 v = {lo, hi};
    bf16x2 r = __builtin_convertvector(v, bf16x2);
    return __builtin_bit_cast(unsigned, r);
}

__device__ __forceinline__ int xslot(int j, int s) { return j < 4 ? (j - 1) * HS + s : 3 * HS + (j - 4) * AS + (s - HS); }

#define STAMP(i) do { } while (0)
#define PSTAMP(j, p, i, t) do { } while (0)

enum { K_X = 0, K_W = 1, K_OUT = 2 };
struct Piece { int kind, raw; unsigned soff, doff, foff, when; };

__device__ __forceinline__ Piece locate(const Params& P, int j, int p) {
    Piece r;
    if (j >= 4 && p >= NP_XI) {
        r.kind = K_OUT; r.raw = 1;
        r.soff = (unsigned)((j - 4) * STASH_BYTES) + (p - NP_XI) * 8192;
        r.doff = p - NP_XI; r.foff = 0;
        const unsigned e = P.ord[NP_IN + NP_XI];
        r.when = (e >> 16) + (((p - NP_XI) * (e & 0xFFFFu)) >> 4);
        return r;
    }
    const unsigned e = P.ord[j < 4 ? p : NP_IN + p];
    r.when = e >> 16;
    const int s = (e >> 4) & 31, kc = e & 15, wp = (e >> 10) & 1 ? (int)(e & 127) : -1;
    r.raw = (e >> 9) & 1;
    if (wp >= 0) {
        r.kind = K_W;
        r.soff = (unsigned)(wp * 32 * (NSH * 2) + (j & 3) * 256);
        r.doff = (unsigned)XS_BYTES + (unsigned)(j - 4) * (unsigned)WSL_BYTES + (wp & 1) * 2 * (unsigned)STRIP_BYTES + (wp >> 1) * 2048;
        r.foff = (unsigned)WFLAG_OFF + ((j - 4) * NWP + (wp & 1) * 64 + (wp >> 1)) * 4;
    } else {
        r.kind = K_X;
        r.soff = (unsigned)(s * 16 * (KD * 2) + kc * 512);
        r.doff = (unsigned)xslot(j, s) * (unsigned)STRIP_BYTES + kc * 8192;
        r.foff = (unsigned)XFLAG_OFF + (j * 32 + s) * XF_STRIDE + kc * 4;
    }
    return r;
}
__device__ __forceinline__ unsigned pf_off(int j, int p) { return (unsigned)PFLAG_OFF + (unsigned)(j * NPMAX + p) * 8u; }

__device__ __forceinline__ void st_w(const u32x4& v, unsigned voff, const GAS char* dst) {
    asm volatile("s_nop 3\n\tglobal_store_dwordx4 %1, %0, %2 sc0 sc1\n\ts_nop 1" :: "v"(v), "v"(voff), "s"(dst) : "memory");
}
#define AGK_UPOL "sc0 sc1"
#define AGK_ULPOL "sc0 sc1"
template <int OFF> __device__ __forceinline__ void st_wo(const u32x4& v, unsigned voff, const GAS char* dst) {
    asm volatile("s_nop 3\n\tglobal_store_dwordx4 %1, %0, %2 offset:%3 " AGK_UPOL "\n\ts_nop 1" :: "v"(v), "v"(voff), "s"(dst), "i"(OFF) : "memory");
}
template <int OFF> __device__ __forceinline__ void ld_c(u32x4& v, unsigned voff, const GAS char* src) {
    asm volatile("s_nop 4\n\tglobal_load_dwordx4 %0, %1, %2 offset:%3 " AGK_ULPOL : "=v"(v) : "v"(voff), "s"(src), "i"(OFF) : "memory");
}
__device__ __forceinline__ void flag_st(const GAS char* f, unsigned val, unsigned zero) {
    asm volatile("s_mov_b64 exec, 1\n\ts_nop 3\n\tglobal_store_dword %0, %1, %2 sc0 sc1\n\ts_mov_b64 exec, -1" :: "v"(zero), "v"(val), "s"(f) : "memory");
}
__device__ __forceinline__ void flag_st64(const GAS char* f, u32x2 val, unsigned zero) {
    asm volatile("s_mov_b64 exec, 1\n\ts_nop 3\n\tglobal_store_dwordx2 %0, %1, %2 sc0 sc1\n\ts_mov_b64 exec, -1" :: "v"(zero), "v"(val), "s"(f) : "memory");
}
__device__ __forceinline__ void add_sys(const GAS char* c, unsigned one, unsigned zero) {
    asm volatile("s_mov_b64 exec, 1\n\ts_nop 3\n\tglobal_atomic_add %0, %1, %2 sc1\n\ts_mov_b64 exec, -1" :: "v"(zero), "v"(one), "s"(c) : "memory");
}
__device__ __forceinline__ unsigned perm(unsigned hi, unsigned lo, unsigned sel) { return __builtin_amdgcn_perm(hi, lo, sel); }
__device__ __forceinline__ unsigned bfi(unsigned m, unsigned a, unsigned b) { return (a & m) | (b & ~m); }
__device__ __forceinline__ unsigned rotr(unsigned v, unsigned n) { return __builtin_amdgcn_alignbit(v, v, n); }
__device__ __forceinline__ unsigned pkmax(unsigned a, unsigned b) {
    typedef unsigned short u16x2 __attribute__((ext_vector_type(2)));
    return __builtin_bit_cast(unsigned, __builtin_elementwise_max(__builtin_bit_cast(u16x2, a), __builtin_bit_cast(u16x2, b)));
}
template <int CTRL> __device__ __forceinline__ unsigned dppmax(unsigned v) { return max(v, (unsigned)__builtin_amdgcn_update_dpp(0, (int)v, CTRL, 0xF, 0xF, true)); }
constexpr unsigned F_RAW = 0x80u;
constexpr int MAXEXC = 16;
__device__ __forceinline__ unsigned ximg0(unsigned lane, unsigned pitch) { return (lane >> 3) * pitch + ((lane & 7) ^ (lane >> 4)) * 16; }
__device__ __forceinline__ unsigned ximg1(unsigned lane, unsigned pitch) { return (8 + (lane >> 3)) * pitch + ((lane & 7) ^ (4 + (lane >> 4))) * 16 - 1024; }
__device__ __forceinline__ unsigned wimg0(unsigned lane, unsigned pitch) { return (lane >> 3) * pitch + ((lane & 7) ^ ((lane >> 3) & 2)) * 16; }
__device__ __forceinline__ unsigned wimg1(unsigned lane, unsigned pitch) { return (8 + (lane >> 3)) * pitch + ((lane & 7) ^ (((lane >> 3) & 2) | 4)) * 16 - 1024; }

__device__ __forceinline__ void fetch_piece(int kind, unsigned soff, const GAS char* sx, const GAS char* sw, const GAS char* ss,
                                            unsigned vx0, unsigned vx1, unsigned vw0, unsigned vw1, unsigned vl, unsigned l) {
    const bool kx = kind == K_X, kw = kind == K_W;
    const GAS char* s = (kx ? sx : kw ? sw : ss) + soff;
    const unsigned a = kx ? vx0 : kw ? vw0 : vl, b = kx ? vx1 : kw ? vw1 : vl;
    const int q1 = kx ? 128 : kw ? 16 * NSH * 2 : 2048, q2 = kx ? 256 : kw ? 128 : 4096;
    glds2(s, a, b, l); glds2(s + q1, a, b, l + 2048); glds2(s + q2, a, b, l + 4096); glds2(s + q1 + q2, a, b, l + 6144);
}

__device__ __forceinline__ unsigned comm_role(const Params& P, LAS char* smem, const gu32* cc, int wslot, int j, int u) {
    const unsigned tstart = (unsigned)now();
    GAS char* myws = (GAS char*)P.ws[0];
    const int lane = threadIdx.x & 63;
    const int npiece = j < 4 ? NP_IN : (int)P.npx;
    const unsigned vlin = lane * 16;
    const unsigned vx0 = ximg0(lane, KD * 2), vx1 = ximg1(lane, KD * 2), vw0 = wimg0(lane, NSH * 2), vw1 = wimg1(lane, NSH * 2);
    const unsigned vblk = (lane >> 5) * 2048 + (lane & 31) * 16, vblw = (lane >> 5) * (unsigned)STRIP_BYTES + (lane & 31) * 16;
    const unsigned vout = ((lane >> 5) * 8 + ((lane >> 4) & 1)) * (NSH * 2) + (lane & 15) * 16;
    const unsigned lds0 = (unsigned)(size_t)smem;
    const GAS char* const srcx = (const GAS char*)P.x;
    const GAS char* const srcw = (const GAS char*)P.w;
    const GAS char* const srcs = myws + STASH_OFF;
#define AGK_FETCH(q, slot) fetch_piece((q).kind, (q).soff, srcx, srcw, srcs, vx0, vx1, vw0, vw1, vlin, lds0 + (slot) * 8192)
    int pA = u;
    unsigned ccv;
    asm volatile("global_load_dword %0, %1, off sc1" : "=v"(ccv) : "v"(cc) : "memory");
    Piece A = locate(P, j, pA), B = locate(P, j, pA + WPL);
    AGK_FETCH(A, 0); AGK_FETCH(B, 1);
    asm volatile("v_mov_b32 v251, 0" ::: "v251", "memory");
    const GAS char* pws = (const GAS char*)P.ws[j];
    const GAS char* outp = (const GAS char*)P.out[j] + (j & 3) * 256;
    asm volatile("" : "+s"(pws), "+s"(outp));
    asm volatile("s_waitcnt vmcnt(8)" : "+v"(ccv) :: "memory");
    const unsigned call = rfl(ccv) + 1u;
    if (wslot == 0) STAMP(0);
    const GAS char* dbuf = pws + (size_t)(call & 1) * BUF_BYTES;
    const GAS char* ocnt = pws + OCNT_OFF + (j & 3) * 256;
    const GAS char* scnt = myws + SCNT_OFF + (j & 3) * 256;
    gu32* tmo = (gu32*)(myws + TMO_OFF);
    const unsigned sneed = call * 8u;
    unsigned one = 1, zero = 0, callv = call, v4 = lane * 4;
    asm volatile("" : "+v"(one), "+v"(zero), "+v"(callv), "+v"(v4));
    int slot = 0;
    bool poison = false, defA = false, defB = false;
    auto stashed = [&](unsigned piece) {
        unsigned sv;
        asm volatile("s_nop 3\n\tv_readlane_b32 %0, v251, %1" : "=s"(sv) : "s"(piece >> 2) : "memory");
        return sv >= sneed;
    };
    for (;;) {
        if (defA) {
            unsigned long long t0 = 0;
            for (unsigned spin = 1; !stashed(A.doff); ++spin) {
                if ((spin & 63) == 0) {
                    if (t0 == 0) t0 = now();
                    if (rfl(ld_agent(tmo)) == call || now() - t0 > DEADLINE_TICKS) { st_agent(tmo, call); poison = true; break; }
                }
                asm volatile("s_sleep 2\n\ts_mov_b64 exec, 0xff\n\ts_nop 3\n\tglobal_load_dword v251, %1, %0 sc1\n\ts_mov_b64 exec, -1\n\ts_waitcnt vmcnt(0)"
                             :: "s"(scnt), "v"(v4) : "v251", "memory");
            }
            AGK_FETCH(A, slot);
            wait_vm<0>();
        }
        if (defB && stashed(B.doff)) { AGK_FETCH(B, slot ^ 1); defB = false; }
        u32x4 v[8];
#pragma unroll
        for (int k = 0; k < 8; ++k) v[k] = *(LAS u32x4*)(smem + slot * 8192 + (k >> 2) * 4096 + (k & 3) * 512 + vblk);
        bool raw = A.raw;
        u32x2 ev = {0, 0};
        unsigned fmt = F_RAW, eoff = 0; unsigned long long bm = 0;
        if (!raw) {
            unsigned mx = pkmax(pkmax(pkmax(v[0].x & 0x7FFF7FFFu, v[0].y & 0x7FFF7FFFu), pkmax(v[0].z & 0x7FFF7FFFu, v[0].w & 0x7FFF7FFFu)),
                                pkmax(pkmax(v[6].x & 0x7FFF7FFFu, v[6].y & 0x7FFF7FFFu), pkmax(v[6].z & 0x7FFF7FFFu, v[6].w & 0x7FFF7FFFu)));
            mx = max(mx >> 16, mx & 0xFFFFu) >> 8;
            mx = dppmax<0x143>(dppmax<0x142>(dppmax<0x118>(dppmax<0x114>(dppmax<0x112>(dppmax<0x111>(mx))))));
            fmt = max((unsigned)__builtin_amdgcn_readlane((int)mx, 63), 7u) - 7u;
            const unsigned B4 = fmt * 0x01010101u;
            unsigned accO = 0, accA = ~0u;
            u32x4 L[4], N[2];
#pragma unroll
            for (int k = 0; k < 8; ++k) {
                const unsigned Ha = perm(v[k].y, v[k].x, 0x07050301u), Hb = perm(v[k].w, v[k].z, 0x07050301u);
                const unsigned da = (Ha | 0x80808080u) - B4, db = (Hb | 0x80808080u) - B4;
                accO |= da | db; accA &= da & db;
                L[(k >> 2) * 2 + (k & 1)][k & 2] = perm(v[k].y, v[k].x, 0x06040200u);
                L[(k >> 2) * 2 + (k & 1)][(k & 2) + 1] = perm(v[k].w, v[k].z, 0x06040200u);
                N[k >> 2][(k & 1) * 2 + ((k >> 1) & 1)] = bfi(0x87878787u, bfi(0x80808080u, Ha, da), rotr(bfi(0x80808080u, Hb, db), 28));
            }
            bm = __ballot(((accO & 0x78787878u) | (~accA & 0x80808080u)) != 0);
            if (bm) {
                unsigned nb = 0, ib = 0;
#pragma unroll
                for (int i = 0; i < 16; ++i) {
                    const unsigned H = (i & 1) ? perm(v[i >> 1].w, v[i >> 1].z, 0x07050301u) : perm(v[i >> 1].y, v[i >> 1].x, 0x07050301u);
                    const bool bad = (((H | 0x80808080u) - B4) & 0xF8F8F8F8u) != 0x80808080u;
                    ev.y = bad ? H : ev.y; ib = bad ? (unsigned)i : ib; nb += bad;
                }
                ev.x = lane | (ib << 8);
                eoff = 1536 + __builtin_amdgcn_mbcnt_hi((unsigned)(bm >> 32), __builtin_amdgcn_mbcnt_lo((unsigned)bm, 0)) * 8;
                const unsigned cnt = __builtin_popcountll(bm);
                raw = cnt > MAXEXC || __ballot(nb > 1) != 0;
                fmt = raw ? F_RAW : fmt | cnt << 8;
            }
            if (!raw) { v[0] = L[0]; v[1] = L[1]; v[2] = N[0]; v[4] = L[2]; v[5] = L[3]; v[6] = N[1]; }
        } else if (poison && A.kind == K_OUT) {
#pragma unroll
            for (int k = 0; k < 8; ++k) v[k] = (u32x4){0x7FC07FC0u, 0x7FC07FC0u, 0x7FC07FC0u, 0x7FC07FC0u};
        }
        const GAS char *d0, *d1; unsigned vd = vblk; int dstep = 512;
        if (A.kind == K_OUT) {
            const unsigned rows = P.orow[A.doff];
            d0 = outp + (rows & 0xFFFFu) * (NSH * 2);
            d1 = outp + (rows >> 16) * (NSH * 2);
            vd = vout; dstep = 2 * NSH * 2;
        } else {
            d0 = dbuf + A.doff; d1 = d0 + (A.kind == K_W ? 4 * (unsigned)STRIP_BYTES : 4096u);
            if (A.kind == K_W) vd = vblw;
        }
        const int pC = pA + 2 * WPL;
        const Piece C = locate(P, j, pC < npiece ? pC : u);
        asm volatile("" :: "v"(v[0]), "v"(v[1]), "v"(v[2]), "v"(v[3]), "v"(v[4]), "v"(v[5]), "v"(v[6]), "v"(v[7]), "v"(ev), "v"(eoff), "s"(d0), "s"(d1), "s"(C.soff) : "memory");
        for (int rest = (int)(A.when - ((unsigned)now() - tstart)); rest > 2; rest -= 3) __builtin_amdgcn_s_sleep(1);
        st_w(v[0], vd, d0);
        st_w(v[1], vd, d0 + dstep);
        st_w(v[2], vd, d0 + 2 * dstep);
        st_w(v[4], vd, d1);
        st_w(v[5], vd, d1 + dstep);
        st_w(v[6], vd, d1 + 2 * dstep);
        if (raw) {
            st_w(v[3], vd, d0 + 3 * dstep);
            st_w(v[7], vd, d1 + 3 * dstep);
        } else if (bm) {
            asm volatile("s_mov_b64 exec, %3\n\ts_nop 3\n\tglobal_store_dwordx2 %0, %1, %2 sc0 sc1\n\ts_mov_b64 exec, -1" :: "v"(eoff), "v"(ev), "s"(d0), "s"(bm) : "memory");
        }
        const bool defC = C.kind == K_OUT && !stashed(C.doff);
        asm volatile("s_mov_b64 exec, 0xff\n\ts_nop 3\n\tglobal_load_dword v251, %1, %0 sc1\n\ts_mov_b64 exec, -1" :: "s"(scnt), "v"(v4) : "v251", "memory");
        AGK_FETCH(C, defC ? 2 * (NCW - wslot) : slot);
        const u32x2 f = {callv, fmt};
        wait_vm<9>();
        if (!A.raw) flag_st64(dbuf + pf_off(j, pA), f, zero);
        if (A.kind == K_OUT) add_sys(ocnt, one, zero);
        else if (raw) flag_st(dbuf + A.foff, callv, zero);
        PSTAMP(j, pA, 0, tstart + A.when); PSTAMP(j, pA, 1, tb); PSTAMP(j, pA, 2, now());
        pA += WPL; A = B; B = C; defA = defB; defB = defC; slot ^= 1;
        if (pA >= npiece) break;
    }
    STAMP(j < 4 ? 1 : 3);
    if (j == 1 && u == 0) {
        if (lane < 4) {
            const gu32* c = (const gu32*)(myws + OCNT_OFF + lane * 256);
            const unsigned need = call * P.out_adds;
            const unsigned long long t0 = now();
            bool bad = false;
            unsigned spin = 0;
            while (ld_sys(c) < need) {
                if ((++spin & 63) == 0 && (ld_agent(tmo) == call || now() - t0 > DEADLINE_TICKS)) { st_agent(tmo, call); bad = true; break; }
                __builtin_amdgcn_s_sleep(1);
            }
            if (bad) {
                __builtin_amdgcn_s_sleep(127);
                __hip_atomic_store((GAS unsigned short*)(P.out[0] + (size_t)(P.rank * MSH) * (NSH * 2)), (unsigned short)0x7FC0, __ATOMIC_RELAXED, __HIP_MEMORY_SCOPE_AGENT);
            }
        }
        STAMP(2);
    }
    return call;
}

__device__ __forceinline__ void unpack_role(const Params& P, GAS char* myws, unsigned call, int w, int nw) {
    const GAS char* buf = myws + (size_t)(call & 1) * BUF_BYTES;
    gu32* tmo = (gu32*)(myws + TMO_OFF);
    const int lane = threadIdx.x & 63;
    const unsigned vblk = (lane >> 5) * 2048 + (lane & 31) * 16, vblw = (lane >> 5) * (unsigned)STRIP_BYTES + (lane & 31) * 16;
    unsigned zero = 0, callv = call;
    asm volatile("" : "+v"(zero), "+v"(callv));
    for (int q = w; q < NPMAX * 7; q += nw) {
        const int p = q / 7, j = 1 + q - p * 7;
        if (p >= (j < 4 ? NP_IN : NP_XI)) continue;
        const Piece A = locate(P, j, p);
        if (A.raw) continue;
        const GAS unsigned long long* pf = (const GAS unsigned long long*)(buf + pf_off(j, p));
        const GAS char* d0 = buf + A.doff;
        const GAS char* d1 = d0 + (A.kind == K_W ? 4 * (unsigned)STRIP_BYTES : 4096u);
        const unsigned vd = A.kind == K_W ? vblw : vblk;
        unsigned long long fv, t0 = 0;
        unsigned spin = 0;
        for (;;) {
            fv = __hip_atomic_load(pf, __ATOMIC_RELAXED, __HIP_MEMORY_SCOPE_SYSTEM);
            if (rfl((unsigned)fv) == call) break;
            if ((++spin & 63) == 0) {
                if (t0 == 0) t0 = now();
                if (rfl(ld_agent(tmo)) == call || now() - t0 > DEADLINE_TICKS) { st_agent(tmo, call); return; }
            }
            __builtin_amdgcn_s_sleep(2);
        }
        PSTAMP(j, p, 3, now()); PSTAMP(j, p, 7, spin);
        const unsigned fmt = rfl((unsigned)(fv >> 32));
        if (fmt & F_RAW) continue;
        const int cnt = fmt >> 8;
        u32x4 L[4], N[2]; u32x2 ex = {0, 0};
        ld_c<1024>(N[0], vd, d0); ld_c<1024>(N[1], vd, d1);
        ld_c<0>(L[0], vd, d0); ld_c<512>(L[1], vd, d0); ld_c<0>(L[2], vd, d1); ld_c<512>(L[3], vd, d1);
        if (cnt) asm volatile("s_nop 4\n\tglobal_load_dwordx2 %0, %1, %2 offset:1536 sc0 sc1" : "=v"(ex) : "v"((lane & (MAXEXC - 1)) * 8), "s"(d0) : "memory");
        asm volatile("s_waitcnt vmcnt(0)" : "+v"(L[0]), "+v"(L[1]), "+v"(L[2]), "+v"(L[3]), "+v"(N[0]), "+v"(N[1]), "+v"(ex) :: "memory");
        PSTAMP(j, p, 4, now());
        const unsigned B4 = (fmt & 127u) * 0x01010101u;
        unsigned H[16];
#pragma unroll
        for (int k = 0; k < 8; ++k) {
            const unsigned n = N[k >> 2][(k & 1) * 2 + ((k >> 1) & 1)];
            H[2 * k] = (n & 0x87878787u) + B4;
            H[2 * k + 1] = (rotr(n, 4) & 0x87878787u) + B4;
        }
        if (cnt) {
            unsigned tag = 255, val = 0;
            for (int e = 0; e < cnt; ++e) {
                const unsigned id = __builtin_amdgcn_readlane(ex.x, e), hv = __builtin_amdgcn_readlane(ex.y, e);
                const bool mine = (id & 63) == (unsigned)lane;
                tag = mine ? id >> 8 : tag; val = mine ? hv : val;
            }
#pragma unroll
            for (int i = 0; i < 16; ++i) H[i] = tag == (unsigned)i ? val : H[i];
        }
#define AGK_DST(k) { const unsigned l0 = L[(k >> 2) * 2 + (k & 1)][k & 2], l1 = L[(k >> 2) * 2 + (k & 1)][(k & 2) + 1]; u32x4 o; \
            o.x = perm(H[2 * k], l0, 0x05010400u); o.y = perm(H[2 * k], l0, 0x07030602u); \
            o.z = perm(H[2 * k + 1], l1, 0x05010400u); o.w = perm(H[2 * k + 1], l1, 0x07030602u); st_wo<(k & 3) * 512>(o, vd, k < 4 ? d0 : d1); }
        AGK_DST(0) AGK_DST(1) AGK_DST(2) AGK_DST(3) AGK_DST(4) AGK_DST(5) AGK_DST(6) AGK_DST(7)
        PSTAMP(j, p, 5, now());
        wait_vm<0>(); flag_st(buf + A.foff, callv, zero);
        PSTAMP(j, p, 6, now());
    }
}

template <int S, int NH, int NSLOT>
__device__ __forceinline__ void tile_role(const Params& P, LAS char* smem, unsigned call, int first_strip, int cg, int ch, bool stash, int win, unsigned lags) {
    constexpr int SPL = S / 4;
    constexpr int NB = SPL + NH, NI = 2 * NB;
    constexpr int RW = NB * 2048, SLOT_BYTES = 4 * RW;
    constexpr int SPW = S / 2, NFR = NH * 2;
    static_assert(SLOT_BYTES * NSLOT <= LDS_BYTES, "LDS ring");
    static_assert(NSLOT <= 8 && (NSLOT - 1) * (NI + 2) < 48, "vmcnt range");
    GAS char* myws = (GAS char*)P.ws[0];
    gu32* tmo = (gu32*)(myws + TMO_OFF);
    const int lane = threadIdx.x & 63;
    const int wv = rfl(threadIdx.x >> 6);
    const bool wremote = cg >= 4;

    if (wv >= 4) {
        const int lw = wv - 4;
        const GAS char* mybuf = myws + (size_t)(call & 1) * BUF_BYTES;
        const unsigned vlin = lane * 16;
        const GAS char* asrc[SPL];
        unsigned av0[SPL], av1[SPL];
        int astep[SPL], scode[SPL];
#pragma unroll
        for (int i = 0; i < SPL; ++i) {
            scode[i] = rfl(P.strip[first_strip + lw * SPL + i]);
            const bool own = scode[i] < 32;
            av0[i] = own ? ximg0(lane, KD * 2) : vlin;
            av1[i] = own ? ximg1(lane, KD * 2) : vlin;
            asrc[i] = (own ? (const GAS char*)P.x + (unsigned)scode[i] * (16 * KD * 2) : mybuf + (unsigned)xslot(scode[i] >> 5, scode[i] & 31) * (unsigned)STRIP_BYTES) + (4096 - 2048 * i);
            astep[i] = own ? 7 : 11;
        }
        const int bhalf = NH == 2 ? (lw >> 1) : ch;
        const int kb0 = NH == 2 ? (lw & 1) * 2 : lw;
        const unsigned bnext = (wremote ? (unsigned)STRIP_BYTES : 16u * NSH * 2) - 2048;
        const unsigned bv0 = wremote ? vlin : wimg0(lane, NSH * 2), bv1 = wremote ? vlin : wimg1(lane, NSH * 2), bv2 = bv0 + bnext, bv3 = bv1 + bnext;
        const GAS char* bsrc = (wremote ? mybuf + XS_BYTES + (unsigned)((cg - 4) * 8 + bhalf * 4 + kb0) * (unsigned)STRIP_BYTES
                                        : (const GAS char*)P.w + cg * 256 + bhalf * 128 + kb0 * 16 * (NSH * 2)) + (4096 - 2048 * SPL);
        const int bstep = wremote ? 11 : 16;
        const unsigned ldsw = (unsigned)(size_t)smem + lw * RW + 4096, ldsend = ldsw + NSLOT * SLOT_BYTES;

        const int fcode = (SPL == 2 && (lane & 16)) ? scode[SPL - 1] : scode[0];
        const GAS char* fx = mybuf + XFLAG_OFF + fcode * XF_STRIDE + (lane & 15) * 4;
        const GAS char* fw = mybuf + WFLAG_OFF + ((wremote ? cg - 4 : 0) * NWP + (kb0 >> 1) * 64 + lane) * 4;
        const bool xo = fcode < 32 || lane >= 16 * SPL;
        const unsigned long long xact = ~__ballot(xo);
        asm volatile("v_mov_b32 v250, %0\n\tv_mov_b32 v251, %1" :: "v"(xo ? ~0u : 0u), "v"(wremote ? 0u : ~0u) : "v250", "v251", "memory");
        int mark = 0, xmark = 0, wmark = 0;
        unsigned npoll = 0;
        auto poll = [&]() {
            if (xmark < NSTEP) { ++npoll; asm volatile("s_mov_b64 exec, %1\n\tglobal_load_dword v250, %0, off sc0 sc1\n\ts_mov_b64 exec, -1" :: "v"(fx), "s"(xact) : "v250", "memory"); }
            if (wmark < NSTEP) { ++npoll; asm volatile("global_load_dword v251, %0, off sc0 sc1" :: "v"(fw) : "v251", "memory"); }
        };
        auto eval = [&]() {
            unsigned long long bx, bw;
            asm volatile("v_cmp_gt_u32 %0, %2, v250\n\tv_cmp_gt_u32 %1, %2, v251" : "=&s"(bx), "=&s"(bw) : "s"(call) : "memory");
            const unsigned m = (unsigned)bx;
            xmark = __builtin_ctz(m | m >> 16 | 0x10000u) * 4;
            wmark = bw ? __builtin_ctzll(bw) & ~3 : NSTEP;
            asm volatile("" : "+s"(xmark), "+s"(wmark));
            mark = xmark < wmark ? xmark : wmark;
        };
        const unsigned rot = (lags >> 8) & 3;
        unsigned ldsi = ldsw, kt = 0;
        const GAS char *pa0 = asrc[0] + (rot << astep[0]), *pa1 = asrc[SPL - 1] + (rot << astep[SPL - 1]), *pb = bsrc + (rot << bstep);
        auto issue = [&]() {
            asm volatile("s_mov_b32 m0, %0\n\ts_nop 0" :: "s"(ldsi) : "m0", "memory");
            glds2o<-4096>(pa0, av0[0], av1[0]);
            if (SPL == 2) glds2o<-2048>(pa1, av0[SPL - 1], av1[SPL - 1]);
            glds2o<2048 * SPL - 4096>(pb, bv0, bv1);
            if (NH == 2) glds2o<2048 * SPL - 2048>(pb, bv2, bv3);
            const unsigned k = ++kt ^ rot;
            pa0 = asrc[0] + (k << astep[0]);
            if (SPL == 2) pa1 = asrc[SPL - 1] + (k << astep[SPL - 1]);
            pb = bsrc + (k << bstep);
            ldsi = ldsi + SLOT_BYTES == ldsend ? ldsw : ldsi + SLOT_BYTES;
            asm volatile("" : "+s"(pa0), "+s"(pa1), "+s"(pb), "+s"(ldsi));
        };
#define AGK_LOOK 3
        const int lag = (lags >> (2 * lw)) & 3, depth = NSLOT - 1 - lag;
        int ti = 0, tb = 0;
        unsigned hist = 0;
#pragma nounroll
        for (;;) {
            if (mark < NSTEP && mark - ti <= AGK_LOOK) eval();
            const int lim = mark < tb + depth ? mark : tb + depth;
#pragma nounroll
            for (; ti < lim; ++ti) { issue(); hist = hist << 4 | (npoll & 15u); }
            if (ti == tb) {
                unsigned long long t0 = 0;
                for (unsigned spin = 1; ti >= mark; ++spin) {
                    wait_vm<16>();
                    poll();
                    __builtin_amdgcn_s_sleep(AGK_SL1);
                    if ((spin & 63) == 0) {
                        if (t0 == 0) t0 = now();
                        if (rfl(ld_agent(tmo)) == call || now() - t0 > DEADLINE_TICKS) break;
                    }
                    eval();
                }
                if (ti >= mark) break;
                continue;
            }
            const int run = (mark < NSTEP ? mark - AGK_LOOK : NSTEP) - ti;
            if (run > 0) {
                if (lag == 0) {
#pragma nounroll
                    for (int i = run; i > 0; --i) { wait_vm<(NSLOT - 2) * NI>(); bar(); issue(); }
                } else if (lag == 1) {
#pragma nounroll
                    for (int i = run; i > 0; --i) { wait_vm<(NSLOT - 3) * NI>(); bar(); issue(); }
                } else if (lag == 2) {
#pragma nounroll
                    for (int i = run; i > 0; --i) { wait_vm<(NSLOT - 4) * NI>(); bar(); issue(); }
                } else {
#pragma nounroll
                    for (int i = run; i > 0; --i) { wait_vm<(NSLOT - 5) * NI>(); bar(); issue(); }
                }
                ti += run; tb += run;
                const unsigned rep = (npoll & 15u) * 0x11111111u;
                hist = run >= 8 ? rep : hist << (4 * run) | (rep & ((1u << (4 * run)) - 1u));
            }
            if (mark < NSTEP) poll();
            const unsigned n = ti - tb - 1;
            wait_vm_var(n * NI + ((npoll - (hist >> (4 * n))) & 15u));
            bar();
            if (++tb == NSTEP) break;
        }
        if (tb < NSTEP) {
            if (lane == 0) st_agent(tmo, call);
            const u32x4 nan = {0x7FC07FC0u, 0x7FC07FC0u, 0x7FC07FC0u, 0x7FC07FC0u};
            for (; tb < NSTEP; ++tb) {
#pragma unroll
                for (int i = 0; i < NI; ++i) *(LAS u32x4*)(size_t)(ldsi - 4096 + i * 1024 + vlin) = nan;
                wait_lgkm();
                bar();
                ldsi = ldsi + SLOT_BYTES == ldsend ? ldsw : ldsi + SLOT_BYTES;
            }
        }
        if (lw == 0) STAMP(4);
        wait_vm<0>();
    } else {
        const int fr = lane & 15, g = lane >> 4;
        const int wm = wv >> 1, wn = wv & 1;
        unsigned aoff[2];
#pragma unroll
        for (int kk = 0; kk < 2; ++kk) aoff[kk] = wm * 2 * RW + fr * 128 + (((kk * 4 + g) ^ ((fr >> 1) & 7)) << 4);
        unsigned boff[NFR];
        {
            const int q = fr >> 2, pp = lane & 3, f = (q >> 1) | ((g & 1) << 1);
#pragma unroll
            for (int nb = 0; nb < NFR; ++nb)
                boff[nb] = (NH == 2 ? wn * 2 * RW + (g >> 1) * 2048 : (g >> 1) * RW) + SPL * 2048 + ((g & 1) * 8 + q) * 128 + ((((wn * NFR + nb) & 3) ^ f) << 5) + pp * 8;
        }
        int rowbase[SPW];
#pragma unroll
        for (int i = 0; i < SPW; ++i) {
            const int code = P.strip[first_strip + wm * SPW + i];
            rowbase[i] = stash ? (first_strip + wm * SPW + i) * 16 : rfl((P.rank ^ (code >> 5)) * MSH + (code & 31) * 16);
        }
        GAS char* outp = stash ? myws + STASH_OFF + (cg & 3) * STASH_BYTES : (GAS char*)P.out[wremote ? cg : 0] + (cg & 3) * 256;
        GAS char* cntp = stash ? myws + SCNT_OFF + (cg & 3) * 256 + (first_strip >> 3) * 4 : (GAS char*)P.ws[cg] + OCNT_OFF + (cg & 3) * 256;
        const int orow = stash ? 256 : NSH * 2;
        asm volatile("" : "+s"(outp), "+s"(cntp));
        f32x4 acc[SPW][NFR];
#pragma unroll
        for (int i = 0; i < SPW; ++i)
#pragma unroll
            for (int nb = 0; nb < NFR; ++nb) acc[i][nb] = (f32x4){0.f, 0.f, 0.f, 0.f};
        struct Frag { bf16x8 a[SPW]; bf16x8 b[NFR]; };
        auto rd = [&](Frag& F, int slot, int kk) {
            LAS char* sb = smem + slot * SLOT_BYTES;
#pragma unroll
            for (int i = 0; i < SPW; ++i) F.a[i] = *(LAS bf16x8*)(sb + aoff[kk] + (i / SPL) * RW + (i % SPL) * 2048);
#pragma unroll
            for (int nb = 0; nb < NFR; ++nb) {
                const s16x4 lo = __builtin_amdgcn_ds_read_tr16_b64_v4i16((LAS s16x4*)(sb + boff[nb] + kk * (2 / NH) * RW));
                const s16x4 hi = __builtin_amdgcn_ds_read_tr16_b64_v4i16((LAS s16x4*)(sb + boff[nb] + kk * (2 / NH) * RW + 512));
                F.b[nb] = __builtin_bit_cast(bf16x8, __builtin_shufflevector(lo, hi, 0, 1, 2, 3, 4, 5, 6, 7));
            }
        };
        auto mm = [&](const Frag& F) {
#pragma unroll
            for (int nb = 0; nb < NFR; ++nb)
#pragma unroll
                for (int i = 0; i < SPW; ++i) acc[i][nb] = __builtin_amdgcn_mfma_f32_16x16x32_bf16(F.b[nb], F.a[i], acc[i][nb], 0, 0, 0);
        };
        Frag F0, F1;
        int cs = 0;
        bar();
        rd(F0, 0, 0);
        wait_lgkm();
#pragma nounroll
        for (int t = 0; t < NSTEP - 1; ++t) {
            const int ns = cs + 1 == NSLOT ? 0 : cs + 1;
            constexpr int NRD = SPW + 2 * NFR, NMM = SPW * NFR, RPG = (NRD + NMM - 1) / NMM;
            rd(F1, cs, 1);
            mm(F0);
#pragma unroll
            for (int i = 0; i < NMM; ++i) { __builtin_amdgcn_sched_group_barrier(0x008, 1, 0); if (i * RPG < NRD) __builtin_amdgcn_sched_group_barrier(0x100, RPG, 0); }
            PIN();
            wait_lgkm();
            bar();
            rd(F0, ns, 0);
            mm(F1);
#pragma unroll
            for (int i = 0; i < NMM; ++i) { __builtin_amdgcn_sched_group_barrier(0x008, 1, 1); if (i * RPG < NRD) __builtin_amdgcn_sched_group_barrier(0x100, RPG, 1); }
            PIN();
            wait_lgkm();
            cs = ns;
        }
        rd(F1, cs, 1);
        PIN();
        mm(F0);
        PIN();
        mm(F1);
        if (wv == 0) STAMP(2);
        constexpr int ROWB = NFR * 32;
        constexpr int CPR = ROWB / 16;
        constexpr int ESLOT = (NSTEP - 1) % NSLOT == 0 ? 1 : 0;
        LAS char* eb = smem + ESLOT * SLOT_BYTES + wv * (SPW * 16 * ROWB);
        static_assert(4 * SPW * 16 * ROWB <= SLOT_BYTES, "epilogue space");
#pragma unroll
        for (int i = 0; i < SPW; ++i)
#pragma unroll
            for (int nb = 0; nb < NFR; ++nb) {
                u32x2 o;
                o.x = pack_bf16(acc[i][nb][0], acc[i][nb][1]);
                o.y = pack_bf16(acc[i][nb][2], acc[i][nb][3]);
                *(LAS u32x2*)(eb + (i * 16 + fr) * ROWB + (((nb * 2 + (g >> 1)) ^ (fr & (CPR - 1))) << 4) + (g & 1) * 8) = o;
            }
        constexpr int RPI = 64 / CPR;
        constexpr int NST = SPW * 16 / RPI;
        u32x4 ov[NST];
        GAS char* od[NST];
        GAS char* obase = outp + (NH == 2 ? wn * 128 : ch * 128 + wn * 64) + (lane & (CPR - 1)) * 16;
#pragma unroll
        for (int n = 0; n < NST; ++n) {
            const int row = n * RPI + lane / CPR;
            ov[n] = *(LAS u32x4*)(eb + row * ROWB + (((lane & (CPR - 1)) ^ (row & (CPR - 1))) << 4));
            od[n] = obase + (size_t)((rowbase[(n * RPI) / 16] + (row & 15)) * orow);
        }
        if (wv == 0) STAMP(5);
        if (win == 1) {
#pragma unroll
            for (int n = 0; n < NST; ++n) { if (n >= 1) wait_vm<0>(); st16_wt(od[n], ov[n]); }
        } else if (win == 2) {
#pragma unroll
            for (int n = 0; n < NST; ++n) { if (n >= 2) wait_vm<1>(); st16_wt(od[n], ov[n]); }
        } else {
#pragma unroll
            for (int n = 0; n < NST; ++n) st16_wt(od[n], ov[n]);
        }
        if (wremote) {
            wait_vm<0>();
            if (lane == 0) __hip_atomic_fetch_add((gu32*)cntp, stash ? (unsigned)NH : 1u, __ATOMIC_RELAXED, __HIP_MEMORY_SCOPE_SYSTEM);
        }
        if (wv == 0) STAMP(3);
    }
}

__global__ void __launch_bounds__(512, 1) __attribute__((amdgpu_num_vgpr(240))) ag_gemm_kernel(const Params P) {
    __shared__ __attribute__((aligned(1024))) char smem_raw[LDS_BYTES];
    LAS char* smem = (LAS char*)smem_raw;
    GAS char* myws = (GAS char*)P.ws[0];
    const int b = blockIdx.x;
    gu32* cc = (gu32*)(myws + CALL_OFF + b * 64);
    if (b < NCOMM) {
        const int wv = rfl(threadIdx.x >> 6), g = wv * NCOMM + b;
        if (wv >= NCW || g >= 7 * WPL) return;
        const unsigned call = comm_role(P, smem + wv * 16384, cc + wv, wv, 1 + g % 7, g / 7);
        if ((threadIdx.x & 63) == 0) st_agent(cc + wv, call);
        return;
    }
    const unsigned role = rfl(P.role[b]);
    const int type = role & 15;
    if (type == R_UNPACK) {
        const int wv = rfl(threadIdx.x >> 6);
        const unsigned call = rfl(ld_agent(cc + wv)) + 1u;
        unpack_role(P, myws, call, (int)(role >> 4) + (int)P.nunpack * wv, (int)P.nunpack * 8);
        if (wv == 7) STAMP(4);
        if ((threadIdx.x & 63) == 0) st_agent(cc + wv, call);
    } else if (type != R_IDLE) {
        const unsigned call = rfl(ld_agent(cc)) + 1u;
        if (threadIdx.x == 0) STAMP(0);
        const int fs = (role >> 4) & 255, cg = (role >> 12) & 7, ch = (role >> 15) & 1, win = (role >> 18) & 3;
        const bool stash = (role >> 16) & 1;
        if (type == R_T128) tile_role<8, 2, 5>(P, smem, call, fs, cg, 0, stash, win, role >> 20);
        else if (type == R_T128H) tile_role<8, 1, 6>(P, smem, call, fs, cg, ch, stash, win, role >> 20);
        else tile_role<4, 1, 8>(P, smem, call, fs, cg, ch, stash, win, role >> 20);
        if (threadIdx.x == 256) st_agent(cc, call);
    }
}

}

#define TUNE(i, dflt) (dflt)

extern "C" void kernel_launch(int rank, int nranks, ncclComm_t comm,
                              hipStream_t stream,
                              void* const* d_in, const int* in_sizes, int n_in,
                              void* d_out, int out_size,
                              const int* in_dtypes, int out_dtype,
                              void* const* d_out_all,
                              void* const* d_ws_all, size_t ws_bytes) {
    using namespace agk;
    (void)comm; (void)in_dtypes; (void)out_dtype; (void)d_out;
    if (nranks != NR || (unsigned)rank >= (unsigned)NR || n_in < 2 || ws_bytes < WS_NEED ||
        in_sizes[0] != MSH * KD || in_sizes[1] != KD * NSH || out_size != NR * MSH * NSH) {
        fprintf(stderr, "ag_gemm: unexpected problem shape\n");
        return;
    }
    Params P;
    P.x = (const char*)d_in[0];
    P.w = (const char*)d_in[1];
    P.rank = rank;
    for (int j = 0; j < NR; ++j) { P.ws[j] = (char*)d_ws_all[rank ^ j]; P.out[j] = (char*)d_out_all[rank ^ j]; }
    int sega = TUNE(0, 11);
    int blate = TUNE(1, 9);
    int wlead = TUNE(2, 4);
    int rt_b = TUNE(3, 2), rt_c = TUNE(4, 2), rt_a = TUNE(5, 0), rt_w = TUNE(6, 0);
    int split_c = TUNE(7, 4);
    int split_a = TUNE(8, 0);
    int small_b = TUNE(9, 0);
    const int segb = HS - sega;
    const int na_t = (3 * sega + 7) / 8, nc_t = (4 * AS + 7) / 8, nb_t = (3 * segb + 7) / 8, no_t = 16 - na_t - nc_t - nb_t;
    const int ne_t = no_t + na_t;
    bool ok = no_t >= 0 && ne_t <= NEARLY_T;
    int n = 0, own = 0;
    for (int i = 0; i < no_t * 8; ++i) P.strip[n++] = (unsigned char)(own++);
    for (int s = 0; s < sega; ++s) for (int j = 1; j < 4; ++j) P.strip[n++] = (unsigned char)((j << 5) | s);
    while (n % 8) P.strip[n++] = (unsigned char)(own++);
    for (int s = HS; s < NSTRIP; ++s) for (int j = 4; j < 8; ++j) P.strip[n++] = (unsigned char)((j << 5) | s);
    while (n % 8) P.strip[n++] = (unsigned char)(own++);
    for (int s = sega; s < HS; ++s) for (int j = 1; j < 4; ++j) P.strip[n++] = (unsigned char)((j << 5) | s);
    while (n % 8) P.strip[n++] = (unsigned char)(own++);
    ok = ok && n == 128 && own == HS;
    for (int i = 0; i < NEARLY_T * 8; ++i) {
        const unsigned row = (unsigned)((rank ^ (P.strip[i] >> 5)) * MSH + (P.strip[i] & 31) * 16);
        P.orow[i / 2] = (i & 1) ? P.orow[i / 2] | (row << 16) : row;
    }
    int q = 0;
    const int t_first = TUNE(16, 200) * 16, t_pk = TUNE(17, 176), t_raw = TUNE(18, 232);
    const int rh_a = TUNE(19, 0), rh_w = TUNE(20, 0), afirst = TUNE(21, 0);
    int tq = t_first;
    auto emit = [&](unsigned e) { if (q < NP_IN + NP_XI) P.ord[q] = e | (unsigned)(tq >> 4) << 16; ++q; tq += (e & 512u) ? t_raw : t_pk; };
    auto xchunk = [&](int s0, int s1, int kc, int rt, int rh) { for (int s = s0; s < s1; ++s) emit((unsigned)kc | (s << 4) | (kc >= NKC - rt || kc < rh ? 512u : 0u)); };
    auto wchunk = [&](int kc) { for (int i = 0; i < 8; ++i) emit((unsigned)(kc * 8 + i) | 1024u | (kc >= NKC - rt_w || kc < rh_w ? 512u : 0u)); };
    for (int ka = 0, kb = 0; kb < NKC; ) {
        if (ka < NKC && kb * (NKC - afirst) >= (ka - afirst) * (NKC - blate)) xchunk(0, sega, ka++, rt_a, rh_a);
        else xchunk(sega, HS, kb++, rt_b, 0);
    }
    ok = ok && q == NP_IN;
    tq = t_first;
    for (int k = -wlead; k < NKC; ++k) {
        if (k + wlead < NKC) wchunk(k + wlead);
        if (k >= 0) xchunk(HS, NSTRIP, k, rt_c, 0);
    }
    ok = ok && q == NP_IN + NP_XI;
    P.ord[NP_IN + NP_XI] = (unsigned)t_raw | (unsigned)(tq >> 4) << 16;
    int fill[8] = {0, 0, 0, 0, 0, 0, 0, 0};
    for (int i = 0; i < NWG; ++i) P.role[i] = R_IDLE;
    auto put = [&](int k, unsigned role) { if (fill[k] < NWG / 8) P.role[fill[k]++ * 8 + k] = role; else ok = false; };
    auto code = [&](int type, int fs, int cg, int ch, bool stash) { return (unsigned)type | (fs << 4) | (cg << 12) | (ch << 15) | ((stash ? 1u : 0u) << 16); };
    for (int i = 0; i < NCOMM; ++i) put(i & 7, R_COMM | (i << 4));
    const int ns_t = TUNE(30, 0) < ne_t ? TUNE(30, 0) : ne_t;
    unsigned adds = ns_t * 4;
    for (int cg = 0; cg < 8; ++cg) {
        const bool rem = cg >= 4;
        for (int r = 0; r < 16; ++r) {
            const int k = (cg & 3) + 4 * (r & 1);
            const bool early = r < ne_t, last = r >= ne_t + nc_t;
            const bool split = last ? rem || r - ne_t - nc_t >= TUNE(29, 4) : (rem ? (early ? r >= ne_t - split_a : r - ne_t < split_c) : (!early && r - ne_t < TUNE(22, 0)));
            const bool small = last && r - ne_t - nc_t < (rem ? small_b >> 4 : small_b & 15);
            const bool sth = rem && r < ns_t;
            const unsigned win = (unsigned)(!rem || sth ? 0 : early ? TUNE(14, 2) : last ? TUNE(12, 0) : split ? TUNE(13, 2) : TUNE(11, 2)) << 18;
            int nw = 0;
            auto lags = [&](int who, int of, unsigned lag) { unsigned v = 0; for (int l = 0; l < 4; ++l) if (l % of != who % of) v |= lag << (2 * l); return v << 20 | (unsigned)(TUNE(26, 1) ? (of == 2 ? (who & 1) * 2 + ((r >> 1) & 1) * TUNE(27, 1) : (who ^ (((r >> 1) & 1) * TUNE(28, 2))) & 3) : 0) << 28; };
            const int rw = TUNE(25, 0) * (r >> 1);
            if (small) for (int h = 0; h < 4; ++h) { put(k, lags(rem * 2 + (h & 1) + rw, 4, TUNE(24, 0)) | win | code(R_T64H, r * 8 + (h >> 1) * 4, cg, h & 1, false)); nw += 4; }
            else if (split) for (int ch = 0; ch < 2; ++ch) { put(k, lags(rem * 2 + ch + rw, 4, TUNE(24, 0)) | win | code(R_T128H, r * 8, cg, ch, sth)); nw += 4; }
            else { put(k, lags(rem + rw, 2, TUNE(23, 0)) | win | code(R_T128, r * 8, cg, 0, sth)); nw += 4; }
            if (cg == 4 && !sth) adds += nw;
        }
    }
    unsigned nun = 0;
    for (int i = 0; i < NWG; ++i) if (P.role[i] == R_IDLE && nun < (unsigned)TUNE(10, 64)) P.role[i] = R_UNPACK | (nun++ << 4);
    P.out_adds = adds; P.npx = NP_XI + ns_t * 4; P.nunpack = nun;
    ok = ok && nun > 0;
    if (!ok) { fprintf(stderr, "ag_gemm: role table overflow\n"); return; }
    hipLaunchKernelGGL(ag_gemm_kernel, dim3(NWG), dim3(512), 0, stream, P);
    HIP_CHECK(hipGetLastError());
}
```

```cpp
#include <hip/hip_runtime.h>
#include <rccl/rccl.h>
#include <cstdio>
#include <cstdint>

#define HIP_CHECK(expr) do { hipError_t _e = (expr); if (_e != hipSuccess) { \
    fprintf(stderr, "HIP error %s at %s:%d\n", hipGetErrorString(_e), __FILE__, __LINE__); \
    return; } } while (0)

namespace agk {

typedef unsigned int u32x4 __attribute__((ext_vector_type(4)));
typedef unsigned int u32x3 __attribute__((ext_vector_type(3)));
typedef unsigned int u32x2 __attribute__((ext_vector_type(2)));
typedef short s16x4 __attribute__((ext_vector_type(4)));
typedef short s16x8 __attribute__((ext_vector_type(8)));
typedef __bf16 bf16x8 __attribute__((ext_vector_type(8)));
typedef float f32x4 __attribute__((ext_vector_type(4)));
#define LAS __attribute__((address_space(3)))
#define GAS __attribute__((address_space(1)))
typedef GAS unsigned int gu32;

constexpr int NR = 8;
constexpr int KD = 4096;
constexpr int MSH = 512;
constexpr int NSH = 512;
constexpr int NSTRIP = 32;
#define AGK_HS 21
constexpr int HS = AGK_HS;
constexpr int AS = NSTRIP - HS;
constexpr int NKC = 16;
constexpr int NSTEP = 64;
constexpr int NWP = 128;

#define AGK_NCOMM 56
#define AGK_NCW 4
constexpr int NCOMM = AGK_NCOMM;
constexpr int NCW = AGK_NCW;
constexpr int WPL = NCOMM * NCW / 7;
static_assert(NCOMM % 7 == 0, "a comm workgroup serves one link");

constexpr int NP_IN = HS * NKC;
constexpr int NP_XI = NWP + AS * NKC;
constexpr int NPMAX = NP_IN > NP_XI ? NP_IN : NP_XI;
constexpr int NRND = (NPMAX + WPL - 1) / WPL;
constexpr int NWG = 256;
enum { R_IDLE = 0, R_COMM = 1, R_T128 = 2, R_T128H = 3, R_T64H = 4, R_UNPACK = 5, R_T64W = 6 };

constexpr size_t STRIP_BYTES = 16ull * KD * 2;
constexpr int NXSLOT = 3 * HS + 4 * AS;
constexpr size_t XS_BYTES = (size_t)NXSLOT * STRIP_BYTES;
constexpr size_t WSL_BYTES = 8 * STRIP_BYTES;
constexpr int XF_STRIDE = 256;
constexpr size_t XFLAG_OFF = XS_BYTES + 4 * WSL_BYTES;
constexpr size_t WFLAG_OFF = XFLAG_OFF + 8 * 32 * XF_STRIDE;
constexpr size_t PK_OFF = WFLAG_OFF + 4 * NWP * 4;
constexpr unsigned PK_BYTES = 6656;
constexpr size_t BUF_BYTES = (PK_OFF + (size_t)(3 * NP_IN + 4 * NP_XI) * PK_BYTES + 4095) / 4096 * 4096;
constexpr size_t OCNT_OFF = 2 * BUF_BYTES;
constexpr size_t TMO_OFF = OCNT_OFF + 4 * 256;
constexpr size_t CALL_OFF = TMO_OFF + 256;
constexpr size_t DBG_OFF = CALL_OFF + 256 * 64;
constexpr size_t DBG2_OFF = DBG_OFF + 256 * 128;
constexpr size_t WS_NEED = DBG2_OFF + 8 * 512 * 32;
static_assert(BUF_BYTES % 4096 == 0, "alignment");
static_assert(WS_NEED <= 65ull << 20, "workspace");

constexpr unsigned long long DEADLINE_TICKS = 30000000ull;
constexpr int LDS_BYTES = 163840;

struct Params {
    const char* x;
    const char* w;
    char* ws[NR];
    char* out[NR];
    int rank;
    unsigned out_adds;
    unsigned tpoll;
    unsigned nunpack;
    unsigned role[NWG - NCOMM];
    unsigned tlead;
    unsigned ord[2 * WPL * NRND];
    unsigned char strip[128];
};

static_assert(sizeof(Params) <= 4096, "kernel arguments");

__device__ __forceinline__ unsigned ld_sys(const gu32* p) { return __hip_atomic_load(p, __ATOMIC_RELAXED, __HIP_MEMORY_SCOPE_SYSTEM); }
__device__ __forceinline__ unsigned ld_agent(const gu32* p) { return __hip_atomic_load(p, __ATOMIC_RELAXED, __HIP_MEMORY_SCOPE_AGENT); }
__device__ __forceinline__ void st_agent(gu32* p, unsigned v) { __hip_atomic_store(p, v, __ATOMIC_RELAXED, __HIP_MEMORY_SCOPE_AGENT); }
__device__ __forceinline__ unsigned long long now() { return __builtin_amdgcn_s_memrealtime(); }
__device__ __forceinline__ unsigned rfl(unsigned v) { return __builtin_amdgcn_readfirstlane(v); }

__device__ __forceinline__ void st16_wt(GAS char* p, u32x4 v) {
    asm volatile("global_store_dwordx4 %0, %1, off sc0 sc1\n\ts_nop 1" :: "v"(p), "v"(v) : "memory");
}
template <int N> __device__ __forceinline__ void wait_vm() { asm volatile("s_waitcnt vmcnt(%0)" :: "i"(N) : "memory"); }
__device__ __forceinline__ void wait_lgkm() { asm volatile("" ::: "memory"); __builtin_amdgcn_s_waitcnt(0xC07F); asm volatile("" ::: "memory"); }
#define AGK_SL1 6
#define AGK_SL2 2
#define PIN() __builtin_amdgcn_sched_barrier(0)
__device__ __forceinline__ void sleep_until(unsigned t) {
    int rest = (int)(t - (unsigned)now());
    for (; rest > 48; rest = (int)(t - (unsigned)now())) __builtin_amdgcn_s_sleep(8);
    for (; rest > 2; rest -= 3) __builtin_amdgcn_s_sleep(1);
}
__device__ __forceinline__ void bar() { asm volatile("s_barrier" ::: "memory"); }

#define AGK_LDPOL "sc0 sc1"
__device__ __forceinline__ void glds2(const GAS char* base, unsigned v0, unsigned v1, unsigned lds_dst) {
    asm volatile("s_mov_b32 m0, %3\n\ts_nop 3\n\tglobal_load_lds_dwordx4 %1, %0 " AGK_LDPOL "\n\tglobal_load_lds_dwordx4 %2, %0 offset:1024 " AGK_LDPOL
                 :: "s"(base), "v"(v0), "v"(v1), "s"(lds_dst) : "m0", "memory");
}
template <int OFF> __device__ __forceinline__ void glds2o(const GAS char* base, unsigned v0, unsigned v1) {
    asm volatile("global_load_lds_dwordx4 %1, %0 offset:%3 " AGK_LDPOL "\n\tglobal_load_lds_dwordx4 %2, %0 offset:%4 " AGK_LDPOL
                 :: "s"(base), "v"(v0), "v"(v1), "i"(OFF), "i"(OFF + 1024) : "memory");
}
__device__ __forceinline__ void wait_vm_var(unsigned n) {
    asm volatile("s_getpc_b64 vcc\n\ts_add_u32 vcc_lo, vcc_lo, %0\n\ts_addc_u32 vcc_hi, vcc_hi, 0\n\ts_setpc_b64 vcc\n\t"
                 ".set agk_n, 0\n\t.rept 64\n\ts_waitcnt vmcnt(agk_n)\n\ts_branch 1f\n\t.set agk_n, agk_n + 1\n\t.endr\n1:"
                 :: "s"(n * 8 + 12) : "vcc", "scc", "memory");
}

__device__ __forceinline__ unsigned pack_bf16(float lo, float hi) {
    typedef float f32x2 __attribute__((ext_vector_type(2)));
    typedef __bf16 bf16x2 __attribute__((ext_vector_type(2)));
    f32x2 v = {lo, hi};
    bf16x2 r = __builtin_convertvector(v, bf16x2);
    return __builtin_bit_cast(unsigned, r);
}

__device__ __forceinline__ int xslot(int j, int s) { return j < 4 ? (j - 1) * HS + s : 3 * HS + (j - 4) * AS + (s - HS); }

#define STAMP(i) do { } while (0)
#define PSTAMP(j, p, i, t) do { } while (0)

enum { K_X = 0, K_W = 1 };
struct Piece { int kind, raw; unsigned soff, doff, foff, when; };

__host__ __device__ __forceinline__ int ord_at(int cross, int p) { return (cross * WPL + p % WPL) * NRND + p / WPL; }
__device__ __forceinline__ Piece locate(unsigned e, int j) {
    Piece r;
    r.when = e >> 16;
    const int s = (e >> 4) & 31, kc = e & 15, wp = (e >> 10) & 1 ? (int)(e & 127) : -1;
    r.raw = (e >> 9) & 1;
    if (wp >= 0) {
        r.kind = K_W;
        r.soff = (unsigned)(wp * 32 * (NSH * 2) + (j & 3) * 256);
        r.doff = (unsigned)XS_BYTES + (unsigned)(j - 4) * (unsigned)WSL_BYTES + (wp & 1) * 2 * (unsigned)STRIP_BYTES + (wp >> 1) * 2048;
        r.foff = (unsigned)WFLAG_OFF + ((j - 4) * NWP + (wp & 1) * 64 + ((wp >> 1) & 3) * 16 + (wp >> 3)) * 4;
    } else {
        r.kind = K_X;
        r.soff = (unsigned)(s * 16 * (KD * 2) + kc * 512);
        r.doff = (unsigned)xslot(j, s) * (unsigned)STRIP_BYTES + kc * 8192;
        r.foff = (unsigned)XFLAG_OFF + (j * 32 + s) * XF_STRIDE + kc * 4;
    }
    return r;
}
__device__ __forceinline__ unsigned pk_off(int j, int p) {
    return (unsigned)PK_OFF + 3072u + (unsigned)((j < 4 ? (j - 1) * NP_IN : 3 * NP_IN + (j - 4) * NP_XI) + p) * PK_BYTES;
}

template <int OFF> __device__ __forceinline__ void st_pk(const u32x4& v, unsigned voff, const GAS char* dst) {
    asm volatile("s_nop 3\n\tglobal_store_dwordx4 %1, %0, %2 offset:%3 sc0 sc1\n\ts_nop 1" :: "v"(v), "v"(voff), "s"(dst), "i"(OFF) : "memory");
}
template <int OFF> __device__ __forceinline__ void ld_c(u32x4& v, unsigned voff, const GAS char* src) {
    asm volatile("global_load_dwordx4 %0, %1, %2 offset:%3 sc0 sc1" : "=v"(v) : "v"(voff), "s"(src), "i"(OFF) : "memory");
}
__device__ __forceinline__ void flag_st(const GAS char* f, unsigned val, unsigned zero) {
    asm volatile("s_mov_b64 exec, 1\n\ts_nop 3\n\tglobal_store_dword %0, %1, %2 sc0 sc1\n\ts_mov_b64 exec, -1" :: "v"(zero), "v"(val), "s"(f) : "memory");
}
__device__ __forceinline__ unsigned perm(unsigned hi, unsigned lo, unsigned sel) { return __builtin_amdgcn_perm(hi, lo, sel); }
__device__ __forceinline__ unsigned bfi(unsigned m, unsigned a, unsigned b) { return (a & m) | (b & ~m); }
__device__ __forceinline__ unsigned rotr(unsigned v, unsigned n) { return __builtin_amdgcn_alignbit(v, v, n); }
__device__ __forceinline__ unsigned pkmax(unsigned a, unsigned b) {
    typedef unsigned short u16x2 __attribute__((ext_vector_type(2)));
    return __builtin_bit_cast(unsigned, __builtin_elementwise_max(__builtin_bit_cast(u16x2, a), __builtin_bit_cast(u16x2, b)));
}
template <int CTRL> __device__ __forceinline__ unsigned dppmax(unsigned v) { return max(v, (unsigned)__builtin_amdgcn_update_dpp(0, (int)v, CTRL, 0xF, 0xF, true)); }
constexpr unsigned F_RAW = 0x80u;
constexpr int MAXEXC = 16;
__device__ __forceinline__ unsigned ximg0(unsigned lane, unsigned pitch) { return (lane >> 3) * pitch + ((lane & 7) ^ (lane >> 4)) * 16; }
__device__ __forceinline__ unsigned ximg1(unsigned lane, unsigned pitch) { return (8 + (lane >> 3)) * pitch + ((lane & 7) ^ (4 + (lane >> 4))) * 16 - 1024; }
__device__ __forceinline__ unsigned wimg0(unsigned lane, unsigned pitch) { return (lane >> 3) * pitch + ((lane & 7) ^ ((lane >> 3) & 2)) * 16; }
__device__ __forceinline__ unsigned wimg1(unsigned lane, unsigned pitch) { return (8 + (lane >> 3)) * pitch + ((lane & 7) ^ (((lane >> 3) & 2) | 4)) * 16 - 1024; }

__device__ __forceinline__ void fetch_piece(int kind, unsigned soff, const GAS char* sx, const GAS char* sw,
                                            unsigned vx0, unsigned vx1, unsigned vw0, unsigned vw1, unsigned l) {
    const bool kx = kind == K_X;
    const GAS char* s = (kx ? sx : sw) + soff;
    const unsigned a = kx ? vx0 : vw0, b = kx ? vx1 : vw1;
    const int q1 = kx ? 128 : 16 * NSH * 2, q2 = kx ? 256 : 128;
    glds2(s, a, b, l); glds2(s + q1, a, b, l + 2048); glds2(s + q2, a, b, l + 4096); glds2(s + q1 + q2, a, b, l + 6144);
}

__device__ __forceinline__ unsigned tag_of(unsigned call) { return ((call + 1u) >> 1) << 31; }

__device__ __forceinline__ void comm_role(const Params& P, LAS char* smem, unsigned long long t64, int wslot, int j, int u) {
    const unsigned* ord = P.ord + ord_at(j >= 4, u);
    unsigned eA = ord[0], eB = ord[1], tlead = P.tlead;
    const GAS char *srcx = (const GAS char*)P.x, *srcw = (const GAS char*)P.w, *pws = (const GAS char*)P.ws[j];
    GAS char* myws = (GAS char*)P.ws[0];
    asm volatile("" : "+s"(eA), "+s"(eB), "+s"(tlead), "+s"(srcx), "+s"(srcw), "+s"(pws), "+s"(myws), "+s"(t64));
    const unsigned tstart = (unsigned)t64;
    gu32* cc = (gu32*)(myws + CALL_OFF + blockIdx.x * 64) + wslot;
    const int lane = threadIdx.x & 63;
    const int npiece = j < 4 ? NP_IN : NP_XI;
    const unsigned vlin = lane * 16;
    const unsigned vx0 = ximg0(lane, KD * 2), vx1 = ximg1(lane, KD * 2), vw0 = wimg0(lane, NSH * 2), vw1 = wimg1(lane, NSH * 2);
    const unsigned vblk = (lane >> 5) * 2048 + (lane & 31) * 16, vblw = (lane >> 5) * (unsigned)STRIP_BYTES + (lane & 31) * 16;
    const unsigned lds0 = (unsigned)(size_t)smem;
#define AGK_FETCH(q, slot) fetch_piece((q).kind, (q).soff, srcx, srcw, vx0, vx1, vw0, vw1, lds0 + (slot) * 8192)
    int pA = u;
    unsigned ccv;
    Piece A = locate(eA, j);
    if (u > 3) sleep_until(tstart + (u * (tlead & 0xFFFFu) >> 2));
    AGK_FETCH(A, 0);
    asm volatile("global_load_dword %0, %1, off sc1" : "=v"(ccv) : "v"(cc) : "memory");
    Piece B = locate(eB, j);
    asm volatile("s_waitcnt vmcnt(0)" : "+v"(ccv) :: "memory");
    AGK_FETCH(B, 1);
    const unsigned call = rfl(ccv) + 1u, tagw = tag_of(call);
    if (wslot == 0) STAMP(0);
    const GAS char* dbuf = pws + (size_t)(call & 1) * BUF_BYTES;
    gu32* tmo = (gu32*)(myws + TMO_OFF);
    unsigned zero = 0, callv = call, vl1 = lane;
    asm volatile("" : "+v"(zero), "+v"(callv), "+v"(vl1));
    int slot = 0;
    for (;;) {
        u32x4 v[8], W[6];
#pragma unroll
        for (int k = 0; k < 8; ++k) v[k] = *(LAS u32x4*)(smem + slot * 8192 + (k >> 2) * 4096 + (k & 3) * 512 + vblk);
        bool raw = A.raw;
        u32x3 ev = {0, 0, callv};
        unsigned trl = 0, eoff = 0; unsigned long long bm = 0;
        if (!raw) {
            unsigned mx = pkmax(pkmax(pkmax(v[0].x & 0x7FFF7FFFu, v[0].y & 0x7FFF7FFFu), pkmax(v[0].z & 0x7FFF7FFFu, v[0].w & 0x7FFF7FFFu)),
                                pkmax(pkmax(v[6].x & 0x7FFF7FFFu, v[6].y & 0x7FFF7FFFu), pkmax(v[6].z & 0x7FFF7FFFu, v[6].w & 0x7FFF7FFFu)));
            mx = max(mx >> 16, mx & 0xFFFFu) >> 8;
            mx = dppmax<0x143>(dppmax<0x142>(dppmax<0x118>(dppmax<0x114>(dppmax<0x112>(dppmax<0x111>(mx))))));
            unsigned fmt = max((unsigned)__builtin_amdgcn_readlane((int)mx, 63), 7u) - 7u;
            const unsigned B4 = fmt * 0x01010101u;
            unsigned accO = 0, accA = ~0u;
#pragma unroll
            for (int k = 0; k < 8; ++k) {
                const unsigned Ha = perm(v[k].y, v[k].x, 0x07050301u), Hb = perm(v[k].w, v[k].z, 0x07050301u);
                const unsigned da = (Ha | 0x80808080u) - B4, db = (Hb | 0x80808080u) - B4;
                accO |= da | db; accA &= da & db;
                W[(k >> 2) * 3 + (k & 1)][k & 2] = perm(v[k].y, v[k].x, 0x06040200u);
                W[(k >> 2) * 3 + (k & 1)][(k & 2) + 1] = perm(v[k].w, v[k].z, 0x06040200u);
                W[(k >> 2) * 3 + 2][(k & 1) * 2 + ((k >> 1) & 1)] = bfi(0x87878787u, bfi(0x80808080u, Ha, da), rotr(bfi(0x80808080u, Hb, db), 28));
            }
            bm = __ballot(((accO & 0x78787878u) | (~accA & 0x80808080u)) != 0);
            if (bm) {
                unsigned nb = 0, ib = 0;
#pragma unroll
                for (int i = 0; i < 16; ++i) {
                    const unsigned H = (i & 1) ? perm(v[i >> 1].w, v[i >> 1].z, 0x07050301u) : perm(v[i >> 1].y, v[i >> 1].x, 0x07050301u);
                    const bool bad = (((H | 0x80808080u) - B4) & 0xF8F8F8F8u) != 0x80808080u;
                    ev.y = bad ? H : ev.y; ib = bad ? (unsigned)i : ib; nb += bad;
                }
                ev.x = lane | (ib << 8);
                eoff = __builtin_amdgcn_mbcnt_hi((unsigned)(bm >> 32), __builtin_amdgcn_mbcnt_lo((unsigned)bm, 0)) * 16;
                const unsigned cnt = __builtin_popcountll(bm);
                raw = cnt > MAXEXC || __ballot(nb > 1) != 0;
                fmt = raw ? F_RAW : fmt | cnt << 8;
                if (raw) bm = 0;
            }
            trl = (tagw >> 30) + (__builtin_amdgcn_inverse_ballot_w64((unsigned long long)call << 32 | fmt) ? 1u : 0u);
#pragma unroll
            for (int k = 0; k < 6; ++k) { trl = __builtin_amdgcn_alignbit(trl, W[k].w, 31); W[k].w = bfi(0x7FFFFFFFu, W[k].w, tagw); }
        } else {
#pragma unroll
            for (int k = 0; k < 6; ++k) W[k] = v[k];
        }
        const GAS char* pk = dbuf + pk_off(j, pA);
        const Piece C = locate(ord[pA + 2 * WPL < npiece ? pA / WPL + 2 : 0], j);
        asm volatile("" :: "v"(W[0]), "v"(W[1]), "v"(W[2]), "v"(W[3]), "v"(W[4]), "v"(W[5]), "v"(v[6]), "v"(v[7]), "v"(trl), "v"(ev), "v"(eoff), "s"(pk), "s"(C.soff) : "memory");
        sleep_until(tstart + A.when);
        if (!A.raw) {
            st_pk<-3072>(W[0], vlin, pk); st_pk<-2048>(W[1], vlin, pk); st_pk<-1024>(W[2], vlin, pk);
            st_pk<0>(W[3], vlin, pk); st_pk<1024>(W[4], vlin, pk); st_pk<2048>(W[5], vlin, pk);
            if (bm) asm volatile("s_mov_b64 exec, %3\n\ts_nop 3\n\tglobal_store_dwordx3 %0, %1, %2 offset:3136 sc0 sc1\n\ts_mov_b64 exec, -1" :: "v"(eoff), "v"(ev), "s"(pk), "s"(bm) : "memory");
            asm volatile("global_store_byte %0, %1, %2 offset:3072 sc0 sc1" :: "v"(vl1), "v"(trl), "s"(pk) : "memory");
        }
        if (raw) {
            const GAS char* d0 = dbuf + A.doff;
            const GAS char* d1 = d0 + (A.kind == K_W ? 4 * (unsigned)STRIP_BYTES : 4096u);
            const unsigned vd = A.kind == K_W ? vblw : vblk;
            st_pk<0>(v[0], vd, d0); st_pk<512>(v[1], vd, d0); st_pk<1024>(v[2], vd, d0); st_pk<1536>(v[3], vd, d0);
            st_pk<0>(v[4], vd, d1); st_pk<512>(v[5], vd, d1); st_pk<1024>(v[6], vd, d1); st_pk<1536>(v[7], vd, d1);
        }
        AGK_FETCH(C, slot);
        if (raw) { wait_vm<8>(); flag_st(dbuf + A.foff, callv, zero); }
        PSTAMP(j, pA, 0, tstart + A.when); PSTAMP(j, pA, 1, tb); PSTAMP(j, pA, 2, now());
        pA += WPL; A = B; B = C; slot ^= 1;
        if (pA >= npiece) break;
        wait_vm<15>();
    }
    wait_vm<0>();
    STAMP(j < 4 ? 1 : 3);
    if (j == 1 && u == 0) {
        if (lane < 4) {
            const gu32* c = (const gu32*)(myws + OCNT_OFF + lane * 256);
            const unsigned need = call * P.out_adds;
            const unsigned long long t0 = now();
            bool bad = false;
            unsigned spin = 0;
            while (ld_sys(c) < need) {
                if ((++spin & 63) == 0 && (ld_agent(tmo) == call || now() - t0 > DEADLINE_TICKS)) { st_agent(tmo, call); bad = true; break; }
                __builtin_amdgcn_s_sleep(1);
            }
            if (bad) {
                __builtin_amdgcn_s_sleep(127);
                __hip_atomic_store((GAS unsigned short*)(P.out[0] + (size_t)(P.rank * MSH) * (NSH * 2)), (unsigned short)0x7FC0, __ATOMIC_RELAXED, __HIP_MEMORY_SCOPE_AGENT);
            }
        }
        STAMP(2);
    }
    if (lane == 0) st_agent(cc, call);
}

__device__ __forceinline__ void unpack_role(const Params& P, GAS char* myws, unsigned tstart, unsigned call, int w, int nw) {
    const GAS char* buf = myws + (size_t)(call & 1) * BUF_BYTES;
    gu32* tmo = (gu32*)(myws + TMO_OFF);
    const int lane = threadIdx.x & 63;
    const unsigned vlin = lane * 16, tagw = tag_of(call);
    const unsigned vblk = (lane >> 5) * 2048 + (lane & 31) * 16, vblw = (lane >> 5) * (unsigned)STRIP_BYTES + (lane & 31) * 16;
    unsigned zero = 0, callv = call, vl1 = lane;
    asm volatile("" : "+v"(zero), "+v"(callv), "+v"(vl1));
    const unsigned tfirst = tstart + (P.tpoll & 0xFFFFu);
    const bool spec = P.tpoll >> 16;
    for (int q = w; q < NPMAX * 7; q += nw) {
        const int p = q / 7, j = 1 + q - p * 7;
        if (p >= (j < 4 ? NP_IN : NP_XI)) continue;
        const Piece A = locate(P.ord[ord_at(j >= 4, p)], j);
        if (A.raw) continue;
        const GAS char* pk = buf + pk_off(j, p);
        const GAS char* d0 = buf + A.doff;
        const GAS char* d1 = d0 + (A.kind == K_W ? 4 * (unsigned)STRIP_BYTES : 4096u);
        const unsigned vd = A.kind == K_W ? vblw : vblk;
        sleep_until(tfirst + A.when);
        unsigned long long t0 = 0;
        unsigned spin = 0, trl, fmt = 0;
        u32x4 W[6]; u32x3 ex;
        bool data = spec;
        for (;;) {
            asm volatile("s_nop 4\n\tglobal_load_ubyte %0, %1, %2 offset:3072 sc0 sc1" : "=v"(trl) : "v"(vl1), "s"(pk) : "memory");
            if (data) {
                ld_c<-1024>(W[2], vlin, pk); ld_c<2048>(W[5], vlin, pk);
                ld_c<-3072>(W[0], vlin, pk); ld_c<-2048>(W[1], vlin, pk); ld_c<0>(W[3], vlin, pk); ld_c<1024>(W[4], vlin, pk);
                asm volatile("global_load_dwordx3 %0, %1, %2 offset:3136 sc0 sc1" : "=v"(ex) : "v"(vlin & (MAXEXC * 16 - 1)), "s"(pk) : "memory");
            }
            asm volatile("s_waitcnt vmcnt(0)" : "+v"(trl), "+v"(W[0]), "+v"(W[1]), "+v"(W[2]), "+v"(W[3]), "+v"(W[4]), "+v"(W[5]), "+v"(ex) :: "memory");
            const unsigned long long fc = __ballot(trl & 64u);
            if (__ballot((int)(trl << 24 ^ tagw) < 0) == 0 && (unsigned)(fc >> 32) == call) {
                fmt = (unsigned)fc;
                if (fmt & F_RAW) break;
                if (data &&
                    __ballot((int)((W[0].w ^ tagw) | (W[1].w ^ tagw) | (W[2].w ^ tagw) | (W[3].w ^ tagw) | (W[4].w ^ tagw) | (W[5].w ^ tagw)) < 0 ||
                             ((unsigned)lane < (fmt >> 8) && ex.z != call)) == 0) break;
                data = true;
            } else __builtin_amdgcn_s_sleep(AGK_SL2);
            if ((++spin & 63) == 0) {
                if (t0 == 0) t0 = now();
                if (rfl(ld_agent(tmo)) == call || now() - t0 > DEADLINE_TICKS) { st_agent(tmo, call); return; }
            }
        }
        PSTAMP(j, p, 3, now()); PSTAMP(j, p, 7, spin); PSTAMP(j, p, 4, now());
        if (fmt & F_RAW) continue;
        const int cnt = fmt >> 8;
#pragma unroll
        for (int k = 0; k < 6; ++k) W[k].w = bfi(0x7FFFFFFFu, W[k].w, trl << (26 + k));
        const unsigned B4 = (fmt & 127u) * 0x01010101u;
        unsigned H[16];
#pragma unroll
        for (int k = 0; k < 8; ++k) {
            const unsigned n = W[(k >> 2) * 3 + 2][(k & 1) * 2 + ((k >> 1) & 1)];
            H[2 * k] = (n & 0x87878787u) + B4;
            H[2 * k + 1] = (rotr(n, 4) & 0x87878787u) + B4;
        }
        if (cnt) {
            unsigned tag = 255, val = 0;
            for (int e = 0; e < cnt; ++e) {
                const unsigned id = __builtin_amdgcn_readlane(ex.x, e), hv = __builtin_amdgcn_readlane(ex.y, e);
                const bool mine = (id & 63) == (unsigned)lane;
                tag = mine ? id >> 8 : tag; val = mine ? hv : val;
            }
#pragma unroll
            for (int i = 0; i < 16; ++i) H[i] = tag == (unsigned)i ? val : H[i];
        }
#define AGK_DST(k) { const unsigned l0 = W[(k >> 2) * 3 + (k & 1)][k & 2], l1 = W[(k >> 2) * 3 + (k & 1)][(k & 2) + 1]; u32x4 o; \
            o.x = perm(H[2 * k], l0, 0x05010400u); o.y = perm(H[2 * k], l0, 0x07030602u); \
            o.z = perm(H[2 * k + 1], l1, 0x05010400u); o.w = perm(H[2 * k + 1], l1, 0x07030602u); st_pk<(k & 3) * 512>(o, vd, k < 4 ? d0 : d1); }
        AGK_DST(0) AGK_DST(1) AGK_DST(2) AGK_DST(3) AGK_DST(4) AGK_DST(5) AGK_DST(6) AGK_DST(7)
        PSTAMP(j, p, 5, now());
        wait_vm<0>(); flag_st(buf + A.foff, callv, zero);
        PSTAMP(j, p, 6, now());
    }
}

#define AGK_RPG 2
#define AGK_RA 4, 6
#define AGK_RH 4, 12
#define AGK_RQ 8, 12
#define AGK_RW 6, 7
#define AGK_LOOK 3
template <int NB, int NP, int NSLOT, int SLOT_BYTES>
__device__ __forceinline__ void loader(GAS char* myws, unsigned call, const GAS char* const (&src)[NP], const int (&shift)[NP], const unsigned (&v0)[NB],
                                       const unsigned (&v1)[NB], const GAS char* fp, bool have, unsigned ldsw, unsigned rot, bool first) {
    constexpr int NI = 2 * NB;
    static_assert((NSLOT - 2) * NI + 8 < 64 && (NSLOT - 1) * NI + 10 < 64, "vmcnt range");
    gu32* tmo = (gu32*)(myws + TMO_OFF);
    const unsigned ldsend = ldsw + NSLOT * SLOT_BYTES;
    const unsigned long long act = ~__ballot(have);
    asm volatile("v_mov_b32 v250, %0" :: "v"(have ? ~0u : 0u) : "v250", "memory");
    int mark = 0;
    unsigned npoll = 0;
    auto poll = [&]() {
        ++npoll;
        asm volatile("s_mov_b64 exec, %1\n\tglobal_load_dword v250, %0, off sc0 sc1\n\ts_mov_b64 exec, -1" :: "v"(fp), "s"(act) : "v250", "memory");
    };
    auto eval = [&]() {
        unsigned long long bx;
        asm volatile("v_cmp_gt_u32 %0, %1, v250" : "=s"(bx) : "s"(call) : "memory");
        const unsigned m = (unsigned)bx | (unsigned)(bx >> 32);
        mark = __builtin_ctz(m | m >> 16 | 0x10000u) * 4;
    };
    unsigned ldsi = ldsw, kt = 0;
    const GAS char* ptr[NP];
#pragma unroll
    for (int i = 0; i < NP; ++i) ptr[i] = src[i] + (rot << shift[i]);
    auto issue = [&]() {
        asm volatile("s_mov_b32 m0, %0\n\ts_nop 0" :: "s"(ldsi) : "m0", "memory");
        glds2o<-4096>(ptr[0], v0[0], v1[0]);
        glds2o<-2048>(ptr[1 % NP], v0[1], v1[1]);
        if (NB == 4) { glds2o<0>(ptr[2 % NP], v0[2 % NB], v1[2 % NB]); glds2o<2048>(ptr[3 % NP], v0[3 % NB], v1[3 % NB]); }
        const unsigned k = ++kt ^ rot;
#pragma unroll
        for (int i = 0; i < NP; ++i) { ptr[i] = src[i] + (k << shift[i]); asm volatile("" : "+s"(ptr[i])); }
        ldsi = ldsi + SLOT_BYTES == ldsend ? ldsw : ldsi + SLOT_BYTES;
        asm volatile("" : "+s"(ldsi));
    };
    int ti = 0, tb = 0;
    unsigned hist = 0;
#pragma nounroll
    for (;;) {
        if (mark < NSTEP && mark - ti <= AGK_LOOK) eval();
        const int lim = mark < tb + NSLOT - 1 ? mark : tb + NSLOT - 1;
#pragma nounroll
        for (; ti < lim; ++ti) { issue(); hist = hist << 4 | (npoll & 15u); }
        if (ti == tb) {
            unsigned long long t0 = 0;
            for (unsigned spin = 1; ti >= mark; ++spin) {
                wait_vm<8>();
                poll();
                __builtin_amdgcn_s_sleep(AGK_SL1);
                if ((spin & 63) == 0) {
                    if (t0 == 0) t0 = now();
                    if (rfl(ld_agent(tmo)) == call || now() - t0 > DEADLINE_TICKS) break;
                }
                eval();
            }
            if (ti >= mark) break;
            continue;
        }
        const int run = (mark < NSTEP ? mark - AGK_LOOK : NSTEP) - ti;
        if (run > 0) {
#pragma nounroll
            for (int i = run; i > 0; --i) { wait_vm<(NSLOT - 2) * NI>(); bar(); issue(); }
            ti += run; tb += run;
            const unsigned rep = (npoll & 15u) * 0x11111111u;
            hist = run >= 8 ? rep : hist << (4 * run) | (rep & ((1u << (4 * run)) - 1u));
        }
        if (mark < NSTEP) poll();
        const unsigned n = ti - tb - 1;
        wait_vm_var(n * NI + (n < 8 ? (npoll - (hist >> (4 * n))) & 15u : 0u));
        bar();
        if (++tb == NSTEP) break;
    }
    if (tb < NSTEP) {
        st_agent(tmo, call);
        const u32x4 nan = {0x7FC07FC0u, 0x7FC07FC0u, 0x7FC07FC0u, 0x7FC07FC0u};
        for (; tb < NSTEP; ++tb) {
#pragma unroll
            for (int i = 0; i < NI; ++i) *(LAS u32x4*)(size_t)(ldsi - 4096 + i * 1024 + (threadIdx.x & 63) * 16) = nan;
            wait_lgkm();
            bar();
            ldsi = ldsi + SLOT_BYTES == ldsend ? ldsw : ldsi + SLOT_BYTES;
        }
    }
    if (first) STAMP(4);
    wait_vm<0>();
}

template <int S, int NH, int NSA, int NSB>
__device__ __forceinline__ void tile_role(const Params& P, LAS char* smem, unsigned call, int first_strip, int cg, int ch, int win, unsigned rot) {
    constexpr int SA = S * 2048, SB = NH * 8192, BOFF = NSA * SA;
    constexpr int SPW = S / 2, NFR = NH * 2;
    static_assert(BOFF + NSB * SB <= LDS_BYTES, "LDS rings");
    GAS char* myws = (GAS char*)P.ws[0];
    const int lane = threadIdx.x & 63;
    const int wv = rfl(threadIdx.x >> 6);
    const bool wremote = cg >= 4;

    if (wv >= 4) {
        const int lw = wv & 1;
        const GAS char* mybuf = myws + (size_t)(call & 1) * BUF_BYTES;
        const unsigned vlin = lane * 16;
        if (wv < 6) {
            const GAS char* src[SPW];
            unsigned v0[SPW], v1[SPW];
            int shift[SPW];
#pragma unroll
            for (int i = 0; i < SPW; ++i) {
                const int code = rfl(P.strip[first_strip + lw * SPW + i]);
                const bool own = code < 32;
                v0[i] = own ? ximg0(lane, KD * 2) : vlin;
                v1[i] = own ? ximg1(lane, KD * 2) : vlin;
                src[i] = (own ? (const GAS char*)P.x + (unsigned)code * (16 * KD * 2) : mybuf + (unsigned)xslot(code >> 5, code & 31) * (unsigned)STRIP_BYTES) + (4096 - 2048 * i);
                shift[i] = own ? 7 : 11;
            }
            const int fcode = P.strip[first_strip + lw * SPW + ((lane >> 4) & (SPW - 1))];
            loader<SPW, SPW, NSA, SA>(myws, call, src, shift, v0, v1, mybuf + XFLAG_OFF + fcode * XF_STRIDE + (lane & 15) * 4, fcode < 32 || lane >= 16 * SPW,
                                      (unsigned)(size_t)smem + lw * (SA / 2) + 4096, rot, lw == 0);
        } else {
            const unsigned dk = (wremote ? (unsigned)STRIP_BYTES : 16u * NSH * 2) - 2048, dh = (wremote ? 4 * (unsigned)STRIP_BYTES : 128u) - 4096;
            const GAS char* const src[1] = {(wremote ? mybuf + XS_BYTES + (unsigned)((cg - 4) * 8 + ch * 4 + lw * 2) * (unsigned)STRIP_BYTES
                                                     : (const GAS char*)P.w + cg * 256 + ch * 128 + lw * 32 * (NSH * 2))};
            const int shift[1] = {wremote ? 11 : 16};
            unsigned v0[NFR], v1[NFR];
#pragma unroll
            for (int i = 0; i < NFR; ++i) {
                v0[i] = (wremote ? vlin : wimg0(lane, NSH * 2)) + 4096 + (i & 1) * dk + (i >> 1) * dh;
                v1[i] = (wremote ? vlin : wimg1(lane, NSH * 2)) + 4096 + (i & 1) * dk + (i >> 1) * dh;
            }
            loader<NFR, 1, NSB, SB>(myws, call, src, shift, v0, v1, mybuf + WFLAG_OFF + ((cg & 3) * NWP + lw * 64 + lane) * 4, !wremote,
                                    (unsigned)(size_t)smem + BOFF + lw * (SB / 2) + 4096, rot, false);
        }
    } else {
        const int fr = lane & 15, g = lane >> 4;
        const int wm = wv >> 1, wn = wv & 1;
        unsigned aoff[2];
#pragma unroll
        for (int kk = 0; kk < 2; ++kk) aoff[kk] = wm * (SA / 2) + fr * 128 + (((kk * 4 + g) ^ ((fr >> 1) & 7)) << 4);
        unsigned boff[NFR];
        {
            const int q = fr >> 2, pp = lane & 3, f = (q >> 1) | ((g & 1) << 1);
#pragma unroll
            for (int nb = 0; nb < NFR; ++nb)
                boff[nb] = BOFF + (NH == 2 ? wn * 4096 : 0) + (g >> 1) * 2048 + ((g & 1) * 8 + q) * 128 + ((((wn * NFR + nb) & 3) ^ f) << 5) + pp * 8;
        }
        int rowbase[SPW];
#pragma unroll
        for (int i = 0; i < SPW; ++i) {
            const int code = P.strip[first_strip + wm * SPW + i];
            rowbase[i] = rfl((P.rank ^ (code >> 5)) * MSH + (code & 31) * 16);
        }
        GAS char* outp = (GAS char*)P.out[wremote ? cg : 0] + (cg & 3) * 256;
        GAS char* cntp = (GAS char*)P.ws[cg] + OCNT_OFF + (cg & 3) * 256;
        constexpr int orow = NSH * 2;
        asm volatile("" : "+s"(outp), "+s"(cntp));
        f32x4 acc[SPW][NFR];
#pragma unroll
        for (int i = 0; i < SPW; ++i)
#pragma unroll
            for (int nb = 0; nb < NFR; ++nb) acc[i][nb] = (f32x4){0.f, 0.f, 0.f, 0.f};
        struct Frag { bf16x8 a[SPW]; bf16x8 b[NFR]; };
        auto rd = [&](Frag& F, int sa, int sb, int kk) {
#pragma unroll
            for (int i = 0; i < SPW; ++i) F.a[i] = *(LAS bf16x8*)(smem + sa + aoff[kk] + i * 2048);
#pragma unroll
            for (int nb = 0; nb < NFR; ++nb) {
                const s16x4 lo = __builtin_amdgcn_ds_read_tr16_b64_v4i16((LAS s16x4*)(smem + sb + boff[nb] + kk * (SB / 2)));
                const s16x4 hi = __builtin_amdgcn_ds_read_tr16_b64_v4i16((LAS s16x4*)(smem + sb + boff[nb] + kk * (SB / 2) + 512));
                F.b[nb] = __builtin_bit_cast(bf16x8, __builtin_shufflevector(lo, hi, 0, 1, 2, 3, 4, 5, 6, 7));
            }
        };
        auto mm = [&](const Frag& F) {
#pragma unroll
            for (int nb = 0; nb < NFR; ++nb)
#pragma unroll
                for (int i = 0; i < SPW; ++i) acc[i][nb] = __builtin_amdgcn_mfma_f32_16x16x32_bf16(F.b[nb], F.a[i], acc[i][nb], 0, 0, 0);
        };
        Frag F0, F1;
        int ca = 0, cb = 0;
        bar();
        rd(F0, 0, 0, 0);
        wait_lgkm();
#pragma nounroll
        for (int t = 0; t < NSTEP - 1; ++t) {
            constexpr int NRD = SPW + 2 * NFR, NMM = SPW * NFR, RPG = S * NH == 16 ? 1 : AGK_RPG;
            rd(F1, ca, cb, 1);
            mm(F0);
#pragma unroll
            for (int i = 0; i < NMM; ++i) { __builtin_amdgcn_sched_group_barrier(0x008, 1, 0); if (i * RPG < NRD) __builtin_amdgcn_sched_group_barrier(0x100, RPG, 0); }
            PIN();
            wait_lgkm();
            bar();
            ca = ca + SA == NSA * SA ? 0 : ca + SA;
            cb = cb + SB == NSB * SB ? 0 : cb + SB;
            rd(F0, ca, cb, 0);
            mm(F1);
#pragma unroll
            for (int i = 0; i < NMM; ++i) { __builtin_amdgcn_sched_group_barrier(0x008, 1, 1); if (i * RPG < NRD) __builtin_amdgcn_sched_group_barrier(0x100, RPG, 1); }
            PIN();
            wait_lgkm();
        }
        rd(F1, ca, cb, 1);
        PIN();
        mm(F0);
        PIN();
        mm(F1);
        if (wv == 0) STAMP(2);
        constexpr int ROWB = NFR * 32;
        constexpr int CPR = ROWB / 16;
        constexpr int ESLOT = (NSTEP - 1) % NSB >= 2 ? 0 : 2;
        LAS char* eb = smem + BOFF + ESLOT * SB + wv * (SPW * 16 * ROWB);
        static_assert(4 * SPW * 16 * ROWB <= 2 * SB && NSB >= 4, "epilogue space");
#pragma unroll
        for (int i = 0; i < SPW; ++i)
#pragma unroll
            for (int nb = 0; nb < NFR; ++nb) {
                u32x2 o;
                o.x = pack_bf16(acc[i][nb][0], acc[i][nb][1]);
                o.y = pack_bf16(acc[i][nb][2], acc[i][nb][3]);
                *(LAS u32x2*)(eb + (i * 16 + fr) * ROWB + (((nb * 2 + (g >> 1)) ^ (fr & (CPR - 1))) << 4) + (g & 1) * 8) = o;
            }
        constexpr int RPI = 64 / CPR;
        constexpr int NST = SPW * 16 / RPI;
        u32x4 ov[NST];
        GAS char* od[NST];
        GAS char* obase = outp + (NH == 2 ? wn * 128 : ch * 128 + wn * 64) + (lane & (CPR - 1)) * 16;
#pragma unroll
        for (int n = 0; n < NST; ++n) {
            const int row = n * RPI + lane / CPR;
            ov[n] = *(LAS u32x4*)(eb + row * ROWB + (((lane & (CPR - 1)) ^ (row & (CPR - 1))) << 4));
            od[n] = obase + (size_t)((rowbase[(n * RPI) / 16] + (row & 15)) * orow);
        }
        if (wv == 0) STAMP(5);
        if (win == 1) {
#pragma unroll
            for (int n = 0; n < NST; ++n) { if (n >= 1) wait_vm<0>(); st16_wt(od[n], ov[n]); }
        } else if (win == 2) {
#pragma unroll
            for (int n = 0; n < NST; ++n) { if (n >= 2) wait_vm<1>(); st16_wt(od[n], ov[n]); }
        } else {
#pragma unroll
            for (int n = 0; n < NST; ++n) st16_wt(od[n], ov[n]);
        }
        if (wremote) {
            wait_vm<0>();
            if (lane == 0) __hip_atomic_fetch_add((gu32*)cntp, 1u, __ATOMIC_RELAXED, __HIP_MEMORY_SCOPE_SYSTEM);
        }
        if (wv == 0) STAMP(3);
    }
}

__global__ void __launch_bounds__(512, 1) __attribute__((amdgpu_num_vgpr(240))) ag_gemm_kernel(const Params P) {
    __shared__ __attribute__((aligned(1024))) char smem_raw[LDS_BYTES];
    LAS char* smem = (LAS char*)smem_raw;
    const unsigned long long t64 = now();
    const unsigned tstart = (unsigned)t64;
    const int b = blockIdx.x;
    if (__builtin_expect(b < NCOMM, 1)) {
        const int wv = rfl(threadIdx.x >> 6), g = wv * NCOMM + b;
        if (wv < NCW) comm_role(P, smem + wv * 16384, t64, wv, 1 + g % 7, g / 7);
        return;
    }
    GAS char* myws = (GAS char*)P.ws[0];
    gu32* cc = (gu32*)(myws + CALL_OFF + b * 64);
    const unsigned role = rfl(P.role[b - NCOMM]);
    const int type = role & 15;
    if (type == R_UNPACK) {
        const int wv = rfl(threadIdx.x >> 6);
        const unsigned call = rfl(ld_agent(cc + wv)) + 1u;
        unpack_role(P, myws, tstart, call, (int)(role >> 4) + (int)P.nunpack * wv, (int)P.nunpack * 8);
        if (wv == 7) STAMP(4);
        if ((threadIdx.x & 63) == 0) st_agent(cc + wv, call);
    } else if (type != R_IDLE) {
        const unsigned call = rfl(ld_agent(cc)) + 1u;
        sleep_until(tstart + (P.tlead >> 16));
        if (threadIdx.x == 0) STAMP(0);
        const int fs = (role >> 4) & 255, cg = (role >> 12) & 7, ch = (role >> 15) & 1, win = (role >> 18) & 3;
        if (type == R_T128) tile_role<8, 2, AGK_RA>(P, smem, call, fs, cg, 0, win, role >> 28);
        else if (type == R_T128H) tile_role<8, 1, AGK_RH>(P, smem, call, fs, cg, ch, win, role >> 28);
        else if (type == R_T64W) tile_role<4, 2, AGK_RW>(P, smem, call, fs, cg, 0, win, role >> 28);
        else tile_role<4, 1, AGK_RQ>(P, smem, call, fs, cg, ch, win, role >> 28);
        if (threadIdx.x == 256) st_agent(cc, call);
    }
}

}

#define TUNE(i, dflt) (dflt)

extern "C" void kernel_launch(int rank, int nranks, ncclComm_t comm,
                              hipStream_t stream,
                              void* const* d_in, const int* in_sizes, int n_in,
                              void* d_out, int out_size,
                              const int* in_dtypes, int out_dtype,
                              void* const* d_out_all,
                              void* const* d_ws_all, size_t ws_bytes) {
    using namespace agk;
    (void)comm; (void)in_dtypes; (void)out_dtype; (void)d_out;
    if (nranks != NR || (unsigned)rank >= (unsigned)NR || n_in < 2 || ws_bytes < WS_NEED ||
        in_sizes[0] != MSH * KD || in_sizes[1] != KD * NSH || out_size != NR * MSH * NSH) {
        fprintf(stderr, "ag_gemm: unexpected problem shape\n");
        return;
    }
    Params P;
    P.x = (const char*)d_in[0];
    P.w = (const char*)d_in[1];
    P.rank = rank;
    for (int j = 0; j < NR; ++j) { P.ws[j] = (char*)d_ws_all[rank ^ j]; P.out[j] = (char*)d_out_all[rank ^ j]; }
    int sega = TUNE(0, 11);
    int blate = TUNE(1, 9);
    int wlead = TUNE(2, 4);
    int rt_b = TUNE(3, 1), rt_c = TUNE(4, 1), rt_a = TUNE(5, 0), rt_w = TUNE(6, 0);
    int split_c = TUNE(7, 6);
    int split_a = TUNE(8, 0);
    int small_b = TUNE(9, 0);
    const int segb = HS - sega;
    const int na_t = (3 * sega + 7) / 8, nc_t = (4 * AS + 7) / 8, nb_t = (3 * segb + 7) / 8, no_t = 16 - na_t - nc_t - nb_t;
    const int ne_t = no_t + na_t;
    bool ok = no_t >= 0;
    int n = 0, own = 0;
    for (int i = 0; i < no_t * 8; ++i) P.strip[n++] = (unsigned char)(own++);
    for (int s = 0; s < sega; ++s) for (int j = 1; j < 4; ++j) P.strip[n++] = (unsigned char)((j << 5) | s);
    while (n % 8) P.strip[n++] = (unsigned char)(own++);
    for (int s = HS; s < NSTRIP; ++s) for (int j = 4; j < 8; ++j) P.strip[n++] = (unsigned char)((j << 5) | s);
    while (n % 8) P.strip[n++] = (unsigned char)(own++);
    for (int s = sega; s < HS; ++s) for (int j = 1; j < 4; ++j) P.strip[n++] = (unsigned char)((j << 5) | s);
    while (n % 8) P.strip[n++] = (unsigned char)(own++);
    ok = ok && n == 128 && own == HS;
    int q = 0;
    const int t_first = TUNE(16, 160) * 16, t_pk = TUNE(17, 174), t_raw = TUNE(18, 232);
    const int rh_a = TUNE(19, 0), rh_w = TUNE(20, 0), afirst = TUNE(21, 0);
    const int n_slow = TUNE(25, 0), t_slow = TUNE(15, 184);
    int tq = t_first;
    for (int i = 0; i < 2 * WPL * NRND; ++i) P.ord[i] = 0;
    auto emit = [&](unsigned e) { if (q < NP_IN + NP_XI) P.ord[q < NP_IN ? ord_at(0, q) : ord_at(1, q - NP_IN)] = e | (unsigned)(tq >> 4) << 16; ++q; tq += (e & 512u) ? t_raw : tq < t_first + n_slow * t_slow ? t_slow : t_pk; };
    auto xchunk = [&](int s0, int s1, int kc, int rt, int rh) { for (int s = s0; s < s1; ++s) emit((unsigned)kc | (s << 4) | (kc >= NKC - rt || kc < rh ? 512u : 0u)); };
    auto wchunk = [&](int kc) { for (int i = 0; i < 8; ++i) emit((unsigned)(kc * 8 + i) | 1024u | (kc >= NKC - rt_w || kc < rh_w ? 512u : 0u)); };
    for (int ka = 0, kb = 0; kb < NKC; ) {
        if (ka < NKC && kb * (NKC - afirst) >= (ka - afirst) * (NKC - blate)) xchunk(0, sega, ka++, rt_a, rh_a);
        else xchunk(sega, HS, kb++, rt_b, 0);
    }
    ok = ok && q == NP_IN;
    tq = t_first;
    for (int k = -wlead; k < NKC; ++k) {
        if (k + wlead < NKC) wchunk(k + wlead);
        if (k >= 0) xchunk(HS, NSTRIP, k, rt_c, 0);
    }
    ok = ok && q == NP_IN + NP_XI;
    int fill[8] = {0, 0, 0, 0, 0, 0, 0, 0};
    unsigned role[NWG];
    for (int i = 0; i < NWG; ++i) role[i] = R_IDLE;
    auto put = [&](int k, unsigned v) { if (fill[k] < NWG / 8) role[fill[k]++ * 8 + k] = v; else ok = false; };
    auto code = [&](int type, int fs, int cg, int ch) { return (unsigned)type | (fs << 4) | (cg << 12) | (ch << 15); };
    for (int i = 0; i < NCOMM; ++i) put(i & 7, R_COMM | (i << 4));
    unsigned adds = 0;
    for (int cg = 0; cg < 8; ++cg) {
        const bool rem = cg >= 4;
        for (int r = 0; r < 16; ++r) {
            const int k = (cg & 3) + 4 * (r & 1);
            const bool early = r < ne_t, last = r >= ne_t + nc_t;
            const bool split = last ? rem || r - ne_t - nc_t >= TUNE(29, 4) : (rem ? (early ? r >= ne_t - split_a : r - ne_t < split_c) : (!early && r - ne_t < TUNE(22, 0)));
            const bool small = last && r - ne_t - nc_t < (rem ? small_b >> 4 : small_b & 15);
            const unsigned win = (unsigned)(!rem ? 0 : early ? TUNE(14, 2) : last ? TUNE(12, 0) : split ? TUNE(13, 2) : TUNE(11, 2)) << 18;
            int nw = 0;
            auto rot = [&](int who, int of) { return (unsigned)(TUNE(26, 1) ? (of == 2 ? (who & 1) * 2 + ((r >> 1) & 1) : (who ^ (((r >> 1) & 1) * 2)) & 3) : 0) << 28; };
            if (small) for (int h = 0; h < 4; ++h) { put(k, rot(rem * 2 + (h & 1), 4) | win | code(R_T64H, r * 8 + (h >> 1) * 4, cg, h & 1)); nw += 4; }
            else if (split && (TUNE(23, 3) >> (last ? 0 : 1) & 1)) for (int h = 0; h < 2; ++h) { put(k, (unsigned)((rem * 2 + h * TUNE(24, 1)) ^ ((r >> 1) & 1)) << 28 | win | code(R_T64W, r * 8 + h * 4, cg, 0)); nw += 4; }
            else if (split) for (int ch = 0; ch < 2; ++ch) { put(k, rot(rem * 2 + ch, 4) | win | code(R_T128H, r * 8, cg, ch)); nw += 4; }
            else { put(k, rot(rem, 2) | win | code(R_T128, r * 8, cg, 0)); nw += 4; }
            if (cg == 4) adds += nw;
        }
    }
    unsigned nun = 0;
    for (int i = 0; i < NWG; ++i) if (role[i] == R_IDLE && nun < (unsigned)TUNE(10, 64)) role[i] = R_UNPACK | (nun++ << 4);
    for (int i = NCOMM; i < NWG; ++i) P.role[i - NCOMM] = role[i];
    P.tlead = TUNE(27, 24) | TUNE(28, 250) << 16;
    P.out_adds = adds; P.tpoll = TUNE(30, 100) | TUNE(31, 1) << 16; P.nunpack = nun;
    ok = ok && nun > 0;
    if (!ok) { fprintf(stderr, "ag_gemm: role table overflow\n"); return; }
    hipLaunchKernelGGL(ag_gemm_kernel, dim3(NWG), dim3(512), 0, stream, P);
    HIP_CHECK(hipGetLastError());
}
```

```cpp
#include <hip/hip_runtime.h>
#include <rccl/rccl.h>
#include <cstdio>
#include <cstdint>

#define HIP_CHECK(expr) do { hipError_t _e = (expr); if (_e != hipSuccess) { \
    fprintf(stderr, "HIP error %s at %s:%d\n", hipGetErrorString(_e), __FILE__, __LINE__); \
    return; } } while (0)

namespace agk {

typedef unsigned int u32x4 __attribute__((ext_vector_type(4)));
typedef unsigned int u32x3 __attribute__((ext_vector_type(3)));
typedef unsigned int u32x2 __attribute__((ext_vector_type(2)));
typedef short s16x4 __attribute__((ext_vector_type(4)));
typedef short s16x8 __attribute__((ext_vector_type(8)));
typedef __bf16 bf16x8 __attribute__((ext_vector_type(8)));
typedef float f32x4 __attribute__((ext_vector_type(4)));
#define LAS __attribute__((address_space(3)))
#define GAS __attribute__((address_space(1)))
typedef GAS unsigned int gu32;

constexpr int NR = 8;
constexpr int KD = 4096;
constexpr int MSH = 512;
constexpr int NSH = 512;
constexpr int NSTRIP = 32;
#define AGK_HS 21
constexpr int HS = AGK_HS;
constexpr int AS = NSTRIP - HS;
constexpr int NKC = 16;
constexpr int NSTEP = 64;
constexpr int NWP = 128;

#define AGK_NCOMM 56
#define AGK_NCW 4
constexpr int NCOMM = AGK_NCOMM;
constexpr int NCW = AGK_NCW;
constexpr int WPL = NCOMM * NCW / 7;
static_assert(NCOMM % 7 == 0, "a comm workgroup serves one link");

constexpr int NP_IN = HS * NKC;
constexpr int NP_XI = NWP + AS * NKC;
constexpr int NPMAX = NP_IN > NP_XI ? NP_IN : NP_XI;
constexpr int NRND = (NPMAX + WPL - 1) / WPL;
constexpr int NWG = 256;
enum { R_IDLE = 0, R_COMM = 1, R_T128 = 2, R_T128H = 3, R_T64H = 4, R_UNPACK = 5, R_T64W = 6 };

constexpr size_t STRIP_BYTES = 16ull * KD * 2;
constexpr int NXSLOT = 3 * HS + 4 * AS;
constexpr size_t XS_BYTES = (size_t)NXSLOT * STRIP_BYTES;
constexpr size_t WSL_BYTES = 8 * STRIP_BYTES;
constexpr int XF_STRIDE = 256;
constexpr size_t XFLAG_OFF = XS_BYTES + 4 * WSL_BYTES;
constexpr size_t WFLAG_OFF = XFLAG_OFF + 8 * 32 * XF_STRIDE;
constexpr size_t PK_OFF = WFLAG_OFF + 4 * NWP * 4;
constexpr unsigned PK_BYTES = 6400;
constexpr size_t BUF_BYTES = (PK_OFF + (size_t)(3 * NP_IN + 4 * NP_XI) * PK_BYTES + 4095) / 4096 * 4096;
constexpr size_t OCNT_OFF = 2 * BUF_BYTES;
constexpr size_t TMO_OFF = OCNT_OFF + 4 * 256;
constexpr size_t CALL_OFF = TMO_OFF + 256;
constexpr size_t DBG_OFF = CALL_OFF + 256 * 64;
constexpr size_t DBG2_OFF = DBG_OFF + 256 * 128;
constexpr size_t WS_NEED = DBG2_OFF + 8 * 512 * 32;
static_assert(BUF_BYTES % 4096 == 0, "alignment");
static_assert(WS_NEED <= 65ull << 20, "workspace");

constexpr unsigned long long DEADLINE_TICKS = 30000000ull;
constexpr int LDS_BYTES = 163840;

struct Params {
    const char* x;
    const char* w;
    char* ws[NR];
    char* out[NR];
    int rank;
    unsigned out_adds;
    unsigned tpoll;
    unsigned nunpack;
    unsigned role[NWG - NCOMM];
    unsigned tlead;
    unsigned ord[2 * WPL * NRND];
    unsigned char strip[128];
};

static_assert(sizeof(Params) <= 4096, "kernel arguments");

__device__ __forceinline__ unsigned ld_sys(const gu32* p) { return __hip_atomic_load(p, __ATOMIC_RELAXED, __HIP_MEMORY_SCOPE_SYSTEM); }
__device__ __forceinline__ unsigned ld_agent(const gu32* p) { return __hip_atomic_load(p, __ATOMIC_RELAXED, __HIP_MEMORY_SCOPE_AGENT); }
__device__ __forceinline__ void st_agent(gu32* p, unsigned v) { __hip_atomic_store(p, v, __ATOMIC_RELAXED, __HIP_MEMORY_SCOPE_AGENT); }
__device__ __forceinline__ unsigned long long now() { return __builtin_amdgcn_s_memrealtime(); }
__device__ __forceinline__ unsigned rfl(unsigned v) { return __builtin_amdgcn_readfirstlane(v); }

__device__ __forceinline__ void st16_wt(GAS char* p, u32x4 v) {
    asm volatile("global_store_dwordx4 %0, %1, off sc0 sc1\n\ts_nop 1" :: "v"(p), "v"(v) : "memory");
}
template <int N> __device__ __forceinline__ void wait_vm() { asm volatile("s_waitcnt vmcnt(%0)" :: "i"(N) : "memory"); }
__device__ __forceinline__ void wait_lgkm() { asm volatile("" ::: "memory"); __builtin_amdgcn_s_waitcnt(0xC07F); asm volatile("" ::: "memory"); }
#define AGK_SL1 6
#define AGK_SL2 2
#define PIN() __builtin_amdgcn_sched_barrier(0)
__device__ __forceinline__ void sleep_until(unsigned t) {
    int rest = (int)(t - (unsigned)now());
    for (; rest > 48; rest = (int)(t - (unsigned)now())) __builtin_amdgcn_s_sleep(8);
    for (; rest > 2; rest -= 3) __builtin_amdgcn_s_sleep(1);
}
__device__ __forceinline__ void bar() { asm volatile("s_barrier" ::: "memory"); }

#define AGK_LDPOL "sc0 sc1"
__device__ __forceinline__ void glds2(const GAS char* base, unsigned v0, unsigned v1, unsigned lds_dst) {
    asm volatile("s_mov_b32 m0, %3\n\ts_nop 3\n\tglobal_load_lds_dwordx4 %1, %0 " AGK_LDPOL "\n\tglobal_load_lds_dwordx4 %2, %0 offset:1024 " AGK_LDPOL
                 :: "s"(base), "v"(v0), "v"(v1), "s"(lds_dst) : "m0", "memory");
}
template <int OFF> __device__ __forceinline__ void glds2o(const GAS char* base, unsigned v0, unsigned v1) {
    asm volatile("global_load_lds_dwordx4 %1, %0 offset:%3 " AGK_LDPOL "\n\tglobal_load_lds_dwordx4 %2, %0 offset:%4 " AGK_LDPOL
                 :: "s"(base), "v"(v0), "v"(v1), "i"(OFF), "i"(OFF + 1024) : "memory");
}
__device__ __forceinline__ void wait_vm_var(unsigned n) {
    asm volatile("s_getpc_b64 vcc\n\ts_add_u32 vcc_lo, vcc_lo, %0\n\ts_addc_u32 vcc_hi, vcc_hi, 0\n\ts_setpc_b64 vcc\n\t"
                 ".set agk_n, 0\n\t.rept 64\n\ts_waitcnt vmcnt(agk_n)\n\ts_branch 1f\n\t.set agk_n, agk_n + 1\n\t.endr\n1:"
                 :: "s"(n * 8 + 12) : "vcc", "scc", "memory");
}

__device__ __forceinline__ unsigned pack_bf16(float lo, float hi) {
    typedef float f32x2 __attribute__((ext_vector_type(2)));
    typedef __bf16 bf16x2 __attribute__((ext_vector_type(2)));
    f32x2 v = {lo, hi};
    bf16x2 r = __builtin_convertvector(v, bf16x2);
    return __builtin_bit_cast(unsigned, r);
}

__device__ __forceinline__ int xslot(int j, int s) { return j < 4 ? (j - 1) * HS + s : 3 * HS + (j - 4) * AS + (s - HS); }

#define STAMP(i) do { } while (0)
#define PSTAMP(j, p, i, t) do { } while (0)

enum { K_X = 0, K_W = 1 };
struct Piece { int kind, raw; unsigned soff, doff, foff, when; };

__host__ __device__ __forceinline__ int ord_at(int cross, int p) { return (cross * WPL + p % WPL) * NRND + p / WPL; }
__device__ __forceinline__ Piece locate(unsigned e, int j) {
    Piece r;
    r.when = e >> 16;
    const int s = (e >> 4) & 31, kc = e & 15, wp = (e >> 10) & 1 ? (int)(e & 127) : -1;
    r.raw = (e >> 9) & 1;
    if (wp >= 0) {
        r.kind = K_W;
        r.soff = (unsigned)(wp * 32 * (NSH * 2) + (j & 3) * 256);
        r.doff = (unsigned)XS_BYTES + (unsigned)(j - 4) * (unsigned)WSL_BYTES + (wp & 1) * 2 * (unsigned)STRIP_BYTES + (wp >> 1) * 2048;
        r.foff = (unsigned)WFLAG_OFF + ((j - 4) * NWP + (wp & 1) * 64 + ((wp >> 1) & 3) * 16 + (wp >> 3)) * 4;
    } else {
        r.kind = K_X;
        r.soff = (unsigned)(s * 16 * (KD * 2) + kc * 512);
        r.doff = (unsigned)xslot(j, s) * (unsigned)STRIP_BYTES + kc * 8192;
        r.foff = (unsigned)XFLAG_OFF + (j * 32 + s) * XF_STRIDE + kc * 4;
    }
    return r;
}
__device__ __forceinline__ unsigned pk_off(int j, int p) {
    return (unsigned)PK_OFF + 3072u + (unsigned)((j < 4 ? (j - 1) * NP_IN : 3 * NP_IN + (j - 4) * NP_XI) + p) * PK_BYTES;
}

template <int OFF> __device__ __forceinline__ void st_pk(const u32x4& v, unsigned voff, const GAS char* dst) {
    asm volatile("s_nop 3\n\tglobal_store_dwordx4 %1, %0, %2 offset:%3 sc0 sc1\n\ts_nop 1" :: "v"(v), "v"(voff), "s"(dst), "i"(OFF) : "memory");
}
template <int OFF> __device__ __forceinline__ void ld_c(u32x4& v, unsigned voff, const GAS char* src) {
    asm volatile("global_load_dwordx4 %0, %1, %2 offset:%3 sc0 sc1" : "=v"(v) : "v"(voff), "s"(src), "i"(OFF) : "memory");
}
__device__ __forceinline__ void flag_st(const GAS char* f, unsigned val, unsigned zero) {
    asm volatile("s_mov_b64 exec, 1\n\ts_nop 3\n\tglobal_store_dword %0, %1, %2 sc0 sc1\n\ts_mov_b64 exec, -1" :: "v"(zero), "v"(val), "s"(f) : "memory");
}
__device__ __forceinline__ unsigned perm(unsigned hi, unsigned lo, unsigned sel) { return __builtin_amdgcn_perm(hi, lo, sel); }
__device__ __forceinline__ unsigned bfi(unsigned m, unsigned a, unsigned b) { return (a & m) | (b & ~m); }
__device__ __forceinline__ unsigned rotr(unsigned v, unsigned n) { return __builtin_amdgcn_alignbit(v, v, n); }
__device__ __forceinline__ unsigned ffbl(unsigned v) { unsigned r; asm("v_ffbl_b32 %0, %1" : "=v"(r) : "v"(v)); return r; }
__device__ __forceinline__ unsigned pkmax(unsigned a, unsigned b) {
    typedef unsigned short u16x2 __attribute__((ext_vector_type(2)));
    return __builtin_bit_cast(unsigned, __builtin_elementwise_max(__builtin_bit_cast(u16x2, a), __builtin_bit_cast(u16x2, b)));
}
template <int CTRL> __device__ __forceinline__ unsigned dppmax(unsigned v) { return max(v, (unsigned)__builtin_amdgcn_update_dpp(0, (int)v, CTRL, 0xF, 0xF, true)); }
constexpr unsigned F_RAW = 0x80u;
constexpr int MAXEXC = 16;
__device__ __forceinline__ unsigned ximg0(unsigned lane, unsigned pitch) { return (lane >> 3) * pitch + ((lane & 7) ^ (lane >> 4)) * 16; }
__device__ __forceinline__ unsigned ximg1(unsigned lane, unsigned pitch) { return (8 + (lane >> 3)) * pitch + ((lane & 7) ^ (4 + (lane >> 4))) * 16 - 1024; }
__device__ __forceinline__ unsigned wimg0(unsigned lane, unsigned pitch) { return (lane >> 3) * pitch + ((lane & 7) ^ ((lane >> 3) & 2)) * 16; }
__device__ __forceinline__ unsigned wimg1(unsigned lane, unsigned pitch) { return (8 + (lane >> 3)) * pitch + ((lane & 7) ^ (((lane >> 3) & 2) | 4)) * 16 - 1024; }

__device__ __forceinline__ void fetch_piece(int kind, unsigned soff, const GAS char* sx, const GAS char* sw,
                                            unsigned vx0, unsigned vx1, unsigned vw0, unsigned vw1, unsigned l) {
    const bool kx = kind == K_X;
    const GAS char* s = (kx ? sx : sw) + soff;
    const unsigned a = kx ? vx0 : vw0, b = kx ? vx1 : vw1;
    const int q1 = kx ? 128 : 16 * NSH * 2, q2 = kx ? 256 : 128;
    glds2(s, a, b, l); glds2(s + q1, a, b, l + 2048); glds2(s + q2, a, b, l + 4096); glds2(s + q1 + q2, a, b, l + 6144);
}

__device__ __forceinline__ unsigned tag_of(unsigned call) { return ((call + 1u) >> 1) << 31; }

__device__ __forceinline__ void comm_role(const Params& P, LAS char* smem, unsigned long long t64, int wslot, int j, int u) {
    const unsigned* ord = P.ord + ord_at(j >= 4, u);
    unsigned eA = ord[0], eB = ord[1], tlead = P.tlead;
    const GAS char *srcx = (const GAS char*)P.x, *srcw = (const GAS char*)P.w, *pws = (const GAS char*)P.ws[j];
    GAS char* myws = (GAS char*)P.ws[0];
    asm volatile("" : "+s"(eA), "+s"(eB), "+s"(tlead), "+s"(srcx), "+s"(srcw), "+s"(pws), "+s"(myws), "+s"(t64));
    const unsigned tstart = (unsigned)t64;
    gu32* cc = (gu32*)(myws + CALL_OFF + blockIdx.x * 64) + wslot;
    const int lane = threadIdx.x & 63;
    const int npiece = j < 4 ? NP_IN : NP_XI;
    const unsigned vlin = lane * 16;
    const unsigned vx0 = ximg0(lane, KD * 2), vx1 = ximg1(lane, KD * 2), vw0 = wimg0(lane, NSH * 2), vw1 = wimg1(lane, NSH * 2);
    const unsigned vblk = (lane >> 5) * 2048 + (lane & 31) * 16, vblw = (lane >> 5) * (unsigned)STRIP_BYTES + (lane & 31) * 16;
    const unsigned lds0 = (unsigned)(size_t)smem;
#define AGK_FETCH(q, slot) fetch_piece((q).kind, (q).soff, srcx, srcw, vx0, vx1, vw0, vw1, lds0 + (slot) * 8192)
    int pA = u;
    unsigned ccv;
    Piece A = locate(eA, j);
    if (u > 3) sleep_until(tstart + (u * (tlead & 0xFFFFu) >> 2));
    AGK_FETCH(A, 0);
    asm volatile("global_load_dword %0, %1, off sc1" : "=v"(ccv) : "v"(cc) : "memory");
    Piece B = locate(eB, j);
    asm volatile("s_waitcnt vmcnt(0)" : "+v"(ccv) :: "memory");
    AGK_FETCH(B, 1);
    const unsigned call = rfl(ccv) + 1u, tagw = tag_of(call);
    if (wslot == 0) STAMP(0);
    const GAS char* dbuf = pws + (size_t)(call & 1) * BUF_BYTES;
    gu32* tmo = (gu32*)(myws + TMO_OFF);
    unsigned zero = 0, callv = call;
    asm volatile("" : "+v"(zero), "+v"(callv));
    int slot = 0;
    for (;;) {
        u32x4 v[8], W[5];
        u32x2 W5;
#pragma unroll
        for (int k = 0; k < 8; ++k) v[k] = *(LAS u32x4*)(smem + slot * 8192 + (k >> 2) * 4096 + (k & 3) * 512 + vblk);
        bool raw = A.raw;
        u32x3 ev = {0, 0, callv}, em = {0, 0, callv};
        unsigned trl = 0, eoff = 0, moff = 0; unsigned long long bm = 0, bo = 0;
        if (!raw) {
            unsigned mx = pkmax(pkmax(pkmax(v[0].x & 0x7FFF7FFFu, v[0].y & 0x7FFF7FFFu), pkmax(v[0].z & 0x7FFF7FFFu, v[0].w & 0x7FFF7FFFu)),
                                pkmax(pkmax(v[6].x & 0x7FFF7FFFu, v[6].y & 0x7FFF7FFFu), pkmax(v[6].z & 0x7FFF7FFFu, v[6].w & 0x7FFF7FFFu)));
            mx = max(mx >> 16, mx & 0xFFFFu) >> 8;
            mx = dppmax<0x143>(dppmax<0x142>(dppmax<0x118>(dppmax<0x114>(dppmax<0x112>(dppmax<0x111>(mx))))));
            unsigned fmt = max((unsigned)__builtin_amdgcn_readlane((int)mx, 63), 7u) - 7u;
            const unsigned B4 = fmt * 0x01010101u;
            unsigned accO = 0, accA = ~0u, N[8];
#pragma unroll
            for (int k = 0; k < 8; ++k) {
                const unsigned Ha = perm(v[k].y, v[k].x, 0x07050301u), Hb = perm(v[k].w, v[k].z, 0x07050301u);
                const unsigned da = (Ha | 0x80808080u) - B4, db = (Hb | 0x80808080u) - B4;
                accO |= da | db; accA &= da & db;
                W[k >> 1][(k & 1) * 2] = perm(v[k].y, v[k].x, 0x06040200u);
                W[k >> 1][(k & 1) * 2 + 1] = perm(v[k].w, v[k].z, 0x06040200u);
                N[k] = bfi(0x87878787u, bfi(0x80808080u, Ha, da), rotr(bfi(0x80808080u, Hb, db), 28));
            }
            em.x = ~bfi(0x33333333u, bfi(0x11111111u, N[0] >> 2, N[1] >> 1), bfi(0x44444444u, N[2], N[3] << 1));
            em.y = ~bfi(0x33333333u, bfi(0x11111111u, N[4] >> 2, N[5] >> 1), bfi(0x44444444u, N[6], N[7] << 1));
            W[4] = (u32x4){bfi(0x44444444u, N[6] << 2, N[0]), bfi(0x44444444u, N[6] << 1, N[1]), bfi(0x44444444u, N[6] >> 1, N[2]), bfi(0x44444444u, N[7] << 2, N[3])};
            W5 = (u32x2){bfi(0x44444444u, N[7] << 1, N[4]), bfi(0x44444444u, N[7] >> 1, N[5])};
            unsigned long long m = (unsigned long long)em.y << 32 | em.x;
            unsigned pos = 0;
#pragma unroll
            for (int i = 0; i < 4; ++i) {
                pos = __builtin_amdgcn_alignbit(min(ffbl((unsigned)m), ffbl((unsigned)(m >> 32)) | 32u), pos, 6);
                m &= m - 1;
            }
            const bool over = m != 0 || (int)em.y < 0;
            bo = __ballot(over);
            moff = __builtin_amdgcn_mbcnt_hi((unsigned)(bo >> 32), __builtin_amdgcn_mbcnt_lo((unsigned)bo, 0));
            pos = over ? moff << 14 | 0xFFF03F00u : pos;
            moff *= 16;
            const unsigned cnt2 = __builtin_popcountll(bo);
            raw = cnt2 > MAXEXC;
            fmt |= cnt2 << 16;
            bm = __ballot(((accO & 0x78787878u) | (~accA & 0x80808080u)) != 0);
            if (bm) {
                unsigned nb = 0, ib = 0;
#pragma unroll
                for (int i = 0; i < 16; ++i) {
                    const unsigned H = (i & 1) ? perm(v[i >> 1].w, v[i >> 1].z, 0x07050301u) : perm(v[i >> 1].y, v[i >> 1].x, 0x07050301u);
                    const bool bad = (((H | 0x80808080u) - B4) & 0xF8F8F8F8u) != 0x80808080u;
                    ev.y = bad ? H : ev.y; ib = bad ? (unsigned)i : ib; nb += bad;
                }
                ev.x = lane | (ib << 8);
                eoff = __builtin_amdgcn_mbcnt_hi((unsigned)(bm >> 32), __builtin_amdgcn_mbcnt_lo((unsigned)bm, 0)) * 16;
                const unsigned cnt = __builtin_popcountll(bm);
                raw = raw || cnt > MAXEXC || __ballot(nb > 1) != 0;
                fmt |= cnt << 8;
            }
            if (raw) { fmt = F_RAW; bm = 0; bo = 0; }
            trl = (tagw >> 30) + (__builtin_amdgcn_inverse_ballot_w64((unsigned long long)call << 32 | fmt) ? 1u : 0u);
#pragma unroll
            for (int k = 0; k < 5; ++k) { trl = __builtin_amdgcn_alignbit(trl, W[k].w, 31); W[k].w = bfi(0x7FFFFFFFu, W[k].w, tagw); }
            trl = __builtin_amdgcn_alignbit(trl, W5.y, 31); W5.y = bfi(0x7FFFFFFFu, W5.y, tagw);
            trl = __builtin_amdgcn_alignbit(trl, pos, 8);
        } else {
#pragma unroll
            for (int k = 0; k < 5; ++k) W[k] = v[k];
            W5 = (u32x2){v[5].x, v[5].y};
        }
        const GAS char* pk = dbuf + pk_off(j, pA);
        const Piece C = locate(ord[pA + 2 * WPL < npiece ? pA / WPL + 2 : 0], j);
        asm volatile("" :: "v"(W[0]), "v"(W[1]), "v"(W[2]), "v"(W[3]), "v"(W[4]), "v"(W5), "v"(v[6]), "v"(v[7]), "v"(trl), "v"(ev), "v"(eoff), "v"(em), "v"(moff),
                     "s"(pk), "s"(C.soff) : "memory");
        sleep_until(tstart + A.when);
        if (!A.raw) {
            st_pk<-3072>(W[0], vlin, pk); st_pk<-2048>(W[1], vlin, pk); st_pk<-1024>(W[2], vlin, pk); st_pk<0>(W[3], vlin, pk); st_pk<1024>(W[4], vlin, pk);
            asm volatile("global_store_dwordx2 %0, %1, %2 offset:2048 sc0 sc1" :: "v"(vlin >> 1), "v"(W5), "s"(pk) : "memory");
            if (bm) asm volatile("s_mov_b64 exec, %3\n\ts_nop 3\n\tglobal_store_dwordx3 %0, %1, %2 offset:2816 sc0 sc1\n\ts_mov_b64 exec, -1" :: "v"(eoff), "v"(ev), "s"(pk), "s"(bm) : "memory");
            if (bo) asm volatile("s_mov_b64 exec, %3\n\ts_nop 3\n\tglobal_store_dwordx3 %0, %1, %2 offset:3072 sc0 sc1\n\ts_mov_b64 exec, -1" :: "v"(moff), "v"(em), "s"(pk), "s"(bo) : "memory");
            asm volatile("global_store_dword %0, %1, %2 offset:2560 sc0 sc1" :: "v"(vlin >> 2), "v"(trl), "s"(pk) : "memory");
        }
        if (raw) {
            const GAS char* d0 = dbuf + A.doff;
            const GAS char* d1 = d0 + (A.kind == K_W ? 4 * (unsigned)STRIP_BYTES : 4096u);
            const unsigned vd = A.kind == K_W ? vblw : vblk;
            st_pk<0>(v[0], vd, d0); st_pk<512>(v[1], vd, d0); st_pk<1024>(v[2], vd, d0); st_pk<1536>(v[3], vd, d0);
            st_pk<0>(v[4], vd, d1); st_pk<512>(v[5], vd, d1); st_pk<1024>(v[6], vd, d1); st_pk<1536>(v[7], vd, d1);
        }
        AGK_FETCH(C, slot);
        if (raw) { wait_vm<8>(); flag_st(dbuf + A.foff, callv, zero); }
        PSTAMP(j, pA, 0, tstart + A.when); PSTAMP(j, pA, 1, tb); PSTAMP(j, pA, 2, now());
        pA += WPL; A = B; B = C; slot ^= 1;
        if (pA >= npiece) break;
        wait_vm<15>();
    }
    wait_vm<0>();
    STAMP(j < 4 ? 1 : 3);
    if (j == 1 && u == 0) {
        if (lane < 4) {
            const gu32* c = (const gu32*)(myws + OCNT_OFF + lane * 256);
            const unsigned need = call * P.out_adds;
            const unsigned long long t0 = now();
            bool bad = false;
            unsigned spin = 0;
            while (ld_sys(c) < need) {
                if ((++spin & 63) == 0 && (ld_agent(tmo) == call || now() - t0 > DEADLINE_TICKS)) { st_agent(tmo, call); bad = true; break; }
                __builtin_amdgcn_s_sleep(1);
            }
            if (bad) {
                __builtin_amdgcn_s_sleep(127);
                __hip_atomic_store((GAS unsigned short*)(P.out[0] + (size_t)(P.rank * MSH) * (NSH * 2)), (unsigned short)0x7FC0, __ATOMIC_RELAXED, __HIP_MEMORY_SCOPE_AGENT);
            }
        }
        STAMP(2);
    }
    if (lane == 0) st_agent(cc, call);
}

__device__ __forceinline__ void unpack_role(const Params& P, GAS char* myws, unsigned tstart, unsigned call, int w, int nw) {
    const GAS char* buf = myws + (size_t)(call & 1) * BUF_BYTES;
    gu32* tmo = (gu32*)(myws + TMO_OFF);
    const int lane = threadIdx.x & 63;
    const unsigned vlin = lane * 16, tagw = tag_of(call);
    const unsigned vblk = (lane >> 5) * 2048 + (lane & 31) * 16, vblw = (lane >> 5) * (unsigned)STRIP_BYTES + (lane & 31) * 16;
    unsigned zero = 0, callv = call;
    asm volatile("" : "+v"(zero), "+v"(callv));
    const unsigned tfirst = tstart + (P.tpoll & 0xFFFFu);
    const bool spec = P.tpoll >> 16;
    for (int q = w; q < NPMAX * 7; q += nw) {
        const int p = q / 7, j = 1 + q - p * 7;
        if (p >= (j < 4 ? NP_IN : NP_XI)) continue;
        const Piece A = locate(P.ord[ord_at(j >= 4, p)], j);
        if (A.raw) continue;
        const GAS char* pk = buf + pk_off(j, p);
        const GAS char* d0 = buf + A.doff;
        const GAS char* d1 = d0 + (A.kind == K_W ? 4 * (unsigned)STRIP_BYTES : 4096u);
        const unsigned vd = A.kind == K_W ? vblw : vblk;
        sleep_until(tfirst + A.when);
        unsigned long long t0 = 0;
        unsigned spin = 0, trl, fmt = 0;
        u32x4 W[5]; u32x2 W5; u32x3 ex;
        bool data = spec;
        for (;;) {
            asm volatile("s_nop 4\n\tglobal_load_dword %0, %1, %2 offset:2560 sc0 sc1" : "=v"(trl) : "v"(vlin >> 2), "s"(pk) : "memory");
            if (data) {
                asm volatile("global_load_dwordx3 %0, %1, %2 offset:2816 sc0 sc1" : "=v"(ex) : "v"(vlin & (MAXEXC * 32 - 1)), "s"(pk) : "memory");
                asm volatile("global_load_dwordx2 %0, %1, %2 offset:2048 sc0 sc1" : "=v"(W5) : "v"(vlin >> 1), "s"(pk) : "memory");
                ld_c<1024>(W[4], vlin, pk);
                ld_c<-3072>(W[0], vlin, pk); ld_c<-2048>(W[1], vlin, pk); ld_c<-1024>(W[2], vlin, pk); ld_c<0>(W[3], vlin, pk);
            }
            asm volatile("s_waitcnt vmcnt(0)" : "+v"(trl), "+v"(W[0]), "+v"(W[1]), "+v"(W[2]), "+v"(W[3]), "+v"(W[4]), "+v"(W5), "+v"(ex) :: "memory");
            const unsigned long long fc = __ballot((int)(trl << 1) < 0);
            if (__ballot((int)(trl ^ tagw) < 0) == 0 && (unsigned)(fc >> 32) == call) {
                fmt = (unsigned)fc;
                if (fmt & F_RAW) break;
                const unsigned used = ((1u << ((fmt >> 8) & 31)) - 1u) | (unsigned)(((1ull << (fmt >> 16)) - 1ull) << MAXEXC);
                if (data && __ballot((int)((W[0].w ^ tagw) | (W[1].w ^ tagw) | (W[2].w ^ tagw) | (W[3].w ^ tagw) | (W[4].w ^ tagw) | (W5.y ^ tagw)) < 0) == 0 &&
                    ((unsigned)__ballot(ex.z != call) & used) == 0) break;
                data = true;
            } else __builtin_amdgcn_s_sleep(AGK_SL2);
            if ((++spin & 63) == 0) {
                if (t0 == 0) t0 = now();
                if (rfl(ld_agent(tmo)) == call || now() - t0 > DEADLINE_TICKS) { st_agent(tmo, call); return; }
            }
        }
        PSTAMP(j, p, 3, now()); PSTAMP(j, p, 7, spin); PSTAMP(j, p, 4, now());
        if (fmt & F_RAW) continue;
        const int cnt = (fmt >> 8) & 31;
#pragma unroll
        for (int k = 0; k < 5; ++k) W[k].w = bfi(0x7FFFFFFFu, W[k].w, trl << (2 + k));
        W5.y = bfi(0x7FFFFFFFu, W5.y, trl << 7);
        const unsigned long long m = 1ull << (trl & 63) | 1ull << ((trl >> 6) & 63) | 1ull << ((trl >> 12) & 63) | 1ull << ((trl >> 18) & 63);
        unsigned m0 = (unsigned)m, m1 = (unsigned)(m >> 32) & 0x7FFFFFFFu;
        if (fmt >> 16) {
            const int src = ((trl >> 4) & 0x3C) + 4 * MAXEXC;
            const unsigned e0 = __builtin_amdgcn_ds_bpermute(src, ex.x), e1 = __builtin_amdgcn_ds_bpermute(src, ex.y);
            const bool over = (trl & 0x83Fu) == 63u;
            m0 = over ? e0 : m0; m1 = over ? e1 : m1;
        }
        unsigned N[8];
        N[0] = bfi(0x44444444u, ~(m0 << 2), W[4].x); N[1] = bfi(0x44444444u, ~(m0 << 1), W[4].y); N[2] = bfi(0x44444444u, ~m0, W[4].z); N[3] = bfi(0x44444444u, ~(m0 >> 1), W[4].w);
        N[4] = bfi(0x44444444u, ~(m1 << 2), W5.x); N[5] = bfi(0x44444444u, ~(m1 << 1), W5.y);
        N[6] = bfi(0x33333333u, bfi(0x11111111u, W[4].x >> 2, W[4].y >> 1), bfi(0x44444444u, ~m1, W[4].z << 1));
        N[7] = bfi(0x33333333u, bfi(0x11111111u, W[4].w >> 2, W5.x >> 1), bfi(0x44444444u, ~(m1 >> 1), W5.y << 1));
        const unsigned B4 = (fmt & 127u) * 0x01010101u;
        unsigned H[16];
#pragma unroll
        for (int k = 0; k < 8; ++k) {
            H[2 * k] = (N[k] & 0x87878787u) + B4;
            H[2 * k + 1] = (rotr(N[k], 4) & 0x87878787u) + B4;
        }
        if (cnt) {
            unsigned tag = 255, val = 0;
            for (int e = 0; e < cnt; ++e) {
                const unsigned id = __builtin_amdgcn_readlane(ex.x, e), hv = __builtin_amdgcn_readlane(ex.y, e);
                const bool mine = (id & 63) == (unsigned)lane;
                tag = mine ? id >> 8 : tag; val = mine ? hv : val;
            }
#pragma unroll
            for (int i = 0; i < 16; ++i) H[i] = tag == (unsigned)i ? val : H[i];
        }
#define AGK_DST(k) { const unsigned l0 = W[k >> 1][(k & 1) * 2], l1 = W[k >> 1][(k & 1) * 2 + 1]; u32x4 o; \
            o.x = perm(H[2 * k], l0, 0x05010400u); o.y = perm(H[2 * k], l0, 0x07030602u); \
            o.z = perm(H[2 * k + 1], l1, 0x05010400u); o.w = perm(H[2 * k + 1], l1, 0x07030602u); st_pk<(k & 3) * 512>(o, vd, k < 4 ? d0 : d1); }
        AGK_DST(0) AGK_DST(1) AGK_DST(2) AGK_DST(3) AGK_DST(4) AGK_DST(5) AGK_DST(6) AGK_DST(7)
        PSTAMP(j, p, 5, now());
        wait_vm<0>(); flag_st(buf + A.foff, callv, zero);
        PSTAMP(j, p, 6, now());
    }
}

#define AGK_RPG 2
#define AGK_RA 4, 6
#define AGK_RH 4, 12
#define AGK_RQ 8, 12
#define AGK_RW 6, 7
#define AGK_LOOK 3
template <int NB, int NP, int NSLOT, int SLOT_BYTES>
__device__ __forceinline__ void loader(GAS char* myws, unsigned call, const GAS char* const (&src)[NP], const int (&shift)[NP], const unsigned (&v0)[NB],
                                       const unsigned (&v1)[NB], const GAS char* fp, bool have, unsigned ldsw, unsigned rot, bool first) {
    constexpr int NI = 2 * NB;
    static_assert((NSLOT - 2) * NI + 8 < 64 && (NSLOT - 1) * NI + 10 < 64, "vmcnt range");
    gu32* tmo = (gu32*)(myws + TMO_OFF);
    const unsigned ldsend = ldsw + NSLOT * SLOT_BYTES;
    const unsigned long long act = ~__ballot(have);
    asm volatile("v_mov_b32 v250, %0" :: "v"(have ? ~0u : 0u) : "v250", "memory");
    int mark = 0;
    unsigned npoll = 0;
    auto poll = [&]() {
        ++npoll;
        asm volatile("s_mov_b64 exec, %1\n\tglobal_load_dword v250, %0, off sc0 sc1\n\ts_mov_b64 exec, -1" :: "v"(fp), "s"(act) : "v250", "memory");
    };
    auto eval = [&]() {
        unsigned long long bx;
        asm volatile("v_cmp_gt_u32 %0, %1, v250" : "=s"(bx) : "s"(call) : "memory");
        const unsigned m = (unsigned)bx | (unsigned)(bx >> 32);
        mark = __builtin_ctz(m | m >> 16 | 0x10000u) * 4;
    };
    unsigned ldsi = ldsw, kt = 0;
    const GAS char* ptr[NP];
#pragma unroll
    for (int i = 0; i < NP; ++i) ptr[i] = src[i] + (rot << shift[i]);
    auto issue = [&]() {
        asm volatile("s_mov_b32 m0, %0\n\ts_nop 0" :: "s"(ldsi) : "m0", "memory");
        glds2o<-4096>(ptr[0], v0[0], v1[0]);
        glds2o<-2048>(ptr[1 % NP], v0[1], v1[1]);
        if (NB == 4) { glds2o<0>(ptr[2 % NP], v0[2 % NB], v1[2 % NB]); glds2o<2048>(ptr[3 % NP], v0[3 % NB], v1[3 % NB]); }
        const unsigned k = ++kt ^ rot;
#pragma unroll
        for (int i = 0; i < NP; ++i) { ptr[i] = src[i] + (k << shift[i]); asm volatile("" : "+s"(ptr[i])); }
        ldsi = ldsi + SLOT_BYTES == ldsend ? ldsw : ldsi + SLOT_BYTES;
        asm volatile("" : "+s"(ldsi));
    };
    int ti = 0, tb = 0;
    unsigned hist = 0;
#pragma nounroll
    for (;;) {
        if (mark < NSTEP && mark - ti <= AGK_LOOK) eval();
        const int lim = mark < tb + NSLOT - 1 ? mark : tb + NSLOT - 1;
#pragma nounroll
        for (; ti < lim; ++ti) { issue(); hist = hist << 4 | (npoll & 15u); }
        if (ti == tb) {
            unsigned long long t0 = 0;
            for (unsigned spin = 1; ti >= mark; ++spin) {
                wait_vm<8>();
                poll();
                __builtin_amdgcn_s_sleep(AGK_SL1);
                if ((spin & 63) == 0) {
                    if (t0 == 0) t0 = now();
                    if (rfl(ld_agent(tmo)) == call || now() - t0 > DEADLINE_TICKS) break;
                }
                eval();
            }
            if (ti >= mark) break;
            continue;
        }
        const int run = (mark < NSTEP ? mark - AGK_LOOK : NSTEP) - ti;
        if (run > 0) {
#pragma nounroll
            for (int i = run; i > 0; --i) { wait_vm<(NSLOT - 2) * NI>(); bar(); issue(); }
            ti += run; tb += run;
            const unsigned rep = (npoll & 15u) * 0x11111111u;
            hist = run >= 8 ? rep : hist << (4 * run) | (rep & ((1u << (4 * run)) - 1u));
        }
        if (mark < NSTEP) poll();
        const unsigned n = ti - tb - 1;
        wait_vm_var(n * NI + (n < 8 ? (npoll - (hist >> (4 * n))) & 15u : 0u));
        bar();
        if (++tb == NSTEP) break;
    }
    if (tb < NSTEP) {
        st_agent(tmo, call);
        const u32x4 nan = {0x7FC07FC0u, 0x7FC07FC0u, 0x7FC07FC0u, 0x7FC07FC0u};
        for (; tb < NSTEP; ++tb) {
#pragma unroll
            for (int i = 0; i < NI; ++i) *(LAS u32x4*)(size_t)(ldsi - 4096 + i * 1024 + (threadIdx.x & 63) * 16) = nan;
            wait_lgkm();
            bar();
            ldsi = ldsi + SLOT_BYTES == ldsend ? ldsw : ldsi + SLOT_BYTES;
        }
    }
    if (first) STAMP(4);
    wait_vm<0>();
}

template <int S, int NH, int NSA, int NSB>
__device__ __forceinline__ void tile_role(const Params& P, LAS char* smem, unsigned call, int first_strip, int cg, int ch, int win, unsigned rot) {
    constexpr int SA = S * 2048, SB = NH * 8192, BOFF = NSA * SA;
    constexpr int SPW = S / 2, NFR = NH * 2;
    static_assert(BOFF + NSB * SB <= LDS_BYTES, "LDS rings");
    GAS char* myws = (GAS char*)P.ws[0];
    const int lane = threadIdx.x & 63;
    const int wv = rfl(threadIdx.x >> 6);
    const bool wremote = cg >= 4;

    if (wv >= 4) {
        const int lw = wv & 1;
        const GAS char* mybuf = myws + (size_t)(call & 1) * BUF_BYTES;
        const unsigned vlin = lane * 16;
        if (wv < 6) {
            const GAS char* src[SPW];
            unsigned v0[SPW], v1[SPW];
            int shift[SPW];
#pragma unroll
            for (int i = 0; i < SPW; ++i) {
                const int code = rfl(P.strip[first_strip + lw * SPW + i]);
                const bool own = code < 32;
                v0[i] = own ? ximg0(lane, KD * 2) : vlin;
                v1[i] = own ? ximg1(lane, KD * 2) : vlin;
                src[i] = (own ? (const GAS char*)P.x + (unsigned)code * (16 * KD * 2) : mybuf + (unsigned)xslot(code >> 5, code & 31) * (unsigned)STRIP_BYTES) + (4096 - 2048 * i);
                shift[i] = own ? 7 : 11;
            }
            const int fcode = P.strip[first_strip + lw * SPW + ((lane >> 4) & (SPW - 1))];
            loader<SPW, SPW, NSA, SA>(myws, call, src, shift, v0, v1, mybuf + XFLAG_OFF + fcode * XF_STRIDE + (lane & 15) * 4, fcode < 32 || lane >= 16 * SPW,
                                      (unsigned)(size_t)smem + lw * (SA / 2) + 4096, rot, lw == 0);
        } else {
            const unsigned dk = (wremote ? (unsigned)STRIP_BYTES : 16u * NSH * 2) - 2048, dh = (wremote ? 4 * (unsigned)STRIP_BYTES : 128u) - 4096;
            const GAS char* const src[1] = {(wremote ? mybuf + XS_BYTES + (unsigned)((cg - 4) * 8 + ch * 4 + lw * 2) * (unsigned)STRIP_BYTES
                                                     : (const GAS char*)P.w + cg * 256 + ch * 128 + lw * 32 * (NSH * 2))};
            const int shift[1] = {wremote ? 11 : 16};
            unsigned v0[NFR], v1[NFR];
#pragma unroll
            for (int i = 0; i < NFR; ++i) {
                v0[i] = (wremote ? vlin : wimg0(lane, NSH * 2)) + 4096 + (i & 1) * dk + (i >> 1) * dh;
                v1[i] = (wremote ? vlin : wimg1(lane, NSH * 2)) + 4096 + (i & 1) * dk + (i >> 1) * dh;
            }
            loader<NFR, 1, NSB, SB>(myws, call, src, shift, v0, v1, mybuf + WFLAG_OFF + ((cg & 3) * NWP + lw * 64 + lane) * 4, !wremote,
                                    (unsigned)(size_t)smem + BOFF + lw * (SB / 2) + 4096, rot, false);
        }
    } else {
        const int fr = lane & 15, g = lane >> 4;
        const int wm = wv >> 1, wn = wv & 1;
        unsigned aoff[2];
#pragma unroll
        for (int kk = 0; kk < 2; ++kk) aoff[kk] = wm * (SA / 2) + fr * 128 + (((kk * 4 + g) ^ ((fr >> 1) & 7)) << 4);
        unsigned boff[NFR];
        {
            const int q = fr >> 2, pp = lane & 3, f = (q >> 1) | ((g & 1) << 1);
#pragma unroll
            for (int nb = 0; nb < NFR; ++nb)
                boff[nb] = BOFF + (NH == 2 ? wn * 4096 : 0) + (g >> 1) * 2048 + ((g & 1) * 8 + q) * 128 + ((((wn * NFR + nb) & 3) ^ f) << 5) + pp * 8;
        }
        int rowbase[SPW];
#pragma unroll
        for (int i = 0; i < SPW; ++i) {
            const int code = P.strip[first_strip + wm * SPW + i];
            rowbase[i] = rfl((P.rank ^ (code >> 5)) * MSH + (code & 31) * 16);
        }
        GAS char* outp = (GAS char*)P.out[wremote ? cg : 0] + (cg & 3) * 256;
        GAS char* cntp = (GAS char*)P.ws[cg] + OCNT_OFF + (cg & 3) * 256;
        constexpr int orow = NSH * 2;
        asm volatile("" : "+s"(outp), "+s"(cntp));
        f32x4 acc[SPW][NFR];
#pragma unroll
        for (int i = 0; i < SPW; ++i)
#pragma unroll
            for (int nb = 0; nb < NFR; ++nb) acc[i][nb] = (f32x4){0.f, 0.f, 0.f, 0.f};
        struct Frag { bf16x8 a[SPW]; bf16x8 b[NFR]; };
        auto rd = [&](Frag& F, int sa, int sb, int kk) {
#pragma unroll
            for (int i = 0; i < SPW; ++i) F.a[i] = *(LAS bf16x8*)(smem + sa + aoff[kk] + i * 2048);
#pragma unroll
            for (int nb = 0; nb < NFR; ++nb) {
                const s16x4 lo = __builtin_amdgcn_ds_read_tr16_b64_v4i16((LAS s16x4*)(smem + sb + boff[nb] + kk * (SB / 2)));
                const s16x4 hi = __builtin_amdgcn_ds_read_tr16_b64_v4i16((LAS s16x4*)(smem + sb + boff[nb] + kk * (SB / 2) + 512));
                F.b[nb] = __builtin_bit_cast(bf16x8, __builtin_shufflevector(lo, hi, 0, 1, 2, 3, 4, 5, 6, 7));
            }
        };
        auto mm = [&](const Frag& F) {
#pragma unroll
            for (int nb = 0; nb < NFR; ++nb)
#pragma unroll
                for (int i = 0; i < SPW; ++i) acc[i][nb] = __builtin_amdgcn_mfma_f32_16x16x32_bf16(F.b[nb], F.a[i], acc[i][nb], 0, 0, 0);
        };
        Frag F0, F1;
        int ca = 0, cb = 0;
        bar();
        rd(F0, 0, 0, 0);
        wait_lgkm();
#pragma nounroll
        for (int t = 0; t < NSTEP - 1; ++t) {
            constexpr int NRD = SPW + 2 * NFR, NMM = SPW * NFR, RPG = S * NH == 16 ? 1 : AGK_RPG;
            rd(F1, ca, cb, 1);
            mm(F0);
#pragma unroll
            for (int i = 0; i < NMM; ++i) { __builtin_amdgcn_sched_group_barrier(0x008, 1, 0); if (i * RPG < NRD) __builtin_amdgcn_sched_group_barrier(0x100, RPG, 0); }
            PIN();
            wait_lgkm();
            bar();
            ca = ca + SA == NSA * SA ? 0 : ca + SA;
            cb = cb + SB == NSB * SB ? 0 : cb + SB;
            rd(F0, ca, cb, 0);
            mm(F1);
#pragma unroll
            for (int i = 0; i < NMM; ++i) { __builtin_amdgcn_sched_group_barrier(0x008, 1, 1); if (i * RPG < NRD) __builtin_amdgcn_sched_group_barrier(0x100, RPG, 1); }
            PIN();
            wait_lgkm();
        }
        rd(F1, ca, cb, 1);
        PIN();
        mm(F0);
        PIN();
        mm(F1);
        if (wv == 0) STAMP(2);
        constexpr int ROWB = NFR * 32;
        constexpr int CPR = ROWB / 16;
        constexpr int ESLOT = (NSTEP - 1) % NSB >= 2 ? 0 : 2;
        LAS char* eb = smem + BOFF + ESLOT * SB + wv * (SPW * 16 * ROWB);
        static_assert(4 * SPW * 16 * ROWB <= 2 * SB && NSB >= 4, "epilogue space");
#pragma unroll
        for (int i = 0; i < SPW; ++i)
#pragma unroll
            for (int nb = 0; nb < NFR; ++nb) {
                u32x2 o;
                o.x = pack_bf16(acc[i][nb][0], acc[i][nb][1]);
                o.y = pack_bf16(acc[i][nb][2], acc[i][nb][3]);
                *(LAS u32x2*)(eb + (i * 16 + fr) * ROWB + (((nb * 2 + (g >> 1)) ^ (fr & (CPR - 1))) << 4) + (g & 1) * 8) = o;
            }
        constexpr int RPI = 64 / CPR;
        constexpr int NST = SPW * 16 / RPI;
        u32x4 ov[NST];
        GAS char* od[NST];
        GAS char* obase = outp + (NH == 2 ? wn * 128 : ch * 128 + wn * 64) + (lane & (CPR - 1)) * 16;
#pragma unroll
        for (int n = 0; n < NST; ++n) {
            const int row = n * RPI + lane / CPR;
            ov[n] = *(LAS u32x4*)(eb + row * ROWB + (((lane & (CPR - 1)) ^ (row & (CPR - 1))) << 4));
            od[n] = obase + (size_t)((rowbase[(n * RPI) / 16] + (row & 15)) * orow);
        }
        if (wv == 0) STAMP(5);
        if (win == 1) {
#pragma unroll
            for (int n = 0; n < NST; ++n) { if (n >= 1) wait_vm<0>(); st16_wt(od[n], ov[n]); }
        } else if (win == 2) {
#pragma unroll
            for (int n = 0; n < NST; ++n) { if (n >= 2) wait_vm<1>(); st16_wt(od[n], ov[n]); }
        } else {
#pragma unroll
            for (int n = 0; n < NST; ++n) st16_wt(od[n], ov[n]);
        }
        if (wremote) {
            wait_vm<0>();
            if (lane == 0) __hip_atomic_fetch_add((gu32*)cntp, 1u, __ATOMIC_RELAXED, __HIP_MEMORY_SCOPE_SYSTEM);
        }
        if (wv == 0) STAMP(3);
    }
}

__global__ void __launch_bounds__(512, 1) __attribute__((amdgpu_num_vgpr(240))) ag_gemm_kernel(const Params P) {
    __shared__ __attribute__((aligned(1024))) char smem_raw[LDS_BYTES];
    LAS char* smem = (LAS char*)smem_raw;
    const unsigned long long t64 = now();
    const unsigned tstart = (unsigned)t64;
    const int b = blockIdx.x;
    if (__builtin_expect(b < NCOMM, 1)) {
        const int wv = rfl(threadIdx.x >> 6), g = wv * NCOMM + b;
        if (wv < NCW) comm_role(P, smem + wv * 16384, t64, wv, 1 + g % 7, g / 7);
        return;
    }
    GAS char* myws = (GAS char*)P.ws[0];
    gu32* cc = (gu32*)(myws + CALL_OFF + b * 64);
    const unsigned role = rfl(P.role[b - NCOMM]);
    const int type = role & 15;
    if (type == R_UNPACK) {
        const int wv = rfl(threadIdx.x >> 6);
        const unsigned call = rfl(ld_agent(cc + wv)) + 1u;
        unpack_role(P, myws, tstart, call, (int)(role >> 4) + (int)P.nunpack * wv, (int)P.nunpack * 8);
        if (wv == 7) STAMP(4);
        if ((threadIdx.x & 63) == 0) st_agent(cc + wv, call);
    } else if (type != R_IDLE) {
        const unsigned call = rfl(ld_agent(cc)) + 1u;
        sleep_until(tstart + (P.tlead >> 16));
        if (threadIdx.x == 0) STAMP(0);
        const int fs = (role >> 4) & 255, cg = (role >> 12) & 7, ch = (role >> 15) & 1, win = (role >> 18) & 3;
        if (type == R_T128) tile_role<8, 2, AGK_RA>(P, smem, call, fs, cg, 0, win, role >> 28);
        else if (type == R_T128H) tile_role<8, 1, AGK_RH>(P, smem, call, fs, cg, ch, win, role >> 28);
        else if (type == R_T64W) tile_role<4, 2, AGK_RW>(P, smem, call, fs, cg, 0, win, role >> 28);
        else tile_role<4, 1, AGK_RQ>(P, smem, call, fs, cg, ch, win, role >> 28);
        if (threadIdx.x == 256) st_agent(cc, call);
    }
}

}

#define TUNE(i, dflt) (dflt)

extern "C" void kernel_launch(int rank, int nranks, ncclComm_t comm,
                              hipStream_t stream,
                              void* const* d_in, const int* in_sizes, int n_in,
                              void* d_out, int out_size,
                              const int* in_dtypes, int out_dtype,
                              void* const* d_out_all,
                              void* const* d_ws_all, size_t ws_bytes) {
    using namespace agk;
    (void)comm; (void)in_dtypes; (void)out_dtype; (void)d_out;
    if (nranks != NR || (unsigned)rank >= (unsigned)NR || n_in < 2 || ws_bytes < WS_NEED ||
        in_sizes[0] != MSH * KD || in_sizes[1] != KD * NSH || out_size != NR * MSH * NSH) {
        fprintf(stderr, "ag_gemm: unexpected problem shape\n");
        return;
    }
    Params P;
    P.x = (const char*)d_in[0];
    P.w = (const char*)d_in[1];
    P.rank = rank;
    for (int j = 0; j < NR; ++j) { P.ws[j] = (char*)d_ws_all[rank ^ j]; P.out[j] = (char*)d_out_all[rank ^ j]; }
    int sega = TUNE(0, 11);
    int blate = TUNE(1, 9);
    int wlead = TUNE(2, 4);
    int rt_b = TUNE(3, 1), rt_c = TUNE(4, 1), rt_a = TUNE(5, 0), rt_w = TUNE(6, 0);
    int split_c = TUNE(7, 6);
    int split_a = TUNE(8, 0);
    int small_b = TUNE(9, 0);
    const int segb = HS - sega;
    const int na_t = (3 * sega + 7) / 8, nc_t = (4 * AS + 7) / 8, nb_t = (3 * segb + 7) / 8, no_t = 16 - na_t - nc_t - nb_t;
    const int ne_t = no_t + na_t;
    bool ok = no_t >= 0;
    int n = 0, own = 0;
    for (int i = 0; i < no_t * 8; ++i) P.strip[n++] = (unsigned char)(own++);
    for (int s = 0; s < sega; ++s) for (int j = 1; j < 4; ++j) P.strip[n++] = (unsigned char)((j << 5) | s);
    while (n % 8) P.strip[n++] = (unsigned char)(own++);
    for (int s = HS; s < NSTRIP; ++s) for (int j = 4; j < 8; ++j) P.strip[n++] = (unsigned char)((j << 5) | s);
    while (n % 8) P.strip[n++] = (unsigned char)(own++);
    for (int s = sega; s < HS; ++s) for (int j = 1; j < 4; ++j) P.strip[n++] = (unsigned char)((j << 5) | s);
    while (n % 8) P.strip[n++] = (unsigned char)(own++);
    ok = ok && n == 128 && own == HS;
    int q = 0;
    const int t_first = TUNE(16, 160) * 16, t_pk = TUNE(17, 168), t_raw = TUNE(18, 232);
    const int rh_a = TUNE(19, 0), rh_w = TUNE(20, 0), afirst = TUNE(21, 0);
    const int n_slow = TUNE(25, 0), t_slow = TUNE(15, 184);
    int tq = t_first;
    for (int i = 0; i < 2 * WPL * NRND; ++i) P.ord[i] = 0;
    auto emit = [&](unsigned e) { if (q < NP_IN + NP_XI) P.ord[q < NP_IN ? ord_at(0, q) : ord_at(1, q - NP_IN)] = e | (unsigned)(tq >> 4) << 16; ++q; tq += (e & 512u) ? t_raw : tq < t_first + n_slow * t_slow ? t_slow : t_pk; };
    auto xchunk = [&](int s0, int s1, int kc, int rt, int rh) { for (int s = s0; s < s1; ++s) emit((unsigned)kc | (s << 4) | (kc >= NKC - rt || kc < rh ? 512u : 0u)); };
    auto wchunk = [&](int kc) { for (int i = 0; i < 8; ++i) emit((unsigned)(kc * 8 + i) | 1024u | (kc >= NKC - rt_w || kc < rh_w ? 512u : 0u)); };
    for (int ka = 0, kb = 0; kb < NKC; ) {
        if (ka < NKC && kb * (NKC - afirst) >= (ka - afirst) * (NKC - blate)) xchunk(0, sega, ka++, rt_a, rh_a);
        else xchunk(sega, HS, kb++, rt_b, 0);
    }
    ok = ok && q == NP_IN;
    tq = t_first;
    for (int k = -wlead; k < NKC; ++k) {
        if (k + wlead < NKC) wchunk(k + wlead);
        if (k >= 0) xchunk(HS, NSTRIP, k, rt_c, 0);
    }
    ok = ok && q == NP_IN + NP_XI;
    int fill[8] = {0, 0, 0, 0, 0, 0, 0, 0};
    unsigned role[NWG];
    for (int i = 0; i < NWG; ++i) role[i] = R_IDLE;
    auto put = [&](int k, unsigned v) { if (fill[k] < NWG / 8) role[fill[k]++ * 8 + k] = v; else ok = false; };
    auto code = [&](int type, int fs, int cg, int ch) { return (unsigned)type | (fs << 4) | (cg << 12) | (ch << 15); };
    for (int i = 0; i < NCOMM; ++i) put(i & 7, R_COMM | (i << 4));
    unsigned adds = 0;
    for (int cg = 0; cg < 8; ++cg) {
        const bool rem = cg >= 4;
        for (int r = 0; r < 16; ++r) {
            const int k = (cg & 3) + 4 * (r & 1);
            const bool early = r < ne_t, last = r >= ne_t + nc_t;
            const bool split = last ? rem || r - ne_t - nc_t >= TUNE(29, 4) : (rem ? (early ? r >= ne_t - split_a : r - ne_t < split_c) : (!early && r - ne_t < TUNE(22, 0)));
            const bool small = last && r - ne_t - nc_t < (rem ? small_b >> 4 : small_b & 15);
            const unsigned win = (unsigned)(!rem ? 0 : early ? TUNE(14, 2) : last ? TUNE(12, 0) : split ? TUNE(13, 2) : TUNE(11, 2)) << 18;
            int nw = 0;
            auto rot = [&](int who, int of) { return (unsigned)(TUNE(26, 1) ? (of == 2 ? (who & 1) * 2 + ((r >> 1) & 1) : (who ^ (((r >> 1) & 1) * 2)) & 3) : 0) << 28; };
            if (small) for (int h = 0; h < 4; ++h) { put(k, rot(rem * 2 + (h & 1), 4) | win | code(R_T64H, r * 8 + (h >> 1) * 4, cg, h & 1)); nw += 4; }
            else if (split && (TUNE(23, 3) >> (last ? 0 : 1) & 1)) for (int h = 0; h < 2; ++h) { put(k, (unsigned)((rem * 2 + h * TUNE(24, 1)) ^ ((r >> 1) & 1)) << 28 | win | code(R_T64W, r * 8 + h * 4, cg, 0)); nw += 4; }
            else if (split) for (int ch = 0; ch < 2; ++ch) { put(k, rot(rem * 2 + ch, 4) | win | code(R_T128H, r * 8, cg, ch)); nw += 4; }
            else { put(k, rot(rem, 2) | win | code(R_T128, r * 8, cg, 0)); nw += 4; }
            if (cg == 4) adds += nw;
        }
    }
    unsigned nun = 0;
    for (int i = 0; i < NWG; ++i) if (role[i] == R_IDLE && nun < (unsigned)TUNE(10, 64)) role[i] = R_UNPACK | (nun++ << 4);
    for (int i = NCOMM; i < NWG; ++i) P.role[i - NCOMM] = role[i];
    P.tlead = TUNE(27, 24) | TUNE(28, 250) << 16;
    P.out_adds = adds; P.tpoll = TUNE(30, 100) | TUNE(31, 1) << 16; P.nunpack = nun;
    ok = ok && nun > 0;
    if (!ok) { fprintf(stderr, "ag_gemm: role table overflow\n"); return; }
    hipLaunchKernelGGL(ag_gemm_kernel, dim3(NWG), dim3(512), 0, stream, P);
    HIP_CHECK(hipGetLastError());
}
```

```cpp
#include <hip/hip_runtime.h>
#include <rccl/rccl.h>
#include <cstdio>
#include <cstdint>

#define HIP_CHECK(expr) do { hipError_t _e = (expr); if (_e != hipSuccess) { \
    fprintf(stderr, "HIP error %s at %s:%d\n", hipGetErrorString(_e), __FILE__, __LINE__); \
    return; } } while (0)

namespace agk {

typedef unsigned int u32x4 __attribute__((ext_vector_type(4)));
typedef unsigned int u32x3 __attribute__((ext_vector_type(3)));
typedef unsigned int u32x2 __attribute__((ext_vector_type(2)));
typedef short s16x4 __attribute__((ext_vector_type(4)));
typedef short s16x8 __attribute__((ext_vector_type(8)));
typedef __bf16 bf16x8 __attribute__((ext_vector_type(8)));
typedef float f32x4 __attribute__((ext_vector_type(4)));
#define LAS __attribute__((address_space(3)))
#define GAS __attribute__((address_space(1)))
typedef GAS unsigned int gu32;

constexpr int NR = 8;
constexpr int KD = 4096;
constexpr int MSH = 512;
constexpr int NSH = 512;
constexpr int NSTRIP = 32;
#define AGK_HS 21
constexpr int HS = AGK_HS;
constexpr int AS = NSTRIP - HS;
constexpr int NKC = 16;
constexpr int NSTEP = 64;
constexpr int NWP = 128;

#define AGK_NCOMM 56
#define AGK_NCW 4
constexpr int NCOMM = AGK_NCOMM;
constexpr int NCW = AGK_NCW;
constexpr int WPL = NCOMM * NCW / 7;
static_assert(NCOMM % 7 == 0, "a comm workgroup serves one link");

constexpr int NP_IN = HS * NKC;
constexpr int NP_XI = NWP + AS * NKC;
constexpr int NPMAX = NP_IN > NP_XI ? NP_IN : NP_XI;
constexpr int NRND = (NPMAX + WPL - 1) / WPL;
constexpr int NWG = 256;
enum { R_IDLE = 0, R_COMM = 1, R_T128 = 2, R_T128H = 3, R_T64H = 4, R_UNPACK = 5, R_T64W = 6 };

constexpr size_t STRIP_BYTES = 16ull * KD * 2;
constexpr int NXSLOT = 3 * HS + 4 * AS;
constexpr size_t XS_BYTES = (size_t)NXSLOT * STRIP_BYTES;
constexpr size_t WSL_BYTES = 8 * STRIP_BYTES;
constexpr int XF_STRIDE = 256;
constexpr size_t XFLAG_OFF = XS_BYTES + 4 * WSL_BYTES;
constexpr size_t WFLAG_OFF = XFLAG_OFF + 8 * 32 * XF_STRIDE;
constexpr size_t PK_OFF = WFLAG_OFF + 4 * NWP * 4;
constexpr unsigned PK_BYTES = 6016;
constexpr int NP_OUT = 56;
constexpr size_t BUF_BYTES = (PK_OFF + (size_t)(3 * NP_IN + 4 * NP_XI + 4 * NP_OUT) * PK_BYTES + 4095) / 4096 * 4096;
constexpr size_t OCNT_OFF = 2 * BUF_BYTES;
constexpr size_t TMO_OFF = OCNT_OFF + 4 * 256;
constexpr size_t CALL_OFF = TMO_OFF + 256;
constexpr size_t DBG_OFF = CALL_OFF + 256 * 64;
constexpr size_t DBG2_OFF = DBG_OFF + 256 * 128;
constexpr size_t WS_NEED = DBG2_OFF + 8 * 512 * 32;
static_assert(BUF_BYTES % 4096 == 0, "alignment");
static_assert(WS_NEED <= 65ull << 20, "workspace");

constexpr unsigned long long DEADLINE_TICKS = 30000000ull;
constexpr int LDS_BYTES = 163840;

struct Params {
    const char* x;
    const char* w;
    char* ws[NR];
    char* out[NR];
    int rank;
    unsigned out_adds;
    unsigned tpoll;
    unsigned nunpack;
    unsigned role[NWG - NCOMM];
    unsigned tlead;
    unsigned ord[2 * WPL * NRND];
    unsigned char strip[128];
};

static_assert(sizeof(Params) <= 4096, "kernel arguments");

__device__ __forceinline__ unsigned ld_sys(const gu32* p) { return __hip_atomic_load(p, __ATOMIC_RELAXED, __HIP_MEMORY_SCOPE_SYSTEM); }
__device__ __forceinline__ unsigned ld_agent(const gu32* p) { return __hip_atomic_load(p, __ATOMIC_RELAXED, __HIP_MEMORY_SCOPE_AGENT); }
__device__ __forceinline__ void st_agent(gu32* p, unsigned v) { __hip_atomic_store(p, v, __ATOMIC_RELAXED, __HIP_MEMORY_SCOPE_AGENT); }
__device__ __forceinline__ unsigned long long now() { return __builtin_amdgcn_s_memrealtime(); }
__device__ __forceinline__ unsigned rfl(unsigned v) { return __builtin_amdgcn_readfirstlane(v); }

__device__ __forceinline__ void st16_wt(GAS char* p, u32x4 v) {
    asm volatile("global_store_dwordx4 %0, %1, off sc0 sc1\n\ts_nop 1" :: "v"(p), "v"(v) : "memory");
}
template <int N> __device__ __forceinline__ void wait_vm() { asm volatile("s_waitcnt vmcnt(%0)" :: "i"(N) : "memory"); }
__device__ __forceinline__ void wait_lgkm() { asm volatile("" ::: "memory"); __builtin_amdgcn_s_waitcnt(0xC07F); asm volatile("" ::: "memory"); }
#define AGK_SL1 6
#define AGK_SL2 2
#define PIN() __builtin_amdgcn_sched_barrier(0)
__device__ __forceinline__ void sleep_until(unsigned t) {
    int rest = (int)(t - (unsigned)now());
    for (; rest > 48; rest = (int)(t - (unsigned)now())) __builtin_amdgcn_s_sleep(8);
    for (; rest > 2; rest -= 3) __builtin_amdgcn_s_sleep(1);
}
__device__ __forceinline__ void bar() { asm volatile("s_barrier" ::: "memory"); }

#define AGK_LDPOL "sc0 sc1"
__device__ __forceinline__ void glds2(const GAS char* base, unsigned v0, unsigned v1, unsigned lds_dst) {
    asm volatile("s_mov_b32 m0, %3\n\ts_nop 3\n\tglobal_load_lds_dwordx4 %1, %0 " AGK_LDPOL "\n\tglobal_load_lds_dwordx4 %2, %0 offset:1024 " AGK_LDPOL
                 :: "s"(base), "v"(v0), "v"(v1), "s"(lds_dst) : "m0", "memory");
}
template <int OFF> __device__ __forceinline__ void glds2o(const GAS char* base, unsigned v0, unsigned v1) {
    asm volatile("global_load_lds_dwordx4 %1, %0 offset:%3 " AGK_LDPOL "\n\tglobal_load_lds_dwordx4 %2, %0 offset:%4 " AGK_LDPOL
                 :: "s"(base), "v"(v0), "v"(v1), "i"(OFF), "i"(OFF + 1024) : "memory");
}
__device__ __forceinline__ void wait_vm_var(unsigned n) {
    asm volatile("s_getpc_b64 vcc\n\ts_add_u32 vcc_lo, vcc_lo, %0\n\ts_addc_u32 vcc_hi, vcc_hi, 0\n\ts_setpc_b64 vcc\n\t"
                 ".set agk_n, 0\n\t.rept 64\n\ts_waitcnt vmcnt(agk_n)\n\ts_branch 1f\n\t.set agk_n, agk_n + 1\n\t.endr\n1:"
                 :: "s"(n * 8 + 12) : "vcc", "scc", "memory");
}

__device__ __forceinline__ unsigned pack_bf16(float lo, float hi) {
    typedef float f32x2 __attribute__((ext_vector_type(2)));
    typedef __bf16 bf16x2 __attribute__((ext_vector_type(2)));
    f32x2 v = {lo, hi};
    bf16x2 r = __builtin_convertvector(v, bf16x2);
    return __builtin_bit_cast(unsigned, r);
}

__device__ __forceinline__ int xslot(int j, int s) { return j < 4 ? (j - 1) * HS + s : 3 * HS + (j - 4) * AS + (s - HS); }

#define STAMP(i) do { } while (0)
#define PSTAMP(j, p, i, t) do { } while (0)

enum { K_X = 0, K_W = 1 };
struct Piece { int kind, raw; unsigned soff, doff, foff, when; };

__host__ __device__ __forceinline__ int ord_at(int cross, int p) { return (cross * WPL + p % WPL) * NRND + p / WPL; }
__device__ __forceinline__ Piece locate(unsigned e, int j) {
    Piece r;
    r.when = e >> 16;
    const int s = (e >> 4) & 31, kc = e & 15, wp = (e >> 10) & 1 ? (int)(e & 127) : -1;
    r.raw = (e >> 9) & 1;
    if (wp >= 0) {
        r.kind = K_W;
        r.soff = (unsigned)(wp * 32 * (NSH * 2) + (j & 3) * 256);
        r.doff = (unsigned)XS_BYTES + (unsigned)(j - 4) * (unsigned)WSL_BYTES + (wp & 1) * 2 * (unsigned)STRIP_BYTES + (wp >> 1) * 2048;
        r.foff = (unsigned)WFLAG_OFF + ((j - 4) * NWP + (wp & 1) * 64 + ((wp >> 1) & 3) * 16 + (wp >> 3)) * 4;
    } else {
        r.kind = K_X;
        r.soff = (unsigned)(s * 16 * (KD * 2) + kc * 512);
        r.doff = (unsigned)xslot(j, s) * (unsigned)STRIP_BYTES + kc * 8192;
        r.foff = (unsigned)XFLAG_OFF + (j * 32 + s) * XF_STRIDE + kc * 4;
    }
    return r;
}
__device__ __forceinline__ unsigned pk_off(int j, int p) {
    return (unsigned)PK_OFF + 3072u + (unsigned)((j < 4 ? (j - 1) * NP_IN : 3 * NP_IN + (j - 4) * NP_XI) + p) * PK_BYTES;
}

__device__ __forceinline__ unsigned pko_off(int j, int p) { return (unsigned)PK_OFF + 3072u + (unsigned)(3 * NP_IN + 4 * NP_XI + (j - 4) * NP_OUT + p) * PK_BYTES; }

template <int OFF> __device__ __forceinline__ void st_pk(const u32x4& v, unsigned voff, const GAS char* dst) {
    asm volatile("s_nop 3\n\tglobal_store_dwordx4 %1, %0, %2 offset:%3 sc0 sc1\n\ts_nop 1" :: "v"(v), "v"(voff), "s"(dst), "i"(OFF) : "memory");
}
template <int OFF> __device__ __forceinline__ void ld_c(u32x4& v, unsigned voff, const GAS char* src) {
    asm volatile("global_load_dwordx4 %0, %1, %2 offset:%3 sc0 sc1" : "=v"(v) : "v"(voff), "s"(src), "i"(OFF) : "memory");
}
__device__ __forceinline__ void flag_st(const GAS char* f, unsigned val, unsigned zero) {
    asm volatile("s_mov_b64 exec, 1\n\ts_nop 3\n\tglobal_store_dword %0, %1, %2 sc0 sc1\n\ts_mov_b64 exec, -1" :: "v"(zero), "v"(val), "s"(f) : "memory");
}
__device__ __forceinline__ unsigned perm(unsigned hi, unsigned lo, unsigned sel) { return __builtin_amdgcn_perm(hi, lo, sel); }
__device__ __forceinline__ unsigned bfi(unsigned m, unsigned a, unsigned b) { return (a & m) | (b & ~m); }
__device__ __forceinline__ unsigned rotr(unsigned v, unsigned n) { return __builtin_amdgcn_alignbit(v, v, n); }
__device__ __forceinline__ unsigned ffbl(unsigned v) { unsigned r; asm("v_ffbl_b32 %0, %1" : "=v"(r) : "v"(v)); return r; }
__device__ __forceinline__ unsigned pkmax(unsigned a, unsigned b) {
    typedef unsigned short u16x2 __attribute__((ext_vector_type(2)));
    return __builtin_bit_cast(unsigned, __builtin_elementwise_max(__builtin_bit_cast(u16x2, a), __builtin_bit_cast(u16x2, b)));
}
template <int CTRL> __device__ __forceinline__ unsigned dppmax(unsigned v) { return max(v, (unsigned)__builtin_amdgcn_update_dpp(0, (int)v, CTRL, 0xF, 0xF, true)); }
constexpr unsigned F_RAW = 0x80u;
constexpr int MAXEXC = 8;
__device__ __forceinline__ unsigned ximg0(unsigned lane, unsigned pitch) { return (lane >> 3) * pitch + ((lane & 7) ^ (lane >> 4)) * 16; }
__device__ __forceinline__ unsigned ximg1(unsigned lane, unsigned pitch) { return (8 + (lane >> 3)) * pitch + ((lane & 7) ^ (4 + (lane >> 4))) * 16 - 1024; }
__device__ __forceinline__ unsigned wimg0(unsigned lane, unsigned pitch) { return (lane >> 3) * pitch + ((lane & 7) ^ ((lane >> 3) & 2)) * 16; }
__device__ __forceinline__ unsigned wimg1(unsigned lane, unsigned pitch) { return (8 + (lane >> 3)) * pitch + ((lane & 7) ^ (((lane >> 3) & 2) | 4)) * 16 - 1024; }

__device__ __forceinline__ void fetch_piece(int kind, unsigned soff, const GAS char* sx, const GAS char* sw,
                                            unsigned vx0, unsigned vx1, unsigned vw0, unsigned vw1, unsigned l) {
    const bool kx = kind == K_X;
    const GAS char* s = (kx ? sx : sw) + soff;
    const unsigned a = kx ? vx0 : vw0, b = kx ? vx1 : vw1;
    const int q1 = kx ? 128 : 16 * NSH * 2, q2 = kx ? 256 : 128;
    glds2(s, a, b, l); glds2(s + q1, a, b, l + 2048); glds2(s + q2, a, b, l + 4096); glds2(s + q1 + q2, a, b, l + 6144);
}

__device__ __forceinline__ unsigned tag_of(unsigned call) { return ((call + 1u) >> 1) << 31; }

struct Packed { u32x4 W[5], ex; u32x2 W5; unsigned trl; bool raw; };
__device__ __forceinline__ void pack(const u32x4 (&v)[8], Packed& Q, unsigned call) {
    const unsigned lane = threadIdx.x & 63, tagw = tag_of(call);
    u32x4 (&W)[5] = Q.W;
    u32x2 W5;
    u32x2 em;
    unsigned trl;
    unsigned mx = pkmax(pkmax(pkmax(v[0].x & 0x7FFF7FFFu, v[0].y & 0x7FFF7FFFu), pkmax(v[0].z & 0x7FFF7FFFu, v[0].w & 0x7FFF7FFFu)),
                        pkmax(pkmax(v[6].x & 0x7FFF7FFFu, v[6].y & 0x7FFF7FFFu), pkmax(v[6].z & 0x7FFF7FFFu, v[6].w & 0x7FFF7FFFu)));
    mx = max(mx >> 16, mx & 0xFFFFu) >> 8;
    mx = dppmax<0x143>(dppmax<0x142>(dppmax<0x118>(dppmax<0x114>(dppmax<0x112>(dppmax<0x111>(mx))))));
    unsigned fmt = max((unsigned)__builtin_amdgcn_readlane((int)mx, 63), 7u) - 7u;
    const unsigned B4 = fmt * 0x01010101u;
    unsigned accO = 0, accA = ~0u, N[8];
#pragma unroll
    for (int k = 0; k < 8; ++k) {
        const unsigned Ha = perm(v[k].y, v[k].x, 0x07050301u), Hb = perm(v[k].w, v[k].z, 0x07050301u);
        const unsigned da = (Ha | 0x80808080u) - B4, db = (Hb | 0x80808080u) - B4;
        accO |= da | db; accA &= da & db;
        W[k >> 1][(k & 1) * 2] = perm(v[k].y, v[k].x, 0x06040200u);
        W[k >> 1][(k & 1) * 2 + 1] = perm(v[k].w, v[k].z, 0x06040200u);
        N[k] = bfi(0x87878787u, bfi(0x80808080u, Ha, da), rotr(bfi(0x80808080u, Hb, db), 28));
    }
    em.x = ~bfi(0x33333333u, bfi(0x11111111u, N[0] >> 2, N[1] >> 1), bfi(0x44444444u, N[2], N[3] << 1));
    em.y = ~bfi(0x33333333u, bfi(0x11111111u, N[4] >> 2, N[5] >> 1), bfi(0x44444444u, N[6], N[7] << 1));
    W[4] = (u32x4){bfi(0x44444444u, N[6] << 2, N[0]), bfi(0x44444444u, N[6] << 1, N[1]), bfi(0x44444444u, N[6] >> 1, N[2]), bfi(0x44444444u, N[7] << 2, N[3])};
    W5 = (u32x2){bfi(0x44444444u, N[7] << 1, N[4]), bfi(0x44444444u, N[7] >> 1, N[5])};
    unsigned long long m = (unsigned long long)em.y << 32 | em.x;
    unsigned pos = 0;
#pragma unroll
    for (int i = 0; i < 4; ++i) {
        pos = __builtin_amdgcn_alignbit(min(ffbl((unsigned)m), ffbl((unsigned)(m >> 32)) | 32u), pos, 6);
        m &= m - 1;
    }
    const bool over = m != 0 || (int)em.y < 0;
    const unsigned long long bo = __ballot(over);
    const unsigned moff = __builtin_amdgcn_mbcnt_hi((unsigned)(bo >> 32), __builtin_amdgcn_mbcnt_lo((unsigned)bo, 0));
    pos = over ? moff << 14 | 0xFFF03F00u : pos;
    const int mto = over ? moff * 4 : 252;
    Q.ex = (u32x4){(unsigned)__builtin_amdgcn_ds_permute(mto, em.x), (unsigned)__builtin_amdgcn_ds_permute(mto, em.y), 0, tagw};
    const unsigned cnt2 = __builtin_popcountll(bo);
    bool raw = cnt2 > MAXEXC;
    fmt |= cnt2 << 16;
    const unsigned long long bm = __ballot(((accO & 0x78787878u) | (~accA & 0x80808080u)) != 0);
    if (bm) {
        unsigned nb = 0, ib = 0, hv = 0;
#pragma unroll
        for (int i = 0; i < 16; ++i) {
            const unsigned H = (i & 1) ? perm(v[i >> 1].w, v[i >> 1].z, 0x07050301u) : perm(v[i >> 1].y, v[i >> 1].x, 0x07050301u);
            const bool bad = (((H | 0x80808080u) - B4) & 0xF8F8F8F8u) != 0x80808080u;
            hv = bad ? H : hv; ib = bad ? (unsigned)i : ib; nb += bad;
        }
        const int eto = nb ? __builtin_amdgcn_mbcnt_hi((unsigned)(bm >> 32), __builtin_amdgcn_mbcnt_lo((unsigned)bm, 0)) * 4 : 252;
        Q.ex.z = __builtin_amdgcn_ds_permute(eto, hv);
        Q.ex.w = __builtin_amdgcn_ds_permute(eto, lane | (ib << 8)) | tagw;
        const unsigned cnt = __builtin_popcountll(bm);
        raw = raw || cnt > MAXEXC || __ballot(nb > 1) != 0;
        fmt |= cnt << 8;
    }
    if (raw) fmt = F_RAW;
    trl = (tagw >> 30) + (__builtin_amdgcn_inverse_ballot_w64((unsigned long long)call << 32 | fmt) ? 1u : 0u);
#pragma unroll
    for (int k = 0; k < 5; ++k) { trl = __builtin_amdgcn_alignbit(trl, W[k].w, 31); W[k].w = bfi(0x7FFFFFFFu, W[k].w, tagw); }
    trl = __builtin_amdgcn_alignbit(trl, W5.y, 31); W5.y = bfi(0x7FFFFFFFu, W5.y, tagw);
    trl = __builtin_amdgcn_alignbit(trl, pos, 8);
    Q.W5 = W5; Q.trl = trl; Q.raw = raw;
}
__device__ __forceinline__ void send_packed(const Packed& Q, const GAS char* pk, unsigned vlin, bool few) {
    st_pk<-3072>(Q.W[0], vlin, pk); st_pk<-2048>(Q.W[1], vlin, pk);
    if (few) wait_vm<1>();
    st_pk<-1024>(Q.W[2], vlin, pk);
    if (few) wait_vm<1>();
    st_pk<0>(Q.W[3], vlin, pk);
    if (few) wait_vm<1>();
    st_pk<1024>(Q.W[4], vlin, pk);
    if (few) wait_vm<1>();
    asm volatile("global_store_dwordx2 %0, %1, %2 offset:2048 sc0 sc1" :: "v"(vlin >> 1), "v"(Q.W5), "s"(pk) : "memory");
    asm volatile("s_mov_b64 exec, 255\n\tglobal_store_dwordx4 %0, %1, %2 offset:2816 sc0 sc1\n\ts_mov_b64 exec, -1" :: "v"(vlin), "v"(Q.ex), "s"(pk) : "memory");
    asm volatile("global_store_dword %0, %1, %2 offset:2560 sc0 sc1" :: "v"(vlin >> 2), "v"(Q.trl), "s"(pk) : "memory");
}

__device__ __forceinline__ void comm_role(const Params& P, LAS char* smem, unsigned long long t64, int wslot, int j, int u) {
    const unsigned* ord = P.ord + ord_at(j >= 4, u);
    unsigned eA = ord[0], eB = ord[1], tlead = P.tlead;
    const GAS char *srcx = (const GAS char*)P.x, *srcw = (const GAS char*)P.w, *pws = (const GAS char*)P.ws[j];
    GAS char* myws = (GAS char*)P.ws[0];
    asm volatile("" : "+s"(eA), "+s"(eB), "+s"(tlead), "+s"(srcx), "+s"(srcw), "+s"(pws), "+s"(myws), "+s"(t64));
    const unsigned tstart = (unsigned)t64;
    gu32* cc = (gu32*)(myws + CALL_OFF + blockIdx.x * 64) + wslot;
    const int lane = threadIdx.x & 63;
    const int npiece = j < 4 ? NP_IN : NP_XI;
    const unsigned vlin = lane * 16;
    const unsigned vx0 = ximg0(lane, KD * 2), vx1 = ximg1(lane, KD * 2), vw0 = wimg0(lane, NSH * 2), vw1 = wimg1(lane, NSH * 2);
    const unsigned vblk = (lane >> 5) * 2048 + (lane & 31) * 16, vblw = (lane >> 5) * (unsigned)STRIP_BYTES + (lane & 31) * 16;
    const unsigned lds0 = (unsigned)(size_t)smem;
#define AGK_FETCH(q, slot) fetch_piece((q).kind, (q).soff, srcx, srcw, vx0, vx1, vw0, vw1, lds0 + (slot) * 8192)
    int pA = u;
    unsigned ccv;
    Piece A = locate(eA, j);
    if (u > 3) sleep_until(tstart + (u * (tlead & 0xFFFFu) >> 2));
    AGK_FETCH(A, 0);
    asm volatile("global_load_dword %0, %1, off sc1" : "=v"(ccv) : "v"(cc) : "memory");
    Piece B = locate(eB, j);
    asm volatile("s_waitcnt vmcnt(0)" : "+v"(ccv) :: "memory");
    AGK_FETCH(B, 1);
    const unsigned call = rfl(ccv) + 1u;
    if (wslot == 0) STAMP(0);
    const GAS char* dbuf = pws + (size_t)(call & 1) * BUF_BYTES;
    gu32* tmo = (gu32*)(myws + TMO_OFF);
    unsigned zero = 0, callv = call;
    asm volatile("" : "+v"(zero), "+v"(callv));
    int slot = 0;
    for (;;) {
        u32x4 v[8];
#pragma unroll
        for (int k = 0; k < 8; ++k) v[k] = *(LAS u32x4*)(smem + slot * 8192 + (k >> 2) * 4096 + (k & 3) * 512 + vblk);
        Packed Q;
        if (!A.raw) pack(v, Q, call);
        else {
#pragma unroll
            for (int k = 0; k < 5; ++k) Q.W[k] = v[k];
            Q.W5 = (u32x2){v[5].x, v[5].y};
            Q.ex = v[6]; Q.trl = 0; Q.raw = true;
        }
        const GAS char* pk = dbuf + pk_off(j, pA);
        const Piece C = locate(ord[pA + 2 * WPL < npiece ? pA / WPL + 2 : 0], j);
        asm volatile("" :: "v"(Q.W[0]), "v"(Q.W[1]), "v"(Q.W[2]), "v"(Q.W[3]), "v"(Q.W[4]), "v"(Q.W5), "v"(v[6]), "v"(v[7]), "v"(Q.trl), "v"(Q.ex),
                     "s"(pk), "s"(C.soff) : "memory");
        sleep_until(tstart + A.when);
        if (!A.raw) send_packed(Q, pk, vlin, false);
        if (Q.raw) {
            const GAS char* d0 = dbuf + A.doff;
            const GAS char* d1 = d0 + (A.kind == K_W ? 4 * (unsigned)STRIP_BYTES : 4096u);
            const unsigned vd = A.kind == K_W ? vblw : vblk;
            st_pk<0>(v[0], vd, d0); st_pk<512>(v[1], vd, d0); st_pk<1024>(v[2], vd, d0); st_pk<1536>(v[3], vd, d0);
            st_pk<0>(v[4], vd, d1); st_pk<512>(v[5], vd, d1); st_pk<1024>(v[6], vd, d1); st_pk<1536>(v[7], vd, d1);
        }
        AGK_FETCH(C, slot);
        if (Q.raw) { wait_vm<8>(); flag_st(dbuf + A.foff, callv, zero); }
        PSTAMP(j, pA, 0, tstart + A.when); PSTAMP(j, pA, 1, tb); PSTAMP(j, pA, 2, now());
        pA += WPL; A = B; B = C; slot ^= 1;
        if (pA >= npiece) break;
        wait_vm<16>();
    }
    wait_vm<0>();
    STAMP(j < 4 ? 1 : 3);
    if (j == 1 && u == 0) {
        if (lane < 4) {
            const gu32* c = (const gu32*)(myws + OCNT_OFF + lane * 256);
            const unsigned need = call * P.out_adds;
            const unsigned long long t0 = now();
            bool bad = false;
            unsigned spin = 0;
            while (ld_sys(c) < need) {
                if ((++spin & 63) == 0 && (ld_agent(tmo) == call || now() - t0 > DEADLINE_TICKS)) { st_agent(tmo, call); bad = true; break; }
                __builtin_amdgcn_s_sleep(1);
            }
            if (bad) {
                __builtin_amdgcn_s_sleep(127);
                __hip_atomic_store((GAS unsigned short*)(P.out[0] + (size_t)(P.rank * MSH) * (NSH * 2)), (unsigned short)0x7FC0, __ATOMIC_RELAXED, __HIP_MEMORY_SCOPE_AGENT);
            }
        }
        STAMP(2);
    }
    if (lane == 0) st_agent(cc, call);
}

__device__ __forceinline__ void out_rows(const Params& P, const GAS char* (&d)[4], int jd, int j, int po) {
#pragma unroll
    for (int i = 0; i < 4; ++i) {
        const int code = rfl(P.strip[(po >> 1) * 4 + i]);
        d[i] = (const GAS char*)P.out[jd] + (unsigned)(((P.rank ^ jd ^ j ^ (code >> 5)) * MSH + (code & 31) * 16) * (NSH * 2) + (j & 3) * 256 + (po & 1) * 128);
    }
}
__device__ __forceinline__ void out_role(const Params& P, unsigned ea, unsigned gs, unsigned call, const GAS char* pk, int j, int po, bool few) {
    const unsigned lane = threadIdx.x & 63;
    u32x4 v[8];
#pragma unroll
    for (int n = 0; n < 8; ++n) v[n] = *(LAS u32x4*)(size_t)(ea + (n >> 2) * gs + (n & 3) * 1024 + (lane >> 3) * 128 + (((lane & 7) ^ (lane >> 3)) << 4));
    Packed Q;
    pack(v, Q, call);
    if (Q.raw) {
        const GAS char* d[4];
        out_rows(P, d, j, j, po);
        const unsigned vA = (lane >> 3) * (NSH * 2) + (lane & 7) * 16, vB = vA + 8 * (NSH * 2);
#pragma unroll
        for (int n = 0; n < 8; ++n) st_pk<0>(v[n], n & 1 ? vB : vA, d[n >> 1]);
        wait_vm<0>();
    }
    send_packed(Q, pk, lane * 16, few);
    wait_vm<0>();
}

__device__ __forceinline__ void unpack_role(const Params& P, GAS char* myws, unsigned tstart, unsigned call, int w, int nw) {
    const GAS char* buf = myws + (size_t)(call & 1) * BUF_BYTES;
    gu32* tmo = (gu32*)(myws + TMO_OFF);
    const int lane = threadIdx.x & 63;
    const unsigned vlin = lane * 16, tagw = tag_of(call);
    const unsigned vblk = (lane >> 5) * 2048 + (lane & 31) * 16, vblw = (lane >> 5) * (unsigned)STRIP_BYTES + (lane & 31) * 16;
    unsigned zero = 0, callv = call;
    asm volatile("" : "+v"(zero), "+v"(callv));
    const unsigned tfirst = tstart + (P.tpoll & 0xFFFFu);
    const bool spec = P.tpoll >> 16;
    for (int q = w; ; q += nw) {
        const int p = q / 7, j = 1 + q - p * 7;
        const bool outp = q >= NPMAX * 7;
        const GAS char *pk, *d[4];
        unsigned vA, vB, foff = 0;
        if (!outp) {
            if (p >= (j < 4 ? NP_IN : NP_XI)) continue;
            const Piece A = locate(P.ord[ord_at(j >= 4, p)], j);
            if (A.raw) continue;
            pk = buf + pk_off(j, p);
            d[0] = buf + A.doff; d[1] = d[0] + 1024;
            d[2] = d[0] + (A.kind == K_W ? 4 * (unsigned)STRIP_BYTES : 4096u); d[3] = d[2] + 1024;
            vA = A.kind == K_W ? vblw : vblk; vB = vA + 512;
            foff = A.foff;
            sleep_until(tfirst + A.when);
        } else {
            const int i = q - NPMAX * 7, jo = 4 + (i & 3), po = (i >> 2) + (int)((P.nunpack >> 16) & 255);
            if (i >= (int)((P.nunpack >> 8) & 255) * 4) break;
            pk = buf + pko_off(jo, i >> 2);
            out_rows(P, d, 0, jo, po);
            vA = (lane >> 3) * (NSH * 2) + (lane & 7) * 16; vB = vA + 8 * (NSH * 2);
        }
        unsigned long long t0 = 0;
        unsigned spin = 0, trl, fmt = 0;
        u32x4 W[5], ex; u32x2 W5;
        bool data = spec && !outp;
        for (;;) {
            asm volatile("s_nop 4\n\tglobal_load_dword %0, %1, %2 offset:2560 sc0 sc1" : "=v"(trl) : "v"(vlin >> 2), "s"(pk) : "memory");
            if (data) {
                ld_c<2816>(ex, vlin & (MAXEXC * 16 - 1), pk);
                asm volatile("global_load_dwordx2 %0, %1, %2 offset:2048 sc0 sc1" : "=v"(W5) : "v"(vlin >> 1), "s"(pk) : "memory");
                ld_c<1024>(W[4], vlin, pk);
                ld_c<-3072>(W[0], vlin, pk); ld_c<-2048>(W[1], vlin, pk); ld_c<-1024>(W[2], vlin, pk); ld_c<0>(W[3], vlin, pk);
            }
            asm volatile("s_waitcnt vmcnt(0)" : "+v"(trl), "+v"(W[0]), "+v"(W[1]), "+v"(W[2]), "+v"(W[3]), "+v"(W[4]), "+v"(W5), "+v"(ex) :: "memory");
            const unsigned long long fc = __ballot((int)(trl << 1) < 0);
            if (__ballot((int)(trl ^ tagw) < 0) == 0 && (unsigned)(fc >> 32) == call) {
                fmt = (unsigned)fc;
                if (fmt & F_RAW) break;
                if (data && __ballot((int)((W[0].w ^ tagw) | (W[1].w ^ tagw) | (W[2].w ^ tagw) | (W[3].w ^ tagw) | (W[4].w ^ tagw) | (W5.y ^ tagw) | (ex.w ^ tagw)) < 0) == 0) break;
                data = true;
            } else __builtin_amdgcn_s_sleep(AGK_SL2);
            if ((++spin & 63) == 0) {
                if (t0 == 0) t0 = now();
                if (rfl(ld_agent(tmo)) == call || now() - t0 > DEADLINE_TICKS) {
                    st_agent(tmo, call);
                    if (outp) st_pk<0>((u32x4){0x7FC07FC0u, 0x7FC07FC0u, 0x7FC07FC0u, 0x7FC07FC0u}, vA, d[0]);
                    fmt = F_RAW;
                    break;
                }
            }
        }
        if (!outp) { PSTAMP(j, p, 3, now()); PSTAMP(j, p, 7, spin); PSTAMP(j, p, 4, now()); }
        if (fmt & F_RAW) continue;
        const int cnt = (fmt >> 8) & 31;
#pragma unroll
        for (int k = 0; k < 5; ++k) W[k].w = bfi(0x7FFFFFFFu, W[k].w, trl << (2 + k));
        W5.y = bfi(0x7FFFFFFFu, W5.y, trl << 7);
        const unsigned long long m = 1ull << (trl & 63) | 1ull << ((trl >> 6) & 63) | 1ull << ((trl >> 12) & 63) | 1ull << ((trl >> 18) & 63);
        unsigned m0 = (unsigned)m, m1 = (unsigned)(m >> 32) & 0x7FFFFFFFu;
        if (fmt >> 16) {
            const int src = (trl >> 4) & 0x3C;
            const unsigned e0 = __builtin_amdgcn_ds_bpermute(src, ex.x), e1 = __builtin_amdgcn_ds_bpermute(src, ex.y);
            const bool over = (trl & 0x83Fu) == 63u;
            m0 = over ? e0 : m0; m1 = over ? e1 : m1;
        }
        unsigned N[8];
        N[0] = bfi(0x44444444u, ~(m0 << 2), W[4].x); N[1] = bfi(0x44444444u, ~(m0 << 1), W[4].y); N[2] = bfi(0x44444444u, ~m0, W[4].z); N[3] = bfi(0x44444444u, ~(m0 >> 1), W[4].w);
        N[4] = bfi(0x44444444u, ~(m1 << 2), W5.x); N[5] = bfi(0x44444444u, ~(m1 << 1), W5.y);
        N[6] = bfi(0x33333333u, bfi(0x11111111u, W[4].x >> 2, W[4].y >> 1), bfi(0x44444444u, ~m1, W[4].z << 1));
        N[7] = bfi(0x33333333u, bfi(0x11111111u, W[4].w >> 2, W5.x >> 1), bfi(0x44444444u, ~(m1 >> 1), W5.y << 1));
        const unsigned B4 = (fmt & 127u) * 0x01010101u;
        unsigned H[16];
#pragma unroll
        for (int k = 0; k < 8; ++k) {
            H[2 * k] = (N[k] & 0x87878787u) + B4;
            H[2 * k + 1] = (rotr(N[k], 4) & 0x87878787u) + B4;
        }
        if (cnt) {
            unsigned tag = 255, val = 0;
            for (int e = 0; e < cnt; ++e) {
                const unsigned id = __builtin_amdgcn_readlane(ex.w, e) & 0xFFFu, hv = __builtin_amdgcn_readlane(ex.z, e);
                const bool mine = (id & 63) == (unsigned)lane;
                tag = mine ? id >> 8 : tag; val = mine ? hv : val;
            }
#pragma unroll
            for (int i = 0; i < 16; ++i) H[i] = tag == (unsigned)i ? val : H[i];
        }
#define AGK_DST(k) { const unsigned l0 = W[k >> 1][(k & 1) * 2], l1 = W[k >> 1][(k & 1) * 2 + 1]; u32x4 o; \
            o.x = perm(H[2 * k], l0, 0x05010400u); o.y = perm(H[2 * k], l0, 0x07030602u); \
            o.z = perm(H[2 * k + 1], l1, 0x05010400u); o.w = perm(H[2 * k + 1], l1, 0x07030602u); st_pk<0>(o, k & 1 ? vB : vA, d[k >> 1]); }
        AGK_DST(0) AGK_DST(1) AGK_DST(2) AGK_DST(3) AGK_DST(4) AGK_DST(5) AGK_DST(6) AGK_DST(7)
        if (outp) continue;
        PSTAMP(j, p, 5, now());
        wait_vm<0>(); flag_st(buf + foff, callv, zero);
        PSTAMP(j, p, 6, now());
    }
    wait_vm<0>();
}

#define AGK_RPG 2
#define AGK_RA 4, 6
#define AGK_RH 4, 12
#define AGK_RQ 8, 12
#define AGK_RW 6, 7
#define AGK_LOOK 3
template <int NB, int NP, int NSLOT, int SLOT_BYTES>
__device__ __forceinline__ void loader(GAS char* myws, unsigned call, const GAS char* const (&src)[NP], const int (&shift)[NP], const unsigned (&v0)[NB],
                                       const unsigned (&v1)[NB], const GAS char* fp, bool have, unsigned ldsw, unsigned rot, bool first) {
    constexpr int NI = 2 * NB;
    static_assert((NSLOT - 2) * NI + 8 < 64 && (NSLOT - 1) * NI + 10 < 64, "vmcnt range");
    gu32* tmo = (gu32*)(myws + TMO_OFF);
    const unsigned ldsend = ldsw + NSLOT * SLOT_BYTES;
    const unsigned long long act = ~__ballot(have);
    asm volatile("v_mov_b32 v250, %0" :: "v"(have ? ~0u : 0u) : "v250", "memory");
    int mark = 0;
    unsigned npoll = 0;
    auto poll = [&]() {
        ++npoll;
        asm volatile("s_mov_b64 exec, %1\n\tglobal_load_dword v250, %0, off sc0 sc1\n\ts_mov_b64 exec, -1" :: "v"(fp), "s"(act) : "v250", "memory");
    };
    auto eval = [&]() {
        unsigned long long bx;
        asm volatile("v_cmp_gt_u32 %0, %1, v250" : "=s"(bx) : "s"(call) : "memory");
        const unsigned m = (unsigned)bx | (unsigned)(bx >> 32);
        mark = __builtin_ctz(m | m >> 16 | 0x10000u) * 4;
    };
    unsigned ldsi = ldsw, kt = 0;
    const GAS char* ptr[NP];
#pragma unroll
    for (int i = 0; i < NP; ++i) ptr[i] = src[i] + (rot << shift[i]);
    auto issue = [&]() {
        asm volatile("s_mov_b32 m0, %0\n\ts_nop 0" :: "s"(ldsi) : "m0", "memory");
        glds2o<-4096>(ptr[0], v0[0], v1[0]);
        glds2o<-2048>(ptr[1 % NP], v0[1], v1[1]);
        if (NB == 4) { glds2o<0>(ptr[2 % NP], v0[2 % NB], v1[2 % NB]); glds2o<2048>(ptr[3 % NP], v0[3 % NB], v1[3 % NB]); }
        const unsigned k = ++kt ^ rot;
#pragma unroll
        for (int i = 0; i < NP; ++i) { ptr[i] = src[i] + (k << shift[i]); asm volatile("" : "+s"(ptr[i])); }
        ldsi = ldsi + SLOT_BYTES == ldsend ? ldsw : ldsi + SLOT_BYTES;
        asm volatile("" : "+s"(ldsi));
    };
    int ti = 0, tb = 0;
    unsigned hist = 0;
#pragma nounroll
    for (;;) {
        if (mark < NSTEP && mark - ti <= AGK_LOOK) eval();
        const int lim = mark < tb + NSLOT - 1 ? mark : tb + NSLOT - 1;
#pragma nounroll
        for (; ti < lim; ++ti) { issue(); hist = hist << 4 | (npoll & 15u); }
        if (ti == tb) {
            unsigned long long t0 = 0;
            for (unsigned spin = 1; ti >= mark; ++spin) {
                wait_vm<8>();
                poll();
                __builtin_amdgcn_s_sleep(AGK_SL1);
                if ((spin & 63) == 0) {
                    if (t0 == 0) t0 = now();
                    if (rfl(ld_agent(tmo)) == call || now() - t0 > DEADLINE_TICKS) break;
                }
                eval();
            }
            if (ti >= mark) break;
            continue;
        }
        const int run = (mark < NSTEP ? mark - AGK_LOOK : NSTEP) - ti;
        if (run > 0) {
#pragma nounroll
            for (int i = run; i > 0; --i) { wait_vm<(NSLOT - 2) * NI>(); bar(); issue(); }
            ti += run; tb += run;
            const unsigned rep = (npoll & 15u) * 0x11111111u;
            hist = run >= 8 ? rep : hist << (4 * run) | (rep & ((1u << (4 * run)) - 1u));
        }
        if (mark < NSTEP) poll();
        const unsigned n = ti - tb - 1;
        wait_vm_var(n * NI + (n < 8 ? (npoll - (hist >> (4 * n))) & 15u : 0u));
        bar();
        if (++tb == NSTEP) break;
    }
    if (tb < NSTEP) {
        st_agent(tmo, call);
        const u32x4 nan = {0x7FC07FC0u, 0x7FC07FC0u, 0x7FC07FC0u, 0x7FC07FC0u};
        for (; tb < NSTEP; ++tb) {
#pragma unroll
            for (int i = 0; i < NI; ++i) *(LAS u32x4*)(size_t)(ldsi - 4096 + i * 1024 + (threadIdx.x & 63) * 16) = nan;
            wait_lgkm();
            bar();
            ldsi = ldsi + SLOT_BYTES == ldsend ? ldsw : ldsi + SLOT_BYTES;
        }
    }
    if (first) STAMP(4);
    wait_vm<0>();
}

template <int S, int NH, int NSA, int NSB>
__device__ __forceinline__ unsigned tile_role(const Params& P, LAS char* smem, unsigned call, int first_strip, int cg, int ch, int win, unsigned rot, bool packed) {
    constexpr int SA = S * 2048, SB = NH * 8192, BOFF = NSA * SA;
    constexpr int SPW = S / 2, NFR = NH * 2;
    static_assert(BOFF + NSB * SB <= LDS_BYTES, "LDS rings");
    GAS char* myws = (GAS char*)P.ws[0];
    const int lane = threadIdx.x & 63;
    const int wv = rfl(threadIdx.x >> 6);
    const bool wremote = cg >= 4;

    if (wv >= 4) {
        const int lw = wv & 1;
        const GAS char* mybuf = myws + (size_t)(call & 1) * BUF_BYTES;
        const unsigned vlin = lane * 16;
        if (wv < 6) {
            const GAS char* src[SPW];
            unsigned v0[SPW], v1[SPW];
            int shift[SPW];
#pragma unroll
            for (int i = 0; i < SPW; ++i) {
                const int code = rfl(P.strip[first_strip + lw * SPW + i]);
                const bool own = code < 32;
                v0[i] = own ? ximg0(lane, KD * 2) : vlin;
                v1[i] = own ? ximg1(lane, KD * 2) : vlin;
                src[i] = (own ? (const GAS char*)P.x + (unsigned)code * (16 * KD * 2) : mybuf + (unsigned)xslot(code >> 5, code & 31) * (unsigned)STRIP_BYTES) + (4096 - 2048 * i);
                shift[i] = own ? 7 : 11;
            }
            const int fcode = P.strip[first_strip + lw * SPW + ((lane >> 4) & (SPW - 1))];
            loader<SPW, SPW, NSA, SA>(myws, call, src, shift, v0, v1, mybuf + XFLAG_OFF + fcode * XF_STRIDE + (lane & 15) * 4, fcode < 32 || lane >= 16 * SPW,
                                      (unsigned)(size_t)smem + lw * (SA / 2) + 4096, rot, lw == 0);
        } else {
            const unsigned dk = (wremote ? (unsigned)STRIP_BYTES : 16u * NSH * 2) - 2048, dh = (wremote ? 4 * (unsigned)STRIP_BYTES : 128u) - 4096;
            const GAS char* const src[1] = {(wremote ? mybuf + XS_BYTES + (unsigned)((cg - 4) * 8 + ch * 4 + lw * 2) * (unsigned)STRIP_BYTES
                                                     : (const GAS char*)P.w + cg * 256 + ch * 128 + lw * 32 * (NSH * 2))};
            const int shift[1] = {wremote ? 11 : 16};
            unsigned v0[NFR], v1[NFR];
#pragma unroll
            for (int i = 0; i < NFR; ++i) {
                v0[i] = (wremote ? vlin : wimg0(lane, NSH * 2)) + 4096 + (i & 1) * dk + (i >> 1) * dh;
                v1[i] = (wremote ? vlin : wimg1(lane, NSH * 2)) + 4096 + (i & 1) * dk + (i >> 1) * dh;
            }
            loader<NFR, 1, NSB, SB>(myws, call, src, shift, v0, v1, mybuf + WFLAG_OFF + ((cg & 3) * NWP + lw * 64 + lane) * 4, !wremote,
                                    (unsigned)(size_t)smem + BOFF + lw * (SB / 2) + 4096, rot, false);
        }
    } else {
        const int fr = lane & 15, g = lane >> 4;
        const int wm = wv >> 1, wn = wv & 1;
        unsigned aoff[2];
#pragma unroll
        for (int kk = 0; kk < 2; ++kk) aoff[kk] = wm * (SA / 2) + fr * 128 + (((kk * 4 + g) ^ ((fr >> 1) & 7)) << 4);
        unsigned boff[NFR];
        {
            const int q = fr >> 2, pp = lane & 3, f = (q >> 1) | ((g & 1) << 1);
#pragma unroll
            for (int nb = 0; nb < NFR; ++nb)
                boff[nb] = BOFF + (NH == 2 ? wn * 4096 : 0) + (g >> 1) * 2048 + ((g & 1) * 8 + q) * 128 + ((((wn * NFR + nb) & 3) ^ f) << 5) + pp * 8;
        }
        int rowbase[SPW];
#pragma unroll
        for (int i = 0; i < SPW; ++i) {
            const int code = P.strip[first_strip + wm * SPW + i];
            rowbase[i] = rfl((P.rank ^ (code >> 5)) * MSH + (code & 31) * 16);
        }
        GAS char* outp = (GAS char*)P.out[wremote ? cg : 0] + (cg & 3) * 256;
        GAS char* cntp = (GAS char*)P.ws[cg] + OCNT_OFF + (cg & 3) * 256;
        constexpr int orow = NSH * 2;
        asm volatile("" : "+s"(outp), "+s"(cntp));
        f32x4 acc[SPW][NFR];
#pragma unroll
        for (int i = 0; i < SPW; ++i)
#pragma unroll
            for (int nb = 0; nb < NFR; ++nb) acc[i][nb] = (f32x4){0.f, 0.f, 0.f, 0.f};
        struct Frag { bf16x8 a[SPW]; bf16x8 b[NFR]; };
        auto rd = [&](Frag& F, int sa, int sb, int kk) {
#pragma unroll
            for (int i = 0; i < SPW; ++i) F.a[i] = *(LAS bf16x8*)(smem + sa + aoff[kk] + i * 2048);
#pragma unroll
            for (int nb = 0; nb < NFR; ++nb) {
                const s16x4 lo = __builtin_amdgcn_ds_read_tr16_b64_v4i16((LAS s16x4*)(smem + sb + boff[nb] + kk * (SB / 2)));
                const s16x4 hi = __builtin_amdgcn_ds_read_tr16_b64_v4i16((LAS s16x4*)(smem + sb + boff[nb] + kk * (SB / 2) + 512));
                F.b[nb] = __builtin_bit_cast(bf16x8, __builtin_shufflevector(lo, hi, 0, 1, 2, 3, 4, 5, 6, 7));
            }
        };
        auto mm = [&](const Frag& F) {
#pragma unroll
            for (int nb = 0; nb < NFR; ++nb)
#pragma unroll
                for (int i = 0; i < SPW; ++i) acc[i][nb] = __builtin_amdgcn_mfma_f32_16x16x32_bf16(F.b[nb], F.a[i], acc[i][nb], 0, 0, 0);
        };
        Frag F0, F1;
        int ca = 0, cb = 0;
        bar();
        rd(F0, 0, 0, 0);
        wait_lgkm();
#pragma nounroll
        for (int t = 0; t < NSTEP - 1; ++t) {
            constexpr int NRD = SPW + 2 * NFR, NMM = SPW * NFR, RPG = S * NH == 16 ? 1 : AGK_RPG;
            rd(F1, ca, cb, 1);
            mm(F0);
#pragma unroll
            for (int i = 0; i < NMM; ++i) { __builtin_amdgcn_sched_group_barrier(0x008, 1, 0); if (i * RPG < NRD) __builtin_amdgcn_sched_group_barrier(0x100, RPG, 0); }
            PIN();
            wait_lgkm();
            bar();
            ca = ca + SA == NSA * SA ? 0 : ca + SA;
            cb = cb + SB == NSB * SB ? 0 : cb + SB;
            rd(F0, ca, cb, 0);
            mm(F1);
#pragma unroll
            for (int i = 0; i < NMM; ++i) { __builtin_amdgcn_sched_group_barrier(0x008, 1, 1); if (i * RPG < NRD) __builtin_amdgcn_sched_group_barrier(0x100, RPG, 1); }
            PIN();
            wait_lgkm();
        }
        rd(F1, ca, cb, 1);
        PIN();
        mm(F0);
        PIN();
        mm(F1);
        if (wv == 0) STAMP(2);
        constexpr int ROWB = NFR * 32;
        constexpr int CPR = ROWB / 16;
        constexpr int ESLOT = (NSTEP - 1) % NSB >= 2 ? 0 : 2;
        LAS char* eb = smem + BOFF + ESLOT * SB + wv * (SPW * 16 * ROWB);
        static_assert(4 * SPW * 16 * ROWB <= 2 * SB && NSB >= 4, "epilogue space");
#pragma unroll
        for (int i = 0; i < SPW; ++i)
#pragma unroll
            for (int nb = 0; nb < NFR; ++nb) {
                u32x2 o;
                o.x = pack_bf16(acc[i][nb][0], acc[i][nb][1]);
                o.y = pack_bf16(acc[i][nb][2], acc[i][nb][3]);
                *(LAS u32x2*)(eb + (i * 16 + fr) * ROWB + (((nb * 2 + (g >> 1)) ^ (fr & (CPR - 1))) << 4) + (g & 1) * 8) = o;
            }
        if (NH == 2 && packed) {
            if (S == 4) { wait_lgkm(); bar(); }
            return S == 4 && wm ? 0u : (unsigned)(size_t)eb;
        }
        constexpr int RPI = 64 / CPR;
        constexpr int NST = SPW * 16 / RPI;
        u32x4 ov[NST];
        GAS char* od[NST];
        GAS char* obase = outp + (NH == 2 ? wn * 128 : ch * 128 + wn * 64) + (lane & (CPR - 1)) * 16;
#pragma unroll
        for (int n = 0; n < NST; ++n) {
            const int row = n * RPI + lane / CPR;
            ov[n] = *(LAS u32x4*)(eb + row * ROWB + (((lane & (CPR - 1)) ^ (row & (CPR - 1))) << 4));
            od[n] = obase + (size_t)((rowbase[(n * RPI) / 16] + (row & 15)) * orow);
        }
        if (wv == 0) STAMP(5);
        if (win == 1) {
#pragma unroll
            for (int n = 0; n < NST; ++n) { if (n >= 1) wait_vm<0>(); st16_wt(od[n], ov[n]); }
        } else if (win == 2) {
#pragma unroll
            for (int n = 0; n < NST; ++n) { if (n >= 2) wait_vm<1>(); st16_wt(od[n], ov[n]); }
        } else {
#pragma unroll
            for (int n = 0; n < NST; ++n) st16_wt(od[n], ov[n]);
        }
        if (wremote) {
            wait_vm<0>();
            if (lane == 0) __hip_atomic_fetch_add((gu32*)cntp, 1u, __ATOMIC_RELAXED, __HIP_MEMORY_SCOPE_SYSTEM);
        }
        if (wv == 0) STAMP(3);
    }
    return 0;
}

__global__ void __launch_bounds__(512, 1) __attribute__((amdgpu_num_vgpr(240))) ag_gemm_kernel(const Params P) {
    __shared__ __attribute__((aligned(1024))) char smem_raw[LDS_BYTES];
    LAS char* smem = (LAS char*)smem_raw;
    const unsigned long long t64 = now();
    const unsigned tstart = (unsigned)t64;
    const int b = blockIdx.x;
    if (__builtin_expect(b < NCOMM, 1)) {
        const int wv = rfl(threadIdx.x >> 6), g = wv * NCOMM + b;
        if (wv < NCW) comm_role(P, smem + wv * 16384, t64, wv, 1 + g % 7, g / 7);
        return;
    }
    GAS char* myws = (GAS char*)P.ws[0];
    gu32* cc = (gu32*)(myws + CALL_OFF + b * 64);
    const unsigned role = rfl(P.role[b - NCOMM]);
    const int type = role & 15;
    if (type == R_UNPACK) {
        const int wv = rfl(threadIdx.x >> 6), nun = P.nunpack & 255;
        const unsigned call = rfl(ld_agent(cc + wv)) + 1u;
        unpack_role(P, myws, tstart, call, (int)(role >> 4) + nun * wv, nun * 8);
        if (wv == 7) STAMP(4);
        if ((threadIdx.x & 63) == 0) st_agent(cc + wv, call);
    } else if (type != R_IDLE) {
        const unsigned call = rfl(ld_agent(cc)) + 1u;
        sleep_until(tstart + (P.tlead >> 16));
        if (threadIdx.x == 0) STAMP(0);
        const int fs = (role >> 4) & 255, cg = (role >> 12) & 7, ch = (role >> 15) & 1, win = (role >> 18) & 3;
        const bool packed = (role >> 16) & 1;
        const int po = (fs >> 1) + rfl(threadIdx.x >> 6);
        const GAS char* pk = (const GAS char*)P.ws[cg] + (size_t)(call & 1) * BUF_BYTES + pko_off(cg, po - (int)((P.nunpack >> 16) & 255));
        asm volatile("" : "+s"(pk));
        unsigned ea;
        if (type == R_T128) ea = tile_role<8, 2, AGK_RA>(P, smem, call, fs, cg, 0, win, role >> 28, packed);
        else if (type == R_T128H) ea = tile_role<8, 1, AGK_RH>(P, smem, call, fs, cg, ch, win, role >> 28, false);
        else if (type == R_T64W) ea = tile_role<4, 2, AGK_RW>(P, smem, call, fs, cg, 0, win, role >> 28, packed);
        else ea = tile_role<4, 1, AGK_RQ>(P, smem, call, fs, cg, ch, win, role >> 28, false);
        if (ea) {
            if (threadIdx.x == 0) STAMP(5);
            out_role(P, ea, type == R_T128 ? 4096 : 8192, call, pk, cg, po, win != 0);
            if (threadIdx.x == 0) STAMP(3);
        }
        if (threadIdx.x == 256) st_agent(cc, call);
    }
}

}

#define TUNE(i, dflt) (dflt)

extern "C" void kernel_launch(int rank, int nranks, ncclComm_t comm,
                              hipStream_t stream,
                              void* const* d_in, const int* in_sizes, int n_in,
                              void* d_out, int out_size,
                              const int* in_dtypes, int out_dtype,
                              void* const* d_out_all,
                              void* const* d_ws_all, size_t ws_bytes) {
    using namespace agk;
    (void)comm; (void)in_dtypes; (void)out_dtype; (void)d_out;
    if (nranks != NR || (unsigned)rank >= (unsigned)NR || n_in < 2 || ws_bytes < WS_NEED ||
        in_sizes[0] != MSH * KD || in_sizes[1] != KD * NSH || out_size != NR * MSH * NSH) {
        fprintf(stderr, "ag_gemm: unexpected problem shape\n");
        return;
    }
    Params P;
    P.x = (const char*)d_in[0];
    P.w = (const char*)d_in[1];
    P.rank = rank;
    for (int j = 0; j < NR; ++j) { P.ws[j] = (char*)d_ws_all[rank ^ j]; P.out[j] = (char*)d_out_all[rank ^ j]; }
    int sega = TUNE(0, 11);
    int blate = TUNE(1, 7);
    int wlead = TUNE(2, 4);
    int rt_b = TUNE(3, 1), rt_c = TUNE(4, 1), rt_a = 0, rt_w = TUNE(6, 0);
    int split_c = TUNE(7, 5);
    int split_a = TUNE(8, 0);
    int small_b = TUNE(9, 0);
    const int segb = HS - sega;
    const int na_t = (3 * sega + 7) / 8, nc_t = (4 * AS + 7) / 8, nb_t = (3 * segb + 7) / 8, no_t = 16 - na_t - nc_t - nb_t;
    const int ne_t = no_t + na_t;
    bool ok = no_t >= 0;
    int n = 0, own = 0;
    for (int i = 0; i < no_t * 8; ++i) P.strip[n++] = (unsigned char)(own++);
    for (int s = 0; s < sega; ++s) for (int j = 1; j < 4; ++j) P.strip[n++] = (unsigned char)((j << 5) | s);
    while (n % 8) P.strip[n++] = (unsigned char)(own++);
    for (int s = HS; s < NSTRIP; ++s) for (int j = 4; j < 8; ++j) P.strip[n++] = (unsigned char)((j << 5) | s);
    while (n % 8) P.strip[n++] = (unsigned char)(own++);
    for (int s = sega; s < HS; ++s) for (int j = 1; j < 4; ++j) P.strip[n++] = (unsigned char)((j << 5) | s);
    while (n % 8) P.strip[n++] = (unsigned char)(own++);
    ok = ok && n == 128 && own == HS;
    int q = 0;
    const int t_first = TUNE(16, 160) * 16, t_pk = TUNE(17, 169), t_raw = TUNE(18, 232);
    const int skew_b = TUNE(20, 16), skew_c = TUNE(21, 12);
    const int opk = TUNE(19, 3);
    const int op1 = opk & 2 ? ne_t + nc_t : ne_t, op0 = !(opk & 1) ? ne_t : op1 > NP_OUT / 4 ? op1 - NP_OUT / 4 : 0;
    const int n_slow = TUNE(25, 0), t_slow = TUNE(15, 184);
    int tq = t_first;
    for (int i = 0; i < 2 * WPL * NRND; ++i) P.ord[i] = 0;
    auto emit = [&](unsigned e) { if (q < NP_IN + NP_XI) P.ord[q < NP_IN ? ord_at(0, q) : ord_at(1, q - NP_IN)] = e | (unsigned)(tq >> 4) << 16; ++q; tq += (e & 512u) ? t_raw : tq < t_first + n_slow * t_slow ? t_slow : t_pk; };
    int nextk[NSTRIP] = {0};
    auto xchunk = [&](int s0, int s1, int rt, int skew) {
        for (int i = s0; i < s1; ++i) {
            int b = -1;
            for (int s = s0; s < s1; ++s) if (nextk[s] < NKC && (b < 0 || nextk[s] * 64 + s * skew < nextk[b] * 64 + b * skew)) b = s;
            emit((unsigned)nextk[b] | (b << 4) | (nextk[b] >= NKC - rt ? 512u : 0u));
            ++nextk[b];
        }
    };
    auto wchunk = [&](int kc) { for (int i = 0; i < 8; ++i) emit((unsigned)(kc * 8 + i) | 1024u | (kc >= NKC - rt_w ? 512u : 0u)); };
    for (int ka = 0, kb = 0; kb < NKC; ) {
        if (ka < NKC && kb * NKC >= ka * (NKC - blate)) { xchunk(0, sega, rt_a, 0); ++ka; }
        else { xchunk(sega, HS, rt_b, skew_b); ++kb; }
    }
    ok = ok && q == NP_IN;
    tq = t_first;
    for (int k = -wlead; k < NKC; ++k) {
        if (k + wlead < NKC) wchunk(k + wlead);
        if (k >= 0) xchunk(HS, NSTRIP, rt_c, skew_c);
    }
    ok = ok && q == NP_IN + NP_XI;
    int fill[8] = {0, 0, 0, 0, 0, 0, 0, 0};
    unsigned role[NWG];
    for (int i = 0; i < NWG; ++i) role[i] = R_IDLE;
    auto put = [&](int k, unsigned v) { if (fill[k] < NWG / 8) role[fill[k]++ * 8 + k] = v; else ok = false; };
    auto code = [&](int type, int fs, int cg, int ch) { return (unsigned)type | (fs << 4) | (cg << 12) | (ch << 15); };
    for (int i = 0; i < NCOMM; ++i) put(i & 7, R_COMM | (i << 4));
    unsigned adds = 0;
    for (int cg = 0; cg < 8; ++cg) {
        const bool rem = cg >= 4;
        for (int r = 0; r < 16; ++r) {
            const int k = (cg & 3) + 4 * (r & 1);
            const bool early = r < ne_t, last = r >= ne_t + nc_t;
            const bool split = last ? rem || r - ne_t - nc_t >= TUNE(29, 3) : (rem ? (early ? r >= ne_t - split_a : TUNE(5, 1) ? r - ne_t >= nc_t - split_c : r - ne_t < split_c) : (!early && r - ne_t < TUNE(22, 0)));
            const bool small = last && r - ne_t - nc_t < (rem ? small_b >> 4 : small_b & 15);
            const unsigned win = (unsigned)(!rem ? 0 : early ? TUNE(14, 2) : last ? TUNE(12, 0) : split ? TUNE(13, 0) : TUNE(11, 0)) << 18 |
                                 (rem && r >= op0 && r < op1 ? 1u << 16 : 0u);
            int nw = 0;
            auto rot = [&](int who, int of) { return (unsigned)(TUNE(26, 1) ? (of == 2 ? (who & 1) * 2 + ((r >> 1) & 1) : (who ^ (((r >> 1) & 1) * 2)) & 3) : 0) << 28; };
            if (small) for (int h = 0; h < 4; ++h) { put(k, rot(rem * 2 + (h & 1), 4) | win | code(R_T64H, r * 8 + (h >> 1) * 4, cg, h & 1)); nw += 4; }
            else if (split && (TUNE(23, 3) >> (last ? 0 : 1) & 1)) for (int h = 0; h < 2; ++h) { put(k, (unsigned)((rem * 2 + h * TUNE(24, 1)) ^ ((r >> 1) & 1)) << 28 | win | code(R_T64W, r * 8 + h * 4, cg, 0)); nw += 4; }
            else if (split) for (int ch = 0; ch < 2; ++ch) { put(k, rot(rem * 2 + ch, 4) | win | code(R_T128H, r * 8, cg, ch)); nw += 4; }
            else { put(k, rot(rem, 2) | win | code(R_T128, r * 8, cg, 0)); nw += 4; }
            if (cg == 4 && !(win >> 16 & 1)) adds += nw;
            ok = ok && !((win >> 16 & 1) && split && !(TUNE(23, 3) & 2));
        }
    }
    unsigned nun = 0;
    for (int i = 0; i < NWG; ++i) if (role[i] == R_IDLE && nun < (unsigned)TUNE(10, 64)) role[i] = R_UNPACK | (nun++ << 4);
    for (int i = NCOMM; i < NWG; ++i) P.role[i - NCOMM] = role[i];
    P.tlead = TUNE(27, 24) | TUNE(28, 250) << 16;
    P.out_adds = adds; P.tpoll = TUNE(30, 100) | TUNE(31, 1) << 16; P.nunpack = nun | (unsigned)(op1 - op0) * 4 << 8 | (unsigned)op0 * 4 << 16;
    ok = ok && nun > 0;
    if (!ok) { fprintf(stderr, "ag_gemm: role table overflow\n"); return; }
    hipLaunchKernelGGL(ag_gemm_kernel, dim3(NWG), dim3(512), 0, stream, P);
    HIP_CHECK(hipGetLastError());
}
```

```cpp
#include <hip/hip_runtime.h>
#include <rccl/rccl.h>
#include <cstdio>
#include <cstdint>

#define HIP_CHECK(expr) do { hipError_t _e = (expr); if (_e != hipSuccess) { \
    fprintf(stderr, "HIP error %s at %s:%d\n", hipGetErrorString(_e), __FILE__, __LINE__); \
    return; } } while (0)

namespace agk {

typedef unsigned int u32x4 __attribute__((ext_vector_type(4)));
typedef unsigned int u32x3 __attribute__((ext_vector_type(3)));
typedef unsigned int u32x2 __attribute__((ext_vector_type(2)));
typedef short s16x4 __attribute__((ext_vector_type(4)));
typedef short s16x8 __attribute__((ext_vector_type(8)));
typedef __bf16 bf16x8 __attribute__((ext_vector_type(8)));
typedef float f32x4 __attribute__((ext_vector_type(4)));
#define LAS __attribute__((address_space(3)))
#define GAS __attribute__((address_space(1)))
typedef GAS unsigned int gu32;

constexpr int NR = 8;
constexpr int KD = 4096;
constexpr int MSH = 512;
constexpr int NSH = 512;
constexpr int NSTRIP = 32;
#define AGK_HS 21
constexpr int HS = AGK_HS;
constexpr int AS = NSTRIP - HS;
constexpr int NKC = 16;
constexpr int NSTEP = 64;
constexpr int NWP = 128;

#define AGK_NCOMM 56
#define AGK_NCW 4
constexpr int NCOMM = AGK_NCOMM;
constexpr int NCW = AGK_NCW;
constexpr int WPL = NCOMM * NCW / 7;
static_assert(NCOMM % 7 == 0, "a comm workgroup serves one link");

constexpr int NP_IN = HS * NKC;
constexpr int NP_XI = NWP + AS * NKC;
constexpr int NPMAX = NP_IN > NP_XI ? NP_IN : NP_XI;
constexpr int NRND = (NPMAX + WPL - 1) / WPL;
constexpr int NWG = 256;
enum { R_IDLE = 0, R_COMM = 1, R_T128 = 2, R_T128H = 3, R_T64H = 4, R_UNPACK = 5, R_T64W = 6 };

constexpr size_t BUF_BYTES = 16ull * KD * 2;
constexpr size_t STRIP_BYTES = 2 * BUF_BYTES;
constexpr int NXSLOT = 3 * HS + 4 * AS;
constexpr size_t XS_BYTES = (size_t)NXSLOT * STRIP_BYTES;
constexpr size_t WSL_BYTES = 8 * STRIP_BYTES;
constexpr int XF_STRIDE = 256;
constexpr size_t XFLAG_OFF = XS_BYTES + 4 * WSL_BYTES;
constexpr size_t WFLAG_OFF = XFLAG_OFF + 8 * 32 * XF_STRIDE;
constexpr size_t PK_OFF = XFLAG_OFF + STRIP_BYTES;
constexpr unsigned PK_BYTES = 6016;
constexpr int NP_OUT = 56;
constexpr unsigned PK_PER = BUF_BYTES / (PK_BYTES + 128);
constexpr size_t OCNT_OFF = PK_OFF + (3 * NP_IN + 4 * NP_XI + 4 * NP_OUT + PK_PER - 1) / PK_PER * STRIP_BYTES;
constexpr size_t TMO_OFF = OCNT_OFF + 4 * 256;
constexpr size_t CALL_OFF = TMO_OFF + 256;
constexpr size_t DBG_OFF = CALL_OFF + 256 * 64;
constexpr size_t DBG2_OFF = DBG_OFF + 256 * 128;
constexpr size_t WS_NEED = DBG2_OFF + 8 * 512 * 32;
static_assert(WFLAG_OFF + 4 * NWP * 4 <= XFLAG_OFF + BUF_BYTES, "flags");
static_assert(WS_NEED <= 65ull << 20, "workspace");

constexpr unsigned long long DEADLINE_TICKS = 30000000ull;
constexpr int LDS_BYTES = 163840;

struct Params {
    const char* x;
    const char* w;
    char* ws[NR];
    char* out[NR];
    int rank;
    unsigned out_adds;
    unsigned tpoll;
    unsigned nunpack;
    unsigned role[NWG - NCOMM];
    unsigned tlead;
    unsigned ord[2 * WPL * NRND];
    unsigned char strip[128];
};

static_assert(sizeof(Params) <= 4096, "kernel arguments");

__device__ __forceinline__ unsigned ld_sys(const gu32* p) { return __hip_atomic_load(p, __ATOMIC_RELAXED, __HIP_MEMORY_SCOPE_SYSTEM); }
__device__ __forceinline__ unsigned ld_agent(const gu32* p) { return __hip_atomic_load(p, __ATOMIC_RELAXED, __HIP_MEMORY_SCOPE_AGENT); }
__device__ __forceinline__ void st_agent(gu32* p, unsigned v) { __hip_atomic_store(p, v, __ATOMIC_RELAXED, __HIP_MEMORY_SCOPE_AGENT); }
__device__ __forceinline__ unsigned long long now() { return __builtin_amdgcn_s_memrealtime(); }
__device__ __forceinline__ unsigned rfl(unsigned v) { return __builtin_amdgcn_readfirstlane(v); }

__device__ __forceinline__ void st16_wt(GAS char* p, u32x4 v) {
    asm volatile("global_store_dwordx4 %0, %1, off sc0 sc1\n\ts_nop 1" :: "v"(p), "v"(v) : "memory");
}
template <int N> __device__ __forceinline__ void wait_vm() { asm volatile("s_waitcnt vmcnt(%0)" :: "i"(N) : "memory"); }
__device__ __forceinline__ void wait_lgkm() { asm volatile("" ::: "memory"); __builtin_amdgcn_s_waitcnt(0xC07F); asm volatile("" ::: "memory"); }
#define AGK_SL1 6
#define AGK_SL2 2
#define PIN() __builtin_amdgcn_sched_barrier(0)
__device__ __forceinline__ void sleep_until(unsigned t) {
    int rest = (int)(t - (unsigned)now());
    for (; rest > 48; rest = (int)(t - (unsigned)now())) __builtin_amdgcn_s_sleep(8);
    for (; rest > 2; rest -= 3) __builtin_amdgcn_s_sleep(1);
}
__device__ __forceinline__ void bar() { asm volatile("s_barrier" ::: "memory"); }

#define AGK_LDPOL "sc0 sc1"
__device__ __forceinline__ void glds2(const GAS char* base, unsigned v0, unsigned v1, unsigned lds_dst) {
    asm volatile("s_mov_b32 m0, %3\n\ts_nop 3\n\tglobal_load_lds_dwordx4 %1, %0 " AGK_LDPOL "\n\tglobal_load_lds_dwordx4 %2, %0 offset:1024 " AGK_LDPOL
                 :: "s"(base), "v"(v0), "v"(v1), "s"(lds_dst) : "m0", "memory");
}
template <int OFF> __device__ __forceinline__ void glds2o(const GAS char* base, unsigned v0, unsigned v1) {
    asm volatile("global_load_lds_dwordx4 %1, %0 offset:%3 " AGK_LDPOL "\n\tglobal_load_lds_dwordx4 %2, %0 offset:%4 " AGK_LDPOL
                 :: "s"(base), "v"(v0), "v"(v1), "i"(OFF), "i"(OFF + 1024) : "memory");
}
__device__ __forceinline__ void wait_vm_var(unsigned n) {
    asm volatile("s_getpc_b64 vcc\n\ts_add_u32 vcc_lo, vcc_lo, %0\n\ts_addc_u32 vcc_hi, vcc_hi, 0\n\ts_setpc_b64 vcc\n\t"
                 ".set agk_n, 0\n\t.rept 64\n\ts_waitcnt vmcnt(agk_n)\n\ts_branch 1f\n\t.set agk_n, agk_n + 1\n\t.endr\n1:"
                 :: "s"(n * 8 + 12) : "vcc", "scc", "memory");
}

__device__ __forceinline__ unsigned pack_bf16(float lo, float hi) {
    typedef float f32x2 __attribute__((ext_vector_type(2)));
    typedef __bf16 bf16x2 __attribute__((ext_vector_type(2)));
    f32x2 v = {lo, hi};
    bf16x2 r = __builtin_convertvector(v, bf16x2);
    return __builtin_bit_cast(unsigned, r);
}

__device__ __forceinline__ int xslot(int j, int s) { return j < 4 ? (j - 1) * HS + s : 3 * HS + (j - 4) * AS + (s - HS); }

#define STAMP(i) do { } while (0)
#define PSTAMP(j, p, i, t) do { } while (0)

enum { K_X = 0, K_W = 1 };
struct Piece { int kind, raw; unsigned soff, doff, foff, when; };

__host__ __device__ __forceinline__ int ord_at(int cross, int p) { return (cross * WPL + p % WPL) * NRND + p / WPL; }
__device__ __forceinline__ Piece locate(unsigned e, int j) {
    Piece r;
    r.when = e >> 16;
    const int s = (e >> 4) & 31, kc = e & 15, wp = (e >> 10) & 1 ? (int)(e & 127) : -1;
    r.raw = (e >> 9) & 1;
    if (wp >= 0) {
        r.kind = K_W;
        r.soff = (unsigned)(wp * 32 * (NSH * 2) + (j & 3) * 256);
        r.doff = (unsigned)XS_BYTES + (unsigned)(j - 4) * (unsigned)WSL_BYTES + (wp & 1) * 2 * (unsigned)STRIP_BYTES + (wp >> 1) * 2048;
        r.foff = (unsigned)WFLAG_OFF + ((j - 4) * NWP + (wp & 1) * 64 + ((wp >> 1) & 3) * 16 + (wp >> 3)) * 4;
    } else {
        r.kind = K_X;
        r.soff = (unsigned)(s * 16 * (KD * 2) + kc * 512);
        r.doff = (unsigned)xslot(j, s) * (unsigned)STRIP_BYTES + kc * 8192;
        r.foff = (unsigned)XFLAG_OFF + (j * 32 + s) * XF_STRIDE + kc * 4;
    }
    return r;
}
__device__ __forceinline__ unsigned pk_off(int j, int p) {
    const unsigned i = (j < 4 ? (j - 1) * NP_IN : 3 * NP_IN + (j - 4) * (NP_XI + NP_OUT)) + p;
    return (unsigned)PK_OFF + 3072u + i * PK_BYTES + i / PK_PER * ((unsigned)STRIP_BYTES - PK_PER * PK_BYTES);
}

__device__ __forceinline__ unsigned long_list(unsigned off, unsigned vlin) {
    return (vlin & 255u) + (vlin & 128u ? (PK_PER - 1) * PK_BYTES - (off - (unsigned)PK_OFF) % (unsigned)STRIP_BYTES / PK_BYTES * (PK_BYTES - 128) : 0u);
}
__device__ __forceinline__ unsigned pko_off(int j, int p) { return pk_off(j, NP_XI + p); }

template <int OFF> __device__ __forceinline__ void st_pk(const u32x4& v, unsigned voff, const GAS char* dst) {
    asm volatile("s_nop 3\n\tglobal_store_dwordx4 %1, %0, %2 offset:%3 sc0 sc1\n\ts_nop 1" :: "v"(v), "v"(voff), "s"(dst), "i"(OFF) : "memory");
}
template <int OFF> __device__ __forceinline__ void ld_c(u32x4& v, unsigned voff, const GAS char* src) {
    asm volatile("global_load_dwordx4 %0, %1, %2 offset:%3 sc0 sc1" : "=v"(v) : "v"(voff), "s"(src), "i"(OFF) : "memory");
}
__device__ __forceinline__ void flag_st(const GAS char* f, unsigned val, unsigned zero) {
    asm volatile("s_mov_b64 exec, 1\n\ts_nop 3\n\tglobal_store_dword %0, %1, %2 sc0 sc1\n\ts_mov_b64 exec, -1" :: "v"(zero), "v"(val), "s"(f) : "memory");
}
__device__ __forceinline__ unsigned perm(unsigned hi, unsigned lo, unsigned sel) { return __builtin_amdgcn_perm(hi, lo, sel); }
__device__ __forceinline__ unsigned bfi(unsigned m, unsigned a, unsigned b) { return (a & m) | (b & ~m); }
__device__ __forceinline__ unsigned rotr(unsigned v, unsigned n) { return __builtin_amdgcn_alignbit(v, v, n); }
__device__ __forceinline__ unsigned ffbl(unsigned v) { unsigned r; asm("v_ffbl_b32 %0, %1" : "=v"(r) : "v"(v)); return r; }
__device__ __forceinline__ unsigned pkmax(unsigned a, unsigned b) {
    typedef unsigned short u16x2 __attribute__((ext_vector_type(2)));
    return __builtin_bit_cast(unsigned, __builtin_elementwise_max(__builtin_bit_cast(u16x2, a), __builtin_bit_cast(u16x2, b)));
}
template <int CTRL> __device__ __forceinline__ unsigned dppmax(unsigned v) { return max(v, (unsigned)__builtin_amdgcn_update_dpp(0, (int)v, CTRL, 0xF, 0xF, true)); }
constexpr unsigned F_RAW = 0x80u;
constexpr int MAXEXC = 8;
__device__ __forceinline__ unsigned ximg0(unsigned lane, unsigned pitch) { return (lane >> 3) * pitch + ((lane & 7) ^ (lane >> 4)) * 16; }
__device__ __forceinline__ unsigned ximg1(unsigned lane, unsigned pitch) { return (8 + (lane >> 3)) * pitch + ((lane & 7) ^ (4 + (lane >> 4))) * 16 - 1024; }
__device__ __forceinline__ unsigned wimg0(unsigned lane, unsigned pitch) { return (lane >> 3) * pitch + ((lane & 7) ^ ((lane >> 3) & 2)) * 16; }
__device__ __forceinline__ unsigned wimg1(unsigned lane, unsigned pitch) { return (8 + (lane >> 3)) * pitch + ((lane & 7) ^ (((lane >> 3) & 2) | 4)) * 16 - 1024; }

__device__ __forceinline__ void fetch_piece(int kind, unsigned soff, const GAS char* sx, const GAS char* sw,
                                            unsigned vx0, unsigned vx1, unsigned vw0, unsigned vw1, unsigned l) {
    const bool kx = kind == K_X;
    const GAS char* s = (kx ? sx : sw) + soff;
    const unsigned a = kx ? vx0 : vw0, b = kx ? vx1 : vw1;
    const int q1 = kx ? 128 : 16 * NSH * 2, q2 = kx ? 256 : 128;
    glds2(s, a, b, l); glds2(s + q1, a, b, l + 2048); glds2(s + q2, a, b, l + 4096); glds2(s + q1 + q2, a, b, l + 6144);
}

__device__ __forceinline__ unsigned tag_of(unsigned call) { return ((call + 1u) >> 1) << 31; }

struct Packed { u32x4 W[5], ex; u32x2 W5; unsigned trl; unsigned long long exl; bool raw; };
__device__ __forceinline__ void pack(const u32x4 (&v)[8], Packed& Q, unsigned call) {
    const unsigned lane = threadIdx.x & 63, tagw = tag_of(call);
    u32x4 (&W)[5] = Q.W;
    u32x2 W5;
    u32x2 em;
    unsigned trl;
    unsigned mx = pkmax(pkmax(pkmax(v[0].x & 0x7FFF7FFFu, v[0].y & 0x7FFF7FFFu), pkmax(v[0].z & 0x7FFF7FFFu, v[0].w & 0x7FFF7FFFu)),
                        pkmax(pkmax(v[6].x & 0x7FFF7FFFu, v[6].y & 0x7FFF7FFFu), pkmax(v[6].z & 0x7FFF7FFFu, v[6].w & 0x7FFF7FFFu)));
    mx = max(mx >> 16, mx & 0xFFFFu) >> 8;
    mx = dppmax<0x143>(dppmax<0x142>(dppmax<0x118>(dppmax<0x114>(dppmax<0x112>(dppmax<0x111>(mx))))));
    unsigned fmt = max((unsigned)__builtin_amdgcn_readlane((int)mx, 63), 7u) - 7u;
    const unsigned B4 = fmt * 0x01010101u;
    unsigned accO = 0, accA = ~0u, N[8];
#pragma unroll
    for (int k = 0; k < 8; ++k) {
        const unsigned Ha = perm(v[k].y, v[k].x, 0x07050301u), Hb = perm(v[k].w, v[k].z, 0x07050301u);
        const unsigned da = (Ha | 0x80808080u) - B4, db = (Hb | 0x80808080u) - B4;
        accO |= da | db; accA &= da & db;
        W[k >> 1][(k & 1) * 2] = perm(v[k].y, v[k].x, 0x06040200u);
        W[k >> 1][(k & 1) * 2 + 1] = perm(v[k].w, v[k].z, 0x06040200u);
        N[k] = bfi(0x87878787u, bfi(0x80808080u, Ha, da), rotr(bfi(0x80808080u, Hb, db), 28));
    }
    em.x = ~bfi(0x33333333u, bfi(0x11111111u, N[0] >> 2, N[1] >> 1), bfi(0x44444444u, N[2], N[3] << 1));
    em.y = ~bfi(0x33333333u, bfi(0x11111111u, N[4] >> 2, N[5] >> 1), bfi(0x44444444u, N[6], N[7] << 1));
    W[4] = (u32x4){bfi(0x44444444u, N[6] << 2, N[0]), bfi(0x44444444u, N[6] << 1, N[1]), bfi(0x44444444u, N[6] >> 1, N[2]), bfi(0x44444444u, N[7] << 2, N[3])};
    W5 = (u32x2){bfi(0x44444444u, N[7] << 1, N[4]), bfi(0x44444444u, N[7] >> 1, N[5])};
    unsigned long long m = (unsigned long long)em.y << 32 | em.x;
    unsigned pos = 0;
#pragma unroll
    for (int i = 0; i < 4; ++i) {
        pos = __builtin_amdgcn_alignbit(min(ffbl((unsigned)m), ffbl((unsigned)(m >> 32)) | 32u), pos, 6);
        m &= m - 1;
    }
    const bool over = m != 0 || (int)em.y < 0;
    const unsigned long long bo = __ballot(over);
    const unsigned moff = __builtin_amdgcn_mbcnt_hi((unsigned)(bo >> 32), __builtin_amdgcn_mbcnt_lo((unsigned)bo, 0));
    pos = over ? moff << 14 | 0xFFF03F00u : pos;
    const int mto = over ? moff * 4 : 252;
    Q.ex = (u32x4){(unsigned)__builtin_amdgcn_ds_permute(mto, em.x), (unsigned)__builtin_amdgcn_ds_permute(mto, em.y), call, tagw};
    const unsigned cnt2 = __builtin_popcountll(bo);
    bool raw = cnt2 > 2 * MAXEXC;
    Q.exl = cnt2 > MAXEXC ? 0xFFFFull : 0xFFull;
    fmt |= cnt2 << 16;
    const unsigned long long bm = __ballot(((accO & 0x78787878u) | (~accA & 0x80808080u)) != 0);
    if (bm) {
        unsigned nb = 0, ib = 0, hv = 0, ib2 = 0, hv2 = 0;
#pragma unroll
        for (int i = 0; i < 16; ++i) {
            const unsigned H = (i & 1) ? perm(v[i >> 1].w, v[i >> 1].z, 0x07050301u) : perm(v[i >> 1].y, v[i >> 1].x, 0x07050301u);
            const bool bad = (((H | 0x80808080u) - B4) & 0xF8F8F8F8u) != 0x80808080u;
            hv2 = bad ? hv : hv2; ib2 = bad ? ib : ib2; hv = bad ? H : hv; ib = bad ? (unsigned)i : ib; nb += bad;
        }
        const int eto = nb ? __builtin_amdgcn_mbcnt_hi((unsigned)(bm >> 32), __builtin_amdgcn_mbcnt_lo((unsigned)bm, 0)) * 4 : 252;
        const unsigned long long b2 = __ballot(nb > 1);
        const unsigned cnt = __builtin_popcountll(bm), cntb = __builtin_popcountll(b2);
        const int eto2 = nb > 1 ? __builtin_amdgcn_mbcnt_hi((unsigned)(b2 >> 32), __builtin_amdgcn_mbcnt_lo((unsigned)b2, cnt)) * 4 : 252;
        const unsigned ez = __builtin_amdgcn_ds_permute(eto, hv) | __builtin_amdgcn_ds_permute(eto2, hv2);
        Q.ex.z = lane < MAXEXC ? ez : call;
        Q.ex.w = __builtin_amdgcn_ds_permute(eto, lane | (ib << 8)) | __builtin_amdgcn_ds_permute(eto2, lane | (ib2 << 8)) | tagw;
        raw = raw || cnt + cntb > MAXEXC || __ballot(nb > 2) != 0;
        fmt |= cnt << 8 | cntb << 24;
    }
    if (raw) fmt = F_RAW;
    trl = (tagw >> 30) + (__builtin_amdgcn_inverse_ballot_w64((unsigned long long)call << 32 | fmt) ? 1u : 0u);
#pragma unroll
    for (int k = 0; k < 5; ++k) { trl = __builtin_amdgcn_alignbit(trl, W[k].w, 31); W[k].w = bfi(0x7FFFFFFFu, W[k].w, tagw); }
    trl = __builtin_amdgcn_alignbit(trl, W5.y, 31); W5.y = bfi(0x7FFFFFFFu, W5.y, tagw);
    trl = __builtin_amdgcn_alignbit(trl, pos, 8);
    Q.W5 = W5; Q.trl = trl; Q.raw = raw;
}
__device__ __forceinline__ void send_packed(const Packed& Q, const GAS char* pk, unsigned off, unsigned vlin, bool few) {
    st_pk<-3072>(Q.W[0], vlin, pk); st_pk<-2048>(Q.W[1], vlin, pk);
    if (few) wait_vm<1>();
    st_pk<-1024>(Q.W[2], vlin, pk);
    if (few) wait_vm<1>();
    st_pk<0>(Q.W[3], vlin, pk);
    if (few) wait_vm<1>();
    st_pk<1024>(Q.W[4], vlin, pk);
    if (few) wait_vm<1>();
    asm volatile("global_store_dwordx2 %0, %1, %2 offset:2048 sc0 sc1" :: "v"(vlin >> 1), "v"(Q.W5), "s"(pk) : "memory");
    unsigned vx = vlin;
    if (__builtin_expect(Q.exl > 255, 0)) vx = long_list(off, vlin);
    asm volatile("s_mov_b64 exec, %3\n\tglobal_store_dwordx4 %0, %1, %2 offset:2816 sc0 sc1\n\ts_mov_b64 exec, -1" :: "v"(vx), "v"(Q.ex), "s"(pk), "s"(Q.exl) : "memory");
    asm volatile("global_store_dword %0, %1, %2 offset:2560 sc0 sc1" :: "v"(vlin >> 2), "v"(Q.trl), "s"(pk) : "memory");
}

__device__ __forceinline__ void comm_role(const Params& P, LAS char* smem, unsigned long long t64, int wslot, int j, int u) {
    const unsigned* ord = P.ord + ord_at(j >= 4, u);
    unsigned eA = ord[0], eB = ord[1], tlead = P.tlead;
    const GAS char *srcx = (const GAS char*)P.x, *srcw = (const GAS char*)P.w, *pws = (const GAS char*)P.ws[j];
    GAS char* myws = (GAS char*)P.ws[0];
    asm volatile("" : "+s"(eA), "+s"(eB), "+s"(tlead), "+s"(srcx), "+s"(srcw), "+s"(pws), "+s"(myws), "+s"(t64));
    const unsigned tstart = (unsigned)t64;
    gu32* cc = (gu32*)(myws + CALL_OFF + blockIdx.x * 64) + wslot;
    const int lane = threadIdx.x & 63;
    const int npiece = j < 4 ? NP_IN : NP_XI;
    const unsigned vlin = lane * 16;
    const unsigned vx0 = ximg0(lane, KD * 2), vx1 = ximg1(lane, KD * 2), vw0 = wimg0(lane, NSH * 2), vw1 = wimg1(lane, NSH * 2);
    const unsigned vblk = (lane >> 5) * 2048 + (lane & 31) * 16, vblw = (lane >> 5) * (unsigned)STRIP_BYTES + (lane & 31) * 16;
    const unsigned lds0 = (unsigned)(size_t)smem;
#define AGK_FETCH(q, slot) fetch_piece((q).kind, (q).soff, srcx, srcw, vx0, vx1, vw0, vw1, lds0 + (slot) * 8192)
    int pA = u;
    unsigned ccv;
    Piece A = locate(eA, j);
    if (u > 3) sleep_until(tstart + (u * (tlead & 0xFFFFu) >> 2));
    AGK_FETCH(A, 0);
    asm volatile("global_load_dword %0, %1, off sc1" : "=v"(ccv) : "v"(cc) : "memory");
    Piece B = locate(eB, j);
    asm volatile("s_waitcnt vmcnt(0)" : "+v"(ccv) :: "memory");
    AGK_FETCH(B, 1);
    const unsigned call = rfl(ccv) + 1u;
    if (wslot == 0) STAMP(0);
    const GAS char* dbuf = pws + (size_t)(call & 1) * BUF_BYTES;
    gu32* tmo = (gu32*)(myws + TMO_OFF);
    unsigned zero = 0, callv = call;
    asm volatile("" : "+v"(zero), "+v"(callv));
    int slot = 0;
    for (;;) {
        u32x4 v[8];
#pragma unroll
        for (int k = 0; k < 8; ++k) v[k] = *(LAS u32x4*)(smem + slot * 8192 + (k >> 2) * 4096 + (k & 3) * 512 + vblk);
        Packed Q;
        if (!A.raw) pack(v, Q, call);
        else {
#pragma unroll
            for (int k = 0; k < 5; ++k) Q.W[k] = v[k];
            Q.W5 = (u32x2){v[5].x, v[5].y};
            Q.ex = v[6]; Q.trl = 0; Q.exl = 0; Q.raw = true;
        }
        const GAS char* pk = dbuf + pk_off(j, pA);
        const Piece C = locate(ord[pA + 2 * WPL < npiece ? pA / WPL + 2 : 0], j);
        asm volatile("" :: "v"(Q.W[0]), "v"(Q.W[1]), "v"(Q.W[2]), "v"(Q.W[3]), "v"(Q.W[4]), "v"(Q.W5), "v"(v[6]), "v"(v[7]), "v"(Q.trl), "v"(Q.ex),
                     "s"(pk), "s"(C.soff) : "memory");
        sleep_until(tstart + A.when);
        if (!A.raw) send_packed(Q, pk, (unsigned)(pk - dbuf), vlin, false);
        if (Q.raw) {
            const GAS char* d0 = dbuf + A.doff;
            const GAS char* d1 = d0 + (A.kind == K_W ? 4 * (unsigned)STRIP_BYTES : 4096u);
            const unsigned vd = A.kind == K_W ? vblw : vblk;
            st_pk<0>(v[0], vd, d0); st_pk<512>(v[1], vd, d0); st_pk<1024>(v[2], vd, d0); st_pk<1536>(v[3], vd, d0);
            st_pk<0>(v[4], vd, d1); st_pk<512>(v[5], vd, d1); st_pk<1024>(v[6], vd, d1); st_pk<1536>(v[7], vd, d1);
        }
        AGK_FETCH(C, slot);
        if (Q.raw) { wait_vm<8>(); flag_st(dbuf + A.foff, callv, zero); }
        PSTAMP(j, pA, 0, tstart + A.when); PSTAMP(j, pA, 1, tb); PSTAMP(j, pA, 2, now());
        pA += WPL; A = B; B = C; slot ^= 1;
        if (pA >= npiece) break;
        wait_vm<16>();
    }
    wait_vm<0>();
    STAMP(j < 4 ? 1 : 3);
    if (j == 1 && u == 0) {
        if (lane < 4) {
            const gu32* c = (const gu32*)(myws + OCNT_OFF + lane * 256);
            const unsigned need = call * P.out_adds;
            const unsigned long long t0 = now();
            bool bad = false;
            unsigned spin = 0;
            while (ld_sys(c) < need) {
                if ((++spin & 63) == 0 && (ld_agent(tmo) == call || now() - t0 > DEADLINE_TICKS)) { st_agent(tmo, call); bad = true; break; }
                __builtin_amdgcn_s_sleep(1);
            }
            if (bad) {
                __builtin_amdgcn_s_sleep(127);
                __hip_atomic_store((GAS unsigned short*)(P.out[0] + (size_t)(P.rank * MSH) * (NSH * 2)), (unsigned short)0x7FC0, __ATOMIC_RELAXED, __HIP_MEMORY_SCOPE_AGENT);
            }
        }
        STAMP(2);
    }
    if (lane == 0) st_agent(cc, call);
}

__device__ __forceinline__ void out_rows(const Params& P, const GAS char* (&d)[4], int jd, int j, int po) {
#pragma unroll
    for (int i = 0; i < 4; ++i) {
        const int code = rfl(P.strip[(po >> 1) * 4 + i]);
        d[i] = (const GAS char*)P.out[jd] + (unsigned)(((P.rank ^ jd ^ j ^ (code >> 5)) * MSH + (code & 31) * 16) * (NSH * 2) + (j & 3) * 256 + (po & 1) * 128);
    }
}
__device__ __forceinline__ void out_role(const Params& P, unsigned ea, unsigned gs, unsigned call, const GAS char* pk, int j, int po, bool few) {
    const unsigned lane = threadIdx.x & 63;
    u32x4 v[8];
#pragma unroll
    for (int n = 0; n < 8; ++n) v[n] = *(LAS u32x4*)(size_t)(ea + (n >> 2) * gs + (n & 3) * 1024 + (lane >> 3) * 128 + (((lane & 7) ^ (lane >> 3)) << 4));
    Packed Q;
    pack(v, Q, call);
    if (Q.raw) {
        const GAS char* d[4];
        out_rows(P, d, j, j, po);
        const unsigned vA = (lane >> 3) * (NSH * 2) + (lane & 7) * 16, vB = vA + 8 * (NSH * 2);
#pragma unroll
        for (int n = 0; n < 8; ++n) st_pk<0>(v[n], n & 1 ? vB : vA, d[n >> 1]);
        wait_vm<0>();
    }
    send_packed(Q, pk, pko_off(j, po - (int)((P.nunpack >> 16) & 255)), lane * 16, few);
    wait_vm<0>();
}

__device__ __forceinline__ void unpack_role(const Params& P, GAS char* myws, unsigned tstart, unsigned call, int w, int nw) {
    const GAS char* buf = myws + (size_t)(call & 1) * BUF_BYTES;
    gu32* tmo = (gu32*)(myws + TMO_OFF);
    const int lane = threadIdx.x & 63;
    const unsigned vlin = lane * 16, tagw = tag_of(call);
    const unsigned vblk = (lane >> 5) * 2048 + (lane & 31) * 16, vblw = (lane >> 5) * (unsigned)STRIP_BYTES + (lane & 31) * 16;
    unsigned zero = 0, callv = call;
    asm volatile("" : "+v"(zero), "+v"(callv));
    const unsigned tfirst = tstart + (P.tpoll & 0xFFFFu);
    const bool spec = P.tpoll >> 16;
    for (int q = w; ; q += nw) {
        const int p = q / 7, j = 1 + q - p * 7;
        const bool outp = q >= NPMAX * 7;
        const GAS char *pk, *d[4];
        unsigned vA, vB, foff = 0;
        if (!outp) {
            if (p >= (j < 4 ? NP_IN : NP_XI)) continue;
            const Piece A = locate(P.ord[ord_at(j >= 4, p)], j);
            if (A.raw) continue;
            pk = buf + pk_off(j, p);
            d[0] = buf + A.doff; d[1] = d[0] + 1024;
            d[2] = d[0] + (A.kind == K_W ? 4 * (unsigned)STRIP_BYTES : 4096u); d[3] = d[2] + 1024;
            vA = A.kind == K_W ? vblw : vblk; vB = vA + 512;
            foff = A.foff;
            sleep_until(tfirst + A.when);
        } else {
            const int i = q - NPMAX * 7, jo = 4 + (i & 3), po = (i >> 2) + (int)((P.nunpack >> 16) & 255);
            if (i >= (int)((P.nunpack >> 8) & 255) * 4) break;
            pk = buf + pko_off(jo, i >> 2);
            out_rows(P, d, 0, jo, po);
            vA = (lane >> 3) * (NSH * 2) + (lane & 7) * 16; vB = vA + 8 * (NSH * 2);
        }
        unsigned long long t0 = 0;
        unsigned spin = 0, trl, fmt = 0, vex = vlin & (MAXEXC * 16 - 1);
        bool more = false;
        u32x4 W[5], ex; u32x2 W5;
        bool data = spec && !outp;
        for (;;) {
            asm volatile("s_nop 4\n\tglobal_load_dword %0, %1, %2 offset:2560 sc0 sc1" : "=v"(trl) : "v"(vlin >> 2), "s"(pk) : "memory");
            if (data) {
                ld_c<2816>(ex, vex, pk);
                asm volatile("global_load_dwordx2 %0, %1, %2 offset:2048 sc0 sc1" : "=v"(W5) : "v"(vlin >> 1), "s"(pk) : "memory");
                ld_c<1024>(W[4], vlin, pk);
                ld_c<-3072>(W[0], vlin, pk); ld_c<-2048>(W[1], vlin, pk); ld_c<-1024>(W[2], vlin, pk); ld_c<0>(W[3], vlin, pk);
            }
            asm volatile("s_waitcnt vmcnt(0)" : "+v"(trl), "+v"(W[0]), "+v"(W[1]), "+v"(W[2]), "+v"(W[3]), "+v"(W[4]), "+v"(W5), "+v"(ex) :: "memory");
            const unsigned long long fc = __ballot((int)(trl << 1) < 0);
            if (__ballot((int)(trl ^ tagw) < 0) == 0 && (unsigned)(fc >> 32) == call) {
                fmt = (unsigned)fc;
                if (fmt & F_RAW) break;
                if (data && __ballot((int)((W[0].w ^ tagw) | (W[1].w ^ tagw) | (W[2].w ^ tagw) | (W[3].w ^ tagw) | (W[4].w ^ tagw) | (W5.y ^ tagw) | (ex.w ^ tagw)) < 0) == 0 &&
                    (((fmt >> 16) & 31) <= MAXEXC || (more && (__ballot(ex.z != call) & 0xFF00u) == 0))) break;
                if (((fmt >> 16) & 31) > MAXEXC) { more = true; vex = long_list((unsigned)(pk - buf), vlin); }
                data = true;
            } else __builtin_amdgcn_s_sleep(AGK_SL2);
            if ((++spin & 63) == 0) {
                if (t0 == 0) t0 = now();
                if (rfl(ld_agent(tmo)) == call || now() - t0 > DEADLINE_TICKS) {
                    st_agent(tmo, call);
                    if (outp) st_pk<0>((u32x4){0x7FC07FC0u, 0x7FC07FC0u, 0x7FC07FC0u, 0x7FC07FC0u}, vA, d[0]);
                    fmt = F_RAW;
                    break;
                }
            }
        }
        if (!outp) { PSTAMP(j, p, 3, now()); PSTAMP(j, p, 7, spin); PSTAMP(j, p, 4, now()); }
        if (fmt & F_RAW) continue;
        const int cnt = (fmt >> 8) & 31;
#pragma unroll
        for (int k = 0; k < 5; ++k) W[k].w = bfi(0x7FFFFFFFu, W[k].w, trl << (2 + k));
        W5.y = bfi(0x7FFFFFFFu, W5.y, trl << 7);
        const unsigned long long m = 1ull << (trl & 63) | 1ull << ((trl >> 6) & 63) | 1ull << ((trl >> 12) & 63) | 1ull << ((trl >> 18) & 63);
        unsigned m0 = (unsigned)m, m1 = (unsigned)(m >> 32) & 0x7FFFFFFFu;
        if (fmt & 0x1F0000u) {
            const int src = (trl >> 4) & 0x3C;
            const unsigned e0 = __builtin_amdgcn_ds_bpermute(src, ex.x), e1 = __builtin_amdgcn_ds_bpermute(src, ex.y);
            const bool over = (trl & 0x83Fu) == 63u;
            m0 = over ? e0 : m0; m1 = over ? e1 : m1;
        }
        unsigned N[8];
        N[0] = bfi(0x44444444u, ~(m0 << 2), W[4].x); N[1] = bfi(0x44444444u, ~(m0 << 1), W[4].y); N[2] = bfi(0x44444444u, ~m0, W[4].z); N[3] = bfi(0x44444444u, ~(m0 >> 1), W[4].w);
        N[4] = bfi(0x44444444u, ~(m1 << 2), W5.x); N[5] = bfi(0x44444444u, ~(m1 << 1), W5.y);
        N[6] = bfi(0x33333333u, bfi(0x11111111u, W[4].x >> 2, W[4].y >> 1), bfi(0x44444444u, ~m1, W[4].z << 1));
        N[7] = bfi(0x33333333u, bfi(0x11111111u, W[4].w >> 2, W5.x >> 1), bfi(0x44444444u, ~(m1 >> 1), W5.y << 1));
        const unsigned B4 = (fmt & 127u) * 0x01010101u;
        unsigned H[16];
#pragma unroll
        for (int k = 0; k < 8; ++k) {
            H[2 * k] = (N[k] & 0x87878787u) + B4;
            H[2 * k + 1] = (rotr(N[k], 4) & 0x87878787u) + B4;
        }
        for (int e = 0, e1 = cnt; e < e1; e1 = cnt + (int)(fmt >> 24)) {
            unsigned tag = 255, val = 0;
            for (; e < e1; ++e) {
                const unsigned id = __builtin_amdgcn_readlane(ex.w, e) & 0xFFFu, hv = __builtin_amdgcn_readlane(ex.z, e);
                const bool mine = (id & 63) == (unsigned)lane;
                tag = mine ? id >> 8 : tag; val = mine ? hv : val;
            }
#pragma unroll
            for (int i = 0; i < 16; ++i) H[i] = tag == (unsigned)i ? val : H[i];
        }
#define AGK_DST(k) { const unsigned l0 = W[k >> 1][(k & 1) * 2], l1 = W[k >> 1][(k & 1) * 2 + 1]; u32x4 o; \
            o.x = perm(H[2 * k], l0, 0x05010400u); o.y = perm(H[2 * k], l0, 0x07030602u); \
            o.z = perm(H[2 * k + 1], l1, 0x05010400u); o.w = perm(H[2 * k + 1], l1, 0x07030602u); st_pk<0>(o, k & 1 ? vB : vA, d[k >> 1]); }
        AGK_DST(0) AGK_DST(1) AGK_DST(2) AGK_DST(3) AGK_DST(4) AGK_DST(5) AGK_DST(6) AGK_DST(7)
        if (outp) continue;
        PSTAMP(j, p, 5, now());
        wait_vm<0>(); flag_st(buf + foff, callv, zero);
        PSTAMP(j, p, 6, now());
    }
    wait_vm<0>();
}

#define AGK_RPG 2
#define AGK_RA 4, 6
#define AGK_RH 4, 12
#define AGK_RQ 8, 12
#define AGK_RW 6, 7
#define AGK_LOOK 3
template <int NB, int NP, int NSLOT, int SLOT_BYTES>
__device__ __forceinline__ void loader(GAS char* myws, unsigned call, const GAS char* const (&src)[NP], const int (&shift)[NP], const unsigned (&v0)[NB],
                                       const unsigned (&v1)[NB], const GAS char* fp, bool have, unsigned ldsw, unsigned rot, bool first) {
    constexpr int NI = 2 * NB;
    static_assert((NSLOT - 2) * NI + 8 < 64 && (NSLOT - 1) * NI + 10 < 64, "vmcnt range");
    gu32* tmo = (gu32*)(myws + TMO_OFF);
    const unsigned ldsend = ldsw + NSLOT * SLOT_BYTES;
    const unsigned long long act = ~__ballot(have);
    asm volatile("v_mov_b32 v250, %0" :: "v"(have ? ~0u : 0u) : "v250", "memory");
    int mark = 0;
    unsigned npoll = 0;
    auto poll = [&]() {
        ++npoll;
        asm volatile("s_mov_b64 exec, %1\n\tglobal_load_dword v250, %0, off sc0 sc1\n\ts_mov_b64 exec, -1" :: "v"(fp), "s"(act) : "v250", "memory");
    };
    auto eval = [&]() {
        unsigned long long bx;
        asm volatile("v_cmp_gt_u32 %0, %1, v250" : "=s"(bx) : "s"(call) : "memory");
        const unsigned m = (unsigned)bx | (unsigned)(bx >> 32);
        mark = __builtin_ctz(m | m >> 16 | 0x10000u) * 4;
    };
    unsigned ldsi = ldsw, kt = 0;
    const GAS char* ptr[NP];
#pragma unroll
    for (int i = 0; i < NP; ++i) ptr[i] = src[i] + (rot << shift[i]);
    auto issue = [&]() {
        asm volatile("s_mov_b32 m0, %0\n\ts_nop 0" :: "s"(ldsi) : "m0", "memory");
        glds2o<-4096>(ptr[0], v0[0], v1[0]);
        glds2o<-2048>(ptr[1 % NP], v0[1], v1[1]);
        if (NB == 4) { glds2o<0>(ptr[2 % NP], v0[2 % NB], v1[2 % NB]); glds2o<2048>(ptr[3 % NP], v0[3 % NB], v1[3 % NB]); }
        const unsigned k = ++kt ^ rot;
#pragma unroll
        for (int i = 0; i < NP; ++i) { ptr[i] = src[i] + (k << shift[i]); asm volatile("" : "+s"(ptr[i])); }
        ldsi = ldsi + SLOT_BYTES == ldsend ? ldsw : ldsi + SLOT_BYTES;
        asm volatile("" : "+s"(ldsi));
    };
    int ti = 0, tb = 0;
    unsigned hist = 0;
#pragma nounroll
    for (;;) {
        if (mark < NSTEP && mark - ti <= AGK_LOOK) eval();
        const int lim = mark < tb + NSLOT - 1 ? mark : tb + NSLOT - 1;
#pragma nounroll
        for (; ti < lim; ++ti) { issue(); hist = hist << 4 | (npoll & 15u); }
        if (ti == tb) {
            unsigned long long t0 = 0;
            for (unsigned spin = 1; ti >= mark; ++spin) {
                wait_vm<8>();
                poll();
                __builtin_amdgcn_s_sleep(AGK_SL1);
                if ((spin & 63) == 0) {
                    if (t0 == 0) t0 = now();
                    if (rfl(ld_agent(tmo)) == call || now() - t0 > DEADLINE_TICKS) break;
                }
                eval();
            }
            if (ti >= mark) break;
            continue;
        }
        const int run = (mark < NSTEP ? mark - AGK_LOOK : NSTEP) - ti;
        if (run > 0) {
#pragma nounroll
            for (int i = run; i > 0; --i) { wait_vm<(NSLOT - 2) * NI>(); bar(); issue(); }
            ti += run; tb += run;
            const unsigned rep = (npoll & 15u) * 0x11111111u;
            hist = run >= 8 ? rep : hist << (4 * run) | (rep & ((1u << (4 * run)) - 1u));
        }
        if (mark < NSTEP) poll();
        const unsigned n = ti - tb - 1;
        wait_vm_var(n * NI + (n < 8 ? (npoll - (hist >> (4 * n))) & 15u : 0u));
        bar();
        if (++tb == NSTEP) break;
    }
    if (tb < NSTEP) {
        st_agent(tmo, call);
        const u32x4 nan = {0x7FC07FC0u, 0x7FC07FC0u, 0x7FC07FC0u, 0x7FC07FC0u};
        for (; tb < NSTEP; ++tb) {
#pragma unroll
            for (int i = 0; i < NI; ++i) *(LAS u32x4*)(size_t)(ldsi - 4096 + i * 1024 + (threadIdx.x & 63) * 16) = nan;
            wait_lgkm();
            bar();
            ldsi = ldsi + SLOT_BYTES == ldsend ? ldsw : ldsi + SLOT_BYTES;
        }
    }
    if (first) STAMP(4);
    wait_vm<0>();
}

template <int S, int NH, int NSA, int NSB>
__device__ __forceinline__ unsigned tile_role(const Params& P, LAS char* smem, unsigned call, int first_strip, int cg, int ch, int win, unsigned rot, bool packed) {
    constexpr int SA = S * 2048, SB = NH * 8192, BOFF = NSA * SA;
    constexpr int SPW = S / 2, NFR = NH * 2;
    static_assert(BOFF + NSB * SB <= LDS_BYTES, "LDS rings");
    GAS char* myws = (GAS char*)P.ws[0];
    const int lane = threadIdx.x & 63;
    const int wv = rfl(threadIdx.x >> 6);
    const bool wremote = cg >= 4;

    if (wv >= 4) {
        const int lw = wv & 1;
        const GAS char* mybuf = myws + (size_t)(call & 1) * BUF_BYTES;
        const unsigned vlin = lane * 16;
        if (wv < 6) {
            const GAS char* src[SPW];
            unsigned v0[SPW], v1[SPW];
            int shift[SPW];
#pragma unroll
            for (int i = 0; i < SPW; ++i) {
                const int code = rfl(P.strip[first_strip + lw * SPW + i]);
                const bool own = code < 32;
                v0[i] = own ? ximg0(lane, KD * 2) : vlin;
                v1[i] = own ? ximg1(lane, KD * 2) : vlin;
                src[i] = (own ? (const GAS char*)P.x + (unsigned)code * (16 * KD * 2) : mybuf + (unsigned)xslot(code >> 5, code & 31) * (unsigned)STRIP_BYTES) + (4096 - 2048 * i);
                shift[i] = own ? 7 : 11;
            }
            const int fcode = P.strip[first_strip + lw * SPW + ((lane >> 4) & (SPW - 1))];
            loader<SPW, SPW, NSA, SA>(myws, call, src, shift, v0, v1, mybuf + XFLAG_OFF + fcode * XF_STRIDE + (lane & 15) * 4, fcode < 32 || lane >= 16 * SPW,
                                      (unsigned)(size_t)smem + lw * (SA / 2) + 4096, rot, lw == 0);
        } else {
            const unsigned dk = (wremote ? (unsigned)STRIP_BYTES : 16u * NSH * 2) - 2048, dh = (wremote ? 4 * (unsigned)STRIP_BYTES : 128u) - 4096;
            const GAS char* const src[1] = {(wremote ? mybuf + XS_BYTES + (unsigned)((cg - 4) * 8 + ch * 4 + lw * 2) * (unsigned)STRIP_BYTES
                                                     : (const GAS char*)P.w + cg * 256 + ch * 128 + lw * 32 * (NSH * 2))};
            const int shift[1] = {wremote ? 11 : 16};
            unsigned v0[NFR], v1[NFR];
#pragma unroll
            for (int i = 0; i < NFR; ++i) {
                v0[i] = (wremote ? vlin : wimg0(lane, NSH * 2)) + 4096 + (i & 1) * dk + (i >> 1) * dh;
                v1[i] = (wremote ? vlin : wimg1(lane, NSH * 2)) + 4096 + (i & 1) * dk + (i >> 1) * dh;
            }
            loader<NFR, 1, NSB, SB>(myws, call, src, shift, v0, v1, mybuf + WFLAG_OFF + ((cg & 3) * NWP + lw * 64 + lane) * 4, !wremote,
                                    (unsigned)(size_t)smem + BOFF + lw * (SB / 2) + 4096, rot, false);
        }
    } else {
        const int fr = lane & 15, g = lane >> 4;
        const int wm = wv >> 1, wn = wv & 1;
        unsigned aoff[2];
#pragma unroll
        for (int kk = 0; kk < 2; ++kk) aoff[kk] = wm * (SA / 2) + fr * 128 + (((kk * 4 + g) ^ ((fr >> 1) & 7)) << 4);
        unsigned boff[NFR];
        {
            const int q = fr >> 2, pp = lane & 3, f = (q >> 1) | ((g & 1) << 1);
#pragma unroll
            for (int nb = 0; nb < NFR; ++nb)
                boff[nb] = BOFF + (NH == 2 ? wn * 4096 : 0) + (g >> 1) * 2048 + ((g & 1) * 8 + q) * 128 + ((((wn * NFR + nb) & 3) ^ f) << 5) + pp * 8;
        }
        int rowbase[SPW];
#pragma unroll
        for (int i = 0; i < SPW; ++i) {
            const int code = P.strip[first_strip + wm * SPW + i];
            rowbase[i] = rfl((P.rank ^ (code >> 5)) * MSH + (code & 31) * 16);
        }
        GAS char* outp = (GAS char*)P.out[wremote ? cg : 0] + (cg & 3) * 256;
        GAS char* cntp = (GAS char*)P.ws[cg] + OCNT_OFF + (cg & 3) * 256;
        constexpr int orow = NSH * 2;
        asm volatile("" : "+s"(outp), "+s"(cntp));
        f32x4 acc[SPW][NFR];
#pragma unroll
        for (int i = 0; i < SPW; ++i)
#pragma unroll
            for (int nb = 0; nb < NFR; ++nb) acc[i][nb] = (f32x4){0.f, 0.f, 0.f, 0.f};
        struct Frag { bf16x8 a[SPW]; bf16x8 b[NFR]; };
        auto rd = [&](Frag& F, int sa, int sb, int kk) {
#pragma unroll
            for (int i = 0; i < SPW; ++i) F.a[i] = *(LAS bf16x8*)(smem + sa + aoff[kk] + i * 2048);
#pragma unroll
            for (int nb = 0; nb < NFR; ++nb) {
                const s16x4 lo = __builtin_amdgcn_ds_read_tr16_b64_v4i16((LAS s16x4*)(smem + sb + boff[nb] + kk * (SB / 2)));
                const s16x4 hi = __builtin_amdgcn_ds_read_tr16_b64_v4i16((LAS s16x4*)(smem + sb + boff[nb] + kk * (SB / 2) + 512));
                F.b[nb] = __builtin_bit_cast(bf16x8, __builtin_shufflevector(lo, hi, 0, 1, 2, 3, 4, 5, 6, 7));
            }
        };
        auto mm = [&](const Frag& F) {
#pragma unroll
            for (int nb = 0; nb < NFR; ++nb)
#pragma unroll
                for (int i = 0; i < SPW; ++i) acc[i][nb] = __builtin_amdgcn_mfma_f32_16x16x32_bf16(F.b[nb], F.a[i], acc[i][nb], 0, 0, 0);
        };
        Frag F0, F1;
        int ca = 0, cb = 0;
        bar();
        rd(F0, 0, 0, 0);
        wait_lgkm();
#pragma nounroll
        for (int t = 0; t < NSTEP - 1; ++t) {
            constexpr int NRD = SPW + 2 * NFR, NMM = SPW * NFR, RPG = S * NH == 16 ? 1 : AGK_RPG;
            rd(F1, ca, cb, 1);
            mm(F0);
#pragma unroll
            for (int i = 0; i < NMM; ++i) { __builtin_amdgcn_sched_group_barrier(0x008, 1, 0); if (i * RPG < NRD) __builtin_amdgcn_sched_group_barrier(0x100, RPG, 0); }
            PIN();
            wait_lgkm();
            bar();
            ca = ca + SA == NSA * SA ? 0 : ca + SA;
            cb = cb + SB == NSB * SB ? 0 : cb + SB;
            rd(F0, ca, cb, 0);
            mm(F1);
#pragma unroll
            for (int i = 0; i < NMM; ++i) { __builtin_amdgcn_sched_group_barrier(0x008, 1, 1); if (i * RPG < NRD) __builtin_amdgcn_sched_group_barrier(0x100, RPG, 1); }
            PIN();
            wait_lgkm();
        }
        rd(F1, ca, cb, 1);
        PIN();
        mm(F0);
        PIN();
        mm(F1);
        if (wv == 0) STAMP(2);
        constexpr int ROWB = NFR * 32;
        constexpr int CPR = ROWB / 16;
        constexpr int ESLOT = (NSTEP - 1) % NSB >= 2 ? 0 : 2;
        LAS char* eb = smem + BOFF + ESLOT * SB + wv * (SPW * 16 * ROWB);
        static_assert(4 * SPW * 16 * ROWB <= 2 * SB && NSB >= 4, "epilogue space");
#pragma unroll
        for (int i = 0; i < SPW; ++i)
#pragma unroll
            for (int nb = 0; nb < NFR; ++nb) {
                u32x2 o;
                o.x = pack_bf16(acc[i][nb][0], acc[i][nb][1]);
                o.y = pack_bf16(acc[i][nb][2], acc[i][nb][3]);
                *(LAS u32x2*)(eb + (i * 16 + fr) * ROWB + (((nb * 2 + (g >> 1)) ^ (fr & (CPR - 1))) << 4) + (g & 1) * 8) = o;
            }
        if (NH == 2 && packed) {
            if (S == 4) { wait_lgkm(); bar(); }
            return S == 4 && wm ? 0u : (unsigned)(size_t)eb;
        }
        constexpr int RPI = 64 / CPR;
        constexpr int NST = SPW * 16 / RPI;
        u32x4 ov[NST];
        GAS char* od[NST];
        GAS char* obase = outp + (NH == 2 ? wn * 128 : ch * 128 + wn * 64) + (lane & (CPR - 1)) * 16;
#pragma unroll
        for (int n = 0; n < NST; ++n) {
            const int row = n * RPI + lane / CPR;
            ov[n] = *(LAS u32x4*)(eb + row * ROWB + (((lane & (CPR - 1)) ^ (row & (CPR - 1))) << 4));
            od[n] = obase + (size_t)((rowbase[(n * RPI) / 16] + (row & 15)) * orow);
        }
        if (wv == 0) STAMP(5);
        if (win == 1) {
#pragma unroll
            for (int n = 0; n < NST; ++n) { if (n >= 1) wait_vm<0>(); st16_wt(od[n], ov[n]); }
        } else if (win == 2) {
#pragma unroll
            for (int n = 0; n < NST; ++n) { if (n >= 2) wait_vm<1>(); st16_wt(od[n], ov[n]); }
        } else {
#pragma unroll
            for (int n = 0; n < NST; ++n) st16_wt(od[n], ov[n]);
        }
        if (wremote) {
            wait_vm<0>();
            if (lane == 0) __hip_atomic_fetch_add((gu32*)cntp, 1u, __ATOMIC_RELAXED, __HIP_MEMORY_SCOPE_SYSTEM);
        }
        if (wv == 0) STAMP(3);
    }
    return 0;
}

__global__ void __launch_bounds__(512, 1) __attribute__((amdgpu_num_vgpr(240))) ag_gemm_kernel(const Params P) {
    __shared__ __attribute__((aligned(1024))) char smem_raw[LDS_BYTES];
    LAS char* smem = (LAS char*)smem_raw;
    const unsigned long long t64 = now();
    const unsigned tstart = (unsigned)t64;
    const int b = blockIdx.x;
    if (__builtin_expect(b < NCOMM, 1)) {
        const int wv = rfl(threadIdx.x >> 6), g = wv * NCOMM + b;
        if (wv < NCW) comm_role(P, smem + wv * 16384, t64, wv, 1 + g % 7, g / 7);
        return;
    }
    GAS char* myws = (GAS char*)P.ws[0];
    gu32* cc = (gu32*)(myws + CALL_OFF + b * 64);
    const unsigned role = rfl(P.role[b - NCOMM]);
    const int type = role & 15;
    if (type == R_UNPACK) {
        const int wv = rfl(threadIdx.x >> 6), nun = P.nunpack & 255;
        const unsigned call = rfl(ld_agent(cc + wv)) + 1u;
        unpack_role(P, myws, tstart, call, (int)(role >> 4) + nun * wv, nun * 8);
        if (wv == 7) STAMP(4);
        if ((threadIdx.x & 63) == 0) st_agent(cc + wv, call);
    } else if (type != R_IDLE) {
        const unsigned call = rfl(ld_agent(cc)) + 1u;
        sleep_until(tstart + (P.tlead >> 16));
        if (threadIdx.x == 0) STAMP(0);
        const int fs = (role >> 4) & 255, cg = (role >> 12) & 7, ch = (role >> 15) & 1, win = (role >> 18) & 3;
        const bool packed = (role >> 16) & 1;
        const int po = (fs >> 1) + rfl(threadIdx.x >> 6);
        const GAS char* pk = (const GAS char*)P.ws[cg] + (size_t)(call & 1) * BUF_BYTES + pko_off(cg, po - (int)((P.nunpack >> 16) & 255));
        asm volatile("" : "+s"(pk));
        unsigned ea;
        if (type == R_T128) ea = tile_role<8, 2, AGK_RA>(P, smem, call, fs, cg, 0, win, role >> 28, packed);
        else if (type == R_T128H) ea = tile_role<8, 1, AGK_RH>(P, smem, call, fs, cg, ch, win, role >> 28, false);
        else if (type == R_T64W) ea = tile_role<4, 2, AGK_RW>(P, smem, call, fs, cg, 0, win, role >> 28, packed);
        else ea = tile_role<4, 1, AGK_RQ>(P, smem, call, fs, cg, ch, win, role >> 28, false);
        if (ea) {
            if (threadIdx.x == 0) STAMP(5);
            out_role(P, ea, type == R_T128 ? 4096 : 8192, call, pk, cg, po, win != 0);
            if (threadIdx.x == 0) STAMP(3);
        }
        if (threadIdx.x == 256) st_agent(cc, call);
    }
}

}

#define TUNE(i, dflt) (dflt)

extern "C" void kernel_launch(int rank, int nranks, ncclComm_t comm,
                              hipStream_t stream,
                              void* const* d_in, const int* in_sizes, int n_in,
                              void* d_out, int out_size,
                              const int* in_dtypes, int out_dtype,
                              void* const* d_out_all,
                              void* const* d_ws_all, size_t ws_bytes) {
    using namespace agk;
    (void)comm; (void)in_dtypes; (void)out_dtype; (void)d_out;
    if (nranks != NR || (unsigned)rank >= (unsigned)NR || n_in < 2 || ws_bytes < WS_NEED ||
        in_sizes[0] != MSH * KD || in_sizes[1] != KD * NSH || out_size != NR * MSH * NSH) {
        fprintf(stderr, "ag_gemm: unexpected problem shape\n");
        return;
    }
    Params P;
    P.x = (const char*)d_in[0];
    P.w = (const char*)d_in[1];
    P.rank = rank;
    for (int j = 0; j < NR; ++j) { P.ws[j] = (char*)d_ws_all[rank ^ j]; P.out[j] = (char*)d_out_all[rank ^ j]; }
    int sega = TUNE(0, 11);
    int blate = TUNE(1, 7);
    int wlead = TUNE(2, 4);
    int rt_b = TUNE(3, 1), rt_c = TUNE(4, 1), rt_a = 0, rt_w = TUNE(6, 0);
    int split_c = TUNE(7, 5);
    int split_a = TUNE(8, 0);
    int small_b = TUNE(9, 0);
    const int segb = HS - sega;
    const int na_t = (3 * sega + 7) / 8, nc_t = (4 * AS + 7) / 8, nb_t = (3 * segb + 7) / 8, no_t = 16 - na_t - nc_t - nb_t;
    const int ne_t = no_t + na_t;
    bool ok = no_t >= 0;
    int n = 0, own = 0;
    for (int i = 0; i < no_t * 8; ++i) P.strip[n++] = (unsigned char)(own++);
    for (int s = 0; s < sega; ++s) for (int j = 1; j < 4; ++j) P.strip[n++] = (unsigned char)((j << 5) | s);
    while (n % 8) P.strip[n++] = (unsigned char)(own++);
    for (int s = HS; s < NSTRIP; ++s) for (int j = 4; j < 8; ++j) P.strip[n++] = (unsigned char)((j << 5) | s);
    while (n % 8) P.strip[n++] = (unsigned char)(own++);
    for (int s = sega; s < HS; ++s) for (int j = 1; j < 4; ++j) P.strip[n++] = (unsigned char)((j << 5) | s);
    while (n % 8) P.strip[n++] = (unsigned char)(own++);
    ok = ok && n == 128 && own == HS;
    int q = 0;
    const int t_first = TUNE(16, 160) * 16, t_pk = TUNE(17, 169), t_raw = TUNE(18, 232);
    const int skew_b = TUNE(20, 16), skew_c = TUNE(21, 12);
    const int opk = TUNE(19, 3);
    const int op1 = opk & 2 ? ne_t + nc_t : ne_t, op0 = !(opk & 1) ? ne_t : op1 > NP_OUT / 4 ? op1 - NP_OUT / 4 : 0;
    const int n_slow = TUNE(25, 0), t_slow = TUNE(15, 184);
    int tq = t_first;
    for (int i = 0; i < 2 * WPL * NRND; ++i) P.ord[i] = 0;
    auto emit = [&](unsigned e) { if (q < NP_IN + NP_XI) P.ord[q < NP_IN ? ord_at(0, q) : ord_at(1, q - NP_IN)] = e | (unsigned)(tq >> 4) << 16; ++q; tq += (e & 512u) ? t_raw : tq < t_first + n_slow * t_slow ? t_slow : t_pk; };
    int nextk[NSTRIP] = {0};
    auto xchunk = [&](int s0, int s1, int rt, int skew) {
        for (int i = s0; i < s1; ++i) {
            int b = -1;
            for (int s = s0; s < s1; ++s) if (nextk[s] < NKC && (b < 0 || nextk[s] * 64 + s * skew < nextk[b] * 64 + b * skew)) b = s;
            emit((unsigned)nextk[b] | (b << 4) | (nextk[b] >= NKC - rt ? 512u : 0u));
            ++nextk[b];
        }
    };
    auto wchunk = [&](int kc) { for (int i = 0; i < 8; ++i) emit((unsigned)(kc * 8 + i) | 1024u | (kc >= NKC - rt_w ? 512u : 0u)); };
    for (int ka = 0, kb = 0; kb < NKC; ) {
        if (ka < NKC && kb * NKC >= ka * (NKC - blate)) { xchunk(0, sega, rt_a, 0); ++ka; }
        else { xchunk(sega, HS, rt_b, skew_b); ++kb; }
    }
    ok = ok && q == NP_IN;
    tq = t_first;
    for (int k = -wlead; k < NKC; ++k) {
        if (k + wlead < NKC) wchunk(k + wlead);
        if (k >= 0) xchunk(HS, NSTRIP, rt_c, skew_c);
    }
    ok = ok && q == NP_IN + NP_XI;
    int fill[8] = {0, 0, 0, 0, 0, 0, 0, 0};
    unsigned role[NWG];
    for (int i = 0; i < NWG; ++i) role[i] = R_IDLE;
    auto put = [&](int k, unsigned v) { if (fill[k] < NWG / 8) role[fill[k]++ * 8 + k] = v; else ok = false; };
    auto code = [&](int type, int fs, int cg, int ch) { return (unsigned)type | (fs << 4) | (cg << 12) | (ch << 15); };
    for (int i = 0; i < NCOMM; ++i) put(i & 7, R_COMM | (i << 4));
    unsigned adds = 0;
    for (int cg = 0; cg < 8; ++cg) {
        const bool rem = cg >= 4;
        for (int r = 0; r < 16; ++r) {
            const int k = (cg & 3) + 4 * (r & 1);
            const bool early = r < ne_t, last = r >= ne_t + nc_t;
            const bool split = last ? rem || r - ne_t - nc_t >= TUNE(29, 3) : (rem ? (early ? r >= ne_t - split_a : TUNE(5, 1) ? r - ne_t >= nc_t - split_c : r - ne_t < split_c) : (!early && r - ne_t < TUNE(22, 0)));
            const bool small = last && r - ne_t - nc_t < (rem ? small_b >> 4 : small_b & 15);
            const unsigned win = (unsigned)(!rem ? 0 : early ? TUNE(14, 2) : last ? TUNE(12, 0) : split ? TUNE(13, 0) : TUNE(11, 0)) << 18 |
                                 (rem && r >= op0 && r < op1 ? 1u << 16 : 0u);
            int nw = 0;
            auto rot = [&](int who, int of) { return (unsigned)(TUNE(26, 1) ? (of == 2 ? (who & 1) * 2 + ((r >> 1) & 1) : (who ^ (((r >> 1) & 1) * 2)) & 3) : 0) << 28; };
            if (small) for (int h = 0; h < 4; ++h) { put(k, rot(rem * 2 + (h & 1), 4) | win | code(R_T64H, r * 8 + (h >> 1) * 4, cg, h & 1)); nw += 4; }
            else if (split && (TUNE(23, 3) >> (last ? 0 : 1) & 1)) for (int h = 0; h < 2; ++h) { put(k, (unsigned)((rem * 2 + h * TUNE(24, 1)) ^ ((r >> 1) & 1)) << 28 | win | code(R_T64W, r * 8 + h * 4, cg, 0)); nw += 4; }
            else if (split) for (int ch = 0; ch < 2; ++ch) { put(k, rot(rem * 2 + ch, 4) | win | code(R_T128H, r * 8, cg, ch)); nw += 4; }
            else { put(k, rot(rem, 2) | win | code(R_T128, r * 8, cg, 0)); nw += 4; }
            if (cg == 4 && !(win >> 16 & 1)) adds += nw;
            ok = ok && !((win >> 16 & 1) && split && !(TUNE(23, 3) & 2));
        }
    }
    unsigned nun = 0;
    for (int i = 0; i < NWG; ++i) if (role[i] == R_IDLE && nun < (unsigned)TUNE(10, 64)) role[i] = R_UNPACK | (nun++ << 4);
    for (int i = NCOMM; i < NWG; ++i) P.role[i - NCOMM] = role[i];
    P.tlead = TUNE(27, 24) | TUNE(28, 250) << 16;
    P.out_adds = adds; P.tpoll = TUNE(30, 100) | TUNE(31, 1) << 16; P.nunpack = nun | (unsigned)(op1 - op0) * 4 << 8 | (unsigned)op0 * 4 << 16;
    ok = ok && nun > 0;
    if (!ok) { fprintf(stderr, "ag_gemm: role table overflow\n"); return; }
    hipLaunchKernelGGL(ag_gemm_kernel, dim3(NWG), dim3(512), 0, stream, P);
    HIP_CHECK(hipGetLastError());
}
```

```cpp
#include <hip/hip_runtime.h>
#include <rccl/rccl.h>
#include <cstdio>
#include <cstdint>

#define HIP_CHECK(expr) do { hipError_t _e = (expr); if (_e != hipSuccess) { \
    fprintf(stderr, "HIP error %s at %s:%d\n", hipGetErrorString(_e), __FILE__, __LINE__); \
    return; } } while (0)

namespace agk {

typedef unsigned int u32x4 __attribute__((ext_vector_type(4)));
typedef unsigned int u32x3 __attribute__((ext_vector_type(3)));
typedef unsigned int u32x2 __attribute__((ext_vector_type(2)));
typedef short s16x4 __attribute__((ext_vector_type(4)));
typedef short s16x8 __attribute__((ext_vector_type(8)));
typedef __bf16 bf16x8 __attribute__((ext_vector_type(8)));
typedef float f32x4 __attribute__((ext_vector_type(4)));
#define LAS __attribute__((address_space(3)))
#define GAS __attribute__((address_space(1)))
typedef GAS unsigned int gu32;

constexpr int NR = 8;
constexpr int KD = 4096;
constexpr int MSH = 512;
constexpr int NSH = 512;
constexpr int NSTRIP = 32;
#define AGK_HS 21
constexpr int HS = AGK_HS;
constexpr int AS = NSTRIP - HS;
constexpr int NKC = 16;
constexpr int NSTEP = 64;
constexpr int NWP = 128;

#define AGK_NCOMM 56
#define AGK_NCW 4
constexpr int NCOMM = AGK_NCOMM;
constexpr int NCW = AGK_NCW;
constexpr int WPL = NCOMM * NCW / 7;
static_assert(NCOMM % 7 == 0, "a comm workgroup serves one link");

constexpr int NP_IN = HS * NKC;
constexpr int NP_XI = NWP + AS * NKC;
constexpr int NPMAX = NP_IN > NP_XI ? NP_IN : NP_XI;
constexpr int NRND = (NPMAX + WPL - 1) / WPL;
constexpr int NWG = 256;
enum { R_IDLE = 0, R_COMM = 1, R_T128 = 2, R_T128H = 3, R_T64H = 4, R_UNPACK = 5, R_T64W = 6 };

constexpr size_t BUF_BYTES = 16ull * KD * 2;
constexpr size_t STRIP_BYTES = 2 * BUF_BYTES;
constexpr int NXSLOT = 3 * HS + 4 * AS;
constexpr size_t XS_BYTES = (size_t)NXSLOT * STRIP_BYTES;
constexpr size_t WSL_BYTES = 8 * STRIP_BYTES;
constexpr int XF_STRIDE = 256;
constexpr size_t XFLAG_OFF = XS_BYTES + 4 * WSL_BYTES;
constexpr size_t WFLAG_OFF = XFLAG_OFF + 8 * 32 * XF_STRIDE;
constexpr size_t PK_OFF = XFLAG_OFF + STRIP_BYTES;
constexpr unsigned PK_BYTES = 5888;
constexpr int NP_OUT = 56;
constexpr unsigned PK_PER = BUF_BYTES / (PK_BYTES + 128);
constexpr size_t OCNT_OFF = PK_OFF + (3 * NP_IN + 4 * NP_XI + 4 * NP_OUT + PK_PER - 1) / PK_PER * STRIP_BYTES;
constexpr size_t TMO_OFF = OCNT_OFF + 4 * 256;
constexpr size_t CALL_OFF = TMO_OFF + 256;
constexpr size_t DBG_OFF = CALL_OFF + 256 * 64;
constexpr size_t DBG2_OFF = DBG_OFF + 256 * 128;
constexpr size_t WS_NEED = DBG2_OFF + 8 * 512 * 32;
static_assert(WFLAG_OFF + 4 * NWP * 4 <= XFLAG_OFF + BUF_BYTES, "flags");
static_assert(WS_NEED <= 65ull << 20, "workspace");

constexpr unsigned long long DEADLINE_TICKS = 30000000ull;
constexpr int LDS_BYTES = 163840;

struct Params {
    const char* x;
    const char* w;
    char* ws[NR];
    char* out[NR];
    int rank;
    unsigned out_adds;
    unsigned tpoll;
    unsigned nunpack;
    unsigned role[NWG - NCOMM];
    unsigned tlead;
    unsigned ord[2 * WPL * NRND];
    unsigned char strip[128];
};

static_assert(sizeof(Params) <= 4096, "kernel arguments");

__device__ __forceinline__ unsigned ld_sys(const gu32* p) { return __hip_atomic_load(p, __ATOMIC_RELAXED, __HIP_MEMORY_SCOPE_SYSTEM); }
__device__ __forceinline__ unsigned ld_agent(const gu32* p) { return __hip_atomic_load(p, __ATOMIC_RELAXED, __HIP_MEMORY_SCOPE_AGENT); }
__device__ __forceinline__ void st_agent(gu32* p, unsigned v) { __hip_atomic_store(p, v, __ATOMIC_RELAXED, __HIP_MEMORY_SCOPE_AGENT); }
__device__ __forceinline__ unsigned long long now() { return __builtin_amdgcn_s_memrealtime(); }
__device__ __forceinline__ unsigned rfl(unsigned v) { return __builtin_amdgcn_readfirstlane(v); }

__device__ __forceinline__ void st16_wt(GAS char* p, u32x4 v) {
    asm volatile("global_store_dwordx4 %0, %1, off sc0 sc1\n\ts_nop 1" :: "v"(p), "v"(v) : "memory");
}
template <int N> __device__ __forceinline__ void wait_vm() { asm volatile("s_waitcnt vmcnt(%0)" :: "i"(N) : "memory"); }
__device__ __forceinline__ void wait_lgkm() { asm volatile("" ::: "memory"); __builtin_amdgcn_s_waitcnt(0xC07F); asm volatile("" ::: "memory"); }
#define AGK_SL1 6
#define AGK_SL2 2
#define PIN() __builtin_amdgcn_sched_barrier(0)
__device__ __forceinline__ void sleep_until(unsigned t) {
    int rest = (int)(t - (unsigned)now());
    for (; rest > 48; rest = (int)(t - (unsigned)now())) __builtin_amdgcn_s_sleep(8);
    for (; rest > 2; rest -= 3) __builtin_amdgcn_s_sleep(1);
}
__device__ __forceinline__ void bar() { asm volatile("s_barrier" ::: "memory"); }

#define AGK_LDPOL "sc0 sc1"
__device__ __forceinline__ void glds2(const GAS char* base, unsigned v0, unsigned v1, unsigned lds_dst) {
    asm volatile("s_mov_b32 m0, %3\n\ts_nop 3\n\tglobal_load_lds_dwordx4 %1, %0 " AGK_LDPOL "\n\tglobal_load_lds_dwordx4 %2, %0 offset:1024 " AGK_LDPOL
                 :: "s"(base), "v"(v0), "v"(v1), "s"(lds_dst) : "m0", "memory");
}
template <int OFF> __device__ __forceinline__ void glds2o(const GAS char* base, unsigned v0, unsigned v1) {
    asm volatile("global_load_lds_dwordx4 %1, %0 offset:%3 " AGK_LDPOL "\n\tglobal_load_lds_dwordx4 %2, %0 offset:%4 " AGK_LDPOL
                 :: "s"(base), "v"(v0), "v"(v1), "i"(OFF), "i"(OFF + 1024) : "memory");
}
__device__ __forceinline__ void wait_vm_var(unsigned n) {
    asm volatile("s_getpc_b64 vcc\n\ts_add_u32 vcc_lo, vcc_lo, %0\n\ts_addc_u32 vcc_hi, vcc_hi, 0\n\ts_setpc_b64 vcc\n\t"
                 ".set agk_n, 0\n\t.rept 64\n\ts_waitcnt vmcnt(agk_n)\n\ts_branch 1f\n\t.set agk_n, agk_n + 1\n\t.endr\n1:"
                 :: "s"(n * 8 + 12) : "vcc", "scc", "memory");
}

__device__ __forceinline__ unsigned pack_bf16(float lo, float hi) {
    typedef float f32x2 __attribute__((ext_vector_type(2)));
    typedef __bf16 bf16x2 __attribute__((ext_vector_type(2)));
    f32x2 v = {lo, hi};
    bf16x2 r = __builtin_convertvector(v, bf16x2);
    return __builtin_bit_cast(unsigned, r);
}

__device__ __forceinline__ int xslot(int j, int s) { return j < 4 ? (j - 1) * HS + s : 3 * HS + (j - 4) * AS + (s - HS); }

#define STAMP(i) do { } while (0)
#define PSTAMP(j, p, i, t) do { } while (0)

enum { K_X = 0, K_W = 1 };
struct Piece { int kind, raw; unsigned soff, doff, foff, when; };

__host__ __device__ __forceinline__ int ord_at(int cross, int p) { return (cross * WPL + p % WPL) * NRND + p / WPL; }
__device__ __forceinline__ Piece locate(unsigned e, int j) {
    Piece r;
    r.when = e >> 16;
    const int s = (e >> 4) & 31, kc = e & 15, wp = (e >> 10) & 1 ? (int)(e & 127) : -1;
    r.raw = (e >> 9) & 1;
    if (wp >= 0) {
        r.kind = K_W;
        r.soff = (unsigned)(wp * 32 * (NSH * 2) + (j & 3) * 256);
        r.doff = (unsigned)XS_BYTES + (unsigned)(j - 4) * (unsigned)WSL_BYTES + (wp & 1) * 2 * (unsigned)STRIP_BYTES + (wp >> 1) * 2048;
        r.foff = (unsigned)WFLAG_OFF + ((j - 4) * NWP + (wp & 1) * 64 + ((wp >> 1) & 3) * 16 + (wp >> 3)) * 4;
    } else {
        r.kind = K_X;
        r.soff = (unsigned)(s * 16 * (KD * 2) + kc * 512);
        r.doff = (unsigned)xslot(j, s) * (unsigned)STRIP_BYTES + kc * 8192;
        r.foff = (unsigned)XFLAG_OFF + (j * 32 + s) * XF_STRIDE + kc * 4;
    }
    return r;
}
__device__ __forceinline__ unsigned pk_off(int j, int p) {
    const unsigned i = (j < 4 ? (j - 1) * NP_IN : 3 * NP_IN + (j - 4) * (NP_XI + NP_OUT)) + p;
    return (unsigned)PK_OFF + 3072u + i * PK_BYTES + i / PK_PER * ((unsigned)STRIP_BYTES - PK_PER * PK_BYTES);
}

__device__ __forceinline__ unsigned long_list(unsigned off) { return PK_PER * PK_BYTES - 3072u - (off - (unsigned)PK_OFF) % (unsigned)STRIP_BYTES / PK_BYTES * (PK_BYTES - 128); }
__device__ __forceinline__ unsigned pko_off(int j, int p) { return pk_off(j, NP_XI + p); }

template <int OFF> __device__ __forceinline__ void st_pk(const u32x4& v, unsigned voff, const GAS char* dst) {
    asm volatile("s_nop 3\n\tglobal_store_dwordx4 %1, %0, %2 offset:%3 sc0 sc1\n\ts_nop 1" :: "v"(v), "v"(voff), "s"(dst), "i"(OFF) : "memory");
}
template <int OFF> __device__ __forceinline__ void ld_c(u32x4& v, unsigned voff, const GAS char* src) {
    asm volatile("global_load_dwordx4 %0, %1, %2 offset:%3 sc0 sc1" : "=v"(v) : "v"(voff), "s"(src), "i"(OFF) : "memory");
}
__device__ __forceinline__ void flag_st(const GAS char* f, unsigned val, unsigned zero) {
    asm volatile("s_mov_b64 exec, 1\n\ts_nop 3\n\tglobal_store_dword %0, %1, %2 sc0 sc1\n\ts_mov_b64 exec, -1" :: "v"(zero), "v"(val), "s"(f) : "memory");
}
__device__ __forceinline__ unsigned perm(unsigned hi, unsigned lo, unsigned sel) { return __builtin_amdgcn_perm(hi, lo, sel); }
__device__ __forceinline__ unsigned bfi(unsigned m, unsigned a, unsigned b) { return (a & m) | (b & ~m); }
__device__ __forceinline__ unsigned rotr(unsigned v, unsigned n) { return __builtin_amdgcn_alignbit(v, v, n); }
__device__ __forceinline__ unsigned ffbl(unsigned v) { unsigned r; asm("v_ffbl_b32 %0, %1" : "=v"(r) : "v"(v)); return r; }
__device__ __forceinline__ unsigned pkmax(unsigned a, unsigned b) {
    typedef unsigned short u16x2 __attribute__((ext_vector_type(2)));
    return __builtin_bit_cast(unsigned, __builtin_elementwise_max(__builtin_bit_cast(u16x2, a), __builtin_bit_cast(u16x2, b)));
}
template <int CTRL> __device__ __forceinline__ unsigned dppmax(unsigned v) { return max(v, (unsigned)__builtin_amdgcn_update_dpp(0, (int)v, CTRL, 0xF, 0xF, true)); }
constexpr unsigned F_RAW = 0x80u;
constexpr int MAXEXC = 8, NLIST = 128, NLONG = 32;
template <int CTRL, int ROWS> __device__ __forceinline__ unsigned dppadd(unsigned v) { return v + (unsigned)__builtin_amdgcn_update_dpp(0, (int)v, CTRL, ROWS, 0xF, true); }
__device__ __forceinline__ unsigned ximg0(unsigned lane, unsigned pitch) { return (lane >> 3) * pitch + ((lane & 7) ^ (lane >> 4)) * 16; }
__device__ __forceinline__ unsigned ximg1(unsigned lane, unsigned pitch) { return (8 + (lane >> 3)) * pitch + ((lane & 7) ^ (4 + (lane >> 4))) * 16 - 1024; }
__device__ __forceinline__ unsigned wimg0(unsigned lane, unsigned pitch) { return (lane >> 3) * pitch + ((lane & 7) ^ ((lane >> 3) & 2)) * 16; }
__device__ __forceinline__ unsigned wimg1(unsigned lane, unsigned pitch) { return (8 + (lane >> 3)) * pitch + ((lane & 7) ^ (((lane >> 3) & 2) | 4)) * 16 - 1024; }

__device__ __forceinline__ void fetch_piece(int kind, unsigned soff, const GAS char* sx, const GAS char* sw,
                                            unsigned vx0, unsigned vx1, unsigned vw0, unsigned vw1, unsigned l) {
    const bool kx = kind == K_X;
    const GAS char* s = (kx ? sx : sw) + soff;
    const unsigned a = kx ? vx0 : vw0, b = kx ? vx1 : vw1;
    const int q1 = kx ? 128 : 16 * NSH * 2, q2 = kx ? 256 : 128;
    glds2(s, a, b, l); glds2(s + q1, a, b, l + 2048); glds2(s + q2, a, b, l + 4096); glds2(s + q1 + q2, a, b, l + 6144);
}

__device__ __forceinline__ unsigned tag_of(unsigned call) { return ((call + 1u) >> 1) << 31; }

struct Packed { u32x4 W[5]; u32x2 W5; unsigned trl, ext; bool raw, more; };
__device__ __forceinline__ void pack(const u32x4 (&v)[8], Packed& Q, unsigned call, unsigned lds) {
    const unsigned lane = threadIdx.x & 63, tagw = tag_of(call);
    u32x4 (&W)[5] = Q.W;
    u32x2 W5;
    u32x2 em;
    unsigned trl;
    unsigned mx = pkmax(pkmax(pkmax(v[0].x & 0x7FFF7FFFu, v[0].y & 0x7FFF7FFFu), pkmax(v[0].z & 0x7FFF7FFFu, v[0].w & 0x7FFF7FFFu)),
                        pkmax(pkmax(v[6].x & 0x7FFF7FFFu, v[6].y & 0x7FFF7FFFu), pkmax(v[6].z & 0x7FFF7FFFu, v[6].w & 0x7FFF7FFFu)));
    mx = max(mx >> 16, mx & 0xFFFFu) >> 8;
    mx = dppmax<0x143>(dppmax<0x142>(dppmax<0x118>(dppmax<0x114>(dppmax<0x112>(dppmax<0x111>(mx))))));
    unsigned fmt = max((unsigned)__builtin_amdgcn_readlane((int)mx, 63), 7u) - 7u;
    const unsigned B4 = fmt * 0x01010101u;
    unsigned accO = 0, accA = ~0u, N[8];
#pragma unroll
    for (int k = 0; k < 8; ++k) {
        const unsigned Ha = perm(v[k].y, v[k].x, 0x07050301u), Hb = perm(v[k].w, v[k].z, 0x07050301u);
        const unsigned da = (Ha | 0x80808080u) - B4, db = (Hb | 0x80808080u) - B4;
        accO |= da | db; accA &= da & db;
        W[k >> 1][(k & 1) * 2] = perm(v[k].y, v[k].x, 0x06040200u);
        W[k >> 1][(k & 1) * 2 + 1] = perm(v[k].w, v[k].z, 0x06040200u);
        N[k] = bfi(0x87878787u, bfi(0x80808080u, Ha, da), rotr(bfi(0x80808080u, Hb, db), 28));
    }
    em.x = ~bfi(0x33333333u, bfi(0x11111111u, N[0] >> 2, N[1] >> 1), bfi(0x44444444u, N[2], N[3] << 1));
    em.y = ~bfi(0x33333333u, bfi(0x11111111u, N[4] >> 2, N[5] >> 1), bfi(0x44444444u, N[6], N[7] << 1));
    W[4] = (u32x4){bfi(0x44444444u, N[6] << 2, N[0]), bfi(0x44444444u, N[6] << 1, N[1]), bfi(0x44444444u, N[6] >> 1, N[2]), bfi(0x44444444u, N[7] << 2, N[3])};
    W5 = (u32x2){bfi(0x44444444u, N[7] << 1, N[4]), bfi(0x44444444u, N[7] >> 1, N[5])};
    bool raw = false;
    unsigned nfront = 0;
    const unsigned long long bm = __ballot(((accO & 0x78787878u) | (~accA & 0x80808080u)) != 0);
    if (bm) {
        unsigned nb = 0, ib = 0, hv = 0, ib2 = 0, hv2 = 0;
#pragma unroll
        for (int i = 0; i < 16; ++i) {
            const unsigned H = (i & 1) ? perm(v[i >> 1].w, v[i >> 1].z, 0x07050301u) : perm(v[i >> 1].y, v[i >> 1].x, 0x07050301u);
            const bool bad = (((H | 0x80808080u) - B4) & 0xF8F8F8F8u) != 0x80808080u;
            hv2 = bad ? hv : hv2; ib2 = bad ? ib : ib2; hv = bad ? H : hv; ib = bad ? (unsigned)i : ib; nb += bad;
        }
        const unsigned long long b2 = __ballot(nb > 1);
        const unsigned cnt = __builtin_popcountll(bm), cntb = __builtin_popcountll(b2);
        raw = cnt + cntb > MAXEXC || __ballot(nb > 2) != 0;
        auto put = [&](unsigned e, unsigned h, unsigned i) { *(LAS u32x2*)(size_t)(lds + e * 8) = (u32x2){(h & 0xFFFu) | (h << 4 & 0xFFF0000u), h >> 24 | i << 8 | lane << 16}; };
        if (nb && !raw) put(__builtin_amdgcn_mbcnt_hi((unsigned)(bm >> 32), __builtin_amdgcn_mbcnt_lo((unsigned)bm, 0)), hv, ib);
        if (nb > 1 && !raw) put(__builtin_amdgcn_mbcnt_hi((unsigned)(b2 >> 32), __builtin_amdgcn_mbcnt_lo((unsigned)b2, cnt)), hv2, ib2);
        fmt |= cnt << 8 | cntb << 12;
        nfront = (cnt + cntb) * 4;
    }
    const unsigned c = __builtin_popcount(em.x) + __builtin_popcount(em.y);
    const unsigned inc = dppadd<0x143, 0xC>(dppadd<0x142, 0xA>(dppadd<0x118, 0xF>(dppadd<0x114, 0xF>(dppadd<0x112, 0xF>(dppadd<0x111, 0xF>(c))))));
    const unsigned npos = __builtin_amdgcn_readlane(inc, 63);
    raw = raw || nfront + npos > NLIST + NLONG;
    Q.more = !raw && nfront + npos > NLIST;
    fmt |= npos << 16;
    if (!raw) {
        unsigned at = lds + (nfront + inc - c) * 2;
        for (unsigned long long m = (unsigned long long)em.y << 32 | em.x; m != 0; m &= m - 1, at += 2)
            *(LAS unsigned short*)(size_t)at = (unsigned short)(min(ffbl((unsigned)m), ffbl((unsigned)(m >> 32)) | 32u) | lane << 6);
    }
    asm volatile("" ::: "memory");
    const unsigned two = *(LAS unsigned*)(size_t)(lds + lane * 4);
    Q.ext = *(LAS unsigned*)(size_t)(lds + NLIST * 2 + (lane & 15) * 4);
    const unsigned pos = bfi(0x000FFF00u, two << 8, two << 4);
    if (raw) fmt = F_RAW;
    trl = (tagw >> 30) + (__builtin_amdgcn_inverse_ballot_w64((unsigned long long)call << 32 | fmt) ? 1u : 0u);
#pragma unroll
    for (int k = 0; k < 5; ++k) { trl = __builtin_amdgcn_alignbit(trl, W[k].w, 31); W[k].w = bfi(0x7FFFFFFFu, W[k].w, tagw); }
    trl = __builtin_amdgcn_alignbit(trl, W5.y, 31); W5.y = bfi(0x7FFFFFFFu, W5.y, tagw);
    trl = __builtin_amdgcn_alignbit(trl, pos, 8);
    Q.W5 = W5; Q.trl = trl; Q.raw = raw;
}
__device__ __forceinline__ void send_packed(const Packed& Q, const GAS char* pk, unsigned off, unsigned call, unsigned vlin, bool few) {
    st_pk<-3072>(Q.W[0], vlin, pk); st_pk<-2048>(Q.W[1], vlin, pk);
    if (few) wait_vm<1>();
    st_pk<-1024>(Q.W[2], vlin, pk);
    if (few) wait_vm<1>();
    st_pk<0>(Q.W[3], vlin, pk);
    if (few) wait_vm<1>();
    st_pk<1024>(Q.W[4], vlin, pk);
    if (few) wait_vm<1>();
    asm volatile("global_store_dwordx2 %0, %1, %2 offset:2048 sc0 sc1" :: "v"(vlin >> 1), "v"(Q.W5), "s"(pk) : "memory");
    if (__builtin_expect(Q.more, 0))
        asm volatile("s_mov_b64 exec, 0xffff\n\ts_nop 3\n\tglobal_store_dwordx2 %0, %1, %2 sc0 sc1\n\ts_mov_b64 exec, -1" :: "v"(vlin >> 1), "v"((u32x2){Q.ext, call}), "s"(pk + long_list(off)) : "memory");
    asm volatile("global_store_dword %0, %1, %2 offset:2560 sc0 sc1" :: "v"(vlin >> 2), "v"(Q.trl), "s"(pk) : "memory");
}

__device__ __forceinline__ void comm_role(const Params& P, LAS char* smem, unsigned long long t64, int wslot, int j, int u) {
    const unsigned* ord = P.ord + ord_at(j >= 4, u);
    unsigned eA = ord[0], eB = ord[1], tlead = P.tlead;
    const GAS char *srcx = (const GAS char*)P.x, *srcw = (const GAS char*)P.w, *pws = (const GAS char*)P.ws[j];
    GAS char* myws = (GAS char*)P.ws[0];
    asm volatile("" : "+s"(eA), "+s"(eB), "+s"(tlead), "+s"(srcx), "+s"(srcw), "+s"(pws), "+s"(myws), "+s"(t64));
    const unsigned tstart = (unsigned)t64;
    gu32* cc = (gu32*)(myws + CALL_OFF + blockIdx.x * 64) + wslot;
    const int lane = threadIdx.x & 63;
    const int npiece = j < 4 ? NP_IN : NP_XI;
    const unsigned vlin = lane * 16;
    const unsigned vx0 = ximg0(lane, KD * 2), vx1 = ximg1(lane, KD * 2), vw0 = wimg0(lane, NSH * 2), vw1 = wimg1(lane, NSH * 2);
    const unsigned vblk = (lane >> 5) * 2048 + (lane & 31) * 16, vblw = (lane >> 5) * (unsigned)STRIP_BYTES + (lane & 31) * 16;
    const unsigned lds0 = (unsigned)(size_t)smem;
#define AGK_FETCH(q, slot) fetch_piece((q).kind, (q).soff, srcx, srcw, vx0, vx1, vw0, vw1, lds0 + (slot) * 8192)
    int pA = u;
    unsigned ccv;
    Piece A = locate(eA, j);
    if (u > 3) sleep_until(tstart + (u * (tlead & 0xFFFFu) >> 2));
    AGK_FETCH(A, 0);
    asm volatile("global_load_dword %0, %1, off sc1" : "=v"(ccv) : "v"(cc) : "memory");
    Piece B = locate(eB, j);
    asm volatile("s_waitcnt vmcnt(0)" : "+v"(ccv) :: "memory");
    AGK_FETCH(B, 1);
    const unsigned call = rfl(ccv) + 1u;
    if (wslot == 0) STAMP(0);
    const GAS char* dbuf = pws + (size_t)(call & 1) * BUF_BYTES;
    gu32* tmo = (gu32*)(myws + TMO_OFF);
    unsigned zero = 0, callv = call;
    asm volatile("" : "+v"(zero), "+v"(callv));
    int slot = 0;
    for (;;) {
        u32x4 v[8];
#pragma unroll
        for (int k = 0; k < 8; ++k) v[k] = *(LAS u32x4*)(smem + slot * 8192 + (k >> 2) * 4096 + (k & 3) * 512 + vblk);
        Packed Q;
        if (!A.raw) pack(v, Q, call, lds0 + 65536);
        else {
#pragma unroll
            for (int k = 0; k < 5; ++k) Q.W[k] = v[k];
            Q.W5 = (u32x2){v[5].x, v[5].y};
            Q.trl = Q.ext = 0; Q.more = false; Q.raw = true;
        }
        const GAS char* pk = dbuf + pk_off(j, pA);
        const Piece C = locate(ord[pA + 2 * WPL < npiece ? pA / WPL + 2 : 0], j);
        asm volatile("" :: "v"(Q.W[0]), "v"(Q.W[1]), "v"(Q.W[2]), "v"(Q.W[3]), "v"(Q.W[4]), "v"(Q.W5), "v"(v[6]), "v"(v[7]), "v"(Q.trl), "v"(Q.ext),
                     "s"(pk), "s"(C.soff) : "memory");
        sleep_until(tstart + A.when);
        if (!A.raw) send_packed(Q, pk, (unsigned)(pk - dbuf), call, vlin, false);
        if (Q.raw) {
            const GAS char* d0 = dbuf + A.doff;
            const GAS char* d1 = d0 + (A.kind == K_W ? 4 * (unsigned)STRIP_BYTES : 4096u);
            const unsigned vd = A.kind == K_W ? vblw : vblk;
            st_pk<0>(v[0], vd, d0); st_pk<512>(v[1], vd, d0); st_pk<1024>(v[2], vd, d0); st_pk<1536>(v[3], vd, d0);
            st_pk<0>(v[4], vd, d1); st_pk<512>(v[5], vd, d1); st_pk<1024>(v[6], vd, d1); st_pk<1536>(v[7], vd, d1);
        }
        AGK_FETCH(C, slot);
        if (Q.raw) { wait_vm<8>(); flag_st(dbuf + A.foff, callv, zero); }
        PSTAMP(j, pA, 0, tstart + A.when); PSTAMP(j, pA, 1, tb); PSTAMP(j, pA, 2, now());
        pA += WPL; A = B; B = C; slot ^= 1;
        if (pA >= npiece) break;
        wait_vm<15>();
    }
    wait_vm<0>();
    STAMP(j < 4 ? 1 : 3);
    if (j == 1 && u == 0) {
        if (lane < 4) {
            const gu32* c = (const gu32*)(myws + OCNT_OFF + lane * 256);
            const unsigned need = call * P.out_adds;
            const unsigned long long t0 = now();
            bool bad = false;
            unsigned spin = 0;
            while (ld_sys(c) < need) {
                if ((++spin & 63) == 0 && (ld_agent(tmo) == call || now() - t0 > DEADLINE_TICKS)) { st_agent(tmo, call); bad = true; break; }
                __builtin_amdgcn_s_sleep(1);
            }
            if (bad) {
                __builtin_amdgcn_s_sleep(127);
                __hip_atomic_store((GAS unsigned short*)(P.out[0] + (size_t)(P.rank * MSH) * (NSH * 2)), (unsigned short)0x7FC0, __ATOMIC_RELAXED, __HIP_MEMORY_SCOPE_AGENT);
            }
        }
        STAMP(2);
    }
    if (lane == 0) st_agent(cc, call);
}

__device__ __forceinline__ void out_rows(const Params& P, const GAS char* (&d)[4], int jd, int j, int po) {
#pragma unroll
    for (int i = 0; i < 4; ++i) {
        const int code = rfl(P.strip[(po >> 1) * 4 + i]);
        d[i] = (const GAS char*)P.out[jd] + (unsigned)(((P.rank ^ jd ^ j ^ (code >> 5)) * MSH + (code & 31) * 16) * (NSH * 2) + (j & 3) * 256 + (po & 1) * 128);
    }
}
__device__ __forceinline__ void out_role(const Params& P, unsigned ea, unsigned gs, unsigned call, const GAS char* pk, int j, int po, bool few) {
    const unsigned lane = threadIdx.x & 63;
    u32x4 v[8];
#pragma unroll
    for (int n = 0; n < 8; ++n) v[n] = *(LAS u32x4*)(size_t)(ea + (n >> 2) * gs + (n & 3) * 1024 + (lane >> 3) * 128 + (((lane & 7) ^ (lane >> 3)) << 4));
    Packed Q;
    pack(v, Q, call, ea);
    if (Q.raw) {
        const GAS char* d[4];
        out_rows(P, d, j, j, po);
        const unsigned vA = (lane >> 3) * (NSH * 2) + (lane & 7) * 16, vB = vA + 8 * (NSH * 2);
#pragma unroll
        for (int n = 0; n < 8; ++n) st_pk<0>(v[n], n & 1 ? vB : vA, d[n >> 1]);
        wait_vm<0>();
    }
    send_packed(Q, pk, pko_off(j, po - (int)((P.nunpack >> 16) & 255)), call, lane * 16, few);
    wait_vm<0>();
}

__device__ __forceinline__ void unpack_role(const Params& P, GAS char* myws, unsigned lds, unsigned tstart, unsigned call, int w, int nw) {
    const GAS char* buf = myws + (size_t)(call & 1) * BUF_BYTES;
    gu32* tmo = (gu32*)(myws + TMO_OFF);
    const int lane = threadIdx.x & 63;
    const unsigned vlin = lane * 16, tagw = tag_of(call);
    const unsigned vblk = (lane >> 5) * 2048 + (lane & 31) * 16, vblw = (lane >> 5) * (unsigned)STRIP_BYTES + (lane & 31) * 16;
    unsigned zero = 0, callv = call;
    asm volatile("" : "+v"(zero), "+v"(callv));
    const unsigned tfirst = tstart + (P.tpoll & 0xFFFFu);
    const unsigned spec = P.tpoll >> 16;
    for (int q = w; ; q += nw) {
        const int p = q / 7, j = 1 + q - p * 7;
        const bool outp = q >= NPMAX * 7;
        const GAS char *pk, *d[4];
        unsigned vA, vB, foff = 0;
        if (!outp) {
            if (p >= (j < 4 ? NP_IN : NP_XI)) continue;
            const Piece A = locate(P.ord[ord_at(j >= 4, p)], j);
            if (A.raw) continue;
            pk = buf + pk_off(j, p);
            d[0] = buf + A.doff; d[1] = d[0] + 1024;
            d[2] = d[0] + (A.kind == K_W ? 4 * (unsigned)STRIP_BYTES : 4096u); d[3] = d[2] + 1024;
            vA = A.kind == K_W ? vblw : vblk; vB = vA + 512;
            foff = A.foff;
            sleep_until(tfirst + A.when);
        } else {
            const int i = q - NPMAX * 7, jo = 4 + (i & 3), po = (i >> 2) + (int)((P.nunpack >> 16) & 255);
            if (i >= (int)((P.nunpack >> 8) & 255) * 4) break;
            pk = buf + pko_off(jo, i >> 2);
            out_rows(P, d, 0, jo, po);
            vA = (lane >> 3) * (NSH * 2) + (lane & 7) * 16; vB = vA + 8 * (NSH * 2);
        }
        unsigned long long t0 = 0;
        unsigned spin = 0, trl, fmt = 0;
        bool more = false;
        u32x4 W[5]; u32x2 W5, ex = {0, 0};
        *(LAS u32x2*)(size_t)(lds + lane * 8) = ex;
        bool data = (spec >> (outp ? 1 : 0)) & 1;
        for (;;) {
            asm volatile("s_nop 4\n\tglobal_load_dword %0, %1, %2 offset:2560 sc0 sc1" : "=v"(trl) : "v"(vlin >> 2), "s"(pk) : "memory");
            if (data) {
                if (more) asm volatile("global_load_dwordx2 %0, %1, %2 sc0 sc1" : "+v"(ex) : "v"(((vlin >> 1) & 127u) + long_list((unsigned)(pk - buf))), "s"(pk) : "memory");
                asm volatile("global_load_dwordx2 %0, %1, %2 offset:2048 sc0 sc1" : "=v"(W5) : "v"(vlin >> 1), "s"(pk) : "memory");
                ld_c<1024>(W[4], vlin, pk);
                ld_c<-3072>(W[0], vlin, pk); ld_c<-2048>(W[1], vlin, pk); ld_c<-1024>(W[2], vlin, pk); ld_c<0>(W[3], vlin, pk);
            }
            asm volatile("s_waitcnt vmcnt(0)" : "+v"(trl), "+v"(W[0]), "+v"(W[1]), "+v"(W[2]), "+v"(W[3]), "+v"(W[4]), "+v"(W5), "+v"(ex) :: "memory");
            const unsigned long long fc = __ballot((int)(trl << 1) < 0);
            if (__ballot((int)(trl ^ tagw) < 0) == 0 && (unsigned)(fc >> 32) == call) {
                fmt = (unsigned)fc;
                if (fmt & F_RAW) break;
                more = ((fmt >> 8) & 15) * 4 + ((fmt >> 12) & 15) * 4 + (fmt >> 16) > NLIST;
                if (data && __ballot((int)((W[0].w ^ tagw) | (W[1].w ^ tagw) | (W[2].w ^ tagw) | (W[3].w ^ tagw) | (W[4].w ^ tagw) | (W5.y ^ tagw)) < 0) == 0 &&
                    (!more || __ballot(ex.y != call) == 0)) break;
                data = true;
            } else __builtin_amdgcn_s_sleep(AGK_SL2);
            if ((++spin & 63) == 0) {
                if (t0 == 0) t0 = now();
                if (rfl(ld_agent(tmo)) == call || now() - t0 > DEADLINE_TICKS) {
                    st_agent(tmo, call);
                    if (outp) st_pk<0>((u32x4){0x7FC07FC0u, 0x7FC07FC0u, 0x7FC07FC0u, 0x7FC07FC0u}, vA, d[0]);
                    fmt = F_RAW;
                    break;
                }
            }
        }
        if (!outp) { PSTAMP(j, p, 3, now()); PSTAMP(j, p, 7, spin); PSTAMP(j, p, 4, now()); }
        if (fmt & F_RAW) continue;
        const int cnt = (fmt >> 8) & 15, cnta = cnt + ((fmt >> 12) & 15);
        auto mark = [&](unsigned e, unsigned ps) {
            __hip_atomic_fetch_or((LAS unsigned*)(size_t)(lds + ((ps >> 3) & 0x1FCu)), e - cnta * 4 < (fmt >> 16) ? 1u << (ps & 31) : 0u, __ATOMIC_RELAXED, __HIP_MEMORY_SCOPE_WAVEFRONT);
        };
        mark(lane * 2, trl); mark(lane * 2 + 1, trl >> 12);
        if (more) { mark(NLIST + (lane & 15) * 2, ex.x); mark(NLIST + (lane & 15) * 2 + 1, ex.x >> 16); }
        asm volatile("" ::: "memory");
        const u32x2 mm = *(LAS u32x2*)(size_t)(lds + lane * 8);
        const unsigned m0 = mm.x, m1 = mm.y;
#pragma unroll
        for (int k = 0; k < 5; ++k) W[k].w = bfi(0x7FFFFFFFu, W[k].w, trl << (2 + k));
        W5.y = bfi(0x7FFFFFFFu, W5.y, trl << 7);
        unsigned N[8];
        N[0] = bfi(0x44444444u, ~(m0 << 2), W[4].x); N[1] = bfi(0x44444444u, ~(m0 << 1), W[4].y); N[2] = bfi(0x44444444u, ~m0, W[4].z); N[3] = bfi(0x44444444u, ~(m0 >> 1), W[4].w);
        N[4] = bfi(0x44444444u, ~(m1 << 2), W5.x); N[5] = bfi(0x44444444u, ~(m1 << 1), W5.y);
        N[6] = bfi(0x33333333u, bfi(0x11111111u, W[4].x >> 2, W[4].y >> 1), bfi(0x44444444u, ~m1, W[4].z << 1));
        N[7] = bfi(0x33333333u, bfi(0x11111111u, W[4].w >> 2, W5.x >> 1), bfi(0x44444444u, ~(m1 >> 1), W5.y << 1));
        const unsigned B4 = (fmt & 127u) * 0x01010101u;
        unsigned H[16];
#pragma unroll
        for (int k = 0; k < 8; ++k) {
            H[2 * k] = (N[k] & 0x87878787u) + B4;
            H[2 * k + 1] = (rotr(N[k], 4) & 0x87878787u) + B4;
        }
        for (int e = 0, e1 = cnt; e < e1; e1 = cnta) {
            unsigned tag = 255, val = 0;
            for (; e < e1; ++e) {
                const unsigned f0 = __builtin_amdgcn_readlane(trl, 2 * e), f1 = __builtin_amdgcn_readlane(trl, 2 * e + 1);
                const bool mine = ((f1 >> 12) & 63) == (unsigned)lane;
                tag = mine ? (f1 >> 8) & 15 : tag; val = mine ? (f0 & 0xFFFFFFu) | f1 << 24 : val;
            }
#pragma unroll
            for (int i = 0; i < 16; ++i) H[i] = tag == (unsigned)i ? val : H[i];
        }
#define AGK_DST(k) { const unsigned l0 = W[k >> 1][(k & 1) * 2], l1 = W[k >> 1][(k & 1) * 2 + 1]; u32x4 o; \
            o.x = perm(H[2 * k], l0, 0x05010400u); o.y = perm(H[2 * k], l0, 0x07030602u); \
            o.z = perm(H[2 * k + 1], l1, 0x05010400u); o.w = perm(H[2 * k + 1], l1, 0x07030602u); st_pk<0>(o, k & 1 ? vB : vA, d[k >> 1]); }
        AGK_DST(0) AGK_DST(1) AGK_DST(2) AGK_DST(3) AGK_DST(4) AGK_DST(5) AGK_DST(6) AGK_DST(7)
        if (outp) continue;
        PSTAMP(j, p, 5, now());
        wait_vm<0>(); flag_st(buf + foff, callv, zero);
        PSTAMP(j, p, 6, now());
    }
    wait_vm<0>();
}

#define AGK_RPG 2
#define AGK_RA 4, 6
#define AGK_RH 4, 12
#define AGK_RQ 8, 12
#define AGK_RW 6, 7
#define AGK_LOOK 3
template <int NB, int NP, int NSLOT, int SLOT_BYTES>
__device__ __forceinline__ void loader(GAS char* myws, unsigned call, const GAS char* const (&src)[NP], const int (&shift)[NP], const unsigned (&v0)[NB],
                                       const unsigned (&v1)[NB], const GAS char* fp, bool have, unsigned ldsw, unsigned rot, bool first) {
    constexpr int NI = 2 * NB;
    static_assert((NSLOT - 2) * NI + 8 < 64 && (NSLOT - 1) * NI + 10 < 64, "vmcnt range");
    gu32* tmo = (gu32*)(myws + TMO_OFF);
    const unsigned ldsend = ldsw + NSLOT * SLOT_BYTES;
    const unsigned long long act = ~__ballot(have);
    asm volatile("v_mov_b32 v250, %0" :: "v"(have ? ~0u : 0u) : "v250", "memory");
    int mark = 0;
    unsigned npoll = 0;
    auto poll = [&]() {
        ++npoll;
        asm volatile("s_mov_b64 exec, %1\n\tglobal_load_dword v250, %0, off sc0 sc1\n\ts_mov_b64 exec, -1" :: "v"(fp), "s"(act) : "v250", "memory");
    };
    auto eval = [&]() {
        unsigned long long bx;
        asm volatile("v_cmp_gt_u32 %0, %1, v250" : "=s"(bx) : "s"(call) : "memory");
        const unsigned m = (unsigned)bx | (unsigned)(bx >> 32);
        mark = __builtin_ctz(m | m >> 16 | 0x10000u) * 4;
    };
    unsigned ldsi = ldsw, kt = 0;
    const GAS char* ptr[NP];
#pragma unroll
    for (int i = 0; i < NP; ++i) ptr[i] = src[i] + (rot << shift[i]);
    auto issue = [&]() {
        asm volatile("s_mov_b32 m0, %0\n\ts_nop 0" :: "s"(ldsi) : "m0", "memory");
        glds2o<-4096>(ptr[0], v0[0], v1[0]);
        glds2o<-2048>(ptr[1 % NP], v0[1], v1[1]);
        if (NB == 4) { glds2o<0>(ptr[2 % NP], v0[2 % NB], v1[2 % NB]); glds2o<2048>(ptr[3 % NP], v0[3 % NB], v1[3 % NB]); }
        const unsigned k = ++kt ^ rot;
#pragma unroll
        for (int i = 0; i < NP; ++i) { ptr[i] = src[i] + (k << shift[i]); asm volatile("" : "+s"(ptr[i])); }
        ldsi = ldsi + SLOT_BYTES == ldsend ? ldsw : ldsi + SLOT_BYTES;
        asm volatile("" : "+s"(ldsi));
    };
    int ti = 0, tb = 0;
    unsigned hist = 0;
#pragma nounroll
    for (;;) {
        if (mark < NSTEP && mark - ti <= AGK_LOOK) eval();
        const int lim = mark < tb + NSLOT - 1 ? mark : tb + NSLOT - 1;
#pragma nounroll
        for (; ti < lim; ++ti) { issue(); hist = hist << 4 | (npoll & 15u); }
        if (ti == tb) {
            unsigned long long t0 = 0;
            for (unsigned spin = 1; ti >= mark; ++spin) {
                wait_vm<8>();
                poll();
                __builtin_amdgcn_s_sleep(AGK_SL1);
                if ((spin & 63) == 0) {
                    if (t0 == 0) t0 = now();
                    if (rfl(ld_agent(tmo)) == call || now() - t0 > DEADLINE_TICKS) break;
                }
                eval();
            }
            if (ti >= mark) break;
            continue;
        }
        const int run = (mark < NSTEP ? mark - AGK_LOOK : NSTEP) - ti;
        if (run > 0) {
#pragma nounroll
            for (int i = run; i > 0; --i) { wait_vm<(NSLOT - 2) * NI>(); bar(); issue(); }
            ti += run; tb += run;
            const unsigned rep = (npoll & 15u) * 0x11111111u;
            hist = run >= 8 ? rep : hist << (4 * run) | (rep & ((1u << (4 * run)) - 1u));
        }
        if (mark < NSTEP) poll();
        const unsigned n = ti - tb - 1;
        wait_vm_var(n * NI + (n < 8 ? (npoll - (hist >> (4 * n))) & 15u : 0u));
        bar();
        if (++tb == NSTEP) break;
    }
    if (tb < NSTEP) {
        st_agent(tmo, call);
        const u32x4 nan = {0x7FC07FC0u, 0x7FC07FC0u, 0x7FC07FC0u, 0x7FC07FC0u};
        for (; tb < NSTEP; ++tb) {
#pragma unroll
            for (int i = 0; i < NI; ++i) *(LAS u32x4*)(size_t)(ldsi - 4096 + i * 1024 + (threadIdx.x & 63) * 16) = nan;
            wait_lgkm();
            bar();
            ldsi = ldsi + SLOT_BYTES == ldsend ? ldsw : ldsi + SLOT_BYTES;
        }
    }
    if (first) STAMP(4);
    wait_vm<0>();
}

template <int S, int NH, int NSA, int NSB>
__device__ __forceinline__ unsigned tile_role(const Params& P, LAS char* smem, unsigned call, int first_strip, int cg, int ch, int win, unsigned rot, bool packed) {
    constexpr int SA = S * 2048, SB = NH * 8192, BOFF = NSA * SA;
    constexpr int SPW = S / 2, NFR = NH * 2;
    static_assert(BOFF + NSB * SB <= LDS_BYTES, "LDS rings");
    GAS char* myws = (GAS char*)P.ws[0];
    const int lane = threadIdx.x & 63;
    const int wv = rfl(threadIdx.x >> 6);
    const bool wremote = cg >= 4;

    if (wv >= 4) {
        const int lw = wv & 1;
        const GAS char* mybuf = myws + (size_t)(call & 1) * BUF_BYTES;
        const unsigned vlin = lane * 16;
        if (wv < 6) {
            const GAS char* src[SPW];
            unsigned v0[SPW], v1[SPW];
            int shift[SPW];
#pragma unroll
            for (int i = 0; i < SPW; ++i) {
                const int code = rfl(P.strip[first_strip + lw * SPW + i]);
                const bool own = code < 32;
                v0[i] = own ? ximg0(lane, KD * 2) : vlin;
                v1[i] = own ? ximg1(lane, KD * 2) : vlin;
                src[i] = (own ? (const GAS char*)P.x + (unsigned)code * (16 * KD * 2) : mybuf + (unsigned)xslot(code >> 5, code & 31) * (unsigned)STRIP_BYTES) + (4096 - 2048 * i);
                shift[i] = own ? 7 : 11;
            }
            const int fcode = P.strip[first_strip + lw * SPW + ((lane >> 4) & (SPW - 1))];
            loader<SPW, SPW, NSA, SA>(myws, call, src, shift, v0, v1, mybuf + XFLAG_OFF + fcode * XF_STRIDE + (lane & 15) * 4, fcode < 32 || lane >= 16 * SPW,
                                      (unsigned)(size_t)smem + lw * (SA / 2) + 4096, rot, lw == 0);
        } else {
            const unsigned dk = (wremote ? (unsigned)STRIP_BYTES : 16u * NSH * 2) - 2048, dh = (wremote ? 4 * (unsigned)STRIP_BYTES : 128u) - 4096;
            const GAS char* const src[1] = {(wremote ? mybuf + XS_BYTES + (unsigned)((cg - 4) * 8 + ch * 4 + lw * 2) * (unsigned)STRIP_BYTES
                                                     : (const GAS char*)P.w + cg * 256 + ch * 128 + lw * 32 * (NSH * 2))};
            const int shift[1] = {wremote ? 11 : 16};
            unsigned v0[NFR], v1[NFR];
#pragma unroll
            for (int i = 0; i < NFR; ++i) {
                v0[i] = (wremote ? vlin : wimg0(lane, NSH * 2)) + 4096 + (i & 1) * dk + (i >> 1) * dh;
                v1[i] = (wremote ? vlin : wimg1(lane, NSH * 2)) + 4096 + (i & 1) * dk + (i >> 1) * dh;
            }
            loader<NFR, 1, NSB, SB>(myws, call, src, shift, v0, v1, mybuf + WFLAG_OFF + ((cg & 3) * NWP + lw * 64 + lane) * 4, !wremote,
                                    (unsigned)(size_t)smem + BOFF + lw * (SB / 2) + 4096, rot, false);
        }
    } else {
        const int fr = lane & 15, g = lane >> 4;
        const int wm = wv >> 1, wn = wv & 1;
        unsigned aoff[2];
#pragma unroll
        for (int kk = 0; kk < 2; ++kk) aoff[kk] = wm * (SA / 2) + fr * 128 + (((kk * 4 + g) ^ ((fr >> 1) & 7)) << 4);
        unsigned boff[NFR];
        {
            const int q = fr >> 2, pp = lane & 3, f = (q >> 1) | ((g & 1) << 1);
#pragma unroll
            for (int nb = 0; nb < NFR; ++nb)
                boff[nb] = BOFF + (NH == 2 ? wn * 4096 : 0) + (g >> 1) * 2048 + ((g & 1) * 8 + q) * 128 + ((((wn * NFR + nb) & 3) ^ f) << 5) + pp * 8;
        }
        int rowbase[SPW];
#pragma unroll
        for (int i = 0; i < SPW; ++i) {
            const int code = P.strip[first_strip + wm * SPW + i];
            rowbase[i] = rfl((P.rank ^ (code >> 5)) * MSH + (code & 31) * 16);
        }
        GAS char* outp = (GAS char*)P.out[wremote ? cg : 0] + (cg & 3) * 256;
        GAS char* cntp = (GAS char*)P.ws[cg] + OCNT_OFF + (cg & 3) * 256;
        constexpr int orow = NSH * 2;
        asm volatile("" : "+s"(outp), "+s"(cntp));
        f32x4 acc[SPW][NFR];
#pragma unroll
        for (int i = 0; i < SPW; ++i)
#pragma unroll
            for (int nb = 0; nb < NFR; ++nb) acc[i][nb] = (f32x4){0.f, 0.f, 0.f, 0.f};
        struct Frag { bf16x8 a[SPW]; bf16x8 b[NFR]; };
        auto rd = [&](Frag& F, int sa, int sb, int kk) {
#pragma unroll
            for (int i = 0; i < SPW; ++i) F.a[i] = *(LAS bf16x8*)(smem + sa + aoff[kk] + i * 2048);
#pragma unroll
            for (int nb = 0; nb < NFR; ++nb) {
                const s16x4 lo = __builtin_amdgcn_ds_read_tr16_b64_v4i16((LAS s16x4*)(smem + sb + boff[nb] + kk * (SB / 2)));
                const s16x4 hi = __builtin_amdgcn_ds_read_tr16_b64_v4i16((LAS s16x4*)(smem + sb + boff[nb] + kk * (SB / 2) + 512));
                F.b[nb] = __builtin_bit_cast(bf16x8, __builtin_shufflevector(lo, hi, 0, 1, 2, 3, 4, 5, 6, 7));
            }
        };
        auto mm = [&](const Frag& F) {
#pragma unroll
            for (int nb = 0; nb < NFR; ++nb)
#pragma unroll
                for (int i = 0; i < SPW; ++i) acc[i][nb] = __builtin_amdgcn_mfma_f32_16x16x32_bf16(F.b[nb], F.a[i], acc[i][nb], 0, 0, 0);
        };
        Frag F0, F1;
        int ca = 0, cb = 0;
        bar();
        rd(F0, 0, 0, 0);
        wait_lgkm();
#pragma nounroll
        for (int t = 0; t < NSTEP - 1; ++t) {
            constexpr int NRD = SPW + 2 * NFR, NMM = SPW * NFR, RPG = S * NH == 16 ? 1 : AGK_RPG;
            rd(F1, ca, cb, 1);
            mm(F0);
#pragma unroll
            for (int i = 0; i < NMM; ++i) { __builtin_amdgcn_sched_group_barrier(0x008, 1, 0); if (i * RPG < NRD) __builtin_amdgcn_sched_group_barrier(0x100, RPG, 0); }
            PIN();
            wait_lgkm();
            bar();
            ca = ca + SA == NSA * SA ? 0 : ca + SA;
            cb = cb + SB == NSB * SB ? 0 : cb + SB;
            rd(F0, ca, cb, 0);
            mm(F1);
#pragma unroll
            for (int i = 0; i < NMM; ++i) { __builtin_amdgcn_sched_group_barrier(0x008, 1, 1); if (i * RPG < NRD) __builtin_amdgcn_sched_group_barrier(0x100, RPG, 1); }
            PIN();
            wait_lgkm();
        }
        rd(F1, ca, cb, 1);
        PIN();
        mm(F0);
        PIN();
        mm(F1);
        if (wv == 0) STAMP(2);
        constexpr int ROWB = NFR * 32;
        constexpr int CPR = ROWB / 16;
        constexpr int ESLOT = (NSTEP - 1) % NSB >= 2 ? 0 : 2;
        LAS char* eb = smem + BOFF + ESLOT * SB + wv * (SPW * 16 * ROWB);
        static_assert(4 * SPW * 16 * ROWB <= 2 * SB && NSB >= 4, "epilogue space");
#pragma unroll
        for (int i = 0; i < SPW; ++i)
#pragma unroll
            for (int nb = 0; nb < NFR; ++nb) {
                u32x2 o;
                o.x = pack_bf16(acc[i][nb][0], acc[i][nb][1]);
                o.y = pack_bf16(acc[i][nb][2], acc[i][nb][3]);
                *(LAS u32x2*)(eb + (i * 16 + fr) * ROWB + (((nb * 2 + (g >> 1)) ^ (fr & (CPR - 1))) << 4) + (g & 1) * 8) = o;
            }
        if (NH == 2 && packed) {
            if (S == 4) { wait_lgkm(); bar(); }
            return S == 4 && wm ? 0u : (unsigned)(size_t)eb;
        }
        constexpr int RPI = 64 / CPR;
        constexpr int NST = SPW * 16 / RPI;
        u32x4 ov[NST];
        GAS char* od[NST];
        GAS char* obase = outp + (NH == 2 ? wn * 128 : ch * 128 + wn * 64) + (lane & (CPR - 1)) * 16;
#pragma unroll
        for (int n = 0; n < NST; ++n) {
            const int row = n * RPI + lane / CPR;
            ov[n] = *(LAS u32x4*)(eb + row * ROWB + (((lane & (CPR - 1)) ^ (row & (CPR - 1))) << 4));
            od[n] = obase + (size_t)((rowbase[(n * RPI) / 16] + (row & 15)) * orow);
        }
        if (wv == 0) STAMP(5);
        if (win == 1) {
#pragma unroll
            for (int n = 0; n < NST; ++n) { if (n >= 1) wait_vm<0>(); st16_wt(od[n], ov[n]); }
        } else if (win == 2) {
#pragma unroll
            for (int n = 0; n < NST; ++n) { if (n >= 2) wait_vm<1>(); st16_wt(od[n], ov[n]); }
        } else {
#pragma unroll
            for (int n = 0; n < NST; ++n) st16_wt(od[n], ov[n]);
        }
        if (wremote) {
            wait_vm<0>();
            if (lane == 0) __hip_atomic_fetch_add((gu32*)cntp, 1u, __ATOMIC_RELAXED, __HIP_MEMORY_SCOPE_SYSTEM);
        }
        if (wv == 0) STAMP(3);
    }
    return 0;
}

__global__ void __launch_bounds__(512, 1) __attribute__((amdgpu_num_vgpr(240))) ag_gemm_kernel(const Params P) {
    __shared__ __attribute__((aligned(1024))) char smem_raw[LDS_BYTES];
    LAS char* smem = (LAS char*)smem_raw;
    const unsigned long long t64 = now();
    const unsigned tstart = (unsigned)t64;
    const int b = blockIdx.x;
    if (__builtin_expect(b < NCOMM, 1)) {
        const int wv = rfl(threadIdx.x >> 6), g = wv * NCOMM + b;
        if (wv < NCW) comm_role(P, smem + wv * 16384, t64, wv, 1 + g % 7, g / 7);
        return;
    }
    GAS char* myws = (GAS char*)P.ws[0];
    gu32* cc = (gu32*)(myws + CALL_OFF + b * 64);
    const unsigned role = rfl(P.role[b - NCOMM]);
    const int type = role & 15;
    if (type == R_UNPACK) {
        const int wv = rfl(threadIdx.x >> 6), nun = P.nunpack & 255;
        const unsigned call = rfl(ld_agent(cc + wv)) + 1u;
        unpack_role(P, myws, (unsigned)(size_t)smem + wv * 512, tstart, call, (int)(role >> 4) + nun * wv, nun * 8);
        if (wv == 7) STAMP(4);
        if ((threadIdx.x & 63) == 0) st_agent(cc + wv, call);
    } else if (type != R_IDLE) {
        const unsigned call = rfl(ld_agent(cc)) + 1u;
        sleep_until(tstart + (P.tlead >> 16));
        if (threadIdx.x == 0) STAMP(0);
        const int fs = (role >> 4) & 255, cg = (role >> 12) & 7, ch = (role >> 15) & 1, win = (role >> 18) & 3;
        const bool packed = (role >> 16) & 1;
        const int po = (fs >> 1) + rfl(threadIdx.x >> 6);
        const GAS char* pk = (const GAS char*)P.ws[cg] + (size_t)(call & 1) * BUF_BYTES + pko_off(cg, po - (int)((P.nunpack >> 16) & 255));
        asm volatile("" : "+s"(pk));
        unsigned ea;
        if (type == R_T128) ea = tile_role<8, 2, AGK_RA>(P, smem, call, fs, cg, 0, win, role >> 28, packed);
        else if (type == R_T128H) ea = tile_role<8, 1, AGK_RH>(P, smem, call, fs, cg, ch, win, role >> 28, false);
        else if (type == R_T64W) ea = tile_role<4, 2, AGK_RW>(P, smem, call, fs, cg, 0, win, role >> 28, packed);
        else ea = tile_role<4, 1, AGK_RQ>(P, smem, call, fs, cg, ch, win, role >> 28, false);
        if (ea) {
            if (threadIdx.x == 0) STAMP(5);
            out_role(P, ea, type == R_T128 ? 4096 : 8192, call, pk, cg, po, win != 0);
            if (threadIdx.x == 0) STAMP(3);
        }
        if (threadIdx.x == 256) st_agent(cc, call);
    }
}

}

#define TUNE(i, dflt) (dflt)

extern "C" void kernel_launch(int rank, int nranks, ncclComm_t comm,
                              hipStream_t stream,
                              void* const* d_in, const int* in_sizes, int n_in,
                              void* d_out, int out_size,
                              const int* in_dtypes, int out_dtype,
                              void* const* d_out_all,
                              void* const* d_ws_all, size_t ws_bytes) {
    using namespace agk;
    (void)comm; (void)in_dtypes; (void)out_dtype; (void)d_out;
    if (nranks != NR || (unsigned)rank >= (unsigned)NR || n_in < 2 || ws_bytes < WS_NEED ||
        in_sizes[0] != MSH * KD || in_sizes[1] != KD * NSH || out_size != NR * MSH * NSH) {
        fprintf(stderr, "ag_gemm: unexpected problem shape\n");
        return;
    }
    Params P;
    P.x = (const char*)d_in[0];
    P.w = (const char*)d_in[1];
    P.rank = rank;
    for (int j = 0; j < NR; ++j) { P.ws[j] = (char*)d_ws_all[rank ^ j]; P.out[j] = (char*)d_out_all[rank ^ j]; }
    int sega = TUNE(0, 11);
    int blate = TUNE(1, 7);
    int wlead = TUNE(2, 4);
    int rt_b = TUNE(3, 1), rt_c = TUNE(4, 1), rt_a = 0, rt_w = 0;
    int split_c = TUNE(7, 5);
    int split_a = TUNE(8, 0);
    int small_b = TUNE(9, 0);
    const int segb = HS - sega;
    const int na_t = (3 * sega + 7) / 8, nc_t = (4 * AS + 7) / 8, nb_t = (3 * segb + 7) / 8, no_t = 16 - na_t - nc_t - nb_t;
    const int ne_t = no_t + na_t;
    bool ok = no_t >= 0;
    int n = 0, own = 0;
    for (int i = 0; i < no_t * 8; ++i) P.strip[n++] = (unsigned char)(own++);
    for (int s = 0; s < sega; ++s) for (int j = 1; j < 4; ++j) P.strip[n++] = (unsigned char)((j << 5) | s);
    while (n % 8) P.strip[n++] = (unsigned char)(own++);
    for (int s = HS; s < NSTRIP; ++s) for (int j = 4; j < 8; ++j) P.strip[n++] = (unsigned char)((j << 5) | s);
    while (n % 8) P.strip[n++] = (unsigned char)(own++);
    for (int s = sega; s < HS; ++s) for (int j = 1; j < 4; ++j) P.strip[n++] = (unsigned char)((j << 5) | s);
    while (n % 8) P.strip[n++] = (unsigned char)(own++);
    ok = ok && n == 128 && own == HS;
    int q = 0;
    const int t_first = TUNE(16, 160) * 16, t_pk = TUNE(17, 161), t_raw = TUNE(18, 232);
    const int skew_b = TUNE(20, 16), skew_c = TUNE(21, 12);
    const int opk = TUNE(19, 3);
    const int op1 = opk & 2 ? ne_t + nc_t - TUNE(6, 0) : ne_t, op0 = !(opk & 1) ? ne_t : op1 > NP_OUT / 4 ? op1 - NP_OUT / 4 : 0;
    const int n_slow = TUNE(25, 40), t_slow = TUNE(15, 182);
    int tq = t_first;
    for (int i = 0; i < 2 * WPL * NRND; ++i) P.ord[i] = 0;
    auto emit = [&](unsigned e) { if (q < NP_IN + NP_XI) P.ord[q < NP_IN ? ord_at(0, q) : ord_at(1, q - NP_IN)] = e | (unsigned)(tq >> 4) << 16; ++q; tq += (e & 512u) ? t_raw : tq < t_first + n_slow * t_slow ? t_slow : t_pk; };
    int nextk[NSTRIP] = {0};
    auto xchunk = [&](int s0, int s1, int rt, int skew) {
        for (int i = s0; i < s1; ++i) {
            int b = -1;
            for (int s = s0; s < s1; ++s) if (nextk[s] < NKC && (b < 0 || nextk[s] * 64 + s * skew < nextk[b] * 64 + b * skew)) b = s;
            emit((unsigned)nextk[b] | (b << 4) | (nextk[b] >= NKC - rt ? 512u : 0u));
            ++nextk[b];
        }
    };
    auto wchunk = [&](int kc) { for (int i = 0; i < 8; ++i) emit((unsigned)(kc * 8 + i) | 1024u | (kc >= NKC - rt_w ? 512u : 0u)); };
    for (int ka = 0, kb = 0; kb < NKC; ) {
        if (ka < NKC && kb * NKC >= ka * (NKC - blate)) { xchunk(0, sega, rt_a, 0); ++ka; }
        else { xchunk(sega, HS, rt_b, skew_b); ++kb; }
    }
    ok = ok && q == NP_IN;
    tq = t_first;
    for (int k = -wlead; k < NKC; ++k) {
        if (k + wlead < NKC) wchunk(k + wlead);
        if (k >= 0) xchunk(HS, NSTRIP, rt_c, skew_c);
    }
    ok = ok && q == NP_IN + NP_XI;
    int fill[8] = {0, 0, 0, 0, 0, 0, 0, 0};
    unsigned role[NWG];
    for (int i = 0; i < NWG; ++i) role[i] = R_IDLE;
    auto put = [&](int k, unsigned v) { if (fill[k] < NWG / 8) role[fill[k]++ * 8 + k] = v; else ok = false; };
    auto code = [&](int type, int fs, int cg, int ch) { return (unsigned)type | (fs << 4) | (cg << 12) | (ch << 15); };
    for (int i = 0; i < NCOMM; ++i) put(i & 7, R_COMM | (i << 4));
    unsigned adds = 0;
    for (int cg = 0; cg < 8; ++cg) {
        const bool rem = cg >= 4;
        for (int r = 0; r < 16; ++r) {
            const int k = (cg & 3) + 4 * (r & 1);
            const bool early = r < ne_t, last = r >= ne_t + nc_t;
            const bool split = last ? rem || r - ne_t - nc_t >= TUNE(29, 3) : (rem ? (early ? r >= ne_t - split_a : TUNE(5, 1) ? r - ne_t >= nc_t - split_c : r - ne_t < split_c) : (!early && r - ne_t < TUNE(22, 0)));
            const bool small = last && r - ne_t - nc_t < (rem ? small_b >> 4 : small_b & 15);
            const unsigned win = (unsigned)(!rem ? 0 : early ? TUNE(14, 2) : last ? TUNE(12, 0) : split ? TUNE(13, 0) : TUNE(11, 0)) << 18 |
                                 (rem && r >= op0 && r < op1 ? 1u << 16 : 0u);
            int nw = 0;
            auto rot = [&](int who, int of) { return (unsigned)(TUNE(26, 1) ? (of == 2 ? (who & 1) * 2 + ((r >> 1) & 1) : (who ^ (((r >> 1) & 1) * 2)) & 3) : 0) << 28; };
            if (small) for (int h = 0; h < 4; ++h) { put(k, rot(rem * 2 + (h & 1), 4) | win | code(R_T64H, r * 8 + (h >> 1) * 4, cg, h & 1)); nw += 4; }
            else if (split && (TUNE(23, 3) >> (last ? 0 : 1) & 1)) for (int h = 0; h < 2; ++h) { put(k, (unsigned)((rem * 2 + h * TUNE(24, 1)) ^ ((r >> 1) & 1)) << 28 | win | code(R_T64W, r * 8 + h * 4, cg, 0)); nw += 4; }
            else if (split) for (int ch = 0; ch < 2; ++ch) { put(k, rot(rem * 2 + ch, 4) | win | code(R_T128H, r * 8, cg, ch)); nw += 4; }
            else { put(k, rot(rem, 2) | win | code(R_T128, r * 8, cg, 0)); nw += 4; }
            if (cg == 4 && !(win >> 16 & 1)) adds += nw;
            ok = ok && !((win >> 16 & 1) && split && !(TUNE(23, 3) & 2));
        }
    }
    unsigned nun = 0;
    for (int i = 0; i < NWG; ++i) if (role[i] == R_IDLE && nun < (unsigned)TUNE(10, 64)) role[i] = R_UNPACK | (nun++ << 4);
    for (int i = NCOMM; i < NWG; ++i) P.role[i - NCOMM] = role[i];
    P.tlead = TUNE(27, 24) | TUNE(28, 250) << 16;
    P.out_adds = adds; P.tpoll = TUNE(30, 100) | TUNE(31, 1) << 16; P.nunpack = nun | (unsigned)(op1 - op0) * 4 << 8 | (unsigned)op0 * 4 << 16;
    ok = ok && nun > 0;
    if (!ok) { fprintf(stderr, "ag_gemm: role table overflow\n"); return; }
    hipLaunchKernelGGL(ag_gemm_kernel, dim3(NWG), dim3(512), 0, stream, P);
    HIP_CHECK(hipGetLastError());
}
```

```cpp
#include <hip/hip_runtime.h>
#include <rccl/rccl.h>
#include <cstdio>
#include <cstdint>

#define HIP_CHECK(expr) do { hipError_t _e = (expr); if (_e != hipSuccess) { \
    fprintf(stderr, "HIP error %s at %s:%d\n", hipGetErrorString(_e), __FILE__, __LINE__); \
    return; } } while (0)

namespace agk {

typedef unsigned int u32x4 __attribute__((ext_vector_type(4)));
typedef unsigned int u32x3 __attribute__((ext_vector_type(3)));
typedef unsigned int u32x2 __attribute__((ext_vector_type(2)));
typedef short s16x4 __attribute__((ext_vector_type(4)));
typedef short s16x8 __attribute__((ext_vector_type(8)));
typedef __bf16 bf16x8 __attribute__((ext_vector_type(8)));
typedef float f32x4 __attribute__((ext_vector_type(4)));
#define LAS __attribute__((address_space(3)))
#define GAS __attribute__((address_space(1)))
typedef GAS unsigned int gu32;

constexpr int NR = 8;
constexpr int KD = 4096;
constexpr int MSH = 512;
constexpr int NSH = 512;
constexpr int NSTRIP = 32;
#define AGK_HS 21
constexpr int HS = AGK_HS;
constexpr int AS = NSTRIP - HS;
constexpr int NKC = 16;
constexpr int NSTEP = 64;
constexpr int NWP = 128;

#define AGK_NCOMM 56
#define AGK_NCW 4
constexpr int NCOMM = AGK_NCOMM;
constexpr int NCW = AGK_NCW;
constexpr int WPL = NCOMM * NCW / 7;
static_assert(NCOMM % 7 == 0, "a comm workgroup serves one link");

constexpr int NP_IN = HS * NKC;
constexpr int NP_XI = NWP + AS * NKC;
constexpr int NPMAX = NP_IN > NP_XI ? NP_IN : NP_XI;
constexpr int NRND = (NPMAX + WPL - 1) / WPL;
constexpr int NWG = 256;
enum { R_IDLE = 0, R_COMM = 1, R_T128 = 2, R_T128H = 3, R_T64H = 4, R_UNPACK = 5, R_T64W = 6 };

constexpr size_t BUF_BYTES = 16ull * KD * 2;
constexpr size_t STRIP_BYTES = 2 * BUF_BYTES;
constexpr int NXSLOT = 3 * HS + 4 * AS;
constexpr size_t XS_BYTES = (size_t)NXSLOT * STRIP_BYTES;
constexpr size_t WSL_BYTES = 8 * STRIP_BYTES;
constexpr int XF_STRIDE = 256;
constexpr size_t XFLAG_OFF = XS_BYTES + 4 * WSL_BYTES;
constexpr size_t WFLAG_OFF = XFLAG_OFF + 8 * 32 * XF_STRIDE;
constexpr size_t PK_OFF = XFLAG_OFF + STRIP_BYTES;
constexpr unsigned PK_BYTES = 5888;
constexpr int NP_OUT = 56;
constexpr unsigned PK_PER = BUF_BYTES / (PK_BYTES + 128);
constexpr size_t OCNT_OFF = PK_OFF + (3 * NP_IN + 4 * NP_XI + 4 * NP_OUT + PK_PER - 1) / PK_PER * STRIP_BYTES;
constexpr size_t TMO_OFF = OCNT_OFF + 4 * 256;
constexpr size_t CALL_OFF = TMO_OFF + 256;
constexpr size_t DBG_OFF = CALL_OFF + 256 * 64;
constexpr size_t DBG2_OFF = DBG_OFF + 256 * 128;
constexpr size_t WS_NEED = DBG2_OFF + 8 * 512 * 32;
static_assert(WFLAG_OFF + 4 * NWP * 4 <= XFLAG_OFF + BUF_BYTES, "flags");
static_assert(WS_NEED <= 65ull << 20, "workspace");

constexpr unsigned long long DEADLINE_TICKS = 30000000ull;
constexpr int LDS_BYTES = 163840;

struct Params {
    const char* x;
    const char* w;
    char* ws[NR];
    char* out[NR];
    int rank;
    unsigned out_adds;
    unsigned tpoll;
    unsigned nunpack;
    unsigned role[NWG - NCOMM];
    unsigned tlead;
    unsigned ord[2 * WPL * NRND];
    unsigned char strip[128];
};

static_assert(sizeof(Params) <= 4096, "kernel arguments");

__device__ __forceinline__ unsigned ld_sys(const gu32* p) { return __hip_atomic_load(p, __ATOMIC_RELAXED, __HIP_MEMORY_SCOPE_SYSTEM); }
__device__ __forceinline__ unsigned ld_agent(const gu32* p) { return __hip_atomic_load(p, __ATOMIC_RELAXED, __HIP_MEMORY_SCOPE_AGENT); }
__device__ __forceinline__ void st_agent(gu32* p, unsigned v) { __hip_atomic_store(p, v, __ATOMIC_RELAXED, __HIP_MEMORY_SCOPE_AGENT); }
__device__ __forceinline__ unsigned long long now() { return __builtin_amdgcn_s_memrealtime(); }
__device__ __forceinline__ unsigned rfl(unsigned v) { return __builtin_amdgcn_readfirstlane(v); }

__device__ __forceinline__ void st16_wt(GAS char* p, u32x4 v) {
    asm volatile("global_store_dwordx4 %0, %1, off sc0 sc1\n\ts_nop 1" :: "v"(p), "v"(v) : "memory");
}
template <int N> __device__ __forceinline__ void wait_vm() { asm volatile("s_waitcnt vmcnt(%0)" :: "i"(N) : "memory"); }
__device__ __forceinline__ void wait_lgkm() { asm volatile("" ::: "memory"); __builtin_amdgcn_s_waitcnt(0xC07F); asm volatile("" ::: "memory"); }
#define AGK_SL1 6
#define AGK_SL2 2
#define PIN() __builtin_amdgcn_sched_barrier(0)
__device__ __forceinline__ void sleep_until(unsigned t) {
    int rest = (int)(t - (unsigned)now());
    for (; rest > 48; rest = (int)(t - (unsigned)now())) __builtin_amdgcn_s_sleep(8);
    for (; rest > 2; rest -= 3) __builtin_amdgcn_s_sleep(1);
}
__device__ __forceinline__ void bar() { asm volatile("s_barrier" ::: "memory"); }

#define AGK_LDPOL "sc0 sc1"
__device__ __forceinline__ void glds2(const GAS char* base, unsigned v0, unsigned v1, unsigned lds_dst) {
    asm volatile("s_mov_b32 m0, %3\n\ts_nop 3\n\tglobal_load_lds_dwordx4 %1, %0 " AGK_LDPOL "\n\tglobal_load_lds_dwordx4 %2, %0 offset:1024 " AGK_LDPOL
                 :: "s"(base), "v"(v0), "v"(v1), "s"(lds_dst) : "m0", "memory");
}
template <int OFF> __device__ __forceinline__ void glds2o(const GAS char* base, unsigned v0, unsigned v1) {
    asm volatile("global_load_lds_dwordx4 %1, %0 offset:%3 " AGK_LDPOL "\n\tglobal_load_lds_dwordx4 %2, %0 offset:%4 " AGK_LDPOL
                 :: "s"(base), "v"(v0), "v"(v1), "i"(OFF), "i"(OFF + 1024) : "memory");
}
__device__ __forceinline__ void wait_vm_var(unsigned n) {
    asm volatile("s_getpc_b64 vcc\n\ts_add_u32 vcc_lo, vcc_lo, %0\n\ts_addc_u32 vcc_hi, vcc_hi, 0\n\ts_setpc_b64 vcc\n\t"
                 ".set agk_n, 0\n\t.rept 64\n\ts_waitcnt vmcnt(agk_n)\n\ts_branch 1f\n\t.set agk_n, agk_n + 1\n\t.endr\n1:"
                 :: "s"(n * 8 + 12) : "vcc", "scc", "memory");
}

__device__ __forceinline__ unsigned pack_bf16(float lo, float hi) {
    typedef float f32x2 __attribute__((ext_vector_type(2)));
    typedef __bf16 bf16x2 __attribute__((ext_vector_type(2)));
    f32x2 v = {lo, hi};
    bf16x2 r = __builtin_convertvector(v, bf16x2);
    return __builtin_bit_cast(unsigned, r);
}

__device__ __forceinline__ int xslot(int j, int s) { return j < 4 ? (j - 1) * HS + s : 3 * HS + (j - 4) * AS + (s - HS); }

#define STAMP(i) do { } while (0)
#define PSTAMP(j, p, i, t) do { } while (0)

enum { K_X = 0, K_W = 1 };
struct Piece { int kind, raw; unsigned soff, doff, foff, when; };

__host__ __device__ __forceinline__ int ord_at(int cross, int p) { return (cross * WPL + p % WPL) * NRND + p / WPL; }
__device__ __forceinline__ Piece locate(unsigned e, int j) {
    Piece r;
    r.when = e >> 16;
    const int s = (e >> 4) & 31, kc = e & 15, wp = (e >> 10) & 1 ? (int)(e & 127) : -1;
    r.raw = (e >> 9) & 1;
    if (wp >= 0) {
        r.kind = K_W;
        r.soff = (unsigned)(wp * 32 * (NSH * 2) + (j & 3) * 256);
        r.doff = (unsigned)XS_BYTES + (unsigned)(j - 4) * (unsigned)WSL_BYTES + (wp & 1) * 2 * (unsigned)STRIP_BYTES + (wp >> 1) * 2048;
        r.foff = (unsigned)WFLAG_OFF + ((j - 4) * NWP + (wp & 1) * 64 + ((wp >> 1) & 3) * 16 + (wp >> 3)) * 4;
    } else {
        r.kind = K_X;
        r.soff = (unsigned)(s * 16 * (KD * 2) + kc * 512);
        r.doff = (unsigned)xslot(j, s) * (unsigned)STRIP_BYTES + kc * 8192;
        r.foff = (unsigned)XFLAG_OFF + (j * 32 + s) * XF_STRIDE + kc * 4;
    }
    return r;
}
__device__ __forceinline__ unsigned pk_off(int j, int p) {
    const unsigned i = (j < 4 ? (j - 1) * NP_IN : 3 * NP_IN + (j - 4) * (NP_XI + NP_OUT)) + p;
    return (unsigned)PK_OFF + 3072u + i * PK_BYTES + i / PK_PER * ((unsigned)STRIP_BYTES - PK_PER * PK_BYTES);
}

__device__ __forceinline__ unsigned long_list(unsigned off) { return PK_PER * PK_BYTES - 3072u - (off - (unsigned)PK_OFF) % (unsigned)STRIP_BYTES / PK_BYTES * (PK_BYTES - 128); }
__device__ __forceinline__ unsigned pko_off(int j, int p) { return pk_off(j, NP_XI + p); }

template <int OFF> __device__ __forceinline__ void st_pk(const u32x4& v, unsigned voff, const GAS char* dst) {
    asm volatile("s_nop 3\n\tglobal_store_dwordx4 %1, %0, %2 offset:%3 sc0 sc1\n\ts_nop 1" :: "v"(v), "v"(voff), "s"(dst), "i"(OFF) : "memory");
}
template <int OFF> __device__ __forceinline__ void ld_c(u32x4& v, unsigned voff, const GAS char* src) {
    asm volatile("global_load_dwordx4 %0, %1, %2 offset:%3 sc0 sc1" : "=v"(v) : "v"(voff), "s"(src), "i"(OFF) : "memory");
}
__device__ __forceinline__ void flag_st(const GAS char* f, unsigned val, unsigned zero) {
    asm volatile("s_mov_b64 exec, 1\n\ts_nop 3\n\tglobal_store_dword %0, %1, %2 sc0 sc1\n\ts_mov_b64 exec, -1" :: "v"(zero), "v"(val), "s"(f) : "memory");
}
__device__ __forceinline__ unsigned perm(unsigned hi, unsigned lo, unsigned sel) { return __builtin_amdgcn_perm(hi, lo, sel); }
__device__ __forceinline__ unsigned bfi(unsigned m, unsigned a, unsigned b) { return (a & m) | (b & ~m); }
__device__ __forceinline__ unsigned rotr(unsigned v, unsigned n) { return __builtin_amdgcn_alignbit(v, v, n); }
__device__ __forceinline__ unsigned ffbl(unsigned v) { unsigned r; asm("v_ffbl_b32 %0, %1" : "=v"(r) : "v"(v)); return r; }
__device__ __forceinline__ unsigned pkmax(unsigned a, unsigned b) {
    typedef unsigned short u16x2 __attribute__((ext_vector_type(2)));
    return __builtin_bit_cast(unsigned, __builtin_elementwise_max(__builtin_bit_cast(u16x2, a), __builtin_bit_cast(u16x2, b)));
}
template <int CTRL> __device__ __forceinline__ unsigned dppmax(unsigned v) { return max(v, (unsigned)__builtin_amdgcn_update_dpp(0, (int)v, CTRL, 0xF, 0xF, true)); }
constexpr unsigned F_RAW = 0x80u;
constexpr int MAXEXC = 8, NLIST = 128, NLONG = 32;
template <int CTRL, int ROWS> __device__ __forceinline__ unsigned dppadd(unsigned v) { return v + (unsigned)__builtin_amdgcn_update_dpp(0, (int)v, CTRL, ROWS, 0xF, true); }
__device__ __forceinline__ unsigned ximg0(unsigned lane, unsigned pitch) { return (lane >> 3) * pitch + ((lane & 7) ^ (lane >> 4)) * 16; }
__device__ __forceinline__ unsigned ximg1(unsigned lane, unsigned pitch) { return (8 + (lane >> 3)) * pitch + ((lane & 7) ^ (4 + (lane >> 4))) * 16 - 1024; }
__device__ __forceinline__ unsigned wimg0(unsigned lane, unsigned pitch) { return (lane >> 3) * pitch + ((lane & 7) ^ ((lane >> 3) & 2)) * 16; }
__device__ __forceinline__ unsigned wimg1(unsigned lane, unsigned pitch) { return (8 + (lane >> 3)) * pitch + ((lane & 7) ^ (((lane >> 3) & 2) | 4)) * 16 - 1024; }

__device__ __forceinline__ void fetch_piece(int kind, unsigned soff, const GAS char* sx, const GAS char* sw,
                                            unsigned vx0, unsigned vx1, unsigned vw0, unsigned vw1, unsigned l) {
    const bool kx = kind == K_X;
    const GAS char* s = (kx ? sx : sw) + soff;
    const unsigned a = kx ? vx0 : vw0, b = kx ? vx1 : vw1;
    const int q1 = kx ? 128 : 16 * NSH * 2, q2 = kx ? 256 : 128;
    glds2(s, a, b, l); glds2(s + q1, a, b, l + 2048); glds2(s + q2, a, b, l + 4096); glds2(s + q1 + q2, a, b, l + 6144);
}

__device__ __forceinline__ unsigned tag_of(unsigned call) { return ((call + 1u) >> 1) << 31; }

struct Packed { u32x4 W[5]; u32x2 W5; unsigned trl, ext; bool raw, more; };
__device__ __forceinline__ void pack(const u32x4 (&v)[8], Packed& Q, unsigned call, unsigned lds) {
    const unsigned lane = threadIdx.x & 63, tagw = tag_of(call);
    u32x4 (&W)[5] = Q.W;
    u32x2 W5;
    u32x2 em;
    unsigned trl;
    unsigned mx = pkmax(pkmax(pkmax(v[0].x & 0x7FFF7FFFu, v[0].y & 0x7FFF7FFFu), pkmax(v[0].z & 0x7FFF7FFFu, v[0].w & 0x7FFF7FFFu)),
                        pkmax(pkmax(v[6].x & 0x7FFF7FFFu, v[6].y & 0x7FFF7FFFu), pkmax(v[6].z & 0x7FFF7FFFu, v[6].w & 0x7FFF7FFFu)));
    mx = max(mx >> 16, mx & 0xFFFFu) >> 8;
    mx = dppmax<0x143>(dppmax<0x142>(dppmax<0x118>(dppmax<0x114>(dppmax<0x112>(dppmax<0x111>(mx))))));
    unsigned fmt = max((unsigned)__builtin_amdgcn_readlane((int)mx, 63), 7u) - 7u;
    const unsigned B4 = fmt * 0x01010101u;
    unsigned accO = 0, accA = ~0u, N[8];
#pragma unroll
    for (int k = 0; k < 8; ++k) {
        const unsigned Ha = perm(v[k].y, v[k].x, 0x07050301u), Hb = perm(v[k].w, v[k].z, 0x07050301u);
        const unsigned da = (Ha | 0x80808080u) - B4, db = (Hb | 0x80808080u) - B4;
        accO |= da | db; accA &= da & db;
        W[k >> 1][(k & 1) * 2] = perm(v[k].y, v[k].x, 0x06040200u);
        W[k >> 1][(k & 1) * 2 + 1] = perm(v[k].w, v[k].z, 0x06040200u);
        N[k] = bfi(0x87878787u, bfi(0x80808080u, Ha, da), rotr(bfi(0x80808080u, Hb, db), 28));
    }
    em.x = ~bfi(0x33333333u, bfi(0x11111111u, N[0] >> 2, N[1] >> 1), bfi(0x44444444u, N[2], N[3] << 1));
    em.y = ~bfi(0x33333333u, bfi(0x11111111u, N[4] >> 2, N[5] >> 1), bfi(0x44444444u, N[6], N[7] << 1));
    W[4] = (u32x4){bfi(0x44444444u, N[6] << 2, N[0]), bfi(0x44444444u, N[6] << 1, N[1]), bfi(0x44444444u, N[6] >> 1, N[2]), bfi(0x44444444u, N[7] << 2, N[3])};
    W5 = (u32x2){bfi(0x44444444u, N[7] << 1, N[4]), bfi(0x44444444u, N[7] >> 1, N[5])};
    bool raw = false;
    unsigned nfront = 0;
    const unsigned long long bm = __ballot(((accO & 0x78787878u) | (~accA & 0x80808080u)) != 0);
    if (bm) {
        unsigned nb = 0, ib = 0, hv = 0, ib2 = 0, hv2 = 0;
#pragma unroll
        for (int i = 0; i < 16; ++i) {
            const unsigned H = (i & 1) ? perm(v[i >> 1].w, v[i >> 1].z, 0x07050301u) : perm(v[i >> 1].y, v[i >> 1].x, 0x07050301u);
            const bool bad = (((H | 0x80808080u) - B4) & 0xF8F8F8F8u) != 0x80808080u;
            hv2 = bad ? hv : hv2; ib2 = bad ? ib : ib2; hv = bad ? H : hv; ib = bad ? (unsigned)i : ib; nb += bad;
        }
        const unsigned long long b2 = __ballot(nb > 1);
        const unsigned cnt = __builtin_popcountll(bm), cntb = __builtin_popcountll(b2);
        raw = cnt + cntb > MAXEXC || __ballot(nb > 2) != 0;
        auto put = [&](unsigned e, unsigned h, unsigned i) { *(LAS u32x2*)(size_t)(lds + e * 8) = (u32x2){(h & 0xFFFu) | (h << 4 & 0xFFF0000u), h >> 24 | i << 8 | lane << 16}; };
        if (nb && !raw) put(__builtin_amdgcn_mbcnt_hi((unsigned)(bm >> 32), __builtin_amdgcn_mbcnt_lo((unsigned)bm, 0)), hv, ib);
        if (nb > 1 && !raw) put(__builtin_amdgcn_mbcnt_hi((unsigned)(b2 >> 32), __builtin_amdgcn_mbcnt_lo((unsigned)b2, cnt)), hv2, ib2);
        fmt |= cnt << 8 | cntb << 12;
        nfront = (cnt + cntb) * 4;
    }
    const unsigned c = __builtin_popcount(em.x) + __builtin_popcount(em.y);
    const unsigned inc = dppadd<0x143, 0xC>(dppadd<0x142, 0xA>(dppadd<0x118, 0xF>(dppadd<0x114, 0xF>(dppadd<0x112, 0xF>(dppadd<0x111, 0xF>(c))))));
    const unsigned npos = __builtin_amdgcn_readlane(inc, 63);
    raw = raw || nfront + npos > NLIST + NLONG;
    Q.more = !raw && nfront + npos > NLIST;
    fmt |= npos << 16;
    if (!raw) {
        unsigned at = lds + (nfront + inc - c) * 2;
        for (unsigned long long m = (unsigned long long)em.y << 32 | em.x; m != 0; m &= m - 1, at += 2)
            *(LAS unsigned short*)(size_t)at = (unsigned short)(min(ffbl((unsigned)m), ffbl((unsigned)(m >> 32)) | 32u) | lane << 6);
    }
    asm volatile("" ::: "memory");
    const unsigned two = *(LAS unsigned*)(size_t)(lds + lane * 4);
    Q.ext = *(LAS unsigned*)(size_t)(lds + NLIST * 2 + (lane & 15) * 4);
    const unsigned pos = bfi(0x000FFF00u, two << 8, two << 4);
    if (raw) fmt = F_RAW;
    trl = (tagw >> 30) + (__builtin_amdgcn_inverse_ballot_w64((unsigned long long)call << 32 | fmt) ? 1u : 0u);
#pragma unroll
    for (int k = 0; k < 5; ++k) { trl = __builtin_amdgcn_alignbit(trl, W[k].w, 31); W[k].w = bfi(0x7FFFFFFFu, W[k].w, tagw); }
    trl = __builtin_amdgcn_alignbit(trl, W5.y, 31); W5.y = bfi(0x7FFFFFFFu, W5.y, tagw);
    trl = __builtin_amdgcn_alignbit(trl, pos, 8);
    Q.W5 = W5; Q.trl = trl; Q.raw = raw;
}
__device__ __forceinline__ void send_packed(const Packed& Q, const GAS char* pk, unsigned off, unsigned call, unsigned vlin, bool few) {
    st_pk<-3072>(Q.W[0], vlin, pk); st_pk<-2048>(Q.W[1], vlin, pk);
    if (few) wait_vm<1>();
    st_pk<-1024>(Q.W[2], vlin, pk);
    if (few) wait_vm<1>();
    st_pk<0>(Q.W[3], vlin, pk);
    if (few) wait_vm<1>();
    st_pk<1024>(Q.W[4], vlin, pk);
    if (few) wait_vm<1>();
    asm volatile("global_store_dwordx2 %0, %1, %2 offset:2048 sc0 sc1" :: "v"(vlin >> 1), "v"(Q.W5), "s"(pk) : "memory");
    if (__builtin_expect(Q.more, 0))
        asm volatile("s_mov_b64 exec, 0xffff\n\ts_nop 3\n\tglobal_store_dwordx2 %0, %1, %2 sc0 sc1\n\ts_mov_b64 exec, -1" :: "v"(vlin >> 1), "v"((u32x2){Q.ext, call}), "s"(pk + long_list(off)) : "memory");
    asm volatile("global_store_dword %0, %1, %2 offset:2560 sc0 sc1" :: "v"(vlin >> 2), "v"(Q.trl), "s"(pk) : "memory");
}

__device__ __forceinline__ void comm_role(const Params& P, LAS char* smem, unsigned long long t64, int wslot, int j, int u) {
    const unsigned* ord = P.ord + ord_at(j >= 4, u);
    unsigned eA = ord[0], eB = ord[1], eC = ord[2], tlead = P.tlead;
    const GAS char *srcx = (const GAS char*)P.x, *srcw = (const GAS char*)P.w, *pws = (const GAS char*)P.ws[j];
    GAS char* myws = (GAS char*)P.ws[0];
    asm volatile("" : "+s"(eA), "+s"(eB), "+s"(eC), "+s"(tlead), "+s"(srcx), "+s"(srcw), "+s"(pws), "+s"(myws), "+s"(t64));
    const unsigned tstart = (unsigned)t64;
    gu32* cc = (gu32*)(myws + CALL_OFF + blockIdx.x * 64) + wslot;
    const int lane = threadIdx.x & 63;
    const int npiece = j < 4 ? NP_IN : NP_XI;
    const unsigned vlin = lane * 16;
    const unsigned vx0 = ximg0(lane, KD * 2), vx1 = ximg1(lane, KD * 2), vw0 = wimg0(lane, NSH * 2), vw1 = wimg1(lane, NSH * 2);
    const unsigned vblk = (lane >> 5) * 2048 + (lane & 31) * 16, vblw = (lane >> 5) * (unsigned)STRIP_BYTES + (lane & 31) * 16;
    const unsigned lds0 = (unsigned)(size_t)smem;
#define AGK_FETCH(q, slot) fetch_piece((q).kind, (q).soff, srcx, srcw, vx0, vx1, vw0, vw1, lds0 + (slot) * 8192)
    int pA = u;
    unsigned ccv;
    Piece A = locate(eA, j);
    if (u > 3) sleep_until(tstart + (u * (tlead & 0xFFFFu) >> 2));
    AGK_FETCH(A, 0);
    asm volatile("global_load_dword %0, %1, off sc1" : "=v"(ccv) : "v"(cc) : "memory");
    Piece B = locate(eB, j);
    asm volatile("s_waitcnt vmcnt(0)" : "+v"(ccv) :: "memory");
    AGK_FETCH(B, 1);
    const unsigned call = rfl(ccv) + 1u;
    if (wslot == 0) STAMP(0);
    const GAS char* dbuf = pws + (size_t)(call & 1) * BUF_BYTES;
    gu32* tmo = (gu32*)(myws + TMO_OFF);
    unsigned zero = 0, callv = call;
    asm volatile("" : "+v"(zero), "+v"(callv));
    int slot = 0;
    for (;;) {
        u32x4 v[8];
#pragma unroll
        for (int k = 0; k < 8; ++k) v[k] = *(LAS u32x4*)(smem + slot * 8192 + (k >> 2) * 4096 + (k & 3) * 512 + vblk);
        Packed Q;
        if (!A.raw) pack(v, Q, call, lds0 + 65536);
        else {
#pragma unroll
            for (int k = 0; k < 5; ++k) Q.W[k] = v[k];
            Q.W5 = (u32x2){v[5].x, v[5].y};
            Q.trl = Q.ext = 0; Q.more = false; Q.raw = true;
        }
        const GAS char* pk = dbuf + pk_off(j, pA);
        const Piece C = locate(eC, j);
        asm volatile("" :: "v"(Q.W[0]), "v"(Q.W[1]), "v"(Q.W[2]), "v"(Q.W[3]), "v"(Q.W[4]), "v"(Q.W5), "v"(v[6]), "v"(v[7]), "v"(Q.trl), "v"(Q.ext),
                     "s"(pk), "s"(C.soff) : "memory");
        sleep_until(tstart + A.when);
        if (!A.raw) send_packed(Q, pk, (unsigned)(pk - dbuf), call, vlin, false);
        if (Q.raw) {
            const GAS char* d0 = dbuf + A.doff;
            const GAS char* d1 = d0 + (A.kind == K_W ? 4 * (unsigned)STRIP_BYTES : 4096u);
            const unsigned vd = A.kind == K_W ? vblw : vblk;
            st_pk<0>(v[0], vd, d0); st_pk<512>(v[1], vd, d0); st_pk<1024>(v[2], vd, d0); st_pk<1536>(v[3], vd, d0);
            st_pk<0>(v[4], vd, d1); st_pk<512>(v[5], vd, d1); st_pk<1024>(v[6], vd, d1); st_pk<1536>(v[7], vd, d1);
        }
        AGK_FETCH(C, slot);
        int iD = pA + 3 * WPL < npiece ? pA / WPL + 3 : 0;
        asm volatile("" : "+s"(iD) :: "memory");
        eC = ord[iD];
        if (Q.raw) { wait_vm<8>(); flag_st(dbuf + A.foff, callv, zero); }
        PSTAMP(j, pA, 0, tstart + A.when); PSTAMP(j, pA, 1, tb); PSTAMP(j, pA, 2, now());
        pA += WPL; A = B; B = C; slot ^= 1;
        if (pA >= npiece) break;
        wait_vm<15>();
    }
    wait_vm<0>();
    STAMP(j < 4 ? 1 : 3);
    if (j == 1 && u == 0) {
        if (lane < 4) {
            const gu32* c = (const gu32*)(myws + OCNT_OFF + lane * 256);
            const unsigned need = call * P.out_adds;
            const unsigned long long t0 = now();
            bool bad = false;
            unsigned spin = 0;
            while (ld_sys(c) < need) {
                if ((++spin & 63) == 0 && (ld_agent(tmo) == call || now() - t0 > DEADLINE_TICKS)) { st_agent(tmo, call); bad = true; break; }
                __builtin_amdgcn_s_sleep(1);
            }
            if (bad) {
                __builtin_amdgcn_s_sleep(127);
                __hip_atomic_store((GAS unsigned short*)(P.out[0] + (size_t)(P.rank * MSH) * (NSH * 2)), (unsigned short)0x7FC0, __ATOMIC_RELAXED, __HIP_MEMORY_SCOPE_AGENT);
            }
        }
        STAMP(2);
    }
    if (lane == 0) st_agent(cc, call);
}

__device__ __forceinline__ void out_rows(const Params& P, const GAS char* (&d)[4], int jd, int j, int po) {
#pragma unroll
    for (int i = 0; i < 4; ++i) {
        const int code = rfl(P.strip[(po >> 1) * 4 + i]);
        d[i] = (const GAS char*)P.out[jd] + (unsigned)(((P.rank ^ jd ^ j ^ (code >> 5)) * MSH + (code & 31) * 16) * (NSH * 2) + (j & 3) * 256 + (po & 1) * 128);
    }
}
__device__ __forceinline__ void out_role(const Params& P, unsigned ea, unsigned gs, unsigned call, const GAS char* pk, int j, int po, bool few) {
    const unsigned lane = threadIdx.x & 63;
    u32x4 v[8];
#pragma unroll
    for (int n = 0; n < 8; ++n) v[n] = *(LAS u32x4*)(size_t)(ea + (n >> 2) * gs + (n & 3) * 1024 + (lane >> 3) * 128 + (((lane & 7) ^ (lane >> 3)) << 4));
    Packed Q;
    pack(v, Q, call, ea);
    if (Q.raw) {
        const GAS char* d[4];
        out_rows(P, d, j, j, po);
        const unsigned vA = (lane >> 3) * (NSH * 2) + (lane & 7) * 16, vB = vA + 8 * (NSH * 2);
#pragma unroll
        for (int n = 0; n < 8; ++n) st_pk<0>(v[n], n & 1 ? vB : vA, d[n >> 1]);
        wait_vm<0>();
    }
    send_packed(Q, pk, pko_off(j, po - (int)((P.nunpack >> 16) & 255)), call, lane * 16, few);
    wait_vm<0>();
}

__device__ __forceinline__ void unpack_role(const Params& P, GAS char* myws, unsigned lds, unsigned tstart, unsigned call, int w, int nw) {
    const GAS char* buf = myws + (size_t)(call & 1) * BUF_BYTES;
    gu32* tmo = (gu32*)(myws + TMO_OFF);
    const int lane = threadIdx.x & 63;
    const unsigned vlin = lane * 16, tagw = tag_of(call);
    const unsigned vblk = (lane >> 5) * 2048 + (lane & 31) * 16, vblw = (lane >> 5) * (unsigned)STRIP_BYTES + (lane & 31) * 16;
    unsigned zero = 0, callv = call;
    asm volatile("" : "+v"(zero), "+v"(callv));
    const unsigned tfirst = tstart + (P.tpoll & 0xFFFFu);
    const unsigned spec = P.tpoll >> 16;
    for (int q = w; ; q += nw) {
        const int p = q / 7, j = 1 + q - p * 7;
        const bool outp = q >= NPMAX * 7;
        const GAS char *pk, *d[4];
        unsigned vA, vB, foff = 0;
        if (!outp) {
            if (p >= (j < 4 ? NP_IN : NP_XI)) continue;
            const Piece A = locate(P.ord[ord_at(j >= 4, p)], j);
            if (A.raw) continue;
            pk = buf + pk_off(j, p);
            d[0] = buf + A.doff; d[1] = d[0] + 1024;
            d[2] = d[0] + (A.kind == K_W ? 4 * (unsigned)STRIP_BYTES : 4096u); d[3] = d[2] + 1024;
            vA = A.kind == K_W ? vblw : vblk; vB = vA + 512;
            foff = A.foff;
            sleep_until(tfirst + A.when);
        } else {
            const int i = q - NPMAX * 7, jo = 4 + (i & 3), po = (i >> 2) + (int)((P.nunpack >> 16) & 255);
            if (i >= (int)((P.nunpack >> 8) & 255) * 4) break;
            pk = buf + pko_off(jo, i >> 2);
            out_rows(P, d, 0, jo, po);
            vA = (lane >> 3) * (NSH * 2) + (lane & 7) * 16; vB = vA + 8 * (NSH * 2);
        }
        unsigned long long t0 = 0;
        unsigned spin = 0, trl, fmt = 0;
        bool more = false;
        u32x4 W[5]; u32x2 W5, ex = {0, 0};
        *(LAS u32x2*)(size_t)(lds + lane * 8) = ex;
        bool data = (spec >> (outp ? 1 : 0)) & 1;
        for (;;) {
            asm volatile("s_nop 4\n\tglobal_load_dword %0, %1, %2 offset:2560 sc0 sc1" : "=v"(trl) : "v"(vlin >> 2), "s"(pk) : "memory");
            if (data) {
                if (more) asm volatile("global_load_dwordx2 %0, %1, %2 sc0 sc1" : "+v"(ex) : "v"(((vlin >> 1) & 127u) + long_list((unsigned)(pk - buf))), "s"(pk) : "memory");
                asm volatile("global_load_dwordx2 %0, %1, %2 offset:2048 sc0 sc1" : "=v"(W5) : "v"(vlin >> 1), "s"(pk) : "memory");
                ld_c<1024>(W[4], vlin, pk);
                ld_c<-3072>(W[0], vlin, pk); ld_c<-2048>(W[1], vlin, pk); ld_c<-1024>(W[2], vlin, pk); ld_c<0>(W[3], vlin, pk);
            }
            asm volatile("s_waitcnt vmcnt(0)" : "+v"(trl), "+v"(W[0]), "+v"(W[1]), "+v"(W[2]), "+v"(W[3]), "+v"(W[4]), "+v"(W5), "+v"(ex) :: "memory");
            const unsigned long long fc = __ballot((int)(trl << 1) < 0);
            if (__ballot((int)(trl ^ tagw) < 0) == 0 && (unsigned)(fc >> 32) == call) {
                fmt = (unsigned)fc;
                if (fmt & F_RAW) break;
                more = ((fmt >> 8) & 15) * 4 + ((fmt >> 12) & 15) * 4 + (fmt >> 16) > NLIST;
                if (data && __ballot((int)((W[0].w ^ tagw) | (W[1].w ^ tagw) | (W[2].w ^ tagw) | (W[3].w ^ tagw) | (W[4].w ^ tagw) | (W5.y ^ tagw)) < 0) == 0 &&
                    (!more || __ballot(ex.y != call) == 0)) break;
                data = true;
            } else __builtin_amdgcn_s_sleep(AGK_SL2);
            if ((++spin & 63) == 0) {
                if (t0 == 0) t0 = now();
                if (rfl(ld_agent(tmo)) == call || now() - t0 > DEADLINE_TICKS) {
                    st_agent(tmo, call);
                    if (outp) st_pk<0>((u32x4){0x7FC07FC0u, 0x7FC07FC0u, 0x7FC07FC0u, 0x7FC07FC0u}, vA, d[0]);
                    fmt = F_RAW;
                    break;
                }
            }
        }
        if (!outp) { PSTAMP(j, p, 3, now()); PSTAMP(j, p, 7, spin); PSTAMP(j, p, 4, now()); }
        if (fmt & F_RAW) continue;
        const int cnt = (fmt >> 8) & 15, cnta = cnt + ((fmt >> 12) & 15);
        auto mark = [&](unsigned e, unsigned ps) {
            __hip_atomic_fetch_or((LAS unsigned*)(size_t)(lds + ((ps >> 3) & 0x1FCu)), e - cnta * 4 < (fmt >> 16) ? 1u << (ps & 31) : 0u, __ATOMIC_RELAXED, __HIP_MEMORY_SCOPE_WAVEFRONT);
        };
        mark(lane * 2, trl); mark(lane * 2 + 1, trl >> 12);
        if (more) { mark(NLIST + (lane & 15) * 2, ex.x); mark(NLIST + (lane & 15) * 2 + 1, ex.x >> 16); }
        asm volatile("" ::: "memory");
        const u32x2 mm = *(LAS u32x2*)(size_t)(lds + lane * 8);
        const unsigned m0 = mm.x, m1 = mm.y;
#pragma unroll
        for (int k = 0; k < 5; ++k) W[k].w = bfi(0x7FFFFFFFu, W[k].w, trl << (2 + k));
        W5.y = bfi(0x7FFFFFFFu, W5.y, trl << 7);
        unsigned N[8];
        N[0] = bfi(0x44444444u, ~(m0 << 2), W[4].x); N[1] = bfi(0x44444444u, ~(m0 << 1), W[4].y); N[2] = bfi(0x44444444u, ~m0, W[4].z); N[3] = bfi(0x44444444u, ~(m0 >> 1), W[4].w);
        N[4] = bfi(0x44444444u, ~(m1 << 2), W5.x); N[5] = bfi(0x44444444u, ~(m1 << 1), W5.y);
        N[6] = bfi(0x33333333u, bfi(0x11111111u, W[4].x >> 2, W[4].y >> 1), bfi(0x44444444u, ~m1, W[4].z << 1));
        N[7] = bfi(0x33333333u, bfi(0x11111111u, W[4].w >> 2, W5.x >> 1), bfi(0x44444444u, ~(m1 >> 1), W5.y << 1));
        const unsigned B4 = (fmt & 127u) * 0x01010101u;
        unsigned H[16];
#pragma unroll
        for (int k = 0; k < 8; ++k) {
            H[2 * k] = (N[k] & 0x87878787u) + B4;
            H[2 * k + 1] = (rotr(N[k], 4) & 0x87878787u) + B4;
        }
        for (int e = 0, e1 = cnt; e < e1; e1 = cnta) {
            unsigned tag = 255, val = 0;
            for (; e < e1; ++e) {
                const unsigned f0 = __builtin_amdgcn_readlane(trl, 2 * e), f1 = __builtin_amdgcn_readlane(trl, 2 * e + 1);
                const bool mine = ((f1 >> 12) & 63) == (unsigned)lane;
                tag = mine ? (f1 >> 8) & 15 : tag; val = mine ? (f0 & 0xFFFFFFu) | f1 << 24 : val;
            }
#pragma unroll
            for (int i = 0; i < 16; ++i) H[i] = tag == (unsigned)i ? val : H[i];
        }
#define AGK_DST(k) { const unsigned l0 = W[k >> 1][(k & 1) * 2], l1 = W[k >> 1][(k & 1) * 2 + 1]; u32x4 o; \
            o.x = perm(H[2 * k], l0, 0x05010400u); o.y = perm(H[2 * k], l0, 0x07030602u); \
            o.z = perm(H[2 * k + 1], l1, 0x05010400u); o.w = perm(H[2 * k + 1], l1, 0x07030602u); st_pk<0>(o, k & 1 ? vB : vA, d[k >> 1]); }
        AGK_DST(0) AGK_DST(1) AGK_DST(2) AGK_DST(3) AGK_DST(4) AGK_DST(5) AGK_DST(6) AGK_DST(7)
        if (outp) continue;
        PSTAMP(j, p, 5, now());
        wait_vm<0>(); flag_st(buf + foff, callv, zero);
        PSTAMP(j, p, 6, now());
    }
    wait_vm<0>();
}

#define AGK_RPG 2
#define AGK_RA 4, 6
#define AGK_RH 4, 12
#define AGK_RQ 8, 12
#define AGK_RW 6, 7
#define AGK_LOOK 3
template <int NB, int NP, int NSLOT, int SLOT_BYTES>
__device__ __forceinline__ void loader(GAS char* myws, unsigned call, const GAS char* const (&src)[NP], const int (&shift)[NP], const unsigned (&v0)[NB],
                                       const unsigned (&v1)[NB], const GAS char* fp, bool have, unsigned ldsw, unsigned rot, bool first) {
    constexpr int NI = 2 * NB;
    static_assert((NSLOT - 2) * NI + 8 < 64 && (NSLOT - 1) * NI + 10 < 64, "vmcnt range");
    gu32* tmo = (gu32*)(myws + TMO_OFF);
    const unsigned ldsend = ldsw + NSLOT * SLOT_BYTES;
    const unsigned long long act = ~__ballot(have);
    asm volatile("v_mov_b32 v250, %0" :: "v"(have ? ~0u : 0u) : "v250", "memory");
    int mark = 0;
    unsigned npoll = 0;
    auto poll = [&]() {
        ++npoll;
        asm volatile("s_mov_b64 exec, %1\n\tglobal_load_dword v250, %0, off sc0 sc1\n\ts_mov_b64 exec, -1" :: "v"(fp), "s"(act) : "v250", "memory");
    };
    auto eval = [&]() {
        unsigned long long bx;
        asm volatile("v_cmp_gt_u32 %0, %1, v250" : "=s"(bx) : "s"(call) : "memory");
        const unsigned m = (unsigned)bx | (unsigned)(bx >> 32);
        mark = __builtin_ctz(m | m >> 16 | 0x10000u) * 4;
    };
    unsigned ldsi = ldsw, kt = 0;
    const GAS char* ptr[NP];
#pragma unroll
    for (int i = 0; i < NP; ++i) ptr[i] = src[i] + (rot << shift[i]);
    auto issue = [&]() {
        asm volatile("s_mov_b32 m0, %0\n\ts_nop 0" :: "s"(ldsi) : "m0", "memory");
        glds2o<-4096>(ptr[0], v0[0], v1[0]);
        glds2o<-2048>(ptr[1 % NP], v0[1], v1[1]);
        if (NB == 4) { glds2o<0>(ptr[2 % NP], v0[2 % NB], v1[2 % NB]); glds2o<2048>(ptr[3 % NP], v0[3 % NB], v1[3 % NB]); }
        const unsigned k = ++kt ^ rot;
#pragma unroll
        for (int i = 0; i < NP; ++i) { ptr[i] = src[i] + (k << shift[i]); asm volatile("" : "+s"(ptr[i])); }
        ldsi = ldsi + SLOT_BYTES == ldsend ? ldsw : ldsi + SLOT_BYTES;
        asm volatile("" : "+s"(ldsi));
    };
    int ti = 0, tb = 0;
    unsigned hist = 0;
#pragma nounroll
    for (;;) {
        if (mark < NSTEP && mark - ti <= AGK_LOOK) eval();
        const int lim = mark < tb + NSLOT - 1 ? mark : tb + NSLOT - 1;
#pragma nounroll
        for (; ti < lim; ++ti) { issue(); hist = hist << 4 | (npoll & 15u); }
        if (ti == tb) {
            unsigned long long t0 = 0;
            for (unsigned spin = 1; ti >= mark; ++spin) {
                wait_vm<8>();
                poll();
                __builtin_amdgcn_s_sleep(AGK_SL1);
                if ((spin & 63) == 0) {
                    if (t0 == 0) t0 = now();
                    if (rfl(ld_agent(tmo)) == call || now() - t0 > DEADLINE_TICKS) break;
                }
                eval();
            }
            if (ti >= mark) break;
            continue;
        }
        const int run = (mark < NSTEP ? mark - AGK_LOOK : NSTEP) - ti;
        if (run > 0) {
#pragma nounroll
            for (int i = run; i > 0; --i) { wait_vm<(NSLOT - 2) * NI>(); bar(); issue(); }
            ti += run; tb += run;
            const unsigned rep = (npoll & 15u) * 0x11111111u;
            hist = run >= 8 ? rep : hist << (4 * run) | (rep & ((1u << (4 * run)) - 1u));
        }
        if (mark < NSTEP) poll();
        const unsigned n = ti - tb - 1;
        wait_vm_var(n * NI + (n < 8 ? (npoll - (hist >> (4 * n))) & 15u : 0u));
        bar();
        if (++tb == NSTEP) break;
    }
    if (tb < NSTEP) {
        st_agent(tmo, call);
        const u32x4 nan = {0x7FC07FC0u, 0x7FC07FC0u, 0x7FC07FC0u, 0x7FC07FC0u};
        for (; tb < NSTEP; ++tb) {
#pragma unroll
            for (int i = 0; i < NI; ++i) *(LAS u32x4*)(size_t)(ldsi - 4096 + i * 1024 + (threadIdx.x & 63) * 16) = nan;
            wait_lgkm();
            bar();
            ldsi = ldsi + SLOT_BYTES == ldsend ? ldsw : ldsi + SLOT_BYTES;
        }
    }
    if (first) STAMP(4);
    wait_vm<0>();
}

template <int S, int NH, int NSA, int NSB>
__device__ __forceinline__ unsigned tile_role(const Params& P, LAS char* smem, unsigned call, int first_strip, int cg, int ch, int win, unsigned rot, bool packed) {
    constexpr int SA = S * 2048, SB = NH * 8192, BOFF = NSA * SA;
    constexpr int SPW = S / 2, NFR = NH * 2;
    static_assert(BOFF + NSB * SB <= LDS_BYTES, "LDS rings");
    GAS char* myws = (GAS char*)P.ws[0];
    const int lane = threadIdx.x & 63;
    const int wv = rfl(threadIdx.x >> 6);
    const bool wremote = cg >= 4;

    if (wv >= 4) {
        const int lw = wv & 1;
        const GAS char* mybuf = myws + (size_t)(call & 1) * BUF_BYTES;
        const unsigned vlin = lane * 16;
        if (wv < 6) {
            const GAS char* src[SPW];
            unsigned v0[SPW], v1[SPW];
            int shift[SPW];
#pragma unroll
            for (int i = 0; i < SPW; ++i) {
                const int code = rfl(P.strip[first_strip + lw * SPW + i]);
                const bool own = code < 32;
                v0[i] = own ? ximg0(lane, KD * 2) : vlin;
                v1[i] = own ? ximg1(lane, KD * 2) : vlin;
                src[i] = (own ? (const GAS char*)P.x + (unsigned)code * (16 * KD * 2) : mybuf + (unsigned)xslot(code >> 5, code & 31) * (unsigned)STRIP_BYTES) + (4096 - 2048 * i);
                shift[i] = own ? 7 : 11;
            }
            const int fcode = P.strip[first_strip + lw * SPW + ((lane >> 4) & (SPW - 1))];
            loader<SPW, SPW, NSA, SA>(myws, call, src, shift, v0, v1, mybuf + XFLAG_OFF + fcode * XF_STRIDE + (lane & 15) * 4, fcode < 32 || lane >= 16 * SPW,
                                      (unsigned)(size_t)smem + lw * (SA / 2) + 4096, rot, lw == 0);
        } else {
            const unsigned dk = (wremote ? (unsigned)STRIP_BYTES : 16u * NSH * 2) - 2048, dh = (wremote ? 4 * (unsigned)STRIP_BYTES : 128u) - 4096;
            const GAS char* const src[1] = {(wremote ? mybuf + XS_BYTES + (unsigned)((cg - 4) * 8 + ch * 4 + lw * 2) * (unsigned)STRIP_BYTES
                                                     : (const GAS char*)P.w + cg * 256 + ch * 128 + lw * 32 * (NSH * 2))};
            const int shift[1] = {wremote ? 11 : 16};
            unsigned v0[NFR], v1[NFR];
#pragma unroll
            for (int i = 0; i < NFR; ++i) {
                v0[i] = (wremote ? vlin : wimg0(lane, NSH * 2)) + 4096 + (i & 1) * dk + (i >> 1) * dh;
                v1[i] = (wremote ? vlin : wimg1(lane, NSH * 2)) + 4096 + (i & 1) * dk + (i >> 1) * dh;
            }
            loader<NFR, 1, NSB, SB>(myws, call, src, shift, v0, v1, mybuf + WFLAG_OFF + ((cg & 3) * NWP + lw * 64 + lane) * 4, !wremote,
                                    (unsigned)(size_t)smem + BOFF + lw * (SB / 2) + 4096, rot, false);
        }
    } else {
        const int fr = lane & 15, g = lane >> 4;
        const int wm = wv >> 1, wn = wv & 1;
        unsigned aoff[2];
#pragma unroll
        for (int kk = 0; kk < 2; ++kk) aoff[kk] = wm * (SA / 2) + fr * 128 + (((kk * 4 + g) ^ ((fr >> 1) & 7)) << 4);
        unsigned boff[NFR];
        {
            const int q = fr >> 2, pp = lane & 3, f = (q >> 1) | ((g & 1) << 1);
#pragma unroll
            for (int nb = 0; nb < NFR; ++nb)
                boff[nb] = BOFF + (NH == 2 ? wn * 4096 : 0) + (g >> 1) * 2048 + ((g & 1) * 8 + q) * 128 + ((((wn * NFR + nb) & 3) ^ f) << 5) + pp * 8;
        }
        int rowbase[SPW];
#pragma unroll
        for (int i = 0; i < SPW; ++i) {
            const int code = P.strip[first_strip + wm * SPW + i];
            rowbase[i] = rfl((P.rank ^ (code >> 5)) * MSH + (code & 31) * 16);
        }
        GAS char* outp = (GAS char*)P.out[wremote ? cg : 0] + (cg & 3) * 256;
        GAS char* cntp = (GAS char*)P.ws[cg] + OCNT_OFF + (cg & 3) * 256;
        constexpr int orow = NSH * 2;
        asm volatile("" : "+s"(outp), "+s"(cntp));
        f32x4 acc[SPW][NFR];
#pragma unroll
        for (int i = 0; i < SPW; ++i)
#pragma unroll
            for (int nb = 0; nb < NFR; ++nb) acc[i][nb] = (f32x4){0.f, 0.f, 0.f, 0.f};
        struct Frag { bf16x8 a[SPW]; bf16x8 b[NFR]; };
        auto rd = [&](Frag& F, int sa, int sb, int kk) {
#pragma unroll
            for (int i = 0; i < SPW; ++i) F.a[i] = *(LAS bf16x8*)(smem + sa + aoff[kk] + i * 2048);
#pragma unroll
            for (int nb = 0; nb < NFR; ++nb) {
                const s16x4 lo = __builtin_amdgcn_ds_read_tr16_b64_v4i16((LAS s16x4*)(smem + sb + boff[nb] + kk * (SB / 2)));
                const s16x4 hi = __builtin_amdgcn_ds_read_tr16_b64_v4i16((LAS s16x4*)(smem + sb + boff[nb] + kk * (SB / 2) + 512));
                F.b[nb] = __builtin_bit_cast(bf16x8, __builtin_shufflevector(lo, hi, 0, 1, 2, 3, 4, 5, 6, 7));
            }
        };
        auto mm = [&](const Frag& F) {
#pragma unroll
            for (int nb = 0; nb < NFR; ++nb)
#pragma unroll
                for (int i = 0; i < SPW; ++i) acc[i][nb] = __builtin_amdgcn_mfma_f32_16x16x32_bf16(F.b[nb], F.a[i], acc[i][nb], 0, 0, 0);
        };
        Frag F0, F1;
        int ca = 0, cb = 0;
        bar();
        rd(F0, 0, 0, 0);
        wait_lgkm();
#pragma nounroll
        for (int t = 0; t < NSTEP - 1; ++t) {
            constexpr int NRD = SPW + 2 * NFR, NMM = SPW * NFR, RPG = S * NH == 16 ? 1 : AGK_RPG;
            rd(F1, ca, cb, 1);
            mm(F0);
#pragma unroll
            for (int i = 0; i < NMM; ++i) { __builtin_amdgcn_sched_group_barrier(0x008, 1, 0); if (i * RPG < NRD) __builtin_amdgcn_sched_group_barrier(0x100, RPG, 0); }
            PIN();
            wait_lgkm();
            bar();
            ca = ca + SA == NSA * SA ? 0 : ca + SA;
            cb = cb + SB == NSB * SB ? 0 : cb + SB;
            rd(F0, ca, cb, 0);
            mm(F1);
#pragma unroll
            for (int i = 0; i < NMM; ++i) { __builtin_amdgcn_sched_group_barrier(0x008, 1, 1); if (i * RPG < NRD) __builtin_amdgcn_sched_group_barrier(0x100, RPG, 1); }
            PIN();
            wait_lgkm();
        }
        rd(F1, ca, cb, 1);
        PIN();
        mm(F0);
        PIN();
        mm(F1);
        if (wv == 0) STAMP(2);
        constexpr int ROWB = NFR * 32;
        constexpr int CPR = ROWB / 16;
        constexpr int ESLOT = (NSTEP - 1) % NSB >= 2 ? 0 : 2;
        LAS char* eb = smem + BOFF + ESLOT * SB + wv * (SPW * 16 * ROWB);
        static_assert(4 * SPW * 16 * ROWB <= 2 * SB && NSB >= 4, "epilogue space");
#pragma unroll
        for (int i = 0; i < SPW; ++i)
#pragma unroll
            for (int nb = 0; nb < NFR; ++nb) {
                u32x2 o;
                o.x = pack_bf16(acc[i][nb][0], acc[i][nb][1]);
                o.y = pack_bf16(acc[i][nb][2], acc[i][nb][3]);
                *(LAS u32x2*)(eb + (i * 16 + fr) * ROWB + (((nb * 2 + (g >> 1)) ^ (fr & (CPR - 1))) << 4) + (g & 1) * 8) = o;
            }
        if (NH == 2 && packed) {
            if (S == 4) { wait_lgkm(); bar(); }
            return S == 4 && wm ? 0u : (unsigned)(size_t)eb;
        }
        constexpr int RPI = 64 / CPR;
        constexpr int NST = SPW * 16 / RPI;
        u32x4 ov[NST];
        GAS char* od[NST];
        GAS char* obase = outp + (NH == 2 ? wn * 128 : ch * 128 + wn * 64) + (lane & (CPR - 1)) * 16;
#pragma unroll
        for (int n = 0; n < NST; ++n) {
            const int row = n * RPI + lane / CPR;
            ov[n] = *(LAS u32x4*)(eb + row * ROWB + (((lane & (CPR - 1)) ^ (row & (CPR - 1))) << 4));
            od[n] = obase + (size_t)((rowbase[(n * RPI) / 16] + (row & 15)) * orow);
        }
        if (wv == 0) STAMP(5);
        if (win == 1) {
#pragma unroll
            for (int n = 0; n < NST; ++n) { if (n >= 1) wait_vm<0>(); st16_wt(od[n], ov[n]); }
        } else if (win == 2) {
#pragma unroll
            for (int n = 0; n < NST; ++n) { if (n >= 2) wait_vm<1>(); st16_wt(od[n], ov[n]); }
        } else {
#pragma unroll
            for (int n = 0; n < NST; ++n) st16_wt(od[n], ov[n]);
        }
        if (wremote) {
            wait_vm<0>();
            if (lane == 0) __hip_atomic_fetch_add((gu32*)cntp, 1u, __ATOMIC_RELAXED, __HIP_MEMORY_SCOPE_SYSTEM);
        }
        if (wv == 0) STAMP(3);
    }
    return 0;
}

__global__ void __launch_bounds__(512, 1) __attribute__((amdgpu_num_vgpr(240))) ag_gemm_kernel(const Params P) {
    __shared__ __attribute__((aligned(1024))) char smem_raw[LDS_BYTES];
    LAS char* smem = (LAS char*)smem_raw;
    const unsigned long long t64 = now();
    const unsigned tstart = (unsigned)t64;
    const int b = blockIdx.x;
    if (__builtin_expect(b < NCOMM, 1)) {
        const int wv = rfl(threadIdx.x >> 6), g = wv * NCOMM + b;
        if (wv < NCW) comm_role(P, smem + wv * 16384, t64, wv, 1 + g % 7, g / 7);
        return;
    }
    GAS char* myws = (GAS char*)P.ws[0];
    gu32* cc = (gu32*)(myws + CALL_OFF + b * 64);
    const unsigned role = rfl(P.role[b - NCOMM]);
    const int type = role & 15;
    if (type == R_UNPACK) {
        const int wv = rfl(threadIdx.x >> 6), nun = P.nunpack & 255;
        const unsigned call = rfl(ld_agent(cc + wv)) + 1u;
        unpack_role(P, myws, (unsigned)(size_t)smem + wv * 512, tstart, call, (int)(role >> 4) + nun * wv, nun * 8);
        if (wv == 7) STAMP(4);
        if ((threadIdx.x & 63) == 0) st_agent(cc + wv, call);
    } else if (type != R_IDLE) {
        const unsigned call = rfl(ld_agent(cc)) + 1u;
        sleep_until(tstart + (P.tlead >> 16));
        if (threadIdx.x == 0) STAMP(0);
        const int fs = (role >> 4) & 255, cg = (role >> 12) & 7, ch = (role >> 15) & 1, win = (role >> 18) & 3;
        const bool packed = (role >> 16) & 1;
        const int po = (fs >> 1) + rfl(threadIdx.x >> 6);
        const GAS char* pk = (const GAS char*)P.ws[cg] + (size_t)(call & 1) * BUF_BYTES + pko_off(cg, po - (int)((P.nunpack >> 16) & 255));
        asm volatile("" : "+s"(pk));
        unsigned ea;
        if (type == R_T128) ea = tile_role<8, 2, AGK_RA>(P, smem, call, fs, cg, 0, win, role >> 28, packed);
        else if (type == R_T128H) ea = tile_role<8, 1, AGK_RH>(P, smem, call, fs, cg, ch, win, role >> 28, false);
        else if (type == R_T64W) ea = tile_role<4, 2, AGK_RW>(P, smem, call, fs, cg, 0, win, role >> 28, packed);
        else ea = tile_role<4, 1, AGK_RQ>(P, smem, call, fs, cg, ch, win, role >> 28, false);
        if (ea) {
            if (threadIdx.x == 0) STAMP(5);
            out_role(P, ea, type == R_T128 ? 4096 : 8192, call, pk, cg, po, win != 0);
            if (threadIdx.x == 0) STAMP(3);
        }
        if (threadIdx.x == 256) st_agent(cc, call);
    }
}

}

#define TUNE(i, dflt) (dflt)

extern "C" void kernel_launch(int rank, int nranks, ncclComm_t comm,
                              hipStream_t stream,
                              void* const* d_in, const int* in_sizes, int n_in,
                              void* d_out, int out_size,
                              const int* in_dtypes, int out_dtype,
                              void* const* d_out_all,
                              void* const* d_ws_all, size_t ws_bytes) {
    using namespace agk;
    (void)comm; (void)in_dtypes; (void)out_dtype; (void)d_out;
    if (nranks != NR || (unsigned)rank >= (unsigned)NR || n_in < 2 || ws_bytes < WS_NEED ||
        in_sizes[0] != MSH * KD || in_sizes[1] != KD * NSH || out_size != NR * MSH * NSH) {
        fprintf(stderr, "ag_gemm: unexpected problem shape\n");
        return;
    }
    Params P;
    P.x = (const char*)d_in[0];
    P.w = (const char*)d_in[1];
    P.rank = rank;
    for (int j = 0; j < NR; ++j) { P.ws[j] = (char*)d_ws_all[rank ^ j]; P.out[j] = (char*)d_out_all[rank ^ j]; }
    int sega = TUNE(0, 11);
    int blate = TUNE(1, 7);
    int wlead = TUNE(2, 4);
    int rt_b = TUNE(3, 1), rt_c = TUNE(4, 1), rt_a = 0, rt_w = 0;
    int split_c = TUNE(7, 5);
    int split_a = TUNE(8, 0);
    int small_b = TUNE(9, 0);
    const int segb = HS - sega;
    const int na_t = (3 * sega + 7) / 8, nc_t = (4 * AS + 7) / 8, nb_t = (3 * segb + 7) / 8, no_t = 16 - na_t - nc_t - nb_t;
    const int ne_t = no_t + na_t;
    bool ok = no_t >= 0;
    int n = 0, own = 0;
    for (int i = 0; i < no_t * 8; ++i) P.strip[n++] = (unsigned char)(own++);
    for (int s = 0; s < sega; ++s) for (int j = 1; j < 4; ++j) P.strip[n++] = (unsigned char)((j << 5) | s);
    while (n % 8) P.strip[n++] = (unsigned char)(own++);
    for (int s = HS; s < NSTRIP; ++s) for (int j = 4; j < 8; ++j) P.strip[n++] = (unsigned char)((j << 5) | s);
    while (n % 8) P.strip[n++] = (unsigned char)(own++);
    for (int s = sega; s < HS; ++s) for (int j = 1; j < 4; ++j) P.strip[n++] = (unsigned char)((j << 5) | s);
    while (n % 8) P.strip[n++] = (unsigned char)(own++);
    ok = ok && n == 128 && own == HS;
    int q = 0;
    const int t_first = TUNE(16, 160) * 16, t_pk = TUNE(17, 166), t_raw = TUNE(18, 232);
    const int skew_b = TUNE(20, 16), skew_c = TUNE(21, 12);
    const int opk = TUNE(19, 3);
    const int op1 = opk & 2 ? ne_t + nc_t - TUNE(6, 0) : ne_t, op0 = !(opk & 1) ? ne_t : op1 > NP_OUT / 4 ? op1 - NP_OUT / 4 : 0;
    const int n_slow = TUNE(25, 40), t_slow = TUNE(15, 182);
    int tq = t_first;
    for (int i = 0; i < 2 * WPL * NRND; ++i) P.ord[i] = 0;
    auto emit = [&](unsigned e) { if (q < NP_IN + NP_XI) P.ord[q < NP_IN ? ord_at(0, q) : ord_at(1, q - NP_IN)] = e | (unsigned)(tq >> 4) << 16; ++q; tq += (e & 512u) ? t_raw : tq < t_first + n_slow * t_slow ? t_slow : t_pk; };
    int nextk[NSTRIP] = {0};
    auto xchunk = [&](int s0, int s1, int rt, int skew) {
        for (int i = s0; i < s1; ++i) {
            int b = -1;
            for (int s = s0; s < s1; ++s) if (nextk[s] < NKC && (b < 0 || nextk[s] * 64 + s * skew < nextk[b] * 64 + b * skew)) b = s;
            emit((unsigned)nextk[b] | (b << 4) | (nextk[b] >= NKC - rt ? 512u : 0u));
            ++nextk[b];
        }
    };
    auto wchunk = [&](int kc) { for (int i = 0; i < 8; ++i) emit((unsigned)(kc * 8 + i) | 1024u | (kc >= NKC - rt_w ? 512u : 0u)); };
    for (int ka = 0, kb = 0; kb < NKC; ) {
        if (ka < NKC && kb * NKC >= ka * (NKC - blate)) { xchunk(0, sega, rt_a, 0); ++ka; }
        else { xchunk(sega, HS, rt_b, skew_b); ++kb; }
    }
    ok = ok && q == NP_IN;
    tq = t_first;
    for (int k = -wlead; k < NKC; ++k) {
        if (k + wlead < NKC) wchunk(k + wlead);
        if (k >= 0) xchunk(HS, NSTRIP, rt_c, skew_c);
    }
    ok = ok && q == NP_IN + NP_XI;
    int fill[8] = {0, 0, 0, 0, 0, 0, 0, 0};
    unsigned role[NWG];
    for (int i = 0; i < NWG; ++i) role[i] = R_IDLE;
    auto put = [&](int k, unsigned v) { if (fill[k] < NWG / 8) role[fill[k]++ * 8 + k] = v; else ok = false; };
    auto code = [&](int type, int fs, int cg, int ch) { return (unsigned)type | (fs << 4) | (cg << 12) | (ch << 15); };
    for (int i = 0; i < NCOMM; ++i) put(i & 7, R_COMM | (i << 4));
    unsigned adds = 0;
    for (int cg = 0; cg < 8; ++cg) {
        const bool rem = cg >= 4;
        for (int r = 0; r < 16; ++r) {
            const int k = (cg & 3) + 4 * (r & 1);
            const bool early = r < ne_t, last = r >= ne_t + nc_t;
            const bool split = last ? rem || r - ne_t - nc_t >= TUNE(29, 3) : (rem ? (early ? r >= ne_t - split_a : TUNE(5, 1) ? r - ne_t >= nc_t - split_c : r - ne_t < split_c) : (!early && r - ne_t < TUNE(22, 0)));
            const bool small = last && r - ne_t - nc_t < (rem ? small_b >> 4 : small_b & 15);
            const unsigned win = (unsigned)(!rem ? 0 : early ? TUNE(14, 2) : last ? TUNE(12, 0) : split ? TUNE(13, 0) : TUNE(11, 0)) << 18 |
                                 (rem && r >= op0 && r < op1 ? 1u << 16 : 0u);
            int nw = 0;
            auto rot = [&](int who, int of) { return (unsigned)(TUNE(26, 1) ? (of == 2 ? (who & 1) * 2 + ((r >> 1) & 1) : (who ^ (((r >> 1) & 1) * 2)) & 3) : 0) << 28; };
            if (small) for (int h = 0; h < 4; ++h) { put(k, rot(rem * 2 + (h & 1), 4) | win | code(R_T64H, r * 8 + (h >> 1) * 4, cg, h & 1)); nw += 4; }
            else if (split && (TUNE(23, 3) >> (last ? 0 : 1) & 1)) for (int h = 0; h < 2; ++h) { put(k, (unsigned)((rem * 2 + h * TUNE(24, 1)) ^ ((r >> 1) & 1)) << 28 | win | code(R_T64W, r * 8 + h * 4, cg, 0)); nw += 4; }
            else if (split) for (int ch = 0; ch < 2; ++ch) { put(k, rot(rem * 2 + ch, 4) | win | code(R_T128H, r * 8, cg, ch)); nw += 4; }
            else { put(k, rot(rem, 2) | win | code(R_T128, r * 8, cg, 0)); nw += 4; }
            if (cg == 4 && !(win >> 16 & 1)) adds += nw;
            ok = ok && !((win >> 16 & 1) && split && !(TUNE(23, 3) & 2));
        }
    }
    unsigned nun = 0;
    for (int i = 0; i < NWG; ++i) if (role[i] == R_IDLE && nun < (unsigned)TUNE(10, 64)) role[i] = R_UNPACK | (nun++ << 4);
    for (int i = NCOMM; i < NWG; ++i) P.role[i - NCOMM] = role[i];
    P.tlead = TUNE(27, 24) | TUNE(28, 250) << 16;
    P.out_adds = adds; P.tpoll = TUNE(30, 100) | TUNE(31, 1) << 16; P.nunpack = nun | (unsigned)(op1 - op0) * 4 << 8 | (unsigned)op0 * 4 << 16;
    ok = ok && nun > 0;
    if (!ok) { fprintf(stderr, "ag_gemm: role table overflow\n"); return; }
    hipLaunchKernelGGL(ag_gemm_kernel, dim3(NWG), dim3(512), 0, stream, P);
    HIP_CHECK(hipGetLastError());
}
```
